```python
import numpy as np

kernel_name = "gpu_gemm_ar_m4096_k4096_n32768_bf16_none"


ABSMAX_RTOL = 0.008

COMPUTE_DTYPE = "bfloat16"
EPILOGUE = "none"
IN_QUANT = None
OUT_QUANT = None
IN_DTYPES = {"x": "bfloat16", "w_mat": "bfloat16"}
WORLD_SIZE = 8

def _to_bf16(x):
    u = np.ascontiguousarray(x, dtype=np.float32).view(np.uint32)
    return ((u + 0x7FFF + ((u >> 16) & 1)) & 0xFFFF0000).view(np.float32)

def setup_inputs():

    x = _to_bf16(np.random.randn(4096, 4096).astype(np.float32))
    w_mat = _to_bf16((np.random.randn(4096, 32768) / np.sqrt(4096)).astype(np.float32))
    return {"x": x, "w_mat": w_mat}

def _epilogue(y):
    return y

def reference(x, w_mat):
    y = (x @ w_mat).astype(np.float32)
    return _epilogue(y)
```

```cpp
#include <hip/hip_runtime.h>
#include <rccl/rccl.h>
#include <cstdio>

#define HIP_CHECK(expr) do { hipError_t _e = (expr); if (_e != hipSuccess) { \
    fprintf(stderr, "HIP error %s at %s:%d\n", hipGetErrorString(_e), __FILE__, __LINE__); \
    return; } } while (0)

typedef unsigned u32;
typedef unsigned long long u64;
typedef u32 u32x4 __attribute__((ext_vector_type(4)));
typedef u32 u32x2 __attribute__((ext_vector_type(2)));
typedef unsigned short u16x2 __attribute__((ext_vector_type(2)));
typedef short bf16x8 __attribute__((ext_vector_type(8)));
typedef float f32x4 __attribute__((ext_vector_type(4)));
#define LAS __attribute__((address_space(3)))
#define GAS __attribute__((address_space(1)))
typedef __amdgpu_buffer_rsrc_t rsrc_t;

constexpr int NR = 8;
#define NPULL_PER_XCD 7
#define PULL_WAVES_PER_PEER 16
constexpr int NGEMM_PER_XCD = 16;
constexpr int NPULL = NPULL_PER_XCD * 8;
constexpr int NENC_ONLY_PER_XCD = 32 - NGEMM_PER_XCD - NPULL_PER_XCD;
constexpr int NIN = 1024, NYB = 4096, NSTREAM = NIN + NYB;
constexpr int NTASK = 16 * 288;
constexpr u64 DEADLINE = 100000000ull;

constexpr size_t OFF_CALL = 0;
constexpr size_t OFF_TMO = 131072;
constexpr size_t OFF_RDY = OFF_TMO + 256;
constexpr size_t OFF_GRAN = 196608;
constexpr size_t OFF_DBG = 524288;
constexpr size_t OFF_XF = 1ull << 20;
constexpr size_t OFF_WF = OFF_XF + (32ull << 20);
constexpr size_t OFF_SX = OFF_WF + (32ull << 20);
constexpr size_t OFF_SW = OFF_SX + (4ull << 20);
constexpr size_t OFF_SY = OFF_SW + (32ull << 20);
constexpr size_t WS_NEED = OFF_SY + (32ull << 20);

constexpr int SLOT = 8192, FIXB = 5120, VARDW = 768;
constexpr int STAGE_BYTES = 131072, WLDS = 3136;
constexpr int AUX_SYS = 17, AUX_DEV = 16;

struct Params {
    const char* x; const char* w; char* out; char* ws[NR];
    int rank; int pad;
    u32 ord2tile[64];
    u32 tile2ord[64];
    u32 wgtile[64];
};
__device__ __forceinline__ u32 tab8(const u32 (&t)[64], u32 i) { return (t[(i >> 2) & 63u] >> ((i & 3u) * 8u)) & 0xffu; }

__device__ __forceinline__ u32 rfl(u32 x) { return (u32)__builtin_amdgcn_readfirstlane((int)x); }
__device__ __forceinline__ rsrc_t mk_rsrc(const void* p, u32 bytes) {
    const u64 a = (u64)p; const u64 u = ((u64)rfl((u32)(a >> 32)) << 32) | rfl((u32)a);
    return __builtin_amdgcn_make_buffer_rsrc((void*)u, (short)0, (int)rfl(bytes), 0x00020000);
}
__device__ __forceinline__ u32 vperm(u32 a, u32 b, u32 s) { return __builtin_amdgcn_perm(a, b, s); }
__device__ __forceinline__ u32 wave_incl_scan(u32 x) {
    x += (u32)__builtin_amdgcn_update_dpp(0, (int)x, 0x111, 0xf, 0xf, false);
    x += (u32)__builtin_amdgcn_update_dpp(0, (int)x, 0x112, 0xf, 0xf, false);
    x += (u32)__builtin_amdgcn_update_dpp(0, (int)x, 0x114, 0xf, 0xf, false);
    x += (u32)__builtin_amdgcn_update_dpp(0, (int)x, 0x118, 0xf, 0xf, false);
    x += (u32)__builtin_amdgcn_update_dpp(0, (int)x, 0x142, 0xa, 0xf, false);
    x += (u32)__builtin_amdgcn_update_dpp(0, (int)x, 0x143, 0xc, 0xf, false);
    return x;
}
__device__ __forceinline__ u32 pk_sub_min(u32 tt, u32 r) {
    u16x2 a = __builtin_bit_cast(u16x2, tt) - __builtin_bit_cast(u16x2, r);
    const u16x2 lim = {0x0bff, 0x0bff};
    return __builtin_bit_cast(u32, __builtin_elementwise_min(a, lim));
}

__device__ __forceinline__ u32 encode_block(const u32 (&v)[8][4], rsrc_t slot, LAS u32* lds, int lane) {
    *(LAS u32x4*)(lds + lane * 4) = (u32x4){0, 0, 0, 0};
    *(LAS u32x4*)(lds + 256 + lane * 4) = (u32x4){0, 0, 0, 0};
    u32 s = 0;
#pragma unroll
    for (int g = 0; g < 8; ++g) s = __builtin_amdgcn_udot4(__builtin_amdgcn_alignbit(v[g][0], v[g][0], 31), 0x01000100u, s, false);
    int T = (int)((u32)__builtin_amdgcn_readlane((int)wave_incl_scan(s), 63) >> 10) + 3;
    T = T < 10 ? 10 : (T > 255 ? 255 : T);
    const u32 tt = ((u32)T * 256u + 255u) * 0x10001u;
    u32 sm[16], U[16], l1[4] = {0, 0, 0, 0}, z3[16], c3[17];
    c3[0] = 0;
#pragma unroll
    for (int k = 0; k < 16; ++k) {
        const u32 a = v[k >> 1][(k & 1) * 2], b = v[k >> 1][(k & 1) * 2 + 1];
        const u32 r0 = __builtin_amdgcn_alignbit(a, a, 31), r1 = __builtin_amdgcn_alignbit(b, b, 31);
        sm[k] = vperm(r1, r0, 0x06040200u);
        U[k] = vperm(pk_sub_min(tt, r1), pk_sub_min(tt, r0), 0x07050301u);
        l1[k >> 2] |= vperm(0u, 0x03020100u, U[k]) << (2 * (k & 3));
        z3[k] = vperm(0x83038303u, 0x80008003u, U[k]) & 0x03030303u;
        c3[k + 1] = __builtin_amdgcn_sad_u8(z3[k], 0u, c3[k]);
    }
    const u32 incl = wave_incl_scan(c3[16]);
    const u32 p0 = incl - c3[16];
    const u32 xoff = ((u32)__builtin_amdgcn_readlane((int)incl, 63) + 31u) >> 5;
    u32 nx = 0;
#pragma unroll
    for (int g = 0; g < 8; ++g) {
        u32 x2[2];
#pragma unroll
        for (int h = 0; h < 2; ++h) {
            const int k = 2 * g + h;
            const u32 d8 = U[k] - (vperm(0x83038303u, 0x83028100u, U[k]) & 0x03030303u);
            const u32 d = d8 & 0x07070707u;
            const u32 rk = __umul24(z3[k], 0x010101u);
            x2[h] = (d & 0xffu) | (((d >> 8) & 0xffu) << (rk & 0xffu)) | (((d >> 16) & 0xffu) << ((rk >> 8) & 0xffu)) | ((d >> 24) << ((rk >> 16) & 0xffu));
            if (__builtin_amdgcn_ballot_w64((d8 & 0x08080808u) != 0u) != 0ull) {
                u32 l22 = (u32)lane << 22;
                asm volatile("" : "+v"(l22));
#pragma unroll
                for (int t = 0; t < 4; ++t) {
                    const bool e = (d8 >> (8 * t + 3)) & 1u;
                    const u64 m = __builtin_amdgcn_ballot_w64(e);
                    const u32 at = xoff + nx + __builtin_amdgcn_mbcnt_hi((u32)(m >> 32), __builtin_amdgcn_mbcnt_lo((u32)m, 0u));
                    const u32 val = (v[g][2 * h + (t >> 1)] >> (16 * (t & 1))) & 0xffffu;
                    if (e && at < (u32)VARDW) lds[at] = l22 | ((u32)(g * 8 + h * 4 + t) << 16) | val;
                    nx += (u32)__builtin_popcountll(m);
                }
            }
        }
        const u32 x = x2[0] | (x2[1] << (c3[2 * g + 1] - c3[2 * g]));
        const u32 p = p0 + c3[2 * g];
        const u64 y = (u64)x << (p & 31u);
        __hip_atomic_fetch_or(lds + (p >> 5), (u32)y, __ATOMIC_RELAXED, __HIP_MEMORY_SCOPE_WORKGROUP);
        __hip_atomic_fetch_or(lds + (p >> 5) + 1, (u32)(y >> 32), __ATOMIC_RELAXED, __HIP_MEMORY_SCOPE_WORKGROUP);
    }
    const u32 vd = xoff + nx;
    if (vd > (u32)VARDW) {
#pragma unroll
        for (int g = 0; g < 8; ++g)
            __builtin_amdgcn_raw_buffer_store_b128((u32x4){v[g][0], v[g][1], v[g][2], v[g][3]}, slot, lane * 16 + g * 1024, 0, AUX_SYS);
        return 0x80000000u;
    }
#pragma unroll
    for (int q = 0; q < 4; ++q)
        __builtin_amdgcn_raw_buffer_store_b128((u32x4){sm[4 * q], sm[4 * q + 1], sm[4 * q + 2], sm[4 * q + 3]}, slot, lane * 16 + q * 1024, 0, AUX_SYS);
    __builtin_amdgcn_raw_buffer_store_b128((u32x4){l1[0], l1[1], l1[2], l1[3]}, slot, lane * 16 + 4096, 0, AUX_SYS);
    __builtin_amdgcn_fence(__ATOMIC_ACQ_REL, "wavefront");
    for (u32 c = 0; c < vd; c += 256) {
        const u32x4 t = *(volatile LAS u32x4*)(lds + c + lane * 4);
        if (c + lane * 4 < vd) __builtin_amdgcn_raw_buffer_store_b128(t, slot, FIXB + (int)c * 4 + lane * 16, 0, AUX_SYS);
    }
    __builtin_amdgcn_fence(__ATOMIC_ACQ_REL, "wavefront");
    return (u32)T | (xoff << 8) | (nx << 20);
}

struct Dest { rsrc_t r; u32 A, B, C, LH; };
__device__ __forceinline__ void store_group(const Dest& d, u32 loff, int g, u32x4 x) {
    __builtin_amdgcn_raw_buffer_store_b128(x, d.r, (int)loff, (int)((g & 1 ? d.A : 0u) + (g & 2 ? d.B : 0u) + (g & 4 ? d.C : 0u)), AUX_DEV);
}
__device__ __forceinline__ u32 info_bytes(u32 info) {
    const u32 vd = ((info >> 8) & 0xfffu) + ((info >> 20) & 0x7ffu);
    return (info >> 31) ? (u32)SLOT : (u32)FIXB + ((vd > (u32)VARDW ? (u32)VARDW : vd) * 4u + 15u) / 16u * 16u;
}

template <class Pre>
__device__ __forceinline__ void decode_block(const u32x4 (&d)[6], u32 info, bool ok, rsrc_t src, const Dest& dst, LAS u32* lds, int lane, Pre pre) {
    const u32 loff = (u32)(lane >> 4) * dst.LH + (u32)(lane & 15) * 16u;
    u32x4 o[8];
    u32 xoff = 0, vd = 0;
    if (!ok || (info >> 31)) {
        o[6] = __builtin_amdgcn_raw_buffer_load_b128(src, lane * 16 + 6144, 0, AUX_SYS);
        o[7] = __builtin_amdgcn_raw_buffer_load_b128(src, lane * 16 + 7168, 0, AUX_SYS);
        const u32 nanv = ok ? 0u : 0x7fc07fc0u;
#pragma unroll
        for (int g = 0; g < 6; ++g) o[g] = d[g] | nanv;
        o[6] |= nanv; o[7] |= nanv;
    } else {
        const u32 T = info & 0xffu;
        xoff = (info >> 8) & 0xfffu;
        vd = xoff + ((info >> 20) & 0x7ffu);
        vd = vd > (u32)VARDW ? (u32)VARDW : vd;
        *(volatile LAS u32x4*)(lds + lane * 4) = d[5];
        for (u32 c = 256; c < vd; c += 256)
            *(volatile LAS u32x4*)(lds + c + lane * 4) = __builtin_amdgcn_raw_buffer_load_b128(src, FIXB + (int)c * 4 + lane * 16, 0, AUX_SYS);
        __builtin_amdgcn_fence(__ATOMIC_ACQ_REL, "wavefront");
        const u32 tb = T * 0x01010101u, t1 = tb - 0x03020100u, t2lo = tb - 0x06050400u, t2hi = tb - 0x0a090807u;
        u32 c4[16], z1[16], cn[17];
        cn[0] = 0;
#pragma unroll
        for (int k = 0; k < 16; ++k) {
            c4[k] = ((d[4][k >> 2] >> (2 * (k & 3))) & 0x03030303u) | 0x04040404u;
            z1[k] = vperm(0x00000001u, 0u, c4[k]);
            cn[k + 1] = cn[k] + (u32)__builtin_popcount(z1[k]);
        }
        const u32 n0 = wave_incl_scan(cn[16]) - cn[16];
        u32 wv[8];
#pragma unroll
        for (int g = 0; g < 8; ++g) {
            const u32 p = (n0 + cn[2 * g]) * 3u;
            const u32 lo = lds[p >> 5], hi = lds[(p >> 5) + 1];
            wv[g] = __builtin_amdgcn_alignbit(hi, lo, p & 31u);
        }
#pragma unroll
        for (int k = 0; k < 16; ++k) {
            const int g = k >> 1, h = k & 1;
            const u32 x = h ? __builtin_amdgcn_ubfe(wv[g], (cn[k] - cn[k - 1]) * 3u, 12u) : (wv[g] & 0xfffu);
            const u32 y = (x | (x << 10)) & 0x003f003fu;
            const u32 z = (y | (y << 5)) & 0x07070707u;
            const u32 e2 = vperm(t2hi, t2lo, z);
            const u32 rank = __umul24(z1[k], 0x010101u) << 8;
            const u32 m = vperm(0x000000ffu, 0u, c4[k]);
            const u32 e = vperm(t1, e2, (rank & m) | (c4[k] & ~m));
            const u32 smk = d[k >> 2][k & 3];
            const u32 r0 = vperm(e, smk, 0x05010400u), r1 = vperm(e, smk, 0x07030602u);
            o[g][2 * h] = __builtin_amdgcn_alignbit(r0, r0, 1);
            o[g][2 * h + 1] = __builtin_amdgcn_alignbit(r1, r1, 1);
        }
    }
    pre();
#pragma unroll
    for (int g = 0; g < 8; ++g) store_group(dst, loff, g, o[g]);
    __builtin_amdgcn_fence(__ATOMIC_ACQ_REL, "wavefront");
    u32 c = xoff + (u32)lane;
    do {
        const u32 en = lds[c < (u32)VARDW ? c : 0u], l2 = (en >> 22) & 63u, g = (en >> 19) & 7u;
        const u32 off = (l2 >> 4) * dst.LH + (l2 & 15u) * 16u + (g & 1u) * dst.A + ((g >> 1) & 1u) * dst.B + (g >> 2) * dst.C + ((en >> 16) & 7u) * 2u;
        __builtin_amdgcn_raw_buffer_store_b16((unsigned short)en, dst.r, c < vd ? (int)off : 0x7ffffff0, 0, AUX_DEV);
        c += 64u; xoff += 64u;
    } while (xoff < vd);
    __builtin_amdgcn_fence(__ATOMIC_ACQ_REL, "wavefront");
}

__device__ __forceinline__ Dest dest_x(char* ws, u32 s, u32 b, u32 src) {
    return Dest{mk_rsrc(ws + OFF_XF + (size_t)(((s * 64u + src * 8u + (b & 1u) * 4u) * 32u + (b >> 1) * 2u) * 1024u), 3u * 32768u + 2048u), 1024u, 32768u, 65536u, 256u};
}
__device__ __forceinline__ Dest dest_w(char* ws, u32 s, u32 b, u32 src) {
    return Dest{mk_rsrc(ws + OFF_WF + (size_t)(((s * 64u + src * 8u + (b >> 2)) * 32u + (b & 1u) * 8u + ((b >> 1) & 1u)) * 1024u), 16384u + 6144u + 1024u), 2048u, 4096u, 16384u, 256u};
}
__device__ __forceinline__ Dest dest_y(char* out, u32 tile, u32 b, u32 src) {
    const u32 row = (tile >> 4) * 256u + (b & 1u) * 128u + ((b >> 1) & 3u) * 32u, col = src * 4096u + (tile & 15u) * 256u + (b >> 3) * 128u;
    return Dest{mk_rsrc(out + (size_t)row * 65536u + col * 2u, 31u * 65536u + 256u), 65536u, 131072u, 16u * 65536u, 4u * 65536u};
}

__device__ __forceinline__ bool give_up(GAS u32* tmo, u32 call, u64 t0) {
    if (rfl(__hip_atomic_load(tmo, __ATOMIC_RELAXED, __HIP_MEMORY_SCOPE_AGENT)) == call || __builtin_amdgcn_s_memrealtime() - t0 > DEADLINE) {
        __hip_atomic_store(tmo, call, __ATOMIC_RELAXED, __HIP_MEMORY_SCOPE_AGENT);
        return true;
    }
    return false;
}

#define STAMP_AT(i) do { } while (0)

__global__ void __launch_bounds__(512, 2) gemm_ar_fused(const Params p) {
    __shared__ __attribute__((aligned(16))) char smem[STAGE_BYTES + 8 * WLDS + 16];
    LAS char* lds = (LAS char*)smem;
    const int tid = threadIdx.x, wid = (int)rfl((u32)tid >> 6), lane = tid & 63;
    const u32 rank = (u32)p.rank, xcd = blockIdx.x & 7u, idx = blockIdx.x >> 3;
    char* ws = p.ws[rank & 7u];
    LAS u32* wl = (LAS u32*)(lds + STAGE_BYTES + wid * WLDS);
    GAS u32* tmo = (GAS u32*)(ws + OFF_TMO);
    STAMP_AT(0);
    GAS u32* callp = (GAS u32*)(ws + OFF_CALL + (blockIdx.x * 8u + wid) * 64u);
    const u32 callv = __hip_atomic_load(callp, __ATOMIC_RELAXED, __HIP_MEMORY_SCOPE_AGENT);

    if (idx >= (u32)NGEMM_PER_XCD && idx < (u32)(NGEMM_PER_XCD + NPULL_PER_XCD)) {
        const u32 pw = (u32)wid * NPULL + (idx - NGEMM_PER_XCD) * 8u + xcd;
        if (pw >= 7u * PULL_WAVES_PER_PEER) return;
        const u32 call = rfl(callv) + 1u;
        if (lane == 0) __hip_atomic_store(callp, call, __ATOMIC_RELAXED, __HIP_MEMORY_SCOPE_AGENT);
        const u32 src = (rank + 1u + pw % 7u) & 7u;
        const char* pws = p.ws[src];
        GAS u64* gran = (GAS u64*)(ws + OFF_GRAN) + (size_t)src * NSTREAM;
        auto slot_of = [&](u32 t) -> const char* {
            if (t >= (u32)NIN) return pws + OFF_SY + (size_t)(t - NIN) * SLOT;
            const u32 i = (t >> 6) * 32u + (t & 31u);
            return (t & 32u) ? pws + OFF_SW + (size_t)(rank * 512u + i) * SLOT : pws + OFF_SX + (size_t)i * SLOT;
        };
        auto wait_gran = [&](u32 t, u32& info) -> bool {
            const u64 t0 = __builtin_amdgcn_s_memrealtime();
            for (;;) {
                const u64 g = __hip_atomic_load(gran + t, __ATOMIC_RELAXED, __HIP_MEMORY_SCOPE_SYSTEM);
                if (rfl((u32)g) == call) { info = rfl((u32)(g >> 32)); return true; }
                if (give_up(tmo, call, t0)) { info = 0; return false; }
                __builtin_amdgcn_s_sleep(4);
            }
        };
        u32x4 cur[6], nxt[6];
        auto issue = [&](u32x4 (&d)[6], u32 t, u32 info, bool ok) {
            const rsrc_t r = mk_rsrc(slot_of(t), ok ? info_bytes(info) : 0u);
#pragma unroll
            for (int q = 0; q < 6; ++q) d[q] = __builtin_amdgcn_raw_buffer_load_b128(r, lane * 16 + q * 1024, 0, AUX_SYS);
        };
        u32 t = pw / 7u, info; bool ok = wait_gran(t, info);
        STAMP_AT(1);
        asm volatile("" ::: "memory");
        issue(cur, t, info, ok);
        u64 gn = __hip_atomic_load(gran + t + PULL_WAVES_PER_PEER, __ATOMIC_RELAXED, __HIP_MEMORY_SCOPE_SYSTEM);
        asm volatile("" : "+v"(cur[0]), "+v"(cur[1]), "+v"(cur[2]), "+v"(cur[3]), "+v"(cur[4]), "+v"(cur[5]), "+v"(gn) :: "memory");
        u32 sig = 0xffffffffu;
        for (;;) {
            const u32 tn = t + PULL_WAVES_PER_PEER;
            asm volatile("" : "+v"(cur[0]), "+v"(cur[1]), "+v"(cur[2]), "+v"(cur[3]), "+v"(cur[4]), "+v"(cur[5]) :: "memory");
            const bool pre = tn < (u32)NSTREAM && rfl((u32)gn) == call;
            u32 infon = rfl((u32)(gn >> 32)); bool okn = true;
            if (pre) {
                issue(nxt, tn, infon, true);
                const u32 t2 = tn + PULL_WAVES_PER_PEER;
                gn = __hip_atomic_load(gran + (t2 < (u32)NSTREAM ? t2 : tn), __ATOMIC_RELAXED, __HIP_MEMORY_SCOPE_SYSTEM);
            }
            asm volatile("" ::: "memory");
            const Dest dst = t >= (u32)NIN ? dest_y(p.out, tab8(p.ord2tile, (t - NIN) >> 4), (t - NIN) & 15u, src)
                           : ((t & 32u) ? dest_w(ws, t >> 6, t & 31u, src) : dest_x(ws, t >> 6, t & 31u, src));
            auto signal_prev = [&]() {
                if (sig != 0xffffffffu) {
                    if (pre) asm volatile("s_waitcnt vmcnt(7)" ::: "memory"); else asm volatile("s_waitcnt vmcnt(0)" ::: "memory");
                    if (lane == 0) __hip_atomic_fetch_add((GAS u32*)(ws + OFF_RDY + sig * 128u), 1u, __ATOMIC_RELAXED, __HIP_MEMORY_SCOPE_AGENT);
                }
            };
            decode_block(cur, info, ok, mk_rsrc(slot_of(t), ok ? info_bytes(info) : 0u), dst, wl, lane, signal_prev);
            sig = t < (u32)NIN ? (t >> 5) : 0xffffffffu;
            if (tn >= (u32)NSTREAM) break;
            if (!pre) {
                if (sig != 0xffffffffu) {
                    asm volatile("s_waitcnt vmcnt(0)" ::: "memory");
                    if (lane == 0) __hip_atomic_fetch_add((GAS u32*)(ws + OFF_RDY + sig * 128u), 1u, __ATOMIC_RELAXED, __HIP_MEMORY_SCOPE_AGENT);
                    sig = 0xffffffffu;
                }
                okn = wait_gran(tn, infon);
                asm volatile("" ::: "memory");
                issue(nxt, tn, infon, okn);
                const u32 t2 = tn + PULL_WAVES_PER_PEER;
                gn = __hip_atomic_load(gran + (t2 < (u32)NSTREAM ? t2 : tn), __ATOMIC_RELAXED, __HIP_MEMORY_SCOPE_SYSTEM);
                asm volatile("" : "+v"(nxt[0]), "+v"(nxt[1]), "+v"(nxt[2]), "+v"(nxt[3]), "+v"(nxt[4]), "+v"(nxt[5]), "+v"(gn) :: "memory");
            }
            cur[0] = nxt[0]; cur[1] = nxt[1]; cur[2] = nxt[2]; cur[3] = nxt[3]; cur[4] = nxt[4]; cur[5] = nxt[5];
            t = tn; info = infon; ok = okn;
        }
        STAMP_AT(3);
        return;
    }

    {
        const bool isg = idx < (u32)NGEMM_PER_XCD;
        const u32 eo = ((idx - NGEMM_PER_XCD - NPULL_PER_XCD) * 8u + xcd) * 8u + wid;
        constexpr u32 NEO = NENC_ONLY_PER_XCD * 64, NALL = NEO + NGEMM_PER_XCD * 64;
        u32 task = isg ? NEO + (idx * 8u + xcd) * 8u + wid : eo;
        for (u32 it = 0; task < (u32)NTASK; ++it) {
            const u32 s = task / 288u, u = task % 288u, b = u & 31u;
            u32 v[8][4];
            if (u < 32u) {
                const rsrc_t r = mk_rsrc(p.x + (size_t)(s * 256u + (b >> 1) * 16u) * 1024u + (b & 1u) * 512u, 15u * 1024u + 512u);
#pragma unroll
                for (int g = 0; g < 8; ++g) {
                    const u32x4 a = __builtin_amdgcn_raw_buffer_load_b128(r, (lane & 15) * 1024 + (lane >> 4) * 16 + g * 64, 0, 0);
                    v[g][0] = a[0]; v[g][1] = a[1]; v[g][2] = a[2]; v[g][3] = a[3];
                }
            } else {
                const u32 dd = (rank + (u >> 5)) & 7u;
                const rsrc_t r = mk_rsrc(p.w + (size_t)((b >> 1) * 32u) * 65536u + (dd * 4096u + s * 256u + (b & 1u) * 128u) * 2u, 31u * 65536u + 256u);
                u32x4 a[8];
#pragma unroll
                for (int i = 0; i < 8; ++i) a[i] = __builtin_amdgcn_raw_buffer_load_b128(r, (lane >> 4) * 8 * 65536 + (lane & 15) * 16, i * 65536, 0);
#pragma unroll
                for (int j = 0; j < 8; ++j)
#pragma unroll
                    for (int q = 0; q < 4; ++q) v[j][q] = vperm(a[2 * q + 1][j >> 1], a[2 * q][j >> 1], (j & 1) ? 0x07060302u : 0x05040100u);
            }
            const bool own = u < 32u || u >= 256u, send = u < 256u;
            u32 info = 0;
            if (send) {
                const u32 dd = (rank + (u >> 5)) & 7u;
                char* slot = u < 32u ? ws + OFF_SX + (size_t)(s * 32u + b) * SLOT : ws + OFF_SW + (size_t)(dd * 512u + s * 32u + b) * SLOT;
                info = encode_block(v, mk_rsrc(slot, SLOT), wl, lane);
            }
            if (own) {
                const Dest dst = u < 32u ? dest_x(ws, s, b, rank) : dest_w(ws, s, b, rank);
#pragma unroll
                for (int g = 0; g < 8; ++g) store_group(dst, (u32)lane * 16u, g, (u32x4){v[g][0], v[g][1], v[g][2], v[g][3]});
            }
            const u32 call = rfl(callv) + 1u;
            asm volatile("s_waitcnt vmcnt(0)" ::: "memory");
            if (send) {
                const u32 t = s * 64u + (u < 32u ? 0u : 32u) + b;
                const u32 k = u < 32u ? (u32)lane + 1u : (u >> 5);
                if (lane < (u < 32u ? 7 : 1))
                    __hip_atomic_store((GAS u64*)(p.ws[(rank + k) & 7u] + OFF_GRAN) + (size_t)rank * NSTREAM + t, ((u64)info << 32) | call, __ATOMIC_RELAXED, __HIP_MEMORY_SCOPE_SYSTEM);
            }
            if (own && lane == 0) __hip_atomic_fetch_add((GAS u32*)(ws + OFF_RDY + (s * 2u + (u < 32u ? 0u : 1u)) * 128u), 1u, __ATOMIC_RELAXED, __HIP_MEMORY_SCOPE_AGENT);
            if (it == 0) STAMP_AT(1);
            if (isg) break;
            task = NALL + it * NEO + eo;
        }
        STAMP_AT(2);
        if (lane == 0) __hip_atomic_store(callp, callv + 1u, __ATOMIC_RELAXED, __HIP_MEMORY_SCOPE_AGENT);
        if (!isg) return;
    }

    const u32 call = rfl(callv) + 1u;
    const int wr = wid >> 2, wc = wid & 3;
    constexpr int HTB = 16384;
#define SA(b, h) (((b) * 2 + (h)) * HTB)
#define SB(b, h) ((4 + (b) * 2 + (h)) * HTB)
#define STAGE(bufoff, gbase) do { _Pragma("unroll") for (int _i = 0; _i < 2; ++_i) \
        __builtin_amdgcn_global_load_lds((const GAS u32*)((gbase) + voff + _i * 8192), (LAS u32*)(lds + (bufoff) + ldsw + _i * 8192), 16, 0, 0); } while (0)
#define LDA(dst, b, h) do { _Pragma("unroll") for (int m = 0; m < 4; ++m) _Pragma("unroll") for (int k = 0; k < 2; ++k) dst[m][k] = *(const LAS bf16x8*)(lds + SA(b, h) + aoff + m * 2048 + k * 1024); } while (0)
#define LDB(dst, b, h) do { _Pragma("unroll") for (int n = 0; n < 2; ++n) _Pragma("unroll") for (int k = 0; k < 2; ++k) dst[n][k] = *(const LAS bf16x8*)(lds + SB(b, h) + boff + n * 2048 + k * 1024); } while (0)
#define MMA(ai, bj, At, Bt) do { __builtin_amdgcn_s_setprio(1); _Pragma("unroll") for (int m = 0; m < 4; ++m) _Pragma("unroll") for (int n = 0; n < 2; ++n) _Pragma("unroll") for (int k = 0; k < 2; ++k) \
        acc[ai][bj][m][n] = __builtin_amdgcn_mfma_f32_16x16x32_bf16(Bt[n][k], At[m][k], acc[ai][bj][m][n], 0, 0, 0); __builtin_amdgcn_s_setprio(0); } while (0)
#define WAIT_V(n) asm volatile("s_waitcnt vmcnt(" #n ")" ::: "memory")
#define WAIT_L(n) asm volatile("s_waitcnt lgkmcnt(" #n ")" ::: "memory")
#define BAR __builtin_amdgcn_s_barrier()
#define SCHED __builtin_amdgcn_sched_barrier(0)
    const u32 voff = (u32)tid * 16u, ldsw = (u32)wid * 1024u;
    const int aoff = wr * 8192 + lane * 16, boff = wc * 4096 + lane * 16;
    LAS u32* badw = (LAS u32*)(lds + STAGE_BYTES + 8 * WLDS);
    for (int round = 0; round < 2; ++round) {
        const u32 tile = tab8(p.wgtile, round * 128u + (blockIdx.x & 127u));
        if (wid == 0) {
            const u64 t0 = __builtin_amdgcn_s_memrealtime(); const u32 need = call * 256u; u32 bad = 0;
            GAS u32* cx = (GAS u32*)(ws + OFF_RDY + ((tile >> 4) * 2u) * 128u); GAS u32* cw = (GAS u32*)(ws + OFF_RDY + ((tile & 15u) * 2u + 1u) * 128u);
            for (;;) {
                const u32 a = __hip_atomic_load(cx, __ATOMIC_RELAXED, __HIP_MEMORY_SCOPE_AGENT), b = __hip_atomic_load(cw, __ATOMIC_RELAXED, __HIP_MEMORY_SCOPE_AGENT);
                if ((int)(rfl(a) - need) >= 0 && (int)(rfl(b) - need) >= 0) break;
                if (give_up(tmo, call, t0)) { bad = 1; break; }
                __builtin_amdgcn_s_sleep(16);
            }
            __builtin_amdgcn_fence(__ATOMIC_ACQUIRE, "agent");
            if (lane == 0) *badw = bad;
            asm volatile("s_waitcnt vmcnt(0) lgkmcnt(0)" ::: "memory");
        }
        asm volatile("" ::: "memory"); BAR; asm volatile("" ::: "memory");
        const bool bad = rfl(*(volatile LAS u32*)badw) != 0u;
        STAMP_AT(3 + round * 3);
        const GAS char* cA = (const GAS char*)(ws + OFF_WF) + (size_t)(tile & 15u) * (2u << 20);
        const GAS char* cB = (const GAS char*)(ws + OFF_XF) + (size_t)(tile >> 4) * (2u << 20);
        constexpr size_t kstep = 32768, hstep = 16384;
        f32x4 acc[2][2][4][2];
#pragma unroll
        for (int a = 0; a < 2; ++a)
#pragma unroll
            for (int b = 0; b < 2; ++b)
#pragma unroll
                for (int m = 0; m < 4; ++m)
#pragma unroll
                    for (int n = 0; n < 2; ++n) acc[a][b][m][n] = (f32x4){0.f, 0.f, 0.f, 0.f};
        bf16x8 At[4][2], B0[2][2], B1[2][2];
        STAGE(SB(0, 0), cB); STAGE(SA(0, 0), cA); STAGE(SB(0, 1), cB + hstep); STAGE(SA(0, 1), cA + hstep);
        if (wr == 1) BAR;
        WAIT_V(4); BAR;
        STAGE(SB(1, 0), cB + kstep); STAGE(SA(1, 0), cA + kstep); STAGE(SB(1, 1), cB + hstep + kstep);
        WAIT_V(6); BAR;
        for (int t = 0; t < 64; t += 2) {
            const int t2 = (t + 2) & 63;
            const GAS char* a1 = cA + (size_t)(t + 1) * kstep;
            const GAS char* a2 = cA + (size_t)t2 * kstep; const GAS char* b2 = cB + (size_t)t2 * kstep;
            const GAS char* a3 = a2 + kstep; const GAS char* b3 = b2 + kstep;
            LDB(B0, 0, 0); SCHED; LDA(At, 0, 0); STAGE(SA(1, 1), a1 + hstep);
            WAIT_L(8); BAR; WAIT_L(0); MMA(0, 0, At, B0); BAR; SCHED;
            LDB(B1, 0, 1); STAGE(SB(0, 0), b2);
            BAR; WAIT_L(0); MMA(0, 1, At, B1); BAR;
            LDA(At, 0, 1); STAGE(SA(0, 0), a2);
            BAR; WAIT_L(0); MMA(1, 0, At, B0); BAR; SCHED;
            STAGE(SB(0, 1), b2 + hstep);
            WAIT_V(6); BAR; MMA(1, 1, At, B1); BAR;
            LDB(B0, 1, 0); SCHED; LDA(At, 1, 0); STAGE(SA(0, 1), a2 + hstep);
            WAIT_L(8); BAR; WAIT_L(0); MMA(0, 0, At, B0); BAR; SCHED;
            LDB(B1, 1, 1); STAGE(SB(1, 0), b3);
            BAR; WAIT_L(0); MMA(0, 1, At, B1); BAR;
            LDA(At, 1, 1); STAGE(SA(1, 0), a3);
            BAR; WAIT_L(0); MMA(1, 0, At, B0); BAR; SCHED;
            STAGE(SB(1, 1), b3 + hstep);
            WAIT_V(6); BAR; MMA(1, 1, At, B1); BAR;
        }
        WAIT_V(0);
        if (wr == 0) BAR;
        BAR;
        STAMP_AT(4 + round * 3);
        const u32 ord = tab8(p.tile2ord, tile);
        u32 info2[2];
#pragma unroll
        for (int bj = 0; bj < 2; ++bj) {
            u32 v[8][4];
#pragma unroll
            for (int g = 0; g < 8; ++g)
#pragma unroll
                for (int q = 0; q < 4; ++q) {
                    typedef float f32x2 __attribute__((ext_vector_type(2)));
                    typedef __bf16 bf16x2 __attribute__((ext_vector_type(2)));
                    const f32x2 f = {acc[q >> 1][bj][(q & 1) * 2][g >> 2][g & 3], acc[q >> 1][bj][(q & 1) * 2 + 1][g >> 2][g & 3]};
                    v[g][q] = bad ? 0x7fc07fc0u : __builtin_bit_cast(u32, __builtin_convertvector(f, bf16x2));
                }
            const u32 b = (u32)wid * 2u + bj;
            info2[bj] = encode_block(v, mk_rsrc(ws + OFF_SY + (size_t)(ord * 16u + b) * SLOT, SLOT), wl, lane);
            const Dest dst = dest_y(p.out, tile, b, rank);
            const u32 loff = (u32)(lane >> 4) * dst.LH + (u32)(lane & 15) * 16u;
#pragma unroll
            for (int g = 0; g < 8; ++g) store_group(dst, loff, g, (u32x4){v[g][0], v[g][1], v[g][2], v[g][3]});
        }
        asm volatile("s_waitcnt vmcnt(0)" ::: "memory");
        if (lane < 14)
            __hip_atomic_store((GAS u64*)(p.ws[(rank + 1u + ((u32)lane >> 1)) & 7u] + OFF_GRAN) + (size_t)rank * NSTREAM + NIN + ord * 16u + (u32)wid * 2u + (lane & 1),
                               ((u64)((lane & 1) ? info2[1] : info2[0]) << 32) | call, __ATOMIC_RELAXED, __HIP_MEMORY_SCOPE_SYSTEM);
        STAMP_AT(5 + round * 3);
    }
}

static void build_tables(Params& p) {
    static const int st[2][8][2] = {{{0, 0}, {1, 0}, {0, 1}, {1, 1}, {2, 0}, {2, 1}, {0, 2}, {1, 2}}, {{2, 2}, {3, 0}, {3, 1}, {3, 2}, {0, 3}, {1, 3}, {2, 3}, {3, 3}}};
    float fin[256], prev[128];
    for (int r = 0; r < 2; ++r)
        for (int b = 0; b < 128; ++b) {
            const int c = b & 7, idx = b >> 3, i = st[r][c][0] * 4 + (idx >> 2), j = st[r][c][1] * 4 + (idx & 3), t = i * 16 + j;
            const int last = (2 * i > 2 * j + 1 ? 2 * i : 2 * j + 1) + 1;
            float start = 8.f + 3.05f * last;
            if (r == 1 && prev[b] > start) start = prev[b];
            fin[t] = prev[b] = start + 78.f;
            ((unsigned char*)p.wgtile)[r * 128 + b] = (unsigned char)t;
        }
    int o[256];
    for (int t = 0; t < 256; ++t) { int k = t; while (k > 0 && fin[o[k - 1]] > fin[t]) { o[k] = o[k - 1]; --k; } o[k] = t; }
    for (int k = 0; k < 256; ++k) { ((unsigned char*)p.ord2tile)[k] = (unsigned char)o[k]; ((unsigned char*)p.tile2ord)[o[k]] = (unsigned char)k; }
}

extern "C" void kernel_launch(int rank, int nranks, ncclComm_t comm, hipStream_t stream,
                              void* const* d_in, const int* in_sizes, int n_in, void* d_out, int out_size,
                              const int* in_dtypes, int out_dtype, void* const* d_out_all,
                              void* const* d_ws_all, size_t ws_bytes) {
    if (nranks != NR || (unsigned)rank >= (unsigned)NR || ws_bytes < WS_NEED || n_in != 2) { fprintf(stderr, "unsupported configuration\n"); return; }
    Params p;
    p.x = (const char*)d_in[0]; p.w = (const char*)d_in[1]; p.out = (char*)d_out; p.rank = rank; p.pad = 0;
    for (int i = 0; i < NR; ++i) p.ws[i] = (char*)d_ws_all[i];
    build_tables(p);
    hipLaunchKernelGGL(gemm_ar_fused, dim3(256), dim3(512), 0, stream, p);
    HIP_CHECK(hipGetLastError());
}
```

```cpp
#include <hip/hip_runtime.h>
#include <rccl/rccl.h>
#include <cstdio>

#define HIP_CHECK(expr) do { hipError_t _e = (expr); if (_e != hipSuccess) { \
    fprintf(stderr, "HIP error %s at %s:%d\n", hipGetErrorString(_e), __FILE__, __LINE__); \
    return; } } while (0)

typedef unsigned u32;
typedef unsigned long long u64;
typedef u32 u32x4 __attribute__((ext_vector_type(4)));
typedef u32 u32x2 __attribute__((ext_vector_type(2)));
typedef unsigned short u16x2 __attribute__((ext_vector_type(2)));
typedef short bf16x8 __attribute__((ext_vector_type(8)));
typedef float f32x4 __attribute__((ext_vector_type(4)));
#define LAS __attribute__((address_space(3)))
#define GAS __attribute__((address_space(1)))
typedef __amdgpu_buffer_rsrc_t rsrc_t;

constexpr int NR = 8;
#define PULL_WG_PER_PEER 8
#define WINDOW 8
#define LOOK 6
constexpr int NGEMM = 128, NPULL = 7 * PULL_WG_PER_PEER, NENC = 256 - NGEMM - NPULL;
constexpr int NCIN = 128, NCY = 512, NC = NCIN + NCY;
constexpr int NTASK = 16 * 36;
constexpr int NDEC = 6;
constexpr u64 DEADLINE = 100000000ull;

constexpr int FIXB = 5120, VARDW = 768, CVAR = 8 * FIXB, CHDR = 64, CH = 66560;
constexpr size_t OFF_CALL = 0;
constexpr size_t OFF_TMO = 131072;
constexpr size_t OFF_RDY = OFF_TMO + 256;
constexpr size_t OFF_TICK = OFF_RDY + 32 * 128;
constexpr size_t OFF_GRAN = 196608;
constexpr size_t OFF_DBG = 524288;
constexpr size_t OFF_XF = 1ull << 20;
constexpr size_t OFF_WF = OFF_XF + (32ull << 20);
constexpr size_t OFF_SX = OFF_WF + (32ull << 20);
constexpr size_t OFF_SW = OFF_SX + 64ull * CH;
constexpr size_t OFF_SY = OFF_SW + 512ull * CH;
constexpr size_t WS_NEED = OFF_SY + 512ull * CH;

constexpr int STAGE_BYTES = 131072, WLDS = 3136;
constexpr int CTL_OFF = STAGE_BYTES + 8 * WLDS, CTL_WORDS = 128;
constexpr int NF = 16, NV = 4, VSLOT = 8192, VRING = NF * FIXB;
constexpr int AUX_SYS = 17, AUX_DEV = 16;

struct Params {
    const char* x; const char* w; char* out; char* ws[NR];
    int rank; int pad;
    u32 ord2tile[64];
    u32 tile2ord[64];
    u32 wgtile[64];
};
__device__ __forceinline__ u32 tab8(const u32 (&t)[64], u32 i) { return (t[(i >> 2) & 63u] >> ((i & 3u) * 8u)) & 0xffu; }

__device__ __forceinline__ u32 rfl(u32 x) { return (u32)__builtin_amdgcn_readfirstlane((int)x); }
__device__ __forceinline__ rsrc_t mk_rsrc(const void* p, u32 bytes) {
    const u64 a = (u64)p; const u64 u = ((u64)rfl((u32)(a >> 32)) << 32) | rfl((u32)a);
    return __builtin_amdgcn_make_buffer_rsrc((void*)u, (short)0, (int)rfl(bytes), 0x00020000);
}
__device__ __forceinline__ u32 vperm(u32 a, u32 b, u32 s) { return __builtin_amdgcn_perm(a, b, s); }
__device__ __forceinline__ u32 wave_incl_scan(u32 x) {
    x += (u32)__builtin_amdgcn_update_dpp(0, (int)x, 0x111, 0xf, 0xf, false);
    x += (u32)__builtin_amdgcn_update_dpp(0, (int)x, 0x112, 0xf, 0xf, false);
    x += (u32)__builtin_amdgcn_update_dpp(0, (int)x, 0x114, 0xf, 0xf, false);
    x += (u32)__builtin_amdgcn_update_dpp(0, (int)x, 0x118, 0xf, 0xf, false);
    x += (u32)__builtin_amdgcn_update_dpp(0, (int)x, 0x142, 0xa, 0xf, false);
    x += (u32)__builtin_amdgcn_update_dpp(0, (int)x, 0x143, 0xc, 0xf, false);
    return x;
}
__device__ __forceinline__ u32 pk_sub_min(u32 tt, u32 r) {
    u16x2 a = __builtin_bit_cast(u16x2, tt) - __builtin_bit_cast(u16x2, r);
    const u16x2 lim = {0x0bff, 0x0bff};
    return __builtin_bit_cast(u32, __builtin_elementwise_min(a, lim));
}

__device__ __forceinline__ void encode_block(const u32 (&v)[8][4], rsrc_t ch, u32 b, LAS u32* alloc, LAS u32* lds, int lane) {
    *(LAS u32x4*)(lds + lane * 4) = (u32x4){0, 0, 0, 0};
    *(LAS u32x4*)(lds + 256 + lane * 4) = (u32x4){0, 0, 0, 0};
    u32 s = 0;
#pragma unroll
    for (int g = 0; g < 8; ++g) s = __builtin_amdgcn_udot4(__builtin_amdgcn_alignbit(v[g][0], v[g][0], 31), 0x01000100u, s, false);
    int T = (int)((u32)__builtin_amdgcn_readlane((int)wave_incl_scan(s), 63) >> 10) + 3;
    T = T < 10 ? 10 : (T > 255 ? 255 : T);
    const u32 tt = ((u32)T * 256u + 255u) * 0x10001u;
    u32 sm[16], U[16], l1[4] = {0, 0, 0, 0}, z3[16], c3[17];
    c3[0] = 0;
#pragma unroll
    for (int k = 0; k < 16; ++k) {
        const u32 a = v[k >> 1][(k & 1) * 2], b = v[k >> 1][(k & 1) * 2 + 1];
        const u32 r0 = __builtin_amdgcn_alignbit(a, a, 31), r1 = __builtin_amdgcn_alignbit(b, b, 31);
        sm[k] = vperm(r1, r0, 0x06040200u);
        U[k] = vperm(pk_sub_min(tt, r1), pk_sub_min(tt, r0), 0x07050301u);
        l1[k >> 2] |= vperm(0u, 0x03020100u, U[k]) << (2 * (k & 3));
        z3[k] = vperm(0x83038303u, 0x80008003u, U[k]) & 0x03030303u;
        c3[k + 1] = __builtin_amdgcn_sad_u8(z3[k], 0u, c3[k]);
    }
    const u32 incl = wave_incl_scan(c3[16]);
    const u32 p0 = incl - c3[16];
    const u32 xoff = ((u32)__builtin_amdgcn_readlane((int)incl, 63) + 31u) >> 5;
    u32 nx = 0;
#pragma unroll
    for (int g = 0; g < 8; ++g) {
        u32 x2[2];
#pragma unroll
        for (int h = 0; h < 2; ++h) {
            const int k = 2 * g + h;
            const u32 d8 = U[k] - (vperm(0x83038303u, 0x83028100u, U[k]) & 0x03030303u);
            const u32 d = d8 & 0x07070707u;
            const u32 rk = __umul24(z3[k], 0x010101u);
            x2[h] = (d & 0xffu) | (((d >> 8) & 0xffu) << (rk & 0xffu)) | (((d >> 16) & 0xffu) << ((rk >> 8) & 0xffu)) | ((d >> 24) << ((rk >> 16) & 0xffu));
            if (__builtin_amdgcn_ballot_w64((d8 & 0x08080808u) != 0u) != 0ull) {
                u32 l22 = (u32)lane << 22;
                asm volatile("" : "+v"(l22));
#pragma unroll
                for (int t = 0; t < 4; ++t) {
                    const bool e = (d8 >> (8 * t + 3)) & 1u;
                    const u64 m = __builtin_amdgcn_ballot_w64(e);
                    const u32 at = xoff + nx + __builtin_amdgcn_mbcnt_hi((u32)(m >> 32), __builtin_amdgcn_mbcnt_lo((u32)m, 0u));
                    const u32 val = (v[g][2 * h + (t >> 1)] >> (16 * (t & 1))) & 0xffffu;
                    if (e && at < (u32)VARDW) lds[at] = l22 | ((u32)(g * 8 + h * 4 + t) << 16) | val;
                    nx += (u32)__builtin_popcountll(m);
                }
            }
        }
        const u32 x = x2[0] | (x2[1] << (c3[2 * g + 1] - c3[2 * g]));
        const u32 p = p0 + c3[2 * g];
        const u64 y = (u64)x << (p & 31u);
        __hip_atomic_fetch_or(lds + (p >> 5), (u32)y, __ATOMIC_RELAXED, __HIP_MEMORY_SCOPE_WORKGROUP);
        __hip_atomic_fetch_or(lds + (p >> 5) + 1, (u32)(y >> 32), __ATOMIC_RELAXED, __HIP_MEMORY_SCOPE_WORKGROUP);
    }
    const u32 vd = xoff + nx;
    const bool raw = vd > (u32)VARDW;
    u32 off = 0;
    if (lane == 0) off = __hip_atomic_fetch_add(alloc, raw ? 3072u : (vd * 4u + 15u) & ~15u, __ATOMIC_RELAXED, __HIP_MEMORY_SCOPE_WORKGROUP);
    off = rfl(off) + CHDR;
    const int fo = (int)(b * FIXB), vo = CVAR + (int)off;
    if (lane == 0) __builtin_amdgcn_raw_buffer_store_b64((u32x2){raw ? 0x80000000u : ((u32)T | (xoff << 8) | (nx << 20)), off}, ch, 0, CVAR + (int)b * 8, AUX_SYS);
    if (raw) {
#pragma unroll
        for (int g = 0; g < 8; ++g)
            __builtin_amdgcn_raw_buffer_store_b128((u32x4){v[g][0], v[g][1], v[g][2], v[g][3]}, ch, lane * 16 + (g < 5 ? g : g - 5) * 1024, g < 5 ? fo : vo, AUX_SYS);
        return;
    }
#pragma unroll
    for (int q = 0; q < 4; ++q)
        __builtin_amdgcn_raw_buffer_store_b128((u32x4){sm[4 * q], sm[4 * q + 1], sm[4 * q + 2], sm[4 * q + 3]}, ch, lane * 16 + q * 1024, fo, AUX_SYS);
    __builtin_amdgcn_raw_buffer_store_b128((u32x4){l1[0], l1[1], l1[2], l1[3]}, ch, lane * 16 + 4096, fo, AUX_SYS);
    __builtin_amdgcn_fence(__ATOMIC_ACQ_REL, "wavefront");
    for (u32 c = 0; c < vd; c += 256) {
        const u32x4 t = *(volatile LAS u32x4*)(lds + c + lane * 4);
        if (c + lane * 4 < vd) __builtin_amdgcn_raw_buffer_store_b128(t, ch, (int)c * 4 + lane * 16, vo, AUX_SYS);
    }
    __builtin_amdgcn_fence(__ATOMIC_ACQ_REL, "wavefront");
}
__device__ __forceinline__ bool chunk_done(LAS u32* cnt, int lane, u32& total) {
    u32 n = 0;
    if (lane == 0) n = __hip_atomic_fetch_add(cnt + 1, 1u, __ATOMIC_RELAXED, __HIP_MEMORY_SCOPE_WORKGROUP);
    if (rfl(n) != 7u) return false;
    total = rfl(*(volatile LAS u32*)cnt) + CHDR;
    return true;
}

struct Dest { rsrc_t r; u32 A, B, C, LH; };
__device__ __forceinline__ void store_group(const Dest& d, u32 loff, int g, u32x4 x) {
    __builtin_amdgcn_raw_buffer_store_b128(x, d.r, (int)loff, (int)((g & 1 ? d.A : 0u) + (g & 2 ? d.B : 0u) + (g & 4 ? d.C : 0u)), AUX_DEV);
}

__device__ __forceinline__ void decode_block(const u32x4 (&d)[5], u32 info, bool ok, const Dest& dst, const LAS u32* lds, int lane) {
    const u32 loff = (u32)(lane >> 4) * dst.LH + (u32)(lane & 15) * 16u;
    u32x4 o[8];
    u32 xoff = 0, vd = 0;
    if (!ok || (info >> 31)) {
        const u32 nanv = ok ? 0u : 0x7fc07fc0u;
#pragma unroll
        for (int g = 0; g < 8; ++g) o[g] = (g < 5 ? d[g] : *(const LAS u32x4*)(lds + (g - 5) * 256 + lane * 4)) | nanv;
    } else {
        const u32 T = info & 0xffu;
        xoff = (info >> 8) & 0xfffu;
        vd = xoff + ((info >> 20) & 0x7ffu);
        vd = vd > (u32)VARDW ? (u32)VARDW : vd;
        const u32 tb = T * 0x01010101u, t1 = tb - 0x03020100u, t2lo = tb - 0x06050400u, t2hi = tb - 0x0a090807u;
        u32 c4[16], z1[16], cn[17];
        cn[0] = 0;
#pragma unroll
        for (int k = 0; k < 16; ++k) {
            c4[k] = ((d[4][k >> 2] >> (2 * (k & 3))) & 0x03030303u) | 0x04040404u;
            z1[k] = vperm(0x00000001u, 0u, c4[k]);
            cn[k + 1] = cn[k] + (u32)__builtin_popcount(z1[k]);
        }
        const u32 n0 = wave_incl_scan(cn[16]) - cn[16];
        u32 wv[8];
#pragma unroll
        for (int g = 0; g < 8; ++g) {
            const u32 p = (n0 + cn[2 * g]) * 3u;
            const u32 lo = lds[p >> 5], hi = lds[(p >> 5) + 1];
            wv[g] = __builtin_amdgcn_alignbit(hi, lo, p & 31u);
        }
#pragma unroll
        for (int k = 0; k < 16; ++k) {
            const int g = k >> 1, h = k & 1;
            const u32 x = h ? __builtin_amdgcn_ubfe(wv[g], (cn[k] - cn[k - 1]) * 3u, 12u) : (wv[g] & 0xfffu);
            const u32 y = (x | (x << 10)) & 0x003f003fu;
            const u32 z = (y | (y << 5)) & 0x07070707u;
            const u32 e2 = vperm(t2hi, t2lo, z);
            const u32 rank = __umul24(z1[k], 0x010101u) << 8;
            const u32 m = vperm(0x000000ffu, 0u, c4[k]);
            const u32 e = vperm(t1, e2, (rank & m) | (c4[k] & ~m));
            const u32 smk = d[k >> 2][k & 3];
            const u32 r0 = vperm(e, smk, 0x05010400u), r1 = vperm(e, smk, 0x07030602u);
            o[g][2 * h] = __builtin_amdgcn_alignbit(r0, r0, 1);
            o[g][2 * h + 1] = __builtin_amdgcn_alignbit(r1, r1, 1);
        }
    }
#pragma unroll
    for (int g = 0; g < 8; ++g) store_group(dst, loff, g, o[g]);
    __builtin_amdgcn_fence(__ATOMIC_ACQ_REL, "wavefront");
    for (u32 c = xoff; c < vd; c += 64u) {
        if (c + (u32)lane < vd) {
            const u32 en = lds[c + (u32)lane], l2 = (en >> 22) & 63u, g = (en >> 19) & 7u;
            const u32 off = (l2 >> 4) * dst.LH + (l2 & 15u) * 16u + (g & 1u) * dst.A + ((g >> 1) & 1u) * dst.B + (g >> 2) * dst.C + ((en >> 16) & 7u) * 2u;
            __builtin_amdgcn_raw_buffer_store_b16((unsigned short)en, dst.r, (int)off, 0, AUX_DEV);
        }
    }
}

__device__ __forceinline__ Dest dest_x(char* ws, u32 s, u32 b, u32 src) {
    return Dest{mk_rsrc(ws + OFF_XF + (size_t)(((s * 64u + src * 8u + (b & 1u) * 4u) * 32u + (b >> 1) * 2u) * 1024u), 3u * 32768u + 2048u), 1024u, 32768u, 65536u, 256u};
}
__device__ __forceinline__ Dest dest_w(char* ws, u32 s, u32 b, u32 src) {
    return Dest{mk_rsrc(ws + OFF_WF + (size_t)(((s * 64u + src * 8u + (b >> 2)) * 32u + (b & 1u) * 8u + ((b >> 1) & 1u)) * 1024u), 16384u + 6144u + 1024u), 2048u, 4096u, 16384u, 256u};
}
__device__ __forceinline__ Dest dest_y(char* out, u32 tile, u32 bj, u32 wv, u32 src) {
    const u32 row = (tile >> 4) * 256u + bj * 128u + (wv & 3u) * 32u, col = src * 4096u + (tile & 15u) * 256u + (wv >> 2) * 128u;
    return Dest{mk_rsrc(out + (size_t)row * 65536u + col * 2u, 31u * 65536u + 256u), 65536u, 131072u, 16u * 65536u, 4u * 65536u};
}

__device__ __forceinline__ bool give_up(GAS u32* tmo, u32 call, u64 t0) {
    if (rfl(__hip_atomic_load(tmo, __ATOMIC_RELAXED, __HIP_MEMORY_SCOPE_AGENT)) == call || __builtin_amdgcn_s_memrealtime() - t0 > DEADLINE) {
        __hip_atomic_store(tmo, call, __ATOMIC_RELAXED, __HIP_MEMORY_SCOPE_AGENT);
        return true;
    }
    return false;
}

#define STAMP_AT(i) do { } while (0)

__device__ __forceinline__ u32 lds_ld(LAS u32* p) { return rfl(*(volatile LAS u32*)p); }
__device__ __forceinline__ void dma_piece(rsrc_t r, u32 voff, u32 soff, u32 ldsaddr) {
    u32 keep;
    asm volatile("s_nop 4\n\ts_mov_b32 %0, m0\n\ts_mov_b32 m0, %1\n\ts_nop 0\n\tbuffer_load_dwordx4 %2, %3, %4 offen sc0 sc1 lds\n\ts_mov_b32 m0, %0"
                 : "=&s"(keep) : "s"(ldsaddr), "v"(voff), "s"(r), "s"(soff) : "memory");
}

__global__ void __launch_bounds__(512, 2) gemm_ar_fused(const Params p) {
    __shared__ __attribute__((aligned(16))) char smem[CTL_OFF + CTL_WORDS * 4];
    LAS char* lds = (LAS char*)smem;
    const int tid = threadIdx.x, wid = (int)rfl((u32)tid >> 6), lane = tid & 63;
    const u32 rank = (u32)p.rank, bid = blockIdx.x;
    char* ws = p.ws[rank & 7u];
    LAS u32* wl = (LAS u32*)(lds + STAGE_BYTES + wid * WLDS);
    LAS u32* ctl = (LAS u32*)(lds + CTL_OFF);
    GAS u32* tmo = (GAS u32*)(ws + OFF_TMO);
    STAMP_AT(0);
    GAS u32* callp = (GAS u32*)(ws + OFF_CALL + (bid * 8u + wid) * 64u);
    const u32 callv = __hip_atomic_load(callp, __ATOMIC_RELAXED, __HIP_MEMORY_SCOPE_AGENT);
    if (tid < CTL_WORDS) ctl[tid] = 0u;
    asm volatile("s_waitcnt lgkmcnt(0)" ::: "memory"); __builtin_amdgcn_s_barrier(); asm volatile("" ::: "memory");

    if (bid >= (u32)NGEMM && bid < (u32)(NGEMM + NPULL)) {
        const u32 pj = bid - NGEMM, src = (rank + 1u + pj % 7u) & 7u;
        const char* pws = p.ws[src];
        LAS u32* landed = ctl; LAS u32* qavail = ctl + 1; LAS u32* prog = ctl + 2;
        LAS u32* q = ctl + 16;
        LAS u32* vdone = ctl + 32; LAS u32* ffree = ctl + 48;
        auto chunk_of = [&](u32 c) -> const char* {
            if (c >= (u32)NCIN) return pws + OFF_SY + (size_t)(c - NCIN) * CH;
            const u32 i = (c >> 3) * 4u + (c & 3u);
            return (c & 4u) ? pws + OFF_SW + (size_t)(rank * 64u + i) * CH : pws + OFF_SX + (size_t)i * CH;
        };
        if (wid == 1) {
            const u32 call = rfl(callv) + 1u;
            if (lane == 0) __hip_atomic_store(callp, call, __ATOMIC_RELAXED, __HIP_MEMORY_SCOPE_AGENT);
            GAS u64* gran = (GAS u64*)(ws + OFF_GRAN) + (size_t)src * NC;
            u32 pieces = 0;
            for (u32 k = 0;; ++k) {
                u32 c = pj / 7u;
                if (k) {
                    while (lds_ld(prog) + LOOK < 8u * k) __builtin_amdgcn_s_sleep(2);
                    u32 tk = 0;
                    if (lane == 0) tk = __hip_atomic_fetch_add((GAS u32*)(ws + OFF_TICK + src * 128u), 1u, __ATOMIC_RELAXED, __HIP_MEMORY_SCOPE_AGENT);
                    c = rfl(tk) - (call - 1u) * NC + PULL_WG_PER_PEER;
                }
                u32 fl = 0x80000000u, vb = 0;
                if (c < (u32)NC) {
                    const u64 t0 = __builtin_amdgcn_s_memrealtime();
                    for (;;) {
                        const u64 g = __hip_atomic_load(gran + c, __ATOMIC_RELAXED, __HIP_MEMORY_SCOPE_SYSTEM);
                        if (rfl((u32)g) == call) { fl = c; vb = rfl((u32)(g >> 32)); break; }
                        if (give_up(tmo, call, t0)) { fl = c | 0x40000000u; break; }
                        __builtin_amdgcn_s_sleep(2);
                    }
                    if (k == 0) STAMP_AT(1);
                }
                while (lds_ld(vdone + (k & (NV - 1))) < 8u * (k / NV)) __builtin_amdgcn_s_sleep(2);
                if (!(fl >> 30)) pieces += vb > (u32)VSLOT ? 1u : (vb + 1023u) >> 10;
                if (lane == 0) { *(volatile LAS u32x4*)(q + (k & (NV - 1)) * 4u) = (u32x4){fl, vb, pieces, 0u}; *(volatile LAS u32*)qavail = k + 1u; }
                if (!(fl >> 30)) pieces += 40u;
                if (fl >> 31) break;
            }
        } else if (wid == 0) {
            __builtin_amdgcn_s_setprio(3);
            const u32 l0 = (u32)(size_t)lds, voff = (u32)lane * 16u;
            u32 issued = 0, pub = 0;
            auto drain = [&]() { asm volatile("s_waitcnt vmcnt(0)" ::: "memory"); pub = issued; if (lane == 0) *(volatile LAS u32*)landed = pub; };
            auto piece = [&](rsrc_t r, u32 vo, u32 so, u32 la) {
                dma_piece(r, vo, so, la);
                ++issued;
                asm volatile("s_waitcnt vmcnt(%0)" :: "n"(WINDOW - 1) : "memory");
                if ((int)(issued - (WINDOW - 1) - pub) > 0) { pub = issued - (WINDOW - 1); if (lane == 0) *(volatile LAS u32*)landed = pub; }
            };
            for (u32 k = 0;; ++k) {
                if (lds_ld(qavail) <= k) { drain(); while (lds_ld(qavail) <= k) __builtin_amdgcn_s_sleep(1); }
                const u32x4 e = *(volatile LAS u32x4*)(q + (k & (NV - 1)) * 4u);
                const u32 fl = rfl(e[0]), vb = rfl(e[1]);
                if (fl >> 31) break;
                if (fl >> 30) { if (lane == 0) *(volatile LAS u32*)prog = 8u * k + 8u; continue; }
                const u32 vl = vb > (u32)VSLOT ? (u32)CHDR : vb;
                const rsrc_t r = mk_rsrc(chunk_of(fl), CVAR + vl);
                for (u32 o = 0; o < vl; o += 1024u) piece(r, voff + CVAR + o, 0u, l0 + VRING + (k & (NV - 1)) * VSLOT + o);
                for (u32 b = 0; b < 8u; ++b) {
                    const u32 n = 8u * k + b, sl = n & (NF - 1);
                    if (lane == 0) *(volatile LAS u32*)prog = n + 1u;
                    if (lds_ld(ffree + sl) + NF <= n) { drain(); while (lds_ld(ffree + sl) + NF <= n) __builtin_amdgcn_s_sleep(1); }
#pragma unroll
                    for (u32 j = 0; j < 5u; ++j) piece(r, voff, b * FIXB + j * 1024u, l0 + sl * FIXB + j * 1024u);
                }
            }
            drain();
            STAMP_AT(2);
        } else {
            for (u32 n = (u32)wid - 2u;; n += NDEC) {
                const u32 k = n >> 3, b = n & 7u, sl = n & (NF - 1);
                while (lds_ld(qavail) <= k) __builtin_amdgcn_s_sleep(1);
                const u32x4 e = *(volatile LAS u32x4*)(q + (k & (NV - 1)) * 4u);
                const u32 fl = rfl(e[0]), vb = rfl(e[1]), c = fl & 0xffffu;
                if (fl >> 31) break;
                const bool ok = !(fl >> 30);
                if (ok) { const u32 need = rfl(e[2]) + 5u * b + 5u; while (lds_ld(landed) < need) __builtin_amdgcn_s_sleep(1); }
                asm volatile("" ::: "memory");
                u32x4 d[5];
#pragma unroll
                for (int j = 0; j < 5; ++j) d[j] = *(const LAS u32x4*)(lds + sl * FIXB + j * 1024 + lane * 16);
                LAS u32* vs = (LAS u32*)(lds + VRING + (k & (NV - 1)) * VSLOT);
                const u32x2 h = *(const LAS u32x2*)(vs + b * 2u);
                asm volatile("s_waitcnt lgkmcnt(0)" ::: "memory");
                if (lane == 0) __hip_atomic_fetch_max(ffree + sl, n + 1u, __ATOMIC_RELAXED, __HIP_MEMORY_SCOPE_WORKGROUP);
                asm volatile("" : "+v"(d[0]), "+v"(d[1]), "+v"(d[2]), "+v"(d[3]), "+v"(d[4]) :: "memory");
                const u32 info = rfl(h[0]), off = rfl(h[1]);
                const LAS u32* var = vs + (off < (u32)VSLOT ? off >> 2 : 0u);
                if (ok && vb > (u32)VSLOT) {
                    const rsrc_t r = mk_rsrc(chunk_of(c), CVAR + (vb < (u32)(CH - CVAR) ? vb : (u32)(CH - CVAR)));
#pragma unroll
                    for (int j = 0; j < 3; ++j) *(volatile LAS u32x4*)(wl + j * 256 + lane * 4) = __builtin_amdgcn_raw_buffer_load_b128(r, (int)(CVAR + off) + j * 1024 + lane * 16, 0, AUX_SYS);
                    __builtin_amdgcn_fence(__ATOMIC_ACQ_REL, "wavefront");
                    var = wl;
                }
                const Dest dst = c >= (u32)NCIN ? dest_y(p.out, tab8(p.ord2tile, (c - NCIN) >> 1), (c - NCIN) & 1u, b, src)
                               : ((c & 4u) ? dest_w(ws, c >> 3, (c & 3u) * 8u + b, src) : dest_x(ws, c >> 3, (c & 3u) * 8u + b, src));
                decode_block(d, info, ok, dst, var, lane);
                asm volatile("s_waitcnt lgkmcnt(0)" ::: "memory");
                if (lane == 0) __hip_atomic_fetch_add(vdone + (k & (NV - 1)), 1u, __ATOMIC_RELAXED, __HIP_MEMORY_SCOPE_WORKGROUP);
                if (c < (u32)NCIN) {
                    asm volatile("s_waitcnt vmcnt(0)" ::: "memory");
                    if (lane == 0) __hip_atomic_fetch_add((GAS u32*)(ws + OFF_RDY + (c >> 2) * 128u), 1u, __ATOMIC_RELAXED, __HIP_MEMORY_SCOPE_AGENT);
                }
            }
            STAMP_AT(3);
        }
        return;
    }

    const bool isg = bid < (u32)NGEMM;
    {
        const u32 eo = bid - NGEMM - NPULL;
        u32 task = isg ? NENC + bid : eo;
        for (u32 it = 0; task < (u32)NTASK; ++it) {
            const u32 s = task / 36u, u = task % 36u, b = (u & 3u) * 8u + (u32)wid;
            const u32 dd = (rank + (u >> 2)) & 7u;
            u32 v[8][4];
            if (u < 4u) {
                const rsrc_t r = mk_rsrc(p.x + (size_t)(s * 256u + (b >> 1) * 16u) * 1024u + (b & 1u) * 512u, 15u * 1024u + 512u);
#pragma unroll
                for (int g = 0; g < 8; ++g) {
                    const u32x4 a = __builtin_amdgcn_raw_buffer_load_b128(r, (lane & 15) * 1024 + (lane >> 4) * 16 + g * 64, 0, 0);
                    v[g][0] = a[0]; v[g][1] = a[1]; v[g][2] = a[2]; v[g][3] = a[3];
                }
            } else {
                const rsrc_t r = mk_rsrc(p.w + (size_t)((b >> 1) * 32u) * 65536u + (dd * 4096u + s * 256u + (b & 1u) * 128u) * 2u, 31u * 65536u + 256u);
                u32x4 a[8];
#pragma unroll
                for (int i = 0; i < 8; ++i) a[i] = __builtin_amdgcn_raw_buffer_load_b128(r, (lane >> 4) * 8 * 65536 + (lane & 15) * 16, i * 65536, 0);
#pragma unroll
                for (int j = 0; j < 8; ++j)
#pragma unroll
                    for (int q = 0; q < 4; ++q) v[j][q] = vperm(a[2 * q + 1][j >> 1], a[2 * q][j >> 1], (j & 1) ? 0x07060302u : 0x05040100u);
            }
            const bool own = u < 4u || u >= 32u, send = u < 32u;
            LAS u32* cnt = ctl + 8 + 2u * it;
            if (send) {
                char* ch = u < 4u ? ws + OFF_SX + (size_t)(s * 4u + u) * CH : ws + OFF_SW + (size_t)(dd * 64u + s * 4u + (u & 3u)) * CH;
                encode_block(v, mk_rsrc(ch, CH), (u32)wid, cnt, wl, lane);
            }
            if (own) {
                const Dest dst = u < 4u ? dest_x(ws, s, b, rank) : dest_w(ws, s, b, rank);
#pragma unroll
                for (int g = 0; g < 8; ++g) store_group(dst, (u32)lane * 16u, g, (u32x4){v[g][0], v[g][1], v[g][2], v[g][3]});
            }
            const u32 call = rfl(callv) + 1u;
            asm volatile("s_waitcnt vmcnt(0)" ::: "memory");
            u32 total;
            if (send && chunk_done(cnt, lane, total)) {
                const u32 c = s * 8u + (u < 4u ? 0u : 4u) + (u & 3u);
                const u32 k = u < 4u ? (u32)lane + 1u : (u >> 2);
                if (lane < (u < 4u ? 7 : 1))
                    __hip_atomic_store((GAS u64*)(p.ws[(rank + k) & 7u] + OFF_GRAN) + (size_t)rank * NC + c, ((u64)total << 32) | call, __ATOMIC_RELAXED, __HIP_MEMORY_SCOPE_SYSTEM);
            }
            if (own && lane == 0) __hip_atomic_fetch_add((GAS u32*)(ws + OFF_RDY + (s * 2u + (u < 4u ? 0u : 1u)) * 128u), 1u, __ATOMIC_RELAXED, __HIP_MEMORY_SCOPE_AGENT);
            if (it == 0) STAMP_AT(1);
            if (isg) break;
            task = NENC + NGEMM + it * NENC + eo;
        }
        STAMP_AT(2);
        if (lane == 0) __hip_atomic_store(callp, callv + 1u, __ATOMIC_RELAXED, __HIP_MEMORY_SCOPE_AGENT);
        if (!isg) return;
    }

    const u32 call = rfl(callv) + 1u;
    const int wr = wid >> 2, wc = wid & 3;
    constexpr int HTB = 16384;
#define SA(b, h) (((b) * 2 + (h)) * HTB)
#define SB(b, h) ((4 + (b) * 2 + (h)) * HTB)
#define STAGE(bufoff, gbase) do { _Pragma("unroll") for (int _i = 0; _i < 2; ++_i) \
        __builtin_amdgcn_global_load_lds((const GAS u32*)((gbase) + voff + _i * 8192), (LAS u32*)(lds + (bufoff) + ldsw + _i * 8192), 16, 0, 0); } while (0)
#define LDA(dst, b, h) do { _Pragma("unroll") for (int m = 0; m < 4; ++m) _Pragma("unroll") for (int k = 0; k < 2; ++k) dst[m][k] = *(const LAS bf16x8*)(lds + SA(b, h) + aoff + m * 2048 + k * 1024); } while (0)
#define LDB(dst, b, h) do { _Pragma("unroll") for (int n = 0; n < 2; ++n) _Pragma("unroll") for (int k = 0; k < 2; ++k) dst[n][k] = *(const LAS bf16x8*)(lds + SB(b, h) + boff + n * 2048 + k * 1024); } while (0)
#define MMA(ai, bj, At, Bt) do { __builtin_amdgcn_s_setprio(1); _Pragma("unroll") for (int m = 0; m < 4; ++m) _Pragma("unroll") for (int n = 0; n < 2; ++n) _Pragma("unroll") for (int k = 0; k < 2; ++k) \
        acc[ai][bj][m][n] = __builtin_amdgcn_mfma_f32_16x16x32_bf16(Bt[n][k], At[m][k], acc[ai][bj][m][n], 0, 0, 0); __builtin_amdgcn_s_setprio(0); } while (0)
#define WAIT_V(n) asm volatile("s_waitcnt vmcnt(" #n ")" ::: "memory")
#define WAIT_L(n) asm volatile("s_waitcnt lgkmcnt(" #n ")" ::: "memory")
#define BAR __builtin_amdgcn_s_barrier()
#define SCHED __builtin_amdgcn_sched_barrier(0)
    const u32 voff = (u32)tid * 16u, ldsw = (u32)wid * 1024u;
    const int aoff = wr * 8192 + lane * 16, boff = wc * 4096 + lane * 16;
    LAS u32* badw = ctl;
    for (int round = 0; round < 2; ++round) {
        const u32 tile = tab8(p.wgtile, round * 128u + (blockIdx.x & 127u));
        if (wid == 0) {
            const u64 t0 = __builtin_amdgcn_s_memrealtime(); const u32 need = call * 256u; u32 bad = 0;
            GAS u32* cx = (GAS u32*)(ws + OFF_RDY + ((tile >> 4) * 2u) * 128u); GAS u32* cw = (GAS u32*)(ws + OFF_RDY + ((tile & 15u) * 2u + 1u) * 128u);
            for (;;) {
                const u32 a = __hip_atomic_load(cx, __ATOMIC_RELAXED, __HIP_MEMORY_SCOPE_AGENT), b = __hip_atomic_load(cw, __ATOMIC_RELAXED, __HIP_MEMORY_SCOPE_AGENT);
                if ((int)(rfl(a) - need) >= 0 && (int)(rfl(b) - need) >= 0) break;
                if (give_up(tmo, call, t0)) { bad = 1; break; }
                __builtin_amdgcn_s_sleep(16);
            }
            __builtin_amdgcn_fence(__ATOMIC_ACQUIRE, "agent");
            if (lane == 0) *badw = bad;
            asm volatile("s_waitcnt vmcnt(0) lgkmcnt(0)" ::: "memory");
        }
        asm volatile("" ::: "memory"); BAR; asm volatile("" ::: "memory");
        const bool bad = rfl(*(volatile LAS u32*)badw) != 0u;
        STAMP_AT(3 + round * 3);
        const GAS char* cA = (const GAS char*)(ws + OFF_WF) + (size_t)(tile & 15u) * (2u << 20);
        const GAS char* cB = (const GAS char*)(ws + OFF_XF) + (size_t)(tile >> 4) * (2u << 20);
        constexpr size_t kstep = 32768, hstep = 16384;
        f32x4 acc[2][2][4][2];
#pragma unroll
        for (int a = 0; a < 2; ++a)
#pragma unroll
            for (int b = 0; b < 2; ++b)
#pragma unroll
                for (int m = 0; m < 4; ++m)
#pragma unroll
                    for (int n = 0; n < 2; ++n) acc[a][b][m][n] = (f32x4){0.f, 0.f, 0.f, 0.f};
        bf16x8 At[4][2], B0[2][2], B1[2][2];
        STAGE(SB(0, 0), cB); STAGE(SA(0, 0), cA); STAGE(SB(0, 1), cB + hstep); STAGE(SA(0, 1), cA + hstep);
        if (wr == 1) BAR;
        WAIT_V(4); BAR;
        STAGE(SB(1, 0), cB + kstep); STAGE(SA(1, 0), cA + kstep); STAGE(SB(1, 1), cB + hstep + kstep);
        WAIT_V(6); BAR;
        for (int t = 0; t < 64; t += 2) {
            const int t2 = (t + 2) & 63;
            const GAS char* a1 = cA + (size_t)(t + 1) * kstep;
            const GAS char* a2 = cA + (size_t)t2 * kstep; const GAS char* b2 = cB + (size_t)t2 * kstep;
            const GAS char* a3 = a2 + kstep; const GAS char* b3 = b2 + kstep;
            LDB(B0, 0, 0); SCHED; LDA(At, 0, 0); STAGE(SA(1, 1), a1 + hstep);
            WAIT_L(8); BAR; WAIT_L(0); MMA(0, 0, At, B0); BAR; SCHED;
            LDB(B1, 0, 1); STAGE(SB(0, 0), b2);
            BAR; WAIT_L(0); MMA(0, 1, At, B1); BAR;
            LDA(At, 0, 1); STAGE(SA(0, 0), a2);
            BAR; WAIT_L(0); MMA(1, 0, At, B0); BAR; SCHED;
            STAGE(SB(0, 1), b2 + hstep);
            WAIT_V(6); BAR; MMA(1, 1, At, B1); BAR;
            LDB(B0, 1, 0); SCHED; LDA(At, 1, 0); STAGE(SA(0, 1), a2 + hstep);
            WAIT_L(8); BAR; WAIT_L(0); MMA(0, 0, At, B0); BAR; SCHED;
            LDB(B1, 1, 1); STAGE(SB(1, 0), b3);
            BAR; WAIT_L(0); MMA(0, 1, At, B1); BAR;
            LDA(At, 1, 1); STAGE(SA(1, 0), a3);
            BAR; WAIT_L(0); MMA(1, 0, At, B0); BAR; SCHED;
            STAGE(SB(1, 1), b3 + hstep);
            WAIT_V(6); BAR; MMA(1, 1, At, B1); BAR;
        }
        WAIT_V(0);
        if (wr == 0) BAR;
        BAR;
        STAMP_AT(4 + round * 3);
        const u32 ord = tab8(p.tile2ord, tile);
#pragma unroll
        for (int bj = 0; bj < 2; ++bj) {
            u32 v[8][4];
#pragma unroll
            for (int g = 0; g < 8; ++g)
#pragma unroll
                for (int q = 0; q < 4; ++q) {
                    typedef float f32x2 __attribute__((ext_vector_type(2)));
                    typedef __bf16 bf16x2 __attribute__((ext_vector_type(2)));
                    const f32x2 f = {acc[q >> 1][bj][(q & 1) * 2][g >> 2][g & 3], acc[q >> 1][bj][(q & 1) * 2 + 1][g >> 2][g & 3]};
                    v[g][q] = bad ? 0x7fc07fc0u : __builtin_bit_cast(u32, __builtin_convertvector(f, bf16x2));
                }
            LAS u32* cnt = ctl + 16 + (round * 2 + bj) * 2;
            encode_block(v, mk_rsrc(ws + OFF_SY + (size_t)(ord * 2u + bj) * CH, CH), (u32)wid, cnt, wl, lane);
            asm volatile("s_waitcnt vmcnt(0)" ::: "memory");
            u32 total;
            if (chunk_done(cnt, lane, total) && lane < 7)
                __hip_atomic_store((GAS u64*)(p.ws[(rank + 1u + (u32)lane) & 7u] + OFF_GRAN) + (size_t)rank * NC + NCIN + ord * 2u + bj, ((u64)total << 32) | call, __ATOMIC_RELAXED, __HIP_MEMORY_SCOPE_SYSTEM);
            const Dest dst = dest_y(p.out, tile, bj, (u32)wid, rank);
            const u32 loff = (u32)(lane >> 4) * dst.LH + (u32)(lane & 15) * 16u;
#pragma unroll
            for (int g = 0; g < 8; ++g) store_group(dst, loff, g, (u32x4){v[g][0], v[g][1], v[g][2], v[g][3]});
        }
        STAMP_AT(5 + round * 3);
    }
}

static void build_tables(Params& p) {
    static const int st[2][8][2] = {{{0, 0}, {1, 0}, {0, 1}, {1, 1}, {2, 0}, {2, 1}, {0, 2}, {1, 2}}, {{2, 2}, {3, 0}, {3, 1}, {3, 2}, {0, 3}, {1, 3}, {2, 3}, {3, 3}}};
    float fin[256], prev[128];
    for (int r = 0; r < 2; ++r)
        for (int b = 0; b < 128; ++b) {
            const int c = b & 7, idx = b >> 3, i = st[r][c][0] * 4 + (idx >> 2), j = st[r][c][1] * 4 + (idx & 3), t = i * 16 + j;
            const int last = (2 * i > 2 * j + 1 ? 2 * i : 2 * j + 1) + 1;
            float start = 8.f + 3.05f * last;
            if (r == 1 && prev[b] > start) start = prev[b];
            fin[t] = prev[b] = start + 78.f;
            ((unsigned char*)p.wgtile)[r * 128 + b] = (unsigned char)t;
        }
    int o[256];
    for (int t = 0; t < 256; ++t) { int k = t; while (k > 0 && fin[o[k - 1]] > fin[t]) { o[k] = o[k - 1]; --k; } o[k] = t; }
    for (int k = 0; k < 256; ++k) { ((unsigned char*)p.ord2tile)[k] = (unsigned char)o[k]; ((unsigned char*)p.tile2ord)[o[k]] = (unsigned char)k; }
}

extern "C" void kernel_launch(int rank, int nranks, ncclComm_t comm, hipStream_t stream,
                              void* const* d_in, const int* in_sizes, int n_in, void* d_out, int out_size,
                              const int* in_dtypes, int out_dtype, void* const* d_out_all,
                              void* const* d_ws_all, size_t ws_bytes) {
    if (nranks != NR || (unsigned)rank >= (unsigned)NR || ws_bytes < WS_NEED || n_in != 2) { fprintf(stderr, "unsupported configuration\n"); return; }
    Params p;
    p.x = (const char*)d_in[0]; p.w = (const char*)d_in[1]; p.out = (char*)d_out; p.rank = rank; p.pad = 0;
    for (int i = 0; i < NR; ++i) p.ws[i] = (char*)d_ws_all[i];
    build_tables(p);
    hipLaunchKernelGGL(gemm_ar_fused, dim3(256), dim3(512), 0, stream, p);
    HIP_CHECK(hipGetLastError());
}
```

```cpp
#include <hip/hip_runtime.h>
#include <rccl/rccl.h>
#include <cstdio>

#define HIP_CHECK(expr) do { hipError_t _e = (expr); if (_e != hipSuccess) { \
    fprintf(stderr, "HIP error %s at %s:%d\n", hipGetErrorString(_e), __FILE__, __LINE__); \
    return; } } while (0)

typedef unsigned u32;
typedef unsigned long long u64;
typedef u32 u32x4 __attribute__((ext_vector_type(4)));
typedef u32 u32x2 __attribute__((ext_vector_type(2)));
typedef unsigned short u16x2 __attribute__((ext_vector_type(2)));
typedef short bf16x8 __attribute__((ext_vector_type(8)));
typedef float f32x4 __attribute__((ext_vector_type(4)));
#define LAS __attribute__((address_space(3)))
#define GAS __attribute__((address_space(1)))
typedef __amdgpu_buffer_rsrc_t rsrc_t;

constexpr int NR = 8;
#define PULL_WG_PER_PEER 8
#define WINDOW 8
#define NLOAD 1
#define TAILC 0
#define LOOK 6
#define RHO 4
#define GLDS_AUX 17
#define ENC_FIRST 16
#define ENC_DELAY 150
#define ENC_PACE 256
#define POLL_SLEEP 8
#define WB 12
constexpr int NPULL = 7 * PULL_WG_PER_PEER, NGEMM = 16 * WB;
constexpr int POWN = 16 - RHO, NP = POWN + 8 * RHO;
constexpr int NB = (NP + WB - 1) / WB, N0 = 16 + WB;
constexpr int NCIN = 64 + 4 * NP, NCY = 32 * POWN, NC = NCIN + NCY;
constexpr int NTASK = 16 * 36, TQ0 = N0 + 7 * POWN;
constexpr int NDEC = 7 - NLOAD;
constexpr u64 DEADLINE = 100000000ull;
static_assert(NGEMM + NPULL <= 256 && POWN <= WB && NB <= 8, "roles");

constexpr int FIXB = 5120, CVAR = 8 * FIXB, CHDR = 32, CH = 66560;
constexpr size_t OFF_CALL = 0;
constexpr size_t OFF_TMO = 131072;
constexpr size_t OFF_TICK = OFF_TMO + 256;
constexpr size_t OFF_RDY = OFF_TICK + 16 * 128;
constexpr size_t OFF_FLAG = OFF_RDY + 32 * 128;
constexpr size_t OFF_GRAN = 196608;
constexpr size_t OFF_DBG = 524288;
constexpr size_t OFF_XF = 1ull << 20;
constexpr size_t OFF_WF = OFF_XF + (32ull << 20);
constexpr size_t OFF_SY = OFF_WF + NP * (2ull << 20);
constexpr size_t OFF_SI = OFF_SY + (size_t)NCY * CH;
constexpr size_t WS_NEED = OFF_SI + 2ull * NTASK * CH;

constexpr int A2B = 2064, A3B = 528, VARB = 912, WLDS = A2B + A3B + VARB, N2MAX = 2032, N3MAX = 496, N4MAX = 256;
constexpr int CTL_OFF = 2048, CTL_WORDS = 128, WL_OFF = 2560, BIG_OFF = 30720, LDS_BYTES = BIG_OFF + 131072;
static_assert(WL_OFF + 8 * WLDS <= BIG_OFF && LDS_BYTES <= 163840, "LDS map");
constexpr int NF = 16, NV = 4, VSLOT = 8192, VRING = NF * FIXB;
constexpr int AUX_SYS = 17, AUX_DEV = 16;

struct Params {
    const char* x; const char* w; char* out; char* ws[NR];
    int rank; int pad;
};

__device__ __forceinline__ u32 rfl(u32 x) { return (u32)__builtin_amdgcn_readfirstlane((int)x); }
__device__ __forceinline__ rsrc_t mk_rsrc(const void* p, u32 bytes) {
    const u64 a = (u64)p; const u64 u = ((u64)rfl((u32)(a >> 32)) << 32) | rfl((u32)a);
    return __builtin_amdgcn_make_buffer_rsrc((void*)u, (short)0, (int)rfl(bytes), 0x00020000);
}
__device__ __forceinline__ u32 vperm(u32 a, u32 b, u32 s) { return __builtin_amdgcn_perm(a, b, s); }
__device__ __forceinline__ u32 wave_incl_scan(u32 x) {
    x += (u32)__builtin_amdgcn_update_dpp(0, (int)x, 0x111, 0xf, 0xf, false);
    x += (u32)__builtin_amdgcn_update_dpp(0, (int)x, 0x112, 0xf, 0xf, false);
    x += (u32)__builtin_amdgcn_update_dpp(0, (int)x, 0x114, 0xf, 0xf, false);
    x += (u32)__builtin_amdgcn_update_dpp(0, (int)x, 0x118, 0xf, 0xf, false);
    x += (u32)__builtin_amdgcn_update_dpp(0, (int)x, 0x142, 0xa, 0xf, false);
    x += (u32)__builtin_amdgcn_update_dpp(0, (int)x, 0x143, 0xc, 0xf, false);
    return x;
}
__device__ __forceinline__ u32 pk_sub(u32 a, u32 b) { return __builtin_bit_cast(u32, (u16x2)(__builtin_bit_cast(u16x2, a) - __builtin_bit_cast(u16x2, b))); }
#define WAVE_FENCE() __builtin_amdgcn_fence(__ATOMIC_ACQ_REL, "wavefront")

__device__ __forceinline__ void build_table(LAS u32x2* tab, u32 i, bool unpack) {
    u32 n = 0, dep = 0, cmp = 0x0c0c0c0cu;
#pragma unroll
    for (u32 t = 0; t < 4; ++t) {
        const u32 c = (i >> (2 * t)) & 3u;
        dep |= (c ? 4u + c : n) << (8 * t);
        if (!c) { cmp = (cmp & ~(0xffu << (8 * n))) | (t << (8 * n)); ++n; }
    }
    tab[i] = (u32x2){unpack ? dep : cmp, n};
}
__device__ __forceinline__ u32 code_byte(u32 hi, u32 lo, u32 x) {
    const u32 f = vperm(hi, lo, x);
    return __builtin_amdgcn_udot4((f ^ (f >> 2)) & 0x03030303u, 0x40100401u, 0u, false);
}
#define L1PRED 0
template <int PRED>
__device__ __forceinline__ void or_bytes(LAS char* a, u32 pos, u32 x) {
    const u64 y = (u64)x << ((pos & 3u) * 8u);
    LAS u32* q = (LAS u32*)(a + (pos & ~3u));
    if (PRED == 3) { if (x) {
        __hip_atomic_fetch_or(q, (u32)y, __ATOMIC_RELAXED, __HIP_MEMORY_SCOPE_WORKGROUP);
        __hip_atomic_fetch_or(q + 1, (u32)(y >> 32), __ATOMIC_RELAXED, __HIP_MEMORY_SCOPE_WORKGROUP); }
        return; }
    if (PRED < 2 || (u32)y) __hip_atomic_fetch_or(q, (u32)y, __ATOMIC_RELAXED, __HIP_MEMORY_SCOPE_WORKGROUP);
    if (PRED < 1 || (u32)(y >> 32)) __hip_atomic_fetch_or(q + 1, (u32)(y >> 32), __ATOMIC_RELAXED, __HIP_MEMORY_SCOPE_WORKGROUP);
}
__device__ __forceinline__ u32 get_bytes(const LAS char* a, u32 pos) {
    const LAS u32* q = (const LAS u32*)(a + (pos & ~3u));
    return __builtin_amdgcn_alignbyte(q[1], q[0], pos);
}
template <bool LAST>
__device__ __forceinline__ u32 pack_level(const LAS u32x2* tab, const LAS char* src, u32 n, LAS u32* codes, LAS char* dst, u32 lim, u32 hi, u32 lo, u32 t4, int lane) {
    u32 m = 0;
    for (u32 i0 = 0; i0 < n; i0 += 1024u) {
        const u32 i = i0 + (u32)lane * 16u;
        const u32x4 x = *(const LAS u32x4*)(src + (i < n ? i : 0u));
        u32 cw = code_byte(hi, lo, x[0]) | (code_byte(hi, lo, x[1]) << 8) | (code_byte(hi, lo, x[2]) << 16) | (code_byte(hi, lo, x[3]) << 24);
        const u32 valid = i < n ? n - i : 0u;
        if (valid < 16u) cw |= 0x55555555u << (2u * valid);
        u32x2 e[4]; u32 c[5]; c[0] = 0;
#pragma unroll
        for (int j = 0; j < 4; ++j) { e[j] = tab[(cw >> (8 * j)) & 0xffu]; c[j + 1] = c[j] + e[j][1]; }
        const u32 incl = wave_incl_scan(c[4]), p0 = m + incl - c[4];
        if (i < n) codes[i >> 4] = cw;
        m += (u32)__builtin_amdgcn_readlane((int)incl, 63);
        if (m <= lim) {
#pragma unroll
            for (int j = 0; j < 4; ++j) {
                u32 y = x[j] ^ 0x08080808u;
                if (LAST) y = ((t4 | 0x80808080u) - (y & 0x7f7f7f7fu)) ^ ((t4 ^ ~y) & 0x80808080u);
                or_bytes<3>(dst, p0 + c[j], vperm(0u, y, e[j][0]));
            }
        }
    }
    WAVE_FENCE();
    return m;
}
__device__ __forceinline__ void encode_block(const u32 (&v)[8][4], rsrc_t ch, u32 b, LAS u32* alloc, const LAS u32x2* tab, LAS char* wl, int lane) {
#pragma unroll
    for (int j = 0; j < 4; ++j) if (j < 3 || lane < (WLDS - 3072) / 16) *(LAS u32x4*)(wl + j * 1024 + lane * 16) = (u32x4){0, 0, 0, 0};
    u32 s = 0;
#pragma unroll
    for (int g = 0; g < 8; ++g) s = __builtin_amdgcn_udot4(__builtin_amdgcn_alignbit(v[g][0], v[g][0], 31), 0x01000100u, s, false);
    u32 T = ((u32)__builtin_amdgcn_readlane((int)wave_incl_scan(s), 63) >> 10) + 5u;
    T = T > 255u ? 255u : T;
    const u32 tt = (T * 256u + 255u) * 0x10001u;
    u32 sm[16], U[16], l1[4] = {0, 0, 0, 0}, sel[16], c[17];
    c[0] = 0;
#pragma unroll
    for (int k = 0; k < 16; ++k) {
        const u32 a0 = v[k >> 1][(k & 1) * 2], a1 = v[k >> 1][(k & 1) * 2 + 1];
        const u32 r0 = __builtin_amdgcn_alignbit(a0, a0, 31), r1 = __builtin_amdgcn_alignbit(a1, a1, 31);
        sm[k] = vperm(r1, r0, 0x06040200u);
        U[k] = vperm(pk_sub(tt, r1), pk_sub(tt, r0), 0x07050301u);
        const u32 cb = code_byte(0x00000302u, 0x01000000u, U[k]);
        l1[k >> 2] |= cb << (8 * (k & 3));
        const u32x2 e = tab[cb];
        sel[k] = e[0]; c[k + 1] = c[k] + e[1];
    }
    const u32 incl = wave_incl_scan(c[16]), p0 = incl - c[16];
    const u32 n2 = (u32)__builtin_amdgcn_readlane((int)incl, 63);
    const int fo = (int)(b * FIXB);
    bool raw = n2 > (u32)N2MAX;
    u32 d2 = 0, d3 = 0, vb = 0;
    if (!raw) {
        WAVE_FENCE();
#pragma unroll
        for (int k = 0; k < 16; ++k) or_bytes<L1PRED>(wl, p0 + c[k], vperm(0u, U[k], sel[k]));
        WAVE_FENCE();
        LAS char* var = wl + A2B + A3B;
        d2 = (n2 + 15u) >> 4;
        const u32 n3 = pack_level<false>(tab, wl, n2, (LAS u32*)var, wl + A2B, N3MAX, 0x02010000u, 0x00030000u, 0u, lane);
        raw = n3 > (u32)N3MAX;
        if (!raw) {
            d3 = (n3 + 15u) >> 4;
            const u32 n4 = pack_level<true>(tab, wl + A2B, n3, (LAS u32*)var + d2, var + (d2 + d3) * 4u, N4MAX, 0u, 0x00030201u, T * 0x01010101u, lane);
            raw = n4 > (u32)N4MAX;
            vb = ((d2 + d3) * 4u + n4 + 7u) & ~7u;
        }
    }
    u32 off = 0;
    if (lane == 0) off = __hip_atomic_fetch_add(alloc, raw ? 3072u : vb, __ATOMIC_RELAXED, __HIP_MEMORY_SCOPE_WORKGROUP);
    off = rfl(off) + CHDR;
    const int vo = CVAR + (int)off;
    if (lane == 0) __builtin_amdgcn_raw_buffer_store_b32((raw ? 0u : (T | (d2 << 8) | (d3 << 15))) | (off << 17), ch, 0, CVAR + (int)b * 4, AUX_SYS);
    if (raw) {
#pragma unroll
        for (int g = 0; g < 8; ++g)
            __builtin_amdgcn_raw_buffer_store_b128((u32x4){v[g][0], v[g][1], v[g][2], v[g][3]}, ch, lane * 16 + (g < 5 ? g : g - 5) * 1024, g < 5 ? fo : vo, AUX_SYS);
    } else {
#pragma unroll
        for (int q = 0; q < 4; ++q)
            __builtin_amdgcn_raw_buffer_store_b128((u32x4){sm[4 * q], sm[4 * q + 1], sm[4 * q + 2], sm[4 * q + 3]}, ch, lane * 16 + q * 1024, fo, AUX_SYS);
        __builtin_amdgcn_raw_buffer_store_b128((u32x4){l1[0], l1[1], l1[2], l1[3]}, ch, lane * 16 + 4096, fo, AUX_SYS);
        for (u32 o = (u32)lane * 8u; o < vb; o += 512u)
            __builtin_amdgcn_raw_buffer_store_b64(*(const LAS u32x2*)(wl + A2B + A3B + o), ch, (int)o, vo, AUX_SYS);
    }
    WAVE_FENCE();
}
__device__ __forceinline__ bool chunk_done(LAS u32* cnt, int lane, u32& total) {
    u32 n = 0;
    if (lane == 0) n = __hip_atomic_fetch_add(cnt + 1, 1u, __ATOMIC_RELAXED, __HIP_MEMORY_SCOPE_WORKGROUP);
    if (rfl(n) != 7u) return false;
    total = rfl(*(volatile LAS u32*)cnt) + CHDR;
    return true;
}

struct Dest { rsrc_t r; u32 A, B, C, LH; };
#define AUX_Y AUX_DEV
#define AUX_T AUX_DEV
template <int AUX = AUX_DEV>
__device__ __forceinline__ void store_group(const Dest& d, u32 loff, int g, u32x4 x) {
    __builtin_amdgcn_raw_buffer_store_b128(x, d.r, (int)loff, (int)((g & 1 ? d.A : 0u) + (g & 2 ? d.B : 0u) + (g & 4 ? d.C : 0u)), AUX);
}

__device__ __forceinline__ void unpack_rest(const LAS u32x2* tab, const LAS u32* codes, u32 i0, u32 nd, const LAS char* src, u32 m, u32 mask, u32 t, LAS char* dst, int lane) {
    for (; i0 < nd; i0 += 64u) {
        const u32 i = i0 + (u32)lane;
        const u32 cw = i < nd ? codes[i] : 0x55555555u;
        u32x2 e[4]; u32 c[5]; c[0] = 0;
#pragma unroll
        for (int j = 0; j < 4; ++j) { e[j] = tab[(cw >> (8 * j)) & 0xffu]; c[j + 1] = c[j] + e[j][1]; }
        const u32 incl = wave_incl_scan(c[4]), p0 = m + incl - c[4];
        u32x4 o;
#pragma unroll
        for (int j = 0; j < 4; ++j) o[j] = vperm(t, get_bytes(src, (p0 + c[j]) & mask), e[j][0]);
        if (i < nd) *(LAS u32x4*)(dst + i * 16u) = o;
        m += (u32)__builtin_amdgcn_readlane((int)incl, 63);
    }
}
__device__ __forceinline__ void decode_block(const u32x4 (&d)[5], u32 hdr, const LAS char* var, const LAS u32x2* tab, LAS char* wl, int lane, u32x4 (&o)[8]) {
    const u32 T = hdr & 0xffu, d2 = (hdr >> 8) & 0x7fu, d3 = (hdr >> 15) & 0x1fu;
    auto ex = [&](u32 a, u32 b, u32 c) { return (((T - a) & 0xffu) << 8) | (((T - b) & 0xffu) << 16) | (((T - c) & 0xffu) << 24); };
    const u32 t1 = ex(3, 4, 5), t2 = ex(6, 7, 2), t3 = ex(8, 9, 10);
    const LAS u32* c2 = (const LAS u32*)var;
    const LAS char* r4 = var + (d2 + d3) * 4u;
    const u32 w2 = (u32)lane < d2 ? c2[lane] : 0x55555555u;
    const u32 w3 = (u32)lane < d3 * 4u ? (u32)*(const LAS unsigned char*)(var + d2 * 4u + lane) : 0x55u;
    u32 sel[16], c[17];
    c[0] = 0;
#pragma unroll
    for (int k = 0; k < 16; ++k) { const u32x2 e = tab[(d[4][k >> 2] >> (8 * (k & 3))) & 0xffu]; sel[k] = e[0]; c[k + 1] = c[k] + e[1]; }
    u32x2 e2[4]; u32 f[5]; f[0] = 0;
#pragma unroll
    for (int j = 0; j < 4; ++j) { e2[j] = tab[(w2 >> (8 * j)) & 0xffu]; f[j + 1] = f[j] + e2[j][1]; }
    const u32x2 e3 = tab[w3];
    const u32 pk = c[16] | (f[4] << 11) | (e3[1] << 22);
    const u32 incl = wave_incl_scan(pk), ex0 = incl - pk;
    const u32 p1 = ex0 & 0x7ffu, p2 = (ex0 >> 11) & 0x7ffu, p3 = ex0 >> 22;
    *(LAS u32*)(wl + A2B + lane * 4) = vperm(t3, get_bytes(r4, p3 & 0x1ffu), e3[0]);
    if (d3 > 16u || d2 > 64u) {
        const u32 tot = (u32)__builtin_amdgcn_readlane((int)incl, 63);
        unpack_rest(tab, c2 + d2, 16u, d3, r4, tot >> 22, 0x1ffu, t3, wl + A2B, lane);
        WAVE_FENCE();
        unpack_rest(tab, c2, 64u, d2, wl + A2B, (tot >> 11) & 0x7ffu, 0x1ffu, t2, wl, lane);
    }
    WAVE_FENCE();
    u32x4 a2;
#pragma unroll
    for (int j = 0; j < 4; ++j) a2[j] = vperm(t2, get_bytes(wl + A2B, (p2 + f[j]) & 0x1ffu), e2[j][0]);
    *(LAS u32x4*)(wl + lane * 16) = a2;
    WAVE_FENCE();
#pragma unroll
    for (int k = 0; k < 16; ++k) {
        const u32 e = vperm(t1, get_bytes(wl, p1 + c[k]), sel[k]);
        const u32 smk = d[k >> 2][k & 3];
        const u32 r0 = vperm(e, smk, 0x05010400u), r1 = vperm(e, smk, 0x07030602u);
        o[k >> 1][2 * (k & 1)] = __builtin_amdgcn_alignbit(r0, r0, 1);
        o[k >> 1][2 * (k & 1) + 1] = __builtin_amdgcn_alignbit(r1, r1, 1);
    }
    WAVE_FENCE();
}

__device__ __forceinline__ Dest dest_x(char* ws, u32 s, u32 q, u32 b, u32 src) {
    return Dest{mk_rsrc(ws + OFF_XF + (size_t)((s * 64u + q * 16u + src * 2u) * 32u + b * 4u) * 1024u, 32768u + 4096u), 1024u, 32768u, 2048u, 256u};
}
__device__ __forceinline__ Dest dest_w(char* ws, u32 s, u32 q, u32 b, u32 src) {
    return Dest{mk_rsrc(ws + OFF_WF + (size_t)((s * 64u + q * 16u + src * 2u + (b >> 2)) * 32u + (b & 1u) * 8u + ((b >> 1) & 1u)) * 1024u, 16384u + 6144u + 1024u), 2048u, 4096u, 16384u, 256u};
}
__device__ __forceinline__ Dest dest_y(char* out, u32 i, u32 gp, u32 bj, u32 wv) {
    const u32 row = i * 256u + bj * 128u + (wv & 3u) * 32u, col = gp * 256u + (wv >> 2) * 128u;
    return Dest{mk_rsrc(out + (size_t)row * 65536u + col * 2u, 31u * 65536u + 256u), 65536u, 131072u, 16u * 65536u, 4u * 65536u};
}
__device__ __forceinline__ u32 gpanel(u32 r, u32 j) { return j < (u32)POWN ? r * 16u + j : ((j - POWN) & 7u) * 16u + POWN + ((j - POWN) >> 3); }
__device__ __forceinline__ u32 slot_in(bool isw, u32 s, u32 q, u32 dst) {
    return !isw ? s * 4u + q : (s < (u32)POWN ? 64u + (dst * POWN + s) * 4u + q : 64u + 32u * POWN + (s - POWN) * 4u + q);
}
__device__ __forceinline__ u32 batch_size(u32 bt) { return bt == 0u ? (u32)N0 : (bt == NB - 1u ? (u32)(NP - (NB - 1) * WB) : (u32)WB); }
__device__ __forceinline__ u32 pos_in(bool isw, u32 s, u32 q) {
    if (!isw || s < (u32)WB) return q * N0 + (isw ? 16u : 0u) + s;
    const u32 bt = s / WB;
    return 64u + 4u * WB * bt + q * batch_size(bt) + s - bt * WB;
}
struct InPos { bool isw; u32 s, q, cnt; };
__device__ __forceinline__ InPos in_of(u32 c) {
    if (c < 4u * N0) { const u32 q = c / N0, pp = c - q * N0; return InPos{pp >= 16u, pp >= 16u ? pp - 16u : pp, q, q}; }
    const u32 bt = (c - 64u) / (4u * WB), r = c - 64u - bt * 4u * WB, q = r / batch_size(bt);
    return InPos{true, bt * WB + r - q * batch_size(bt), q, bt * 4u + q};
}

__device__ __forceinline__ void arrive(char* ws, u32 ci, u32 call, int lane) {
    u32 old = 0;
    if (lane == 0) old = __hip_atomic_fetch_add((GAS u32*)(ws + OFF_RDY + ci * 128u), 1u, __ATOMIC_RELAXED, __HIP_MEMORY_SCOPE_AGENT);
    if (rfl(old) + 1u == call * 8u * batch_size(ci >> 2) && lane < 8)
        __hip_atomic_store((GAS u32*)(ws + OFF_FLAG + (u32)lane * 128u + ci * 4u), call, __ATOMIC_RELAXED, __HIP_MEMORY_SCOPE_AGENT);
}

__device__ __forceinline__ bool give_up(GAS u32* tmo, u32 call, u64 t0) {
    if (rfl(__hip_atomic_load(tmo, __ATOMIC_RELAXED, __HIP_MEMORY_SCOPE_AGENT)) == call || __builtin_amdgcn_s_memrealtime() - t0 > DEADLINE) {
        __hip_atomic_store(tmo, call, __ATOMIC_RELAXED, __HIP_MEMORY_SCOPE_AGENT);
        return true;
    }
    return false;
}

#define STAMP_AT(i) do { } while (0)

__device__ __forceinline__ u32 lds_ld(LAS u32* p) { return rfl(*(volatile LAS u32*)p); }
__device__ __forceinline__ void dma_piece(rsrc_t r, u32 voff, u32 soff, u32 ldsaddr) {
    u32 keep;
    asm volatile("s_nop 4\n\ts_mov_b32 %0, m0\n\ts_mov_b32 m0, %1\n\ts_nop 0\n\tbuffer_load_dwordx4 %2, %3, %4 offen sc0 sc1 lds\n\ts_mov_b32 m0, %0"
                 : "=&s"(keep) : "s"(ldsaddr), "v"(voff), "s"(r), "s"(soff) : "memory");
}

__global__ void __launch_bounds__(512, 2) gemm_ar_fused(const Params p) {
    __shared__ __attribute__((aligned(16))) char smem[LDS_BYTES];
    LAS char* lds = (LAS char*)smem + BIG_OFF;
    LAS u32x2* tab = (LAS u32x2*)smem;
    const int tid = threadIdx.x, wid = (int)rfl((u32)tid >> 6), lane = tid & 63;
    const u32 rank = (u32)p.rank, bid = blockIdx.x;
    char* ws = p.ws[rank & 7u];
    LAS char* wl = (LAS char*)smem + WL_OFF + wid * WLDS;
    LAS u32* ctl = (LAS u32*)((LAS char*)smem + CTL_OFF);
    GAS u32* tmo = (GAS u32*)(ws + OFF_TMO);
    STAMP_AT(0);
    GAS u32* callp = (GAS u32*)(ws + OFF_CALL + (bid * 8u + wid) * 64u);
    const u32 callv = __hip_atomic_load(callp, __ATOMIC_RELAXED, __HIP_MEMORY_SCOPE_AGENT);
    const bool ispull = bid >= (u32)NGEMM && bid < (u32)(NGEMM + NPULL);
    if (tid < 256) build_table(tab, (u32)tid, ispull); else if (tid < 256 + CTL_WORDS) ctl[tid - 256] = 0u;
    asm volatile("s_waitcnt lgkmcnt(0)" ::: "memory"); __builtin_amdgcn_s_barrier(); asm volatile("" ::: "memory");

    if (bid >= (u32)(NGEMM + NPULL)) return;
    if (ispull) {
        const u32 pj = bid - NGEMM, src = (rank + 1u + pj % 7u) & 7u;
        const char* pws = p.ws[src];
        LAS u32* qavail = ctl; LAS u32* vready = ctl + 1; LAS u32* prog = ctl + 2; LAS u32* landed = ctl + 4;
        LAS u32* q = ctl + 16;
        LAS u32* vdone = ctl + 32; LAS u32* ffree = ctl + 48;
        auto chunk_of = [&](u32 fl) -> const char* {
            const u32 c = fl & 0xffffu;
            if (c >= (u32)NCIN) return pws + OFF_SY + (size_t)(c - NCIN) * CH;
            const InPos ip = in_of(c);
            return pws + OFF_SI + (size_t)(((fl >> 16) & 1u) * NTASK + slot_in(ip.isw, ip.s, ip.q, rank)) * CH;
        };
        const u32 l0 = (u32)(size_t)lds, voff = (u32)lane * 16u;
        if (wid == 0) {
            const u32 call = rfl(callv) + 1u;
            if (lane == 0) __hip_atomic_store(callp, call, __ATOMIC_RELAXED, __HIP_MEMORY_SCOPE_AGENT);
            GAS u64* gran = (GAS u64*)(ws + OFF_GRAN) + (size_t)((call & 1u) * 8u + src) * NC;
            for (u32 k = 0;; ++k) {
                u32 c = pj / 7u;
                if (k) {
                    while (lds_ld(prog) + LOOK < 8u * k) __builtin_amdgcn_s_sleep(2);
                    u32 tk = 0;
                    if (lane == 0) tk = __hip_atomic_fetch_add((GAS u32*)(ws + OFF_TICK + src * 128u), 1u, __ATOMIC_RELAXED, __HIP_MEMORY_SCOPE_AGENT);
                    c = rfl(tk) - (call - 1u) * NC + PULL_WG_PER_PEER;
                }
                u32 fl = 0x80000000u, vb = 0;
                if (c < (u32)NC) {
                    const u64 t0 = __builtin_amdgcn_s_memrealtime();
                    for (u32 spin = 0;; ++spin) {
                        const u64 g = __hip_atomic_load(gran + c, __ATOMIC_RELAXED, __HIP_MEMORY_SCOPE_SYSTEM);
                        if (rfl((u32)g) == call) { fl = c | ((call & 1u) << 16); vb = rfl((u32)(g >> 32)); break; }
                        if ((spin & 15u) == 15u && give_up(tmo, call, t0)) { fl = c | 0x40000000u; break; }
                        __builtin_amdgcn_s_sleep(1);
                    }
                    if (k == 0) STAMP_AT(1);
                }
                while (lds_ld(vdone + (k & (NV - 1))) < 8u * (k / NV)) __builtin_amdgcn_s_sleep(2);
                if (lane == 0) { *(volatile LAS u32x4*)(q + (k & (NV - 1)) * 4u) = (u32x4){fl, vb, call, 0u}; *(volatile LAS u32*)qavail = k + 1u; }
                if (!(fl >> 30)) {
                    const u32 vl = vb > (u32)VSLOT ? (u32)CHDR : (vb + 15u) & ~15u;
                    const rsrc_t r = mk_rsrc(chunk_of(fl), CVAR + vl);
                    for (u32 o = 0; o < vl; o += 1024u) dma_piece(r, voff + CVAR + o, 0u, l0 + VRING + (k & (NV - 1)) * VSLOT + o);
                    asm volatile("s_waitcnt vmcnt(0)" ::: "memory");
                }
                if (lane == 0) *(volatile LAS u32*)vready = k + 1u;
                if (fl >> 31) break;
            }
        } else if (wid <= NLOAD) {
            __builtin_amdgcn_s_setprio(3);
            LAS u32* mine = landed + (wid - 1);
            u32 issued = 0, pub = 0, kc = 0xffffffffu, fl = 0;
            rsrc_t r = mk_rsrc(pws, 0u);
            auto drain = [&]() { asm volatile("s_waitcnt vmcnt(0)" ::: "memory"); pub = issued; if (lane == 0) *(volatile LAS u32*)mine = pub; };
            for (u32 n = (u32)wid - 1u;; n += NLOAD) {
                const u32 k = n >> 3, sl = n & (NF - 1);
                if (k != kc) {
                    if (lds_ld(qavail) <= k) { drain(); while (lds_ld(qavail) <= k) __builtin_amdgcn_s_sleep(1); }
                    fl = lds_ld(q + (k & (NV - 1)) * 4u);
                    if (fl >> 31) break;
                    r = mk_rsrc(chunk_of(fl), (fl >> 30) ? 0u : (u32)CVAR);
                    kc = k;
                }
                if (wid == 1 && lane == 0) *(volatile LAS u32*)prog = n + 1u;
                if (lds_ld(ffree + sl) + NF <= n) { drain(); while (lds_ld(ffree + sl) + NF <= n) __builtin_amdgcn_s_sleep(1); }
                const bool wide = TAILC && (fl & 0xffffu) >= (u32)(NC - TAILC);
#pragma unroll
                for (u32 j = 0; j < 5u; ++j) {
                    dma_piece(r, voff, (n & 7u) * FIXB + j * 1024u, l0 + sl * FIXB + j * 1024u);
                    ++issued;
                    u32 done;
                    if (wide) { asm volatile("s_waitcnt vmcnt(%0)" :: "n"(2 * WINDOW - 1) : "memory"); done = issued - (2 * WINDOW - 1); }
                    else { asm volatile("s_waitcnt vmcnt(%0)" :: "n"(WINDOW - 1) : "memory"); done = issued - (WINDOW - 1); }
                    if ((wide || (j + 6u - WINDOW % 5u + 1u) % 5u == 0u) && (int)(done - pub) > 0) { pub = done; if (lane == 0) *(volatile LAS u32*)mine = pub; }
                }
            }
            drain();
            STAMP_AT(2);
        } else {
#define SEG(i)
            for (u32 n = (u32)wid - 1u - NLOAD;; n += NDEC) {
                const u32 k = n >> 3, b = n & 7u, sl = n & (NF - 1);
                while (lds_ld(vready) <= k) __builtin_amdgcn_s_sleep(1);
                const u32x4 e = *(volatile LAS u32x4*)(q + (k & (NV - 1)) * 4u);
                const u32 fl = rfl(e[0]), vb = rfl(e[1]), c = fl & 0xffffu;
                if (fl >> 31) break;
                const bool ok = !(fl >> 30);
                { const u32 need = 5u * (n / NLOAD) + 5u; while (lds_ld(landed + n % NLOAD) < need) __builtin_amdgcn_s_sleep(1); }
                asm volatile("" ::: "memory");
                SEG(0);
                u32x4 d[5];
#pragma unroll
                for (int j = 0; j < 5; ++j) d[j] = *(const LAS u32x4*)(lds + sl * FIXB + j * 1024 + lane * 16);
                const LAS char* vs = lds + VRING + (k & (NV - 1)) * VSLOT;
                const u32 h = *(const LAS u32*)(vs + b * 4u);
                asm volatile("s_waitcnt lgkmcnt(0)" ::: "memory");
                if (lane == 0) *(volatile LAS u32*)(ffree + sl) = n + 1u;
                asm volatile("" : "+v"(d[0]), "+v"(d[1]), "+v"(d[2]), "+v"(d[3]), "+v"(d[4]) :: "memory");
                const u32 hdr = rfl(h), off = (hdr >> 20) * 8u;
                const bool far = vb > (u32)VSLOT;
                const LAS char* var = vs + (off < (u32)VSLOT ? off : 0u);
                u32x4 o[8];
                if (!ok) {
#pragma unroll
                    for (int g = 0; g < 8; ++g) o[g] = (u32x4){0x7fc07fc0u, 0x7fc07fc0u, 0x7fc07fc0u, 0x7fc07fc0u};
                } else {
                    if (far) {
                        const rsrc_t r = mk_rsrc(chunk_of(fl), CH);
#pragma unroll
                        for (int j = 0; j < 3; ++j) o[5 + j] = __builtin_amdgcn_raw_buffer_load_b128(r, (int)(CVAR + off) + j * 1024 + lane * 16, 0, AUX_SYS);
                        if (lane < VARB / 16) *(LAS u32x4*)(wl + A2B + A3B + lane * 16) = o[5];
                        WAVE_FENCE();
                        var = wl + A2B + A3B;
                    }
                    if (hdr & 0xffu) decode_block(d, hdr, var, tab, wl, lane, o);
                    else {
#pragma unroll
                        for (int g = 0; g < 5; ++g) o[g] = d[g];
                        if (!far) {
#pragma unroll
                            for (int j = 0; j < 3; ++j) {
                                const u32x2 lo = *(const volatile LAS u32x2*)(var + j * 1024 + lane * 16), hi = *(const volatile LAS u32x2*)(var + j * 1024 + lane * 16 + 8);
                                o[5 + j] = (u32x4){lo[0], lo[1], hi[0], hi[1]};
                            }
                        }
                    }
                }
                SEG(1);
                const bool isy = c >= (u32)NCIN;
                const InPos ip = in_of(isy ? 0u : c);
                const u32 yt = (c - NCIN) >> 1, yi = yt / POWN;
                const Dest dst = isy ? dest_y(p.out, yi & 15u, src * 16u + yt - yi * POWN, (c - NCIN) & 1u, b)
                               : (ip.isw ? dest_w(ws, ip.s, ip.q, b, src) : dest_x(ws, ip.s, ip.q, b, src));
                const u32 loff = (u32)(lane >> 4) * dst.LH + (u32)(lane & 15) * 16u;
                if (AUX_Y != AUX_DEV && isy) {
#pragma unroll
                    for (int g = 0; g < 8; ++g) store_group<AUX_Y>(dst, loff, g, o[g]);
                } else {
#pragma unroll
                    for (int g = 0; g < 8; ++g) store_group(dst, loff, g, o[g]);
                }
                SEG(2);
                if (!isy) asm volatile("s_waitcnt vmcnt(0)" ::: "memory");
                asm volatile("s_waitcnt lgkmcnt(0)" ::: "memory");
                u32 old = 0;
                if (lane == 0) old = __hip_atomic_fetch_add(vdone + (k & (NV - 1)), 1u, __ATOMIC_RELAXED, __HIP_MEMORY_SCOPE_WORKGROUP);
                if (!isy && (rfl(old) & 7u) == 7u) arrive(ws, ip.cnt, rfl(e[2]), lane);
                SEG(3);
            }
            STAMP_AT(3);
        }
        return;
    }

    {
        const u64 tb = __builtin_amdgcn_s_memrealtime();
        for (u32 task = bid, it = 0; task < (u32)NTASK; task += NGEMM, ++it) {
            if (task >= (u32)ENC_FIRST) while (__builtin_amdgcn_s_memrealtime() - tb < (u64)(ENC_DELAY + task * ENC_PACE / 64u)) __builtin_amdgcn_s_sleep(4);
            if (it == 0) STAMP_AT(12);
            u32 s, q, k = 0u, ci; bool isw = true; const u32 b = (u32)wid;
            if (task < 4u * TQ0) {
                q = task / TQ0; ci = q; const u32 t = task - q * TQ0;
                if (t < 16u) { s = t; isw = false; } else if (t < 16u + 8u * POWN) { s = (t - 16u) >> 3; k = ((t - 16u) & 7u) + 1u; } else s = t - 16u - 7u * POWN;
            } else { const InPos ip = in_of(task - 4u * TQ0 + 4u * N0); s = ip.s; q = ip.q; ci = ip.cnt; }
            const u32 dd = (rank + k) & 7u;
            u32 v[8][4];
            if (!isw) {
                const rsrc_t r = mk_rsrc(p.x + (size_t)(s * 256u + b * 32u) * 1024u + q * 256u, 31u * 1024u + 256u);
#pragma unroll
                for (int g = 0; g < 8; ++g) {
                    const u32x4 a = __builtin_amdgcn_raw_buffer_load_b128(r, (lane & 15) * 1024 + (lane >> 4) * 16 + (g & 3) * 64, (g >> 2) * 16384, 0);
                    v[g][0] = a[0]; v[g][1] = a[1]; v[g][2] = a[2]; v[g][3] = a[3];
                }
            } else {
                const rsrc_t r = mk_rsrc(p.w + (size_t)(q * 128u + (b >> 1) * 32u) * 65536u + (gpanel(dd, s) * 256u + (b & 1u) * 128u) * 2u, 31u * 65536u + 256u);
                u32x4 a[8];
#pragma unroll
                for (int i = 0; i < 8; ++i) a[i] = __builtin_amdgcn_raw_buffer_load_b128(r, (lane >> 4) * 8 * 65536 + (lane & 15) * 16, i * 65536, 0);
#pragma unroll
                for (int j = 0; j < 8; ++j)
#pragma unroll
                    for (int e = 0; e < 4; ++e) v[j][e] = vperm(a[2 * e + 1][j >> 1], a[2 * e][j >> 1], (j & 1) ? 0x07060302u : 0x05040100u);
            }
            const bool own = (k & 7u) == 0u, send = k < 8u;
            const u32 call = rfl(callv) + 1u;
            LAS u32* cnt = ctl + 8 + 2u * it;
            if (send) encode_block(v, mk_rsrc(ws + OFF_SI + (size_t)((call & 1u) * NTASK + slot_in(isw, s, q, dd)) * CH, CH), (u32)wid, cnt, tab, wl, lane);
            if (own) {
                const Dest dst = isw ? dest_w(ws, s, q, b, rank) : dest_x(ws, s, q, b, rank);
#pragma unroll
                for (int g = 0; g < 8; ++g) store_group(dst, (u32)lane * 16u, g, (u32x4){v[g][0], v[g][1], v[g][2], v[g][3]});
            }
            if (it == 0) STAMP_AT(14);
            asm volatile("s_waitcnt vmcnt(0)" ::: "memory");
            u32 total;
            if (chunk_done(cnt, lane, total)) {
                if (send && lane < (k ? 1 : 7))
                    __hip_atomic_store((GAS u64*)(p.ws[(rank + (k ? k : (u32)lane + 1u)) & 7u] + OFF_GRAN) + (size_t)((call & 1u) * 8u + rank) * NC + pos_in(isw, s, q),
                                       ((u64)total << 32) | call, __ATOMIC_RELAXED, __HIP_MEMORY_SCOPE_SYSTEM);
                if (own) arrive(ws, ci, call, lane);
            }
            if (it == 0) STAMP_AT(1);
        }
        STAMP_AT(2);
        if (lane == 0) __hip_atomic_store(callp, callv + 1u, __ATOMIC_RELAXED, __HIP_MEMORY_SCOPE_AGENT);
    }

    const u32 call = rfl(callv) + 1u;
    const int wr = wid >> 2, wc = wid & 3;
    constexpr int HTB = 16384;
#define SA(b, h) (((b) * 2 + (h)) * HTB)
#define SB(b, h) ((4 + (b) * 2 + (h)) * HTB)
#define STAGE(bufoff, gbase) do { _Pragma("unroll") for (int _i = 0; _i < 2; ++_i) \
        __builtin_amdgcn_global_load_lds((const GAS u32*)((gbase) + voff + _i * 8192), (LAS u32*)(lds + (bufoff) + ldsw + _i * 8192), 16, 0, GLDS_AUX); } while (0)
#define LDA(dst, b, h) do { _Pragma("unroll") for (int m = 0; m < 4; ++m) _Pragma("unroll") for (int k = 0; k < 2; ++k) dst[m][k] = *(const LAS bf16x8*)(lds + SA(b, h) + aoff + m * 2048 + k * 1024); } while (0)
#define LDB(dst, b, h) do { _Pragma("unroll") for (int n = 0; n < 2; ++n) _Pragma("unroll") for (int k = 0; k < 2; ++k) dst[n][k] = *(const LAS bf16x8*)(lds + SB(b, h) + boff + n * 2048 + k * 1024); } while (0)
#define MMA(ai, bj, At, Bt) do { __builtin_amdgcn_s_setprio(1); _Pragma("unroll") for (int m = 0; m < 4; ++m) _Pragma("unroll") for (int n = 0; n < 2; ++n) _Pragma("unroll") for (int k = 0; k < 2; ++k) \
        acc[ai][bj][m][n] = __builtin_amdgcn_mfma_f32_16x16x32_bf16(Bt[n][k], At[m][k], acc[ai][bj][m][n], 0, 0, 0); __builtin_amdgcn_s_setprio(0); } while (0)
#define WAIT_V(n) asm volatile("s_waitcnt vmcnt(" #n ")" ::: "memory")
#define WAIT_L(n) asm volatile("s_waitcnt lgkmcnt(" #n ")" ::: "memory")
#define BAR __builtin_amdgcn_s_barrier()
#define SCHED __builtin_amdgcn_sched_barrier(0)
    const u32 voff = (u32)tid * 16u, ldsw = (u32)wid * 1024u;
    const int aoff = wr * 8192 + lane * 16, boff = wc * 4096 + lane * 16;
    const u32 idx = bid >> 3, hh = (bid >> 2) & 1u;
    const u32 ti = (bid & 3u) * 4u + (idx < 4u * (WB / 2) ? idx & 3u : 2u * hh + (idx & 1u)), tc = idx < 4u * (WB / 2) ? 2u * (idx >> 2) + hh : WB - 1u;
    u32 nwait = 0, bad = 0;
    for (u32 round = 0; round < (u32)NB; ++round) {
        const u32 tj = round * WB + tc;
        if (tj >= (u32)NP) break;
        const GAS char* cA = (const GAS char*)(ws + OFF_WF) + (size_t)tj * (2u << 20);
        const GAS char* cB = (const GAS char*)(ws + OFF_XF) + (size_t)ti * (2u << 20);
        constexpr size_t kstep = 32768, hstep = 16384;
        f32x4 acc[2][2][4][2];
#pragma unroll
        for (int a = 0; a < 2; ++a)
#pragma unroll
            for (int b = 0; b < 2; ++b)
#pragma unroll
                for (int m = 0; m < 4; ++m)
#pragma unroll
                    for (int n = 0; n < 2; ++n) acc[a][b][m][n] = (f32x4){0.f, 0.f, 0.f, 0.f};
        bf16x8 At[4][2], B0[2][2], B1[2][2];
        for (u32 q0 = 0; q0 < 4u;) {
            if (wid == 0) {
                const u64 t0 = __builtin_amdgcn_s_memrealtime();
                GAS u32* cp = (GAS u32*)(ws + OFF_FLAG + (bid & 7u) * 128u + (round * 4u + ((u32)lane & 3u)) * 4u);
                u32 q1;
                for (;;) {
                    const u32 a = __hip_atomic_load(cp, __ATOMIC_RELAXED, __HIP_MEMORY_SCOPE_AGENT);
                    q1 = (u32)__builtin_ctz(((u32)__builtin_amdgcn_ballot_w64(a != call) & 15u) | 16u);
                    if (q1 > q0) break;
                    if (give_up(tmo, call, t0)) { q1 = 4u | 8u; break; }
                    __builtin_amdgcn_s_sleep(POLL_SLEEP);
                }
                if (GLDS_AUX == 0) __builtin_amdgcn_fence(__ATOMIC_ACQUIRE, "agent");
                if (lane == 0) *(volatile LAS u32*)(ctl + (nwait & 1u)) = q1;
                if (GLDS_AUX == 0) asm volatile("s_waitcnt vmcnt(0)" ::: "memory");
                asm volatile("s_waitcnt lgkmcnt(0)" ::: "memory");
            }
            asm volatile("" ::: "memory"); BAR; asm volatile("" ::: "memory");
            const u32 te = lds_ld(ctl + (nwait & 1u));
            ++nwait; bad |= te >> 3;
            const u32 k0 = q0 * 16u, k1 = (te & 7u) * 16u;
            q0 = te & 7u;
            if (k0 == 0u) STAMP_AT(3 + round * 2);
            const GAS char* pA = cA + (size_t)k0 * kstep; const GAS char* pB = cB + (size_t)k0 * kstep;
            STAGE(SB(0, 0), pB); STAGE(SA(0, 0), pA); STAGE(SB(0, 1), pB + hstep); STAGE(SA(0, 1), pA + hstep);
            if (wr == 1) BAR;
            WAIT_V(4); BAR;
            STAGE(SB(1, 0), pB + kstep); STAGE(SA(1, 0), pA + kstep); STAGE(SB(1, 1), pB + hstep + kstep);
            WAIT_V(6); BAR;
            for (u32 t = k0; t < k1; t += 2u) {
                const u32 t2 = t + 2u < k1 ? t + 2u : t;
                const GAS char* a1 = cA + (size_t)(t + 1u) * kstep;
                const GAS char* a2 = cA + (size_t)t2 * kstep; const GAS char* b2 = cB + (size_t)t2 * kstep;
                const GAS char* a3 = a2 + kstep; const GAS char* b3 = b2 + kstep;
                LDB(B0, 0, 0); SCHED; LDA(At, 0, 0); STAGE(SA(1, 1), a1 + hstep);
                WAIT_L(8); BAR; WAIT_L(0); MMA(0, 0, At, B0); BAR; SCHED;
                LDB(B1, 0, 1); STAGE(SB(0, 0), b2);
                BAR; WAIT_L(0); MMA(0, 1, At, B1); BAR;
                LDA(At, 0, 1); STAGE(SA(0, 0), a2);
                BAR; WAIT_L(0); MMA(1, 0, At, B0); BAR; SCHED;
                STAGE(SB(0, 1), b2 + hstep);
                WAIT_V(6); BAR; MMA(1, 1, At, B1); BAR;
                LDB(B0, 1, 0); SCHED; LDA(At, 1, 0); STAGE(SA(0, 1), a2 + hstep);
                WAIT_L(8); BAR; WAIT_L(0); MMA(0, 0, At, B0); BAR; SCHED;
                LDB(B1, 1, 1); STAGE(SB(1, 0), b3);
                BAR; WAIT_L(0); MMA(0, 1, At, B1); BAR;
                LDA(At, 1, 1); STAGE(SA(1, 0), a3);
                BAR; WAIT_L(0); MMA(1, 0, At, B0); BAR; SCHED;
                STAGE(SB(1, 1), b3 + hstep);
                WAIT_V(6); BAR; MMA(1, 1, At, B1); BAR;
            }
            WAIT_V(0);
            if (wr == 0) BAR;
            BAR;
        }
        STAMP_AT(4 + round * 2);
        const u32 ord = ti * POWN + tj;
#pragma unroll
        for (int bj = 0; bj < 2; ++bj) {
            u32 v[8][4];
#pragma unroll
            for (int g = 0; g < 8; ++g)
#pragma unroll
                for (int q = 0; q < 4; ++q) {
                    typedef float f32x2 __attribute__((ext_vector_type(2)));
                    typedef __bf16 bf16x2 __attribute__((ext_vector_type(2)));
                    const f32x2 f = {acc[q >> 1][bj][(q & 1) * 2][g >> 2][g & 3], acc[q >> 1][bj][(q & 1) * 2 + 1][g >> 2][g & 3]};
                    v[g][q] = bad ? 0x7fc07fc0u : __builtin_bit_cast(u32, __builtin_convertvector(f, bf16x2));
                }
            if (tj < (u32)POWN) {
                LAS u32* cnt = ctl + 32 + bj * 2;
                encode_block(v, mk_rsrc(ws + OFF_SY + (size_t)(ord * 2u + bj) * CH, CH), (u32)wid, cnt, tab, wl, lane);
                asm volatile("s_waitcnt vmcnt(0)" ::: "memory");
                u32 total;
                if (chunk_done(cnt, lane, total) && lane < 7)
                    __hip_atomic_store((GAS u64*)(p.ws[(rank + 1u + (u32)lane) & 7u] + OFF_GRAN) + (size_t)((call & 1u) * 8u + rank) * NC + NCIN + ord * 2u + bj,
                                       ((u64)total << 32) | call, __ATOMIC_RELAXED, __HIP_MEMORY_SCOPE_SYSTEM);
            }
            const Dest dst = dest_y(p.out, ti, gpanel(rank, tj), bj, (u32)wid);
            const u32 loff = (u32)(lane >> 4) * dst.LH + (u32)(lane & 15) * 16u;
#pragma unroll
            for (int g = 0; g < 8; ++g) store_group<AUX_T>(dst, loff, g, (u32x4){v[g][0], v[g][1], v[g][2], v[g][3]});
        }
    }
    STAMP_AT(15);
}

extern "C" void kernel_launch(int rank, int nranks, ncclComm_t comm, hipStream_t stream,
                              void* const* d_in, const int* in_sizes, int n_in, void* d_out, int out_size,
                              const int* in_dtypes, int out_dtype, void* const* d_out_all,
                              void* const* d_ws_all, size_t ws_bytes) {
    if (nranks != NR || (unsigned)rank >= (unsigned)NR || ws_bytes < WS_NEED || n_in != 2) { fprintf(stderr, "unsupported configuration\n"); return; }
    Params p;
    p.x = (const char*)d_in[0]; p.w = (const char*)d_in[1]; p.out = (char*)d_out; p.rank = rank; p.pad = 0;
    for (int i = 0; i < NR; ++i) p.ws[i] = (char*)d_ws_all[i];
    hipLaunchKernelGGL(gemm_ar_fused, dim3(256), dim3(512), 0, stream, p);
    HIP_CHECK(hipGetLastError());
}
```

```cpp
#include <hip/hip_runtime.h>
#include <rccl/rccl.h>
#include <cstdio>

#define HIP_CHECK(expr) do { hipError_t _e = (expr); if (_e != hipSuccess) { \
    fprintf(stderr, "HIP error %s at %s:%d\n", hipGetErrorString(_e), __FILE__, __LINE__); \
    return; } } while (0)

typedef unsigned u32;
typedef unsigned long long u64;
typedef u32 u32x4 __attribute__((ext_vector_type(4)));
typedef u32 u32x2 __attribute__((ext_vector_type(2)));
typedef unsigned short u16x2 __attribute__((ext_vector_type(2)));
typedef short bf16x8 __attribute__((ext_vector_type(8)));
typedef float f32x4 __attribute__((ext_vector_type(4)));
#define LAS __attribute__((address_space(3)))
#define GAS __attribute__((address_space(1)))
typedef __amdgpu_buffer_rsrc_t rsrc_t;

constexpr int NR = 8;
#define PULL_WG_PER_PEER 8
#define WINDOW 8
#define TAILC 0
#define PUSHC 4
#define PUSHW 2
#define LOOK 6
#define RHO 5
#define ENC_FIRST (16 - PUSHC)
#define ENC_DELAY 150
#define ENC_PACE 256
#define POLL_SLEEP 8
#define WB 12
constexpr int NPULL = 7 * PULL_WG_PER_PEER, NGEMM = 16 * WB;
constexpr int POWN = 16 - RHO, NP = POWN + 8 * RHO;
constexpr int NB = (NP + WB - 1) / WB, N0 = 16 + WB;
constexpr int NCIN = 64 + 4 * NP, NCY = 32 * POWN, NC = NCIN + NCY;
constexpr int NTASK = 16 * 36;
constexpr int NLOAD = 1, NDEC = 6;
constexpr u64 DEADLINE = 100000000ull;
constexpr int PUSHB = (8 * PUSHC + PULL_WG_PER_PEER - 1) / PULL_WG_PER_PEER;
static_assert(NGEMM + NPULL <= 256 && NB <= 8 && NP <= 80 && PUSHB <= NDEC, "roles");

constexpr int FIXB = 5120, CVAR = 8 * FIXB, CHDR = 32, CH = 66560;
constexpr size_t OFF_CALL = 0;
constexpr size_t OFF_TMO = 131072;
constexpr size_t OFF_TICK = OFF_TMO + 256;
constexpr size_t OFF_RDY = OFF_TICK + 16 * 128;
constexpr size_t OFF_FLAG = OFF_RDY + 32 * 128;
constexpr size_t OFF_GRAN = 196608;
constexpr size_t OFF_DBG = 524288;
constexpr size_t OFF_XF = 1ull << 20;
constexpr size_t OFF_WF = OFF_XF + (64ull << 20);
constexpr size_t OFF_SY = OFF_WF + NP * (2ull << 20);
constexpr size_t OFF_SI = OFF_SY + (size_t)NCY * CH;
constexpr size_t WS_NEED = OFF_SI + 2ull * NTASK * CH;

constexpr int A2B = 2064, A3B = 528, VARB = 912, WLDS = A2B + A3B + VARB, N2MAX = 2032, N3MAX = 496, N4MAX = 256;
constexpr int CTL_OFF = 2048, CTL_WORDS = 128, WL_OFF = 2560, BIG_OFF = 30720, LDS_BYTES = BIG_OFF + 131072;
static_assert(WL_OFF + 8 * WLDS <= BIG_OFF && LDS_BYTES <= 163840, "LDS map");
constexpr int NF = 16, NV = 4, VSLOT = 8192, VRING = NF * FIXB;
constexpr int AUX_SYS = 17, AUX_DEV = 16, AUX_OUT = 18, AUX_IN = 2;

struct Params {
    const char* x; const char* w; char* out; char* ws[NR];
    int rank; int pad;
    u32 pmap[20];
    u32 task[NTASK / 2];
};

__device__ __forceinline__ u32 rfl(u32 x) { return (u32)__builtin_amdgcn_readfirstlane((int)x); }
__device__ __forceinline__ rsrc_t mk_rsrc(const void* p, u32 bytes) {
    const u64 a = (u64)p; const u64 u = ((u64)rfl((u32)(a >> 32)) << 32) | rfl((u32)a);
    return __builtin_amdgcn_make_buffer_rsrc((void*)u, (short)0, (int)rfl(bytes), 0x00020000);
}
__device__ __forceinline__ u32 vperm(u32 a, u32 b, u32 s) { return __builtin_amdgcn_perm(a, b, s); }
__device__ __forceinline__ u32 wave_incl_scan(u32 x) {
    x += (u32)__builtin_amdgcn_update_dpp(0, (int)x, 0x111, 0xf, 0xf, false);
    x += (u32)__builtin_amdgcn_update_dpp(0, (int)x, 0x112, 0xf, 0xf, false);
    x += (u32)__builtin_amdgcn_update_dpp(0, (int)x, 0x114, 0xf, 0xf, false);
    x += (u32)__builtin_amdgcn_update_dpp(0, (int)x, 0x118, 0xf, 0xf, false);
    x += (u32)__builtin_amdgcn_update_dpp(0, (int)x, 0x142, 0xa, 0xf, false);
    x += (u32)__builtin_amdgcn_update_dpp(0, (int)x, 0x143, 0xc, 0xf, false);
    return x;
}
__device__ __forceinline__ u32 pk_sub(u32 a, u32 b) { return __builtin_bit_cast(u32, (u16x2)(__builtin_bit_cast(u16x2, a) - __builtin_bit_cast(u16x2, b))); }
template <int AUX>
__device__ __forceinline__ void st128(u32x4 x, rsrc_t r, int voff, int soff) {
    __builtin_amdgcn_raw_buffer_store_b128(x, r, voff, soff, AUX);
    asm volatile("s_nop 1" :: "v"(x) : "memory");
}
#define WAVE_FENCE() __builtin_amdgcn_fence(__ATOMIC_ACQ_REL, "wavefront")

__device__ __forceinline__ void build_table(LAS u32x2* tab, u32 i, bool unpack) {
    u32 n = 0, dep = 0, cmp = 0x0c0c0c0cu;
#pragma unroll
    for (u32 t = 0; t < 4; ++t) {
        const u32 c = (i >> (2 * t)) & 3u;
        dep |= (c ? 4u + c : n) << (8 * t);
        if (!c) { cmp = (cmp & ~(0xffu << (8 * n))) | (t << (8 * n)); ++n; }
    }
    tab[i] = (u32x2){unpack ? dep : cmp, n};
}
__device__ __forceinline__ u32 code_byte(u32 hi, u32 lo, u32 x) {
    const u32 f = vperm(hi, lo, x);
    return __builtin_amdgcn_udot4((f ^ (f >> 2)) & 0x03030303u, 0x40100401u, 0u, false);
}
template <bool PRED>
__device__ __forceinline__ void or_bytes(u32 pos, u32 x) {
    const u64 y = (u64)x << ((pos & 3u) * 8u);
    LAS u32* q = (LAS u32*)(size_t)(pos & ~3u);
    if (!PRED || x) {
        __hip_atomic_fetch_or(q, (u32)y, __ATOMIC_RELAXED, __HIP_MEMORY_SCOPE_WORKGROUP);
        __hip_atomic_fetch_or(q + 1, (u32)(y >> 32), __ATOMIC_RELAXED, __HIP_MEMORY_SCOPE_WORKGROUP);
    }
}
__device__ __forceinline__ u32 get_bytes(u32 pos) {
    const LAS u32* q = (const LAS u32*)(size_t)(pos & ~3u);
    return __builtin_amdgcn_alignbyte(q[1], q[0], pos);
}
__device__ __forceinline__ u32 la(const LAS void* p) { return (u32)(size_t)p; }
template <bool LAST>
__device__ __forceinline__ u32 pack_level(const LAS u32x2* tab, const LAS char* src, u32 n, LAS u32* codes, LAS char* dst, u32 lim, u32 hi, u32 lo, u32 t4, int lane) {
    u32 m = 0;
    for (u32 i0 = 0; i0 < n; i0 += 1024u) {
        const u32 i = i0 + (u32)lane * 16u;
        const u32x4 x = *(const LAS u32x4*)(src + (i < n ? i : 0u));
        u32 cw = code_byte(hi, lo, x[0]) | (code_byte(hi, lo, x[1]) << 8) | (code_byte(hi, lo, x[2]) << 16) | (code_byte(hi, lo, x[3]) << 24);
        const u32 valid = i < n ? n - i : 0u;
        if (valid < 16u) cw |= 0x55555555u << (2u * valid);
        u32x2 e[4]; u32 c[5]; c[0] = 0;
#pragma unroll
        for (int j = 0; j < 4; ++j) { e[j] = tab[(cw >> (8 * j)) & 0xffu]; c[j + 1] = c[j] + e[j][1]; }
        const u32 incl = wave_incl_scan(c[4]), p0 = la(dst) + m + incl - c[4];
        if (i < n) codes[i >> 4] = cw;
        m += (u32)__builtin_amdgcn_readlane((int)incl, 63);
        if (m <= lim) {
#pragma unroll
            for (int j = 0; j < 4; ++j) {
                u32 y = x[j] ^ 0x08080808u;
                if (LAST) y = ((t4 | 0x80808080u) - (y & 0x7f7f7f7fu)) ^ ((t4 ^ ~y) & 0x80808080u);
                or_bytes<true>(p0 + c[j], vperm(0u, y, e[j][0]));
            }
        }
    }
    WAVE_FENCE();
    return m;
}
__device__ __forceinline__ void encode_block(const u32 (&v)[8][4], rsrc_t ch, u32 b, LAS u32* alloc, const LAS u32x2* tab, LAS char* wl, int lane) {
#pragma unroll
    for (int j = 0; j < 4; ++j) if (j < 3 || lane < (WLDS - 3072) / 16) *(LAS u32x4*)(wl + j * 1024 + lane * 16) = (u32x4){0, 0, 0, 0};
    u32 s = 0;
#pragma unroll
    for (int g = 0; g < 8; ++g) s = __builtin_amdgcn_udot4(__builtin_amdgcn_alignbit(v[g][0], v[g][0], 31), 0x01000100u, s, false);
    u32 T = ((u32)__builtin_amdgcn_readlane((int)wave_incl_scan(s), 63) >> 10) + 5u;
    T = T > 255u ? 255u : T;
    const u32 tt = (T * 256u + 255u) * 0x10001u;
    const int fo = (int)(b * FIXB);
    u32 U[16], l1[4] = {0, 0, 0, 0}, c[17];
    u32x2 e[16];
#pragma unroll
    for (int q = 0; q < 4; ++q) {
        u32x4 sm;
#pragma unroll
        for (int j = 0; j < 4; ++j) {
            const int k = 4 * q + j;
            const u32 a0 = v[k >> 1][(k & 1) * 2], a1 = v[k >> 1][(k & 1) * 2 + 1];
            const u32 r0 = __builtin_amdgcn_alignbit(a0, a0, 31), r1 = __builtin_amdgcn_alignbit(a1, a1, 31);
            sm[j] = vperm(r1, r0, 0x06040200u);
            U[k] = vperm(pk_sub(tt, r1), pk_sub(tt, r0), 0x07050301u);
            const u32 cb = code_byte(0x00000302u, 0x01000000u, U[k]);
            l1[q] |= cb << (8 * j);
            e[k] = tab[cb];
        }
        st128<AUX_SYS>(sm, ch, lane * 16 + q * 1024, fo);
    }
    st128<AUX_SYS>((u32x4){l1[0], l1[1], l1[2], l1[3]}, ch, lane * 16 + 4096, fo);
    __builtin_amdgcn_sched_barrier(0);
    c[0] = 0;
#pragma unroll
    for (int k = 0; k < 16; ++k) c[k + 1] = c[k] + e[k][1];
    const u32 incl = wave_incl_scan(c[16]), p0 = la(wl) + incl - c[16];
    const u32 n2 = (u32)__builtin_amdgcn_readlane((int)incl, 63);
    bool raw = n2 > (u32)N2MAX;
    u32 d2 = 0, d3 = 0, vb = 0;
    if (!raw) {
        WAVE_FENCE();
#pragma unroll
        for (int k = 0; k < 16; ++k) or_bytes<false>(p0 + c[k], vperm(0u, U[k], e[k][0]));
        WAVE_FENCE();
        LAS char* var = wl + A2B + A3B;
        d2 = (n2 + 15u) >> 4;
        const u32 n3 = pack_level<false>(tab, wl, n2, (LAS u32*)var, wl + A2B, N3MAX, 0x02010000u, 0x00030000u, 0u, lane);
        raw = n3 > (u32)N3MAX;
        if (!raw) {
            d3 = (n3 + 15u) >> 4;
            const u32 n4 = pack_level<true>(tab, wl + A2B, n3, (LAS u32*)var + d2, var + (d2 + d3) * 4u, N4MAX, 0u, 0x00030201u, T * 0x01010101u, lane);
            raw = n4 > (u32)N4MAX;
            vb = ((d2 + d3) * 4u + n4 + 7u) & ~7u;
        }
    }
    u32 off = 0;
    if (lane == 0) off = __hip_atomic_fetch_add(alloc, raw ? 3072u : vb, __ATOMIC_RELAXED, __HIP_MEMORY_SCOPE_WORKGROUP);
    off = rfl(off) + CHDR;
    const int vo = CVAR + (int)off;
    if (lane == 0) __builtin_amdgcn_raw_buffer_store_b32((raw ? 0u : (T | (d2 << 8) | (d3 << 15))) | (off << 17), ch, 0, CVAR + (int)b * 4, AUX_SYS);
    if (raw) {
        asm volatile("s_waitcnt vmcnt(0)" ::: "memory");
#pragma unroll
        for (int g = 0; g < 8; ++g)
            st128<AUX_SYS>((u32x4){v[g][0], v[g][1], v[g][2], v[g][3]}, ch, lane * 16 + (g < 5 ? g : g - 5) * 1024, g < 5 ? fo : vo);
    } else {
        for (u32 o = (u32)lane * 8u; o < vb; o += 512u)
            __builtin_amdgcn_raw_buffer_store_b64(*(const LAS u32x2*)(wl + A2B + A3B + o), ch, (int)o, vo, AUX_SYS);
    }
    WAVE_FENCE();
}
__device__ __forceinline__ bool chunk_done(LAS u32* cnt, int lane, u32& total) {
    u32 n = 0;
    if (lane == 0) n = __hip_atomic_fetch_add(cnt + 1, 1u, __ATOMIC_RELAXED, __HIP_MEMORY_SCOPE_WORKGROUP);
    if (rfl(n) != 7u) return false;
    total = rfl(*(volatile LAS u32*)cnt) + CHDR;
    return true;
}

struct Dest { rsrc_t r; u32 A, B, C, LH; };
__device__ __forceinline__ Dest dest_of(char* base, u32 kind, u32 b) {
    if (kind == 0u) return Dest{mk_rsrc(base + b * 4096u, 32768u + 4096u), 1024u, 32768u, 2048u, 256u};
    if (kind == 1u) return Dest{mk_rsrc(base + (b >> 2) * 32768u + (b & 1u) * 8192u + (b & 2u) * 512u, 16384u + 6144u + 1024u), 2048u, 4096u, 16384u, 256u};
    return Dest{mk_rsrc(base + (b & 3u) * (32u * 65536u) + (b >> 2) * 256u, 31u * 65536u + 256u), 65536u, 131072u, 16u * 65536u, 4u * 65536u};
}
template <int AUX>
__device__ __forceinline__ void store_group(const Dest& d, u32 loff, int g, u32x4 x) {
    st128<AUX>(x, d.r, (int)loff, (int)((g & 1 ? d.A : 0u) + (g & 2 ? d.B : 0u) + (g & 4 ? d.C : 0u)));
}

__device__ __forceinline__ void unpack_rest(const LAS u32x2* tab, const LAS u32* codes, u32 i0, u32 nd, const LAS char* src, u32 m, u32 mask, u32 t, LAS char* dst, int lane) {
    for (; i0 < nd; i0 += 64u) {
        const u32 i = i0 + (u32)lane;
        const u32 cw = i < nd ? codes[i] : 0x55555555u;
        u32x2 e[4]; u32 c[5]; c[0] = 0;
#pragma unroll
        for (int j = 0; j < 4; ++j) { e[j] = tab[(cw >> (8 * j)) & 0xffu]; c[j + 1] = c[j] + e[j][1]; }
        const u32 incl = wave_incl_scan(c[4]), p0 = m + incl - c[4];
        u32x4 o;
#pragma unroll
        for (int j = 0; j < 4; ++j) o[j] = vperm(t, get_bytes(la(src) + ((p0 + c[j]) & mask)), e[j][0]);
        if (i < nd) *(LAS u32x4*)(dst + i * 16u) = o;
        m += (u32)__builtin_amdgcn_readlane((int)incl, 63);
    }
}
#define SCHED __builtin_amdgcn_sched_barrier(0)
__device__ __forceinline__ u32x2 get2(u32 pos) { const LAS u32* q = (const LAS u32*)(size_t)(pos & ~3u); return (u32x2){q[0], q[1]}; }
__device__ __forceinline__ void decode_block(const u32x4 (&d)[5], u32 hdr, const LAS char* var, const LAS u32x2* tab, LAS char* wl, int lane, u32x4 (&o)[8]) {
    const u32 T = hdr & 0xffu, d2 = (hdr >> 8) & 0x7fu, d3 = (hdr >> 15) & 0x1fu;
    auto ex = [&](u32 a, u32 b, u32 c) { return (((T - a) & 0xffu) << 8) | (((T - b) & 0xffu) << 16) | (((T - c) & 0xffu) << 24); };
    const u32 t1 = ex(3, 4, 5), t2 = ex(6, 7, 2), t3 = ex(8, 9, 10);
    const LAS u32* c2 = (const LAS u32*)var;
    const LAS char* r4 = var + (d2 + d3) * 4u;
    u32 w2 = c2[lane], w3 = (u32)*(const LAS unsigned char*)(var + d2 * 4u + lane);
    u32x2 e[16], e2[4], g[16];
#pragma unroll
    for (int k = 0; k < 8; ++k) e[k] = tab[(d[4][k >> 2] >> (8 * (k & 3))) & 0xffu];
    SCHED;
    w2 = (u32)lane < d2 ? w2 : 0x55555555u;
    w3 = (u32)lane < d3 * 4u ? w3 : 0x55u;
    const u32x2 e3 = tab[w3];
#pragma unroll
    for (int j = 0; j < 4; ++j) e2[j] = tab[(w2 >> (8 * j)) & 0xffu];
#pragma unroll
    for (int k = 8; k < 16; ++k) e[k] = tab[(d[4][k >> 2] >> (8 * (k & 3))) & 0xffu];
    SCHED;
    u32 c[17], f[5];
    c[0] = 0; f[0] = 0;
#pragma unroll
    for (int k = 0; k < 16; ++k) c[k + 1] = c[k] + e[k][1];
#pragma unroll
    for (int j = 0; j < 4; ++j) f[j + 1] = f[j] + e2[j][1];
    const u32 pk = c[16] | (f[4] << 11) | (e3[1] << 22);
    const u32 incl = wave_incl_scan(pk), ex0 = incl - pk;
    const u32 p1 = la(wl) + (ex0 & 0x7ffu), p2 = la(wl) + A2B + ((ex0 >> 11) & 0x1ffu), p3 = la(r4) + (ex0 >> 22);
    g[0] = get2(p3);
    SCHED;
#pragma unroll
    for (int j = 0; j < 4; ++j) f[j] += p2;
#pragma unroll
    for (int k = 0; k < 8; ++k) c[k] += p1;
    SCHED;
    *(LAS u32*)(wl + A2B + lane * 4) = vperm(t3, __builtin_amdgcn_alignbyte(g[0][1], g[0][0], p3), e3[0]);
    if (d3 > 16u || d2 > 64u) {
        const u32 tot = (u32)__builtin_amdgcn_readlane((int)incl, 63);
        unpack_rest(tab, c2 + d2, 16u, d3, r4, tot >> 22, 0x1ffu, t3, wl + A2B, lane);
        WAVE_FENCE();
        unpack_rest(tab, c2, 64u, d2, wl + A2B, (tot >> 11) & 0x7ffu, 0x1ffu, t2, wl, lane);
    }
    WAVE_FENCE();
#pragma unroll
    for (int j = 0; j < 4; ++j) g[j] = get2(f[j]);
    SCHED;
#pragma unroll
    for (int k = 8; k < 16; ++k) c[k] += p1;
    SCHED;
    u32x4 a2;
#pragma unroll
    for (int j = 0; j < 4; ++j) a2[j] = vperm(t2, __builtin_amdgcn_alignbyte(g[j][1], g[j][0], f[j]), e2[j][0]);
    *(LAS u32x4*)(wl + lane * 16) = a2;
    WAVE_FENCE();
#pragma unroll
    for (int k = 0; k < 16; ++k) g[k] = get2(c[k]);
    SCHED;
#pragma unroll
    for (int k = 0; k < 16; ++k) {
        const u32 x = vperm(t1, __builtin_amdgcn_alignbyte(g[k][1], g[k][0], c[k]), e[k][0]);
        const u32 smk = d[k >> 2][k & 3];
        const u32 r0 = vperm(x, smk, 0x05010400u), r1 = vperm(x, smk, 0x07030602u);
        o[k >> 1][2 * (k & 1)] = __builtin_amdgcn_alignbit(r0, r0, 1);
        o[k >> 1][2 * (k & 1) + 1] = __builtin_amdgcn_alignbit(r1, r1, 1);
    }
    WAVE_FENCE();
}

__device__ __forceinline__ char* frag_base(char* ws, bool isw, u32 s, u32 q, u32 src, u32 call) {
    return ws + (isw ? OFF_WF : OFF_XF + (size_t)(call & 1u) * (32u << 20)) + (size_t)(s * 64u + q * 16u + src * 2u) * 32768u;
}
__device__ __forceinline__ char* y_base(char* out, u32 i, u32 gp, u32 bj) { return out + (size_t)(i * 256u + bj * 128u) * 65536u + gp * 512u; }
__device__ __forceinline__ u32 pmap_of(const Params& p, u32 j) { return (p.pmap[j >> 2] >> ((j & 3u) * 8u)) & 0xffu; }
__device__ __forceinline__ u32 gpanel(u32 r, u32 pm) { return pm < 0x80u ? r * 16u + pm : (pm & 7u) * 16u + POWN + ((pm & 0x7fu) >> 3); }
__device__ __forceinline__ u32 slot_in(bool isw, u32 pm, u32 q, u32 dst) {
    return !isw ? pm * 4u + q : (pm < 0x80u ? 64u + (dst * POWN + pm) * 4u + q : 64u + 32u * POWN + (pm & 0x7fu) * 4u + q);
}
__host__ __device__ __forceinline__ u32 batch_size(u32 bt) { return bt == 0u ? (u32)N0 : (bt == NB - 1u ? (u32)(NP - (NB - 1) * WB) : (u32)WB); }
struct InPos { bool isw; u32 s, q, cnt; };
__host__ __device__ __forceinline__ InPos in_of(u32 c) {
    if (c < 4u * N0) { const u32 q = c / N0, pp = c - q * N0; return InPos{pp >= 16u, pp >= 16u ? pp - 16u : pp, q, q}; }
    const u32 bt = (c - 64u) / (4u * WB), r = c - 64u - bt * 4u * WB, q = r / batch_size(bt);
    return InPos{true, bt * WB + r - q * batch_size(bt), q, bt * 4u + q};
}

__device__ __forceinline__ void arrive(char* ws, u32 ci, u32 n, u32 call, int lane) {
    u32 old = 0;
    if (lane == 0) old = __hip_atomic_fetch_add((GAS u32*)(ws + OFF_RDY + ci * 128u), n, __ATOMIC_RELAXED, __HIP_MEMORY_SCOPE_SYSTEM);
    if (rfl(old) + n == call * 64u * batch_size(ci >> 2) && lane < 8)
        __hip_atomic_store((GAS u32*)(ws + OFF_FLAG + (u32)lane * 128u + ci * 4u), call, __ATOMIC_RELAXED, __HIP_MEMORY_SCOPE_SYSTEM);
}
__device__ __forceinline__ void load_x(const char* x, u32 s, u32 q, u32 b, int lane, u32x4 (&a)[8]) {
    const rsrc_t r = mk_rsrc(x + (size_t)(s * 256u + b * 32u) * 1024u + q * 256u, 31u * 1024u + 256u);
#pragma unroll
    for (int i = 0; i < 8; ++i) a[i] = __builtin_amdgcn_raw_buffer_load_b128(r, (lane & 15) * 1024 + (lane >> 4) * 16 + (i & 3) * 64, (i >> 2) * 16384, 0);
}

__device__ __forceinline__ bool give_up(GAS u32* tmo, u32 call, u64 t0) {
    if (rfl(__hip_atomic_load(tmo, __ATOMIC_RELAXED, __HIP_MEMORY_SCOPE_AGENT)) == call || __builtin_amdgcn_s_memrealtime() - t0 > DEADLINE) {
        __hip_atomic_store(tmo, call, __ATOMIC_RELAXED, __HIP_MEMORY_SCOPE_AGENT);
        return true;
    }
    return false;
}

#define STAMP_IT 0
#define STAMP_AT(i) do { } while (0)

__device__ __forceinline__ u32 lds_ld(LAS u32* p) { return rfl(*(volatile LAS u32*)p); }
__device__ __forceinline__ void dma_piece(rsrc_t r, u32 voff, u32 soff, u32 ldsaddr) {
    u32 keep;
    asm volatile("s_nop 4\n\ts_mov_b32 %0, m0\n\ts_mov_b32 m0, %1\n\ts_nop 0\n\tbuffer_load_dwordx4 %2, %3, %4 offen sc0 sc1 lds\n\ts_mov_b32 m0, %0"
                 : "=&s"(keep) : "s"(ldsaddr), "v"(voff), "s"(r), "s"(soff) : "memory");
}

__global__ void __launch_bounds__(512, 2) gemm_ar_fused(const Params p) {
    __shared__ __attribute__((aligned(16))) char smem[LDS_BYTES];
    LAS char* lds = (LAS char*)smem + BIG_OFF;
    LAS u32x2* tab = (LAS u32x2*)smem;
    const int tid = threadIdx.x, wid = (int)rfl((u32)tid >> 6), lane = tid & 63;
    const u32 rank = (u32)p.rank, bid = blockIdx.x;
    char* ws = p.ws[rank & 7u];
    LAS char* wl = (LAS char*)smem + WL_OFF + wid * WLDS;
    LAS u32* ctl = (LAS u32*)((LAS char*)smem + CTL_OFF);
    GAS u32* tmo = (GAS u32*)(ws + OFF_TMO);
    STAMP_AT(0);
    if (bid >= (u32)(NGEMM + NPULL)) return;
    GAS u32* callp = (GAS u32*)(ws + OFF_CALL + (bid * 8u + wid) * 64u);
    const u32 callv = __hip_atomic_load(callp, __ATOMIC_RELAXED, __HIP_MEMORY_SCOPE_AGENT);
    auto setup = [&](bool unpack) {
        if (tid < 256) build_table(tab, (u32)tid, unpack); else if (tid < 256 + CTL_WORDS) ctl[tid - 256] = 0u;
        asm volatile("s_waitcnt lgkmcnt(0)" ::: "memory"); __builtin_amdgcn_s_barrier(); asm volatile("" ::: "memory");
    };

    if (bid >= (u32)NGEMM) {
        const u32 pj = bid - NGEMM, src = (rank + 1u + pj % 7u) & 7u;
        const u32 pn = (pj / 7u) * PUSHB + (u32)wid - 2u;
        const bool push = wid >= 2 && wid < 2 + PUSHB && pn < 8u * PUSHC;
        u32x4 pa[8];
        if (push) load_x(p.x, pn >> 3, 0u, pn & 7u, lane, pa);
        setup(true);
        if (push) {
            const u32 call = rfl(callv) + 1u;
            const Dest dst = dest_of(frag_base(p.ws[src], false, pn >> 3, 0u, rank, call), 0u, pn & 7u);
#pragma unroll
            for (int g = 0; g < 8; ++g) asm volatile("" : "+v"(pa[g]));
#pragma unroll
            for (int g = 0; g < 8; ++g) { store_group<AUX_SYS>(dst, (u32)lane * 16u, g, pa[g]); asm volatile("s_waitcnt vmcnt(%0)" :: "n"(PUSHW - 1) : "memory"); }
            STAMP_AT(12);
            if (lane == 0) __hip_atomic_store(callp, call, __ATOMIC_RELAXED, __HIP_MEMORY_SCOPE_AGENT);
            asm volatile("s_waitcnt vmcnt(0)" ::: "memory");
            STAMP_AT(13);
            arrive(p.ws[src], 0u, 1u, call, lane);
        }
        LAS u32* vready = ctl; LAS u32* landed = ctl + 1; LAS u32* endk = ctl + 2; LAS u32* qavail = ctl + 4; LAS u32* prog = ctl + 5;
        LAS u32* vdone = ctl + 8; LAS u32* sdone = ctl + 16; LAS u32* ffree = ctl + 32;
        LAS u32* q = ctl + 64;
        const u32 l0 = la(lds), voff = (u32)lane * 16u;
        if (wid == 0) {
            const char* pws = p.ws[src];
            const u32 call = rfl(callv) + 1u;
            if (lane == 0) __hip_atomic_store(callp, call, __ATOMIC_RELAXED, __HIP_MEMORY_SCOPE_AGENT);
            GAS u64* gran = (GAS u64*)(ws + OFF_GRAN) + (size_t)((call & 1u) * 8u + src) * NC;
            for (u32 k = 0;; ++k) {
                u32 c = PUSHC + pj / 7u;
                if (k) {
                    while (lds_ld(prog) + LOOK < 8u * k) __builtin_amdgcn_s_sleep(2);
                    u32 tk = 0;
                    if (lane == 0) tk = __hip_atomic_fetch_add((GAS u32*)(ws + OFF_TICK + src * 128u), 1u, __ATOMIC_RELAXED, __HIP_MEMORY_SCOPE_AGENT);
                    c = rfl(tk) - (call - 1u) * (NC - PUSHC) + PULL_WG_PER_PEER + PUSHC;
                }
                u32 fl = 2u, vb = 0, kc = 2u;
                const char* from = pws; char* to = ws;
                if (c < (u32)NC) {
                    if (c >= (u32)NCIN) {
                        const u32 yc = c - NCIN;
                        from = pws + OFF_SY + (size_t)yc * CH;
                        to = y_base(p.out, (yc >> 1) & 15u, src * 16u + (yc >> 5), yc & 1u);
                    } else {
                        const InPos ip = in_of(c);
                        from = pws + OFF_SI + (size_t)((call & 1u) * NTASK + slot_in(ip.isw, ip.isw ? pmap_of(p, ip.s) : ip.s, ip.q, rank)) * CH;
                        to = frag_base(ws, ip.isw, ip.s, ip.q, src, call);
                        kc = (ip.isw ? 1u : 0u) | (ip.cnt << 2);
                    }
                    const u64 t0 = __builtin_amdgcn_s_memrealtime();
                    for (u32 spin = 0;; ++spin) {
                        const u64 g = __hip_atomic_load(gran + c, __ATOMIC_RELAXED, __HIP_MEMORY_SCOPE_SYSTEM);
                        if (rfl((u32)g) == call) { fl = c >= (u32)(NC - TAILC) ? 4u : 0u; vb = rfl((u32)(g >> 32)); break; }
                        if ((spin & 15u) == 15u && give_up(tmo, call, t0)) { fl = 1u; break; }
                        __builtin_amdgcn_s_sleep(1);
                    }
                    if (k == 0) STAMP_AT(1);
                }
                while (lds_ld(vdone + (k & (NV - 1))) < 8u * (k / NV)) __builtin_amdgcn_s_sleep(2);
                if (lane == 0) {
                    *(volatile LAS u32x4*)(q + (k & (NV - 1)) * 8u) = (u32x4){(u32)(u64)from, (u32)((u64)from >> 32), fl, vb};
                    *(volatile LAS u32x4*)(q + (k & (NV - 1)) * 8u + 4u) = (u32x4){(u32)(u64)to, (u32)((u64)to >> 32), kc, call};
                    *(volatile LAS u32*)qavail = k + 1u;
                }
                if (fl & 2u) { if (lane == 0) *(volatile LAS u32*)endk = k + 1u; break; }
                if (!(fl & 1u)) {
                    const u32 vl = vb > (u32)VSLOT ? (u32)CHDR : (vb + 15u) & ~15u;
                    const rsrc_t r = mk_rsrc(from, CVAR + vl);
                    for (u32 o = 0; o < vl; o += 1024u) dma_piece(r, voff + CVAR + o, 0u, l0 + VRING + (k & (NV - 1)) * VSLOT + o);
                    asm volatile("s_waitcnt vmcnt(0)" ::: "memory");
                }
                if (lane == 0) *(volatile LAS u32*)vready = k + 1u;
            }
        } else if (wid == 1) {
            __builtin_amdgcn_s_setprio(3);
            u32 issued = 0, pub = 0;
            auto drain = [&]() { asm volatile("s_waitcnt vmcnt(0)" ::: "memory"); pub = issued; if (lane == 0) *(volatile LAS u32*)landed = pub; };
            for (u32 k = 0;; ++k) {
                if (lds_ld(qavail) <= k) { drain(); while (lds_ld(qavail) <= k) __builtin_amdgcn_s_sleep(1); }
                const u32x4 e = *(volatile LAS u32x4*)(q + (k & (NV - 1)) * 8u);
                const u32 fl = rfl(e[2]);
                if (fl & 2u) break;
                const rsrc_t r = mk_rsrc((const void*)(((u64)e[1] << 32) | e[0]), (fl & 1u) ? 0u : (u32)CVAR);
                const bool wide = TAILC && (fl & 4u);
#pragma unroll 1
                for (u32 b = 0; b < 8u; ++b) {
                    const u32 n = 8u * k + b, sl = n & (NF - 1);
                    if (lane == 0) *(volatile LAS u32*)prog = n + 1u;
                    if (lds_ld(ffree + sl) + NF <= n) { drain(); while (lds_ld(ffree + sl) + NF <= n) __builtin_amdgcn_s_sleep(1); }
#pragma unroll
                    for (u32 j = 0; j < 5u; ++j) {
                        dma_piece(r, voff, b * FIXB + j * 1024u, l0 + sl * FIXB + j * 1024u);
                        ++issued;
                        u32 done;
                        if (wide) { asm volatile("s_waitcnt vmcnt(%0)" :: "n"(2 * WINDOW - 1) : "memory"); done = issued - (2 * WINDOW - 1); }
                        else { asm volatile("s_waitcnt vmcnt(%0)" :: "n"(WINDOW - 1) : "memory"); done = issued - (WINDOW - 1); }
                        if ((wide || (j + 7u - WINDOW % 5u) % 5u == 0u) && (int)(done - pub) > 0) { pub = done; if (lane == 0) *(volatile LAS u32*)landed = pub; }
                    }
                }
            }
            drain();
            STAMP_AT(2);
        } else {
#define SEG(i)
            u32 owed = 0xffffffffu, call = 0;
            auto settle = [&]() {
                if (owed != 0xffffffffu) {
                    asm volatile("s_waitcnt vmcnt(0)" ::: "memory");
                    u32 old = 0;
                    if (lane == 0) old = __hip_atomic_fetch_add(sdone + (owed & 7u), 1u, __ATOMIC_RELAXED, __HIP_MEMORY_SCOPE_WORKGROUP);
                    if ((rfl(old) & 7u) == 7u) arrive(ws, owed >> 3, 8u, call, lane);
                    owed = 0xffffffffu;
                }
            };
            for (u32 n = (u32)wid - 2u;; n += NDEC) {
                const u32 k = n >> 3, b = n & 7u, sl = n & (NF - 1);
                bool end = false;
                for (;;) {
                    const u32x4 st = *(volatile LAS u32x4*)ctl;
                    if (rfl(st[0]) > k && rfl(st[1]) >= 5u * n + 5u) break;
                    if (rfl(st[2]) - 1u <= k) { end = true; break; }
                    settle();
                    __builtin_amdgcn_s_sleep(1);
                }
                if (end) break;
                asm volatile("" ::: "memory");
                u32x4 d[5];
                const LAS char* vs = lds + VRING + (k & (NV - 1)) * VSLOT;
                const u32 h = *(const LAS u32*)(vs + b * 4u);
                const u32x4 e0 = *(const LAS u32x4*)(q + (k & (NV - 1)) * 8u), e1 = *(const LAS u32x4*)(q + (k & (NV - 1)) * 8u + 4u);
                d[4] = *(const LAS u32x4*)(lds + sl * FIXB + 4096 + lane * 16);
#pragma unroll
                for (int j = 0; j < 4; ++j) d[j] = *(const LAS u32x4*)(lds + sl * FIXB + j * 1024 + lane * 16);
                asm volatile("s_waitcnt lgkmcnt(0)" ::: "memory");
                if (lane == 0) *(volatile LAS u32*)(ffree + sl) = n + 1u;
                asm volatile("" : "+v"(d[0]), "+v"(d[1]), "+v"(d[2]), "+v"(d[3]), "+v"(d[4]) :: "memory");
                const u32 hdr = rfl(h), off = (hdr >> 20) * 8u, kc = rfl(e1[2]);
                SEG(0);
                call = rfl(e1[3]);
                const bool far = rfl(e0[3]) > (u32)VSLOT;
                const LAS char* var = vs + (off < (u32)VSLOT ? off : 0u);
                u32x4 o[8];
                if (rfl(e0[2]) & 1u) {
#pragma unroll
                    for (int g = 0; g < 8; ++g) o[g] = (u32x4){0x7fc07fc0u, 0x7fc07fc0u, 0x7fc07fc0u, 0x7fc07fc0u};
                } else {
                    if (far) {
                        const rsrc_t r = mk_rsrc((const void*)(((u64)e0[1] << 32) | e0[0]), CH);
#pragma unroll
                        for (int j = 0; j < 3; ++j) o[5 + j] = __builtin_amdgcn_raw_buffer_load_b128(r, (int)(CVAR + off) + j * 1024 + lane * 16, 0, AUX_SYS);
                        if (lane < VARB / 16) *(LAS u32x4*)(wl + A2B + A3B + lane * 16) = o[5];
                        WAVE_FENCE();
                        var = wl + A2B + A3B;
                    }
                    if (hdr & 0xffu) decode_block(d, hdr, var, tab, wl, lane, o);
                    else {
#pragma unroll
                        for (int g = 0; g < 5; ++g) o[g] = d[g];
                        if (!far) {
#pragma unroll
                            for (int j = 0; j < 3; ++j) {
                                const u32x2 lo = *(const volatile LAS u32x2*)(var + j * 1024 + lane * 16), hi = *(const volatile LAS u32x2*)(var + j * 1024 + lane * 16 + 8);
                                o[5 + j] = (u32x4){lo[0], lo[1], hi[0], hi[1]};
                            }
                        }
                    }
                }
                const Dest dst = dest_of((char*)(((u64)e1[1] << 32) | e1[0]), kc & 3u, b);
                asm volatile("s_waitcnt lgkmcnt(0)" ::: "memory");
                if (lane == 0) __hip_atomic_fetch_add(vdone + (k & (NV - 1)), 1u, __ATOMIC_RELAXED, __HIP_MEMORY_SCOPE_WORKGROUP);
                SEG(1);
                settle();
                SEG(3);
                const u32 loff = (u32)(lane >> 4) * dst.LH + (u32)(lane & 15) * 16u;
                if ((kc & 3u) == 2u) {
#pragma unroll
                    for (int g = 0; g < 8; ++g) store_group<AUX_OUT>(dst, loff, g, o[g]);
                } else {
#pragma unroll
                    for (int g = 0; g < 8; ++g) store_group<AUX_DEV>(dst, loff, g, o[g]);
                    owed = (k & 7u) | ((kc >> 2) << 3);
                }
                SEG(2);
            }
            settle();
            STAMP_AT(3);
        }
        return;
    }

    {
        const u64 tb = __builtin_amdgcn_s_memrealtime();
        for (u32 task = bid, it = 0; task < (u32)NTASK; task += NGEMM, ++it) {
            if (task >= (u32)ENC_FIRST) while (__builtin_amdgcn_s_memrealtime() - tb < (u64)(ENC_DELAY + task * ENC_PACE / 64u)) __builtin_amdgcn_s_sleep(4);
            if (it == STAMP_IT) STAMP_AT(12);
            const u32 te = task < 16u - PUSHC ? (task + PUSHC) << 4 : (p.task[task >> 1] >> ((task & 1u) * 16u)) & 0xffffu, c = te >> 4, k = te & 15u, b = (u32)wid;
            const InPos ip = in_of(c);
            const u32 dd = (rank + k) & 7u, pm = ip.isw ? pmap_of(p, ip.s) : ip.s;
            u32x4 a[8];
            if (ip.isw) {
                const rsrc_t r = mk_rsrc(p.w + (size_t)(ip.q * 128u + (b >> 1) * 32u) * 65536u + (gpanel(dd, pm) * 256u + (b & 1u) * 128u) * 2u, 31u * 65536u + 256u);
#pragma unroll
                for (int i = 0; i < 8; ++i) a[i] = __builtin_amdgcn_raw_buffer_load_b128(r, (lane >> 4) * 8 * 65536 + (lane & 15) * 16, i * 65536, AUX_IN);
            } else load_x(p.x, ip.s, ip.q, b, lane, a);
            if (it == 0) setup(false);
            u32 v[8][4];
            if (ip.isw) {
#pragma unroll
                for (int j = 0; j < 8; ++j)
#pragma unroll
                    for (int e = 0; e < 4; ++e) v[j][e] = vperm(a[2 * e + 1][j >> 1], a[2 * e][j >> 1], (j & 1) ? 0x07060302u : 0x05040100u);
            } else {
#pragma unroll
                for (int j = 0; j < 8; ++j)
#pragma unroll
                    for (int e = 0; e < 4; ++e) v[j][e] = a[j][e];
            }
            const bool own = (k & 7u) == 0u, send = k < 8u;
            const u32 call = rfl(callv) + 1u;
            LAS u32* cnt = ctl + 8 + 2u * it;
            if (own) {
                const Dest dst = dest_of(frag_base(ws, ip.isw, ip.s, ip.q, rank, call), ip.isw ? 1u : 0u, b);
#pragma unroll
                for (int g = 0; g < 8; ++g) store_group<AUX_DEV>(dst, (u32)lane * 16u, g, (u32x4){v[g][0], v[g][1], v[g][2], v[g][3]});
            }
            if (send) encode_block(v, mk_rsrc(ws + OFF_SI + (size_t)((call & 1u) * NTASK + slot_in(ip.isw, pm, ip.q, dd)) * CH, CH), b, cnt, tab, wl, lane);
            if (it == STAMP_IT) STAMP_AT(14);
            asm volatile("s_waitcnt vmcnt(0)" ::: "memory");
            u32 total;
            if (chunk_done(cnt, lane, total)) {
                if (send && lane < (k ? 1 : 7))
                    __hip_atomic_store((GAS u64*)(p.ws[(rank + (k ? k : (u32)lane + 1u)) & 7u] + OFF_GRAN) + (size_t)((call & 1u) * 8u + rank) * NC + c,
                                       ((u64)total << 32) | call, __ATOMIC_RELAXED, __HIP_MEMORY_SCOPE_SYSTEM);
                if (own) arrive(ws, ip.cnt, 8u, call, lane);
            }
            if (it == STAMP_IT) STAMP_AT(1);
        }
        STAMP_AT(2);
        if (lane == 0) __hip_atomic_store(callp, callv + 1u, __ATOMIC_RELAXED, __HIP_MEMORY_SCOPE_AGENT);
    }

    const u32 call = rfl(callv) + 1u;
    const int wr = wid >> 2, wc = wid & 3;
    constexpr int HTB = 16384;
#define SA(b, h) (((b) * 2 + (h)) * HTB)
#define SB(b, h) ((4 + (b) * 2 + (h)) * HTB)
#define STAGE(bufoff, gbase) do { _Pragma("unroll") for (int _i = 0; _i < 2; ++_i) \
        __builtin_amdgcn_global_load_lds((const GAS u32*)((gbase) + voff + _i * 8192), (LAS u32*)(lds + (bufoff) + ldsw + _i * 8192), 16, 0, AUX_SYS); } while (0)
#define LDA(dst, b, h) do { _Pragma("unroll") for (int m = 0; m < 4; ++m) _Pragma("unroll") for (int k = 0; k < 2; ++k) dst[m][k] = *(const LAS bf16x8*)(lds + SA(b, h) + aoff + m * 2048 + k * 1024); } while (0)
#define LDB(dst, b, h) do { _Pragma("unroll") for (int n = 0; n < 2; ++n) _Pragma("unroll") for (int k = 0; k < 2; ++k) dst[n][k] = *(const LAS bf16x8*)(lds + SB(b, h) + boff + n * 2048 + k * 1024); } while (0)
#define MMA(ai, bj, At, Bt) do { __builtin_amdgcn_s_setprio(1); _Pragma("unroll") for (int m = 0; m < 4; ++m) _Pragma("unroll") for (int n = 0; n < 2; ++n) _Pragma("unroll") for (int k = 0; k < 2; ++k) \
        acc[ai][bj][m][n] = __builtin_amdgcn_mfma_f32_16x16x32_bf16(Bt[n][k], At[m][k], acc[ai][bj][m][n], 0, 0, 0); __builtin_amdgcn_s_setprio(0); } while (0)
#define WAIT_V(n) asm volatile("s_waitcnt vmcnt(" #n ")" ::: "memory")
#define WAIT_L(n) asm volatile("s_waitcnt lgkmcnt(" #n ")" ::: "memory")
#define BAR __builtin_amdgcn_s_barrier()
    const u32 voff = (u32)tid * 16u, ldsw = (u32)wid * 1024u;
    const int aoff = wr * 8192 + lane * 16, boff = wc * 4096 + lane * 16;
    const u32 idx = bid >> 3, hh = (bid >> 2) & 1u;
    const u32 ti = (bid & 3u) * 4u + (idx < 4u * (WB / 2) ? idx & 3u : 2u * hh + (idx & 1u)), tc = idx < 4u * (WB / 2) ? 2u * (idx >> 2) + hh : WB - 1u;
    constexpr size_t kstep = 32768, hstep = 16384;
    const GAS char* cB = (const GAS char*)frag_base(ws, false, ti, 0u, 0u, call);
    GAS u32* flags = (GAS u32*)(ws + OFF_FLAG + (bid & 7u) * 128u) + ((u32)lane & 3u);
    auto arrived = [&](u32 bt) { return (u32)__builtin_ctz(((u32)__builtin_amdgcn_ballot_w64(__hip_atomic_load(flags + bt * 4u, __ATOMIC_RELAXED, __HIP_MEMORY_SCOPE_SYSTEM) != call) & 15u) | 16u); };
#define STAGE_FIRST(pA, pB) do { STAGE(SB(0, 0), pB); STAGE(SA(0, 0), pA); STAGE(SB(0, 1), (pB) + hstep); STAGE(SA(0, 1), (pA) + hstep); \
        STAGE(SB(1, 0), (pB) + kstep); STAGE(SA(1, 0), (pA) + kstep); STAGE(SB(1, 1), (pB) + hstep + kstep); } while (0)
    u32 nwait = 0, bad = 0, staged = 0;
    for (u32 round = 0; round < (u32)NB; ++round) {
        const u32 tj = round * WB + tc;
        if (tj >= (u32)NP) break;
        const GAS char* cA = (const GAS char*)(ws + OFF_WF) + (size_t)tj * (2u << 20);
        f32x4 acc[2][2][4][2];
#pragma unroll
        for (int a = 0; a < 2; ++a)
#pragma unroll
            for (int b = 0; b < 2; ++b)
#pragma unroll
                for (int m = 0; m < 4; ++m)
#pragma unroll
                    for (int n = 0; n < 2; ++n) acc[a][b][m][n] = (f32x4){0.f, 0.f, 0.f, 0.f};
        bf16x8 At[4][2], B0[2][2], B1[2][2];
        for (u32 q0 = 0; q0 < 4u;) {
            const u32 k0 = q0 * 16u;
            if (!staged) {
                if (wid == 0) {
                    const u64 t0 = __builtin_amdgcn_s_memrealtime();
                    u32 q1;
                    for (;;) {
                        q1 = arrived(round);
                        if (q1 > q0) break;
                        if (give_up(tmo, call, t0)) { q1 = 4u | 8u; break; }
                        __builtin_amdgcn_s_sleep(POLL_SLEEP);
                    }
                    if (lane == 0) *(volatile LAS u32*)(ctl + (nwait & 1u)) = q1;
                    asm volatile("s_waitcnt lgkmcnt(0)" ::: "memory");
                }
                asm volatile("" ::: "memory"); BAR; asm volatile("" ::: "memory");
                const u32 te = lds_ld(ctl + (nwait & 1u));
                ++nwait; bad |= te >> 3;
                staged = te & 7u;
                STAGE_FIRST(cA + (size_t)k0 * kstep, cB + (size_t)k0 * kstep);
            }
            if (k0 == 0u && round < 4u) STAMP_AT(3 + round * 2);
            q0 = staged & 7u;
            const u32 k1 = q0 * 16u;
            if (wr == 1) BAR;
            if (staged >> 3) WAIT_V(22); else WAIT_V(6);
            BAR; BAR;
            for (u32 t = k0; t < k1; t += 2u) {
                const u32 t2 = t + 2u < k1 ? t + 2u : t;
                const GAS char* a1 = cA + (size_t)(t + 1u) * kstep;
                const GAS char* a2 = cA + (size_t)t2 * kstep; const GAS char* b2 = cB + (size_t)t2 * kstep;
                const GAS char* a3 = a2 + kstep; const GAS char* b3 = b2 + kstep;
                LDB(B0, 0, 0); SCHED; LDA(At, 0, 0); STAGE(SA(1, 1), a1 + hstep);
                WAIT_L(8); BAR; WAIT_L(0); MMA(0, 0, At, B0); BAR; SCHED;
                LDB(B1, 0, 1); STAGE(SB(0, 0), b2);
                BAR; WAIT_L(0); MMA(0, 1, At, B1); BAR;
                LDA(At, 0, 1); STAGE(SA(0, 0), a2);
                BAR; WAIT_L(0); MMA(1, 0, At, B0); BAR; SCHED;
                STAGE(SB(0, 1), b2 + hstep);
                WAIT_V(6); BAR; MMA(1, 1, At, B1); BAR;
                LDB(B0, 1, 0); SCHED; LDA(At, 1, 0); STAGE(SA(0, 1), a2 + hstep);
                WAIT_L(8); BAR; WAIT_L(0); MMA(0, 0, At, B0); BAR; SCHED;
                LDB(B1, 1, 1); STAGE(SB(1, 0), b3);
                BAR; WAIT_L(0); MMA(0, 1, At, B1); BAR;
                LDA(At, 1, 1); STAGE(SA(1, 0), a3);
                BAR; WAIT_L(0); MMA(1, 0, At, B0); BAR; SCHED;
                STAGE(SB(1, 1), b3 + hstep);
                WAIT_V(6); BAR; MMA(1, 1, At, B1); BAR;
            }
            const bool last = q0 == 4u, more = !last || tj + WB < (u32)NP;
            if (wid == 0) {
                const u32 q1 = more ? arrived(last ? round + 1u : round) : 0u;
                if (lane == 0) *(volatile LAS u32*)(ctl + (nwait & 1u)) = q1;
                asm volatile("s_waitcnt lgkmcnt(0)" ::: "memory");
            }
            WAIT_V(0);
            if (wr == 0) BAR;
            BAR;
            asm volatile("" ::: "memory");
            staged = lds_ld(ctl + (nwait & 1u));
            ++nwait;
            if (staged <= (last ? 0u : q0)) staged = 0u;
            if (staged) STAGE_FIRST(cA + (last ? (size_t)WB * (2u << 20) : (size_t)k1 * kstep), cB + (last ? (size_t)0 : (size_t)k1 * kstep));
        }
        if (round < 4u) STAMP_AT(4 + round * 2);
        const u32 pm = pmap_of(p, tj), yc = (pm * 16u + ti) * 2u;
        LAS u32* cnt = ctl + 32 + round * 4u;
#pragma unroll
        for (int bj = 0; bj < 2; ++bj) {
            u32 v[8][4];
#pragma unroll
            for (int g = 0; g < 8; ++g)
#pragma unroll
                for (int q = 0; q < 4; ++q) {
                    typedef float f32x2 __attribute__((ext_vector_type(2)));
                    typedef __bf16 bf16x2 __attribute__((ext_vector_type(2)));
                    const f32x2 f = {acc[q >> 1][bj][(q & 1) * 2][g >> 2][g & 3], acc[q >> 1][bj][(q & 1) * 2 + 1][g >> 2][g & 3]};
                    v[g][q] = __builtin_bit_cast(u32, __builtin_convertvector(f, bf16x2)) | (bad ? 0x7fc07fc0u : 0u);
                }
            const Dest dst = dest_of(y_base(p.out, ti, gpanel(rank, pm), bj), 2u, (u32)wid);
            const u32 loff = (u32)(lane >> 4) * dst.LH + (u32)(lane & 15) * 16u;
#pragma unroll
            for (int g = 0; g < 8; ++g) store_group<AUX_OUT>(dst, loff, g, (u32x4){v[g][0], v[g][1], v[g][2], v[g][3]});
            if (pm < 0x80u) encode_block(v, mk_rsrc(ws + OFF_SY + (size_t)(yc + bj) * CH, CH), (u32)wid, cnt + bj * 2, tab, wl, lane);
        }
        if (pm < 0x80u) {
            asm volatile("s_waitcnt vmcnt(0)" ::: "memory");
            u32 total;
            if (chunk_done(cnt, lane, total) && lane < 14)
                __hip_atomic_store((GAS u64*)(p.ws[(rank + 1u + ((u32)lane >> 1)) & 7u] + OFF_GRAN) + (size_t)((call & 1u) * 8u + rank) * NC + NCIN + yc + (lane & 1),
                                   ((u64)((lane & 1) ? *(volatile LAS u32*)(cnt + 2) + CHDR : total) << 32) | call, __ATOMIC_RELAXED, __HIP_MEMORY_SCOPE_SYSTEM);
        } else if (staged) staged |= 8u;
    }
    STAMP_AT(15);
}

static void build_tables(Params& p) {
    unsigned char* pm = (unsigned char*)p.pmap;
    int skip = NP % WB, o = 0, m = 0;
    while (skip && (WB - skip) * 2 < POWN) --skip;
    for (int j = 0; j < 80; ++j) pm[j] = (unsigned char)(j >= NP ? 0x80 : (j % WB >= skip && o < POWN ? o++ : 0x80 | m++));
    unsigned short* t = (unsigned short*)p.task;
    int n = 0;
    for (u32 c = PUSHC; c < (u32)NCIN; ++c) {
        if (c == 16u) for (u32 e = 0; e < (u32)PUSHC; ++e) t[n++] = (unsigned short)(e << 4 | 8u);
        const InPos ip = in_of(c);
        if (ip.isw && pm[ip.s] < 0x80) for (u32 k = 1; k <= 8; ++k) t[n++] = (unsigned short)(c << 4 | k);
        else t[n++] = (unsigned short)(c << 4);
    }
}

extern "C" void kernel_launch(int rank, int nranks, ncclComm_t comm, hipStream_t stream,
                              void* const* d_in, const int* in_sizes, int n_in, void* d_out, int out_size,
                              const int* in_dtypes, int out_dtype, void* const* d_out_all,
                              void* const* d_ws_all, size_t ws_bytes) {
    if (nranks != NR || (unsigned)rank >= (unsigned)NR || ws_bytes < WS_NEED || n_in != 2) { fprintf(stderr, "unsupported configuration\n"); return; }
    Params p;
    p.x = (const char*)d_in[0]; p.w = (const char*)d_in[1]; p.out = (char*)d_out; p.rank = rank; p.pad = 0;
    for (int i = 0; i < NR; ++i) p.ws[i] = (char*)d_ws_all[i];
    build_tables(p);
    hipLaunchKernelGGL(gemm_ar_fused, dim3(256), dim3(512), 0, stream, p);
    HIP_CHECK(hipGetLastError());
}
```

```cpp
#include <hip/hip_runtime.h>
#include <rccl/rccl.h>
#include <cstdio>

#define HIP_CHECK(expr) do { hipError_t _e = (expr); if (_e != hipSuccess) { \
    fprintf(stderr, "HIP error %s at %s:%d\n", hipGetErrorString(_e), __FILE__, __LINE__); \
    return; } } while (0)

typedef unsigned u32;
typedef unsigned long long u64;
typedef u32 u32x4 __attribute__((ext_vector_type(4)));
typedef u32 u32x2 __attribute__((ext_vector_type(2)));
typedef unsigned short u16x2 __attribute__((ext_vector_type(2)));
typedef short bf16x8 __attribute__((ext_vector_type(8)));
typedef float f32x4 __attribute__((ext_vector_type(4)));
#define LAS __attribute__((address_space(3)))
#define GAS __attribute__((address_space(1)))
typedef __amdgpu_buffer_rsrc_t rsrc_t;

constexpr int NR = 8;
#define PULL_WG_PER_PEER 8
#define WINDOW 8
#define TAILC 0
#define PUSHC 5
#define PUSHW 2
#define NLOAD 1
#define PACK_IN 112
#define AFFINE 1
#define LOOK 6
#define RHO 6
#define ENC_FIRST (16 - PUSHC)
#define ENC_DELAY 150
#define ENC_PACE 256
#define POLL_SLEEP 8
#define WB 12
constexpr int NPULL = 7 * PULL_WG_PER_PEER, NGEMM = 16 * WB, NPACK = 256 - NGEMM - NPULL;
constexpr int POWN = 16 - RHO, NP = POWN + 8 * RHO;
constexpr int NB = (NP + WB - 1) / WB, N0 = 16 + WB;
constexpr int NCIN = 64 + 4 * NP, NCY = 32 * POWN, NC = NCIN + NCY;
constexpr int NTASK = 16 * 36;
constexpr int NDEC = 7 - NLOAD;
constexpr u64 DEADLINE = 100000000ull;
constexpr int PUSHB = (8 * PUSHC + PULL_WG_PER_PEER - 1) / PULL_WG_PER_PEER;
constexpr int NLAST = NP - (NB - 1) * WB;
static_assert(NGEMM == 192 && NPACK >= 4 && (PACK_IN + NCY) / NPACK < 126 && NB <= 8 && POWN <= WB && PUSHB <= NDEC, "roles");

constexpr int FIXB = 5120, CVAR = 8 * FIXB, CHDR = 32, CH = 66560;
constexpr size_t OFF_CALL = 0;
constexpr size_t OFF_TMO = 131072;
constexpr size_t OFF_TICK = OFF_TMO + 256;
constexpr size_t OFF_RDY = OFF_TICK + 16 * 128;
constexpr size_t OFF_FLAG = OFF_RDY + 64 * 128;
constexpr size_t OFF_TILE = OFF_FLAG + 8 * 256;
constexpr size_t OFF_GRAN = 196608;
constexpr size_t OFF_DBG = 524288;
constexpr size_t OFF_XF = 1ull << 20;
constexpr size_t OFF_WF = OFF_XF + (64ull << 20);
constexpr size_t OFF_SY = OFF_WF + NP * (2ull << 20);
constexpr size_t OFF_SI = OFF_SY + (size_t)NCY * CH;
constexpr size_t WS_NEED = OFF_SI + 2ull * NTASK * CH;

constexpr int A2B = 2064, A3B = 528, VARB = 912, WLDS = A2B + A3B + VARB, N2MAX = 2032, N3MAX = 496, UBITS = 2016, NPATCH = 31;
constexpr int CTL_OFF = 2048, CTL_WORDS = 128, WL_OFF = 2560, BIG_OFF = 30720, LDS_BYTES = BIG_OFF + 131072;
static_assert(WL_OFF + 8 * WLDS <= BIG_OFF && LDS_BYTES <= 163840, "LDS map");
constexpr int NF = 16, NV = 4, VSLOT = 8192, VRING = NF * FIXB;
#define AUX_MID 2
#define DEC_SLEEP 1
#define AUX_OWN 16
constexpr int AUX_SYS = 17, AUX_DEV = 16, AUX_OUT = 18, AUX_IN = 2;

struct Params {
    int rank; int pad;
    const char* x; const char* w; char* out;
    char* ws[NR];
    u32 role[16];
    u32 task[NTASK / 2];
};

__device__ __forceinline__ u32 rfl(u32 x) { return (u32)__builtin_amdgcn_readfirstlane((int)x); }
__device__ __forceinline__ rsrc_t mk_rsrc(const void* p, u32 bytes) {
    const u64 a = (u64)p; const u64 u = ((u64)rfl((u32)(a >> 32)) << 32) | rfl((u32)a);
    return __builtin_amdgcn_make_buffer_rsrc((void*)u, (short)0, (int)rfl(bytes), 0x00020000);
}
__device__ __forceinline__ u32 vperm(u32 a, u32 b, u32 s) { return __builtin_amdgcn_perm(a, b, s); }
__device__ __forceinline__ u32 wave_incl_scan(u32 x) {
    x += (u32)__builtin_amdgcn_update_dpp(0, (int)x, 0x111, 0xf, 0xf, false);
    x += (u32)__builtin_amdgcn_update_dpp(0, (int)x, 0x112, 0xf, 0xf, false);
    x += (u32)__builtin_amdgcn_update_dpp(0, (int)x, 0x114, 0xf, 0xf, false);
    x += (u32)__builtin_amdgcn_update_dpp(0, (int)x, 0x118, 0xf, 0xf, false);
    x += (u32)__builtin_amdgcn_update_dpp(0, (int)x, 0x142, 0xa, 0xf, false);
    x += (u32)__builtin_amdgcn_update_dpp(0, (int)x, 0x143, 0xc, 0xf, false);
    return x;
}
__device__ __forceinline__ u32 pk_sub(u32 a, u32 b) { return __builtin_bit_cast(u32, (u16x2)(__builtin_bit_cast(u16x2, a) - __builtin_bit_cast(u16x2, b))); }
template <int AUX>
__device__ __forceinline__ void st128(u32x4 x, rsrc_t r, int voff, int soff) {
    __builtin_amdgcn_raw_buffer_store_b128(x, r, voff, soff, AUX);
    asm volatile("s_nop 1" :: "v"(x) : "memory");
}
#define WAVE_FENCE() __builtin_amdgcn_fence(__ATOMIC_ACQ_REL, "wavefront")

__device__ __forceinline__ void build_table(LAS u32x2* tab, u32 i, bool unpack) {
    u32 n = 0, dep = 0, cmp = 0x0c0c0c0cu;
#pragma unroll
    for (u32 t = 0; t < 4; ++t) {
        const u32 c = (i >> (2 * t)) & 3u;
        dep |= (c ? 4u + c : n) << (8 * t);
        if (!c) { cmp = (cmp & ~(0xffu << (8 * n))) | (t << (8 * n)); ++n; }
    }
    tab[i] = (u32x2){unpack ? dep : cmp, n};
}
__device__ __forceinline__ void build_table3(LAS u32x2* tab3, u32 i) {
    u64 r = 0; u32 n = 0, run = 0;
#pragma unroll
    for (u32 t = 0; t < 8; ++t) { if ((i >> t) & 1u) { r |= (u64)run << (8 * n); ++n; run = 0; } else ++run; }
    tab3[i] = (u32x2){(u32)r, (u32)(r >> 32)};
}
__device__ __forceinline__ void or3(LAS u32* q, u64 x, u32 sh) {
    const u64 y = x << sh;
    __hip_atomic_fetch_or(q, (u32)y, __ATOMIC_RELAXED, __HIP_MEMORY_SCOPE_WORKGROUP);
    __hip_atomic_fetch_or(q + 1, (u32)(y >> 32), __ATOMIC_RELAXED, __HIP_MEMORY_SCOPE_WORKGROUP);
    __hip_atomic_fetch_or(q + 2, (u32)(((x >> 32) << sh) >> 32), __ATOMIC_RELAXED, __HIP_MEMORY_SCOPE_WORKGROUP);
}
__device__ __forceinline__ u32 code_byte(u32 hi, u32 lo, u32 x) {
    const u32 f = vperm(hi, lo, x);
    return __builtin_amdgcn_udot4((f ^ (f >> 2)) & 0x03030303u, 0x40100401u, 0u, false);
}
template <bool PRED>
__device__ __forceinline__ void or_bytes(u32 pos, u32 x) {
    const u64 y = (u64)x << ((pos & 3u) * 8u);
    LAS u32* q = (LAS u32*)(size_t)(pos & ~3u);
    if (!PRED || x) {
        __hip_atomic_fetch_or(q, (u32)y, __ATOMIC_RELAXED, __HIP_MEMORY_SCOPE_WORKGROUP);
        __hip_atomic_fetch_or(q + 1, (u32)(y >> 32), __ATOMIC_RELAXED, __HIP_MEMORY_SCOPE_WORKGROUP);
    }
}
__device__ __forceinline__ u32 get_bytes(u32 pos) {
    const LAS u32* q = (const LAS u32*)(size_t)(pos & ~3u);
    return __builtin_amdgcn_alignbyte(q[1], q[0], pos);
}
__device__ __forceinline__ u32 la(const LAS void* p) { return (u32)(size_t)p; }
__device__ __forceinline__ u32 pack_level2(const LAS u32x2* tab, const LAS char* src, u32 n, LAS u32* end, LAS char* dst, int lane) {
    u32 m = 0;
    for (u32 i0 = 0; i0 < n; i0 += 1024u) {
        const u32 i = i0 + (u32)lane * 16u, hi = 0x02010000u, lo = 0x00030000u;
        const u32x4 x = *(const LAS u32x4*)(src + (i < n ? i : 0u));
        u32 cw = code_byte(hi, lo, x[0]) | (code_byte(hi, lo, x[1]) << 8) | (code_byte(hi, lo, x[2]) << 16) | (code_byte(hi, lo, x[3]) << 24);
        const u32 valid = i < n ? n - i : 0u;
        if (valid < 16u) cw |= 0x55555555u << (2u * valid);
        u32x2 e[4]; u32 c[5]; c[0] = 0;
#pragma unroll
        for (int j = 0; j < 4; ++j) { e[j] = tab[(cw >> (8 * j)) & 0xffu]; c[j + 1] = c[j] + e[j][1]; }
        const u32 incl = wave_incl_scan(c[4]), p0 = la(dst) + m + incl - c[4];
        if (i < n) *(end - 1 - (i >> 4)) = cw;
        m += (u32)__builtin_amdgcn_readlane((int)incl, 63);
        if (m <= (u32)N3MAX) {
#pragma unroll
            for (int j = 0; j < 4; ++j) or_bytes<true>(p0 + c[j], vperm(0u, x[j], e[j][0]));
        }
    }
    WAVE_FENCE();
    return m;
}
__device__ __forceinline__ void pack_level3(const LAS char* src, u32 n, LAS u32* ub, LAS u32* pt, u32 T, int lane, u32& bits, u32& np) {
    for (u32 i0 = 0; i0 < n; i0 += 256u) {
        const u32 i = i0 + (u32)lane * 4u;
        const u32 x = *(const LAS u32*)(src + (i < n ? i : 0u));
        u32 cum = 0, pf = 0; u64 m = 0;
#pragma unroll
        for (u32 t = 0; t < 4; ++t) {
            const u32 r = ((x >> (8 * t)) - 8u) & 0xffu;
            const bool valid = i + t < n, big = r > 15u;
            cum += valid && !big ? r : 0u;
            m |= (u64)(valid ? 1u : 0u) << cum;
            cum += valid ? 1u : 0u;
            pf |= valid && big ? 1u << t : 0u;
        }
        const u32 incl = wave_incl_scan(cum), p = bits + incl - cum;
        bits += (u32)__builtin_amdgcn_readlane((int)incl, 63);
        if (bits <= (u32)UBITS && cum) or3(ub + (p >> 5), m, p & 31u);
        if (__builtin_amdgcn_ballot_w64(pf != 0u)) {
#pragma unroll
            for (u32 t = 0; t < 4; ++t) {
                const u64 bm = __builtin_amdgcn_ballot_w64((pf >> t) & 1u);
                const u32 at = np + __builtin_amdgcn_mbcnt_hi((u32)(bm >> 32), __builtin_amdgcn_mbcnt_lo((u32)bm, 0u));
                if ((pf >> t) & 1u) pt[at] = ((i + t) << 8) | ((T - (x >> (8 * t))) & 0xffu);
                np += (u32)__builtin_popcountll(bm);
            }
        }
    }
    WAVE_FENCE();
}
__device__ __forceinline__ void encode_block(const u32 (&v)[8][4], rsrc_t ch, u32 b, LAS u32* alloc, const LAS u32x2* tab, LAS char* wl, int lane) {
#pragma unroll
    for (int j = 0; j < 4; ++j) if (j < 3 || lane < (WLDS - 3072) / 16) *(LAS u32x4*)(wl + j * 1024 + lane * 16) = (u32x4){0, 0, 0, 0};
    u32 s = 0;
#pragma unroll
    for (int g = 0; g < 8; ++g) s = __builtin_amdgcn_udot4(__builtin_amdgcn_alignbit(v[g][0], v[g][0], 31), 0x01000100u, s, false);
    u32 T = ((u32)__builtin_amdgcn_readlane((int)wave_incl_scan(s), 63) >> 10) + 5u;
    T = T > 255u ? 255u : (T < 23u ? 23u : T);
    const u32 tt = (T * 256u + 255u) * 0x10001u;
    const int fo = (int)(b * FIXB);
    u32 U[16], l1[4] = {0, 0, 0, 0}, c[17];
    u32x2 e[16];
#pragma unroll
    for (int q = 0; q < 4; ++q) {
        u32x4 sm;
#pragma unroll
        for (int j = 0; j < 4; ++j) {
            const int k = 4 * q + j;
            const u32 a0 = v[k >> 1][(k & 1) * 2], a1 = v[k >> 1][(k & 1) * 2 + 1];
            const u32 r0 = __builtin_amdgcn_alignbit(a0, a0, 31), r1 = __builtin_amdgcn_alignbit(a1, a1, 31);
            sm[j] = vperm(r1, r0, 0x06040200u);
            U[k] = vperm(pk_sub(tt, r1), pk_sub(tt, r0), 0x07050301u);
            const u32 cb = code_byte(0x00000302u, 0x01000000u, U[k]);
            l1[q] |= cb << (8 * j);
            e[k] = tab[cb];
        }
        st128<AUX_SYS>(sm, ch, lane * 16 + q * 1024, fo);
    }
    st128<AUX_SYS>((u32x4){l1[0], l1[1], l1[2], l1[3]}, ch, lane * 16 + 4096, fo);
    __builtin_amdgcn_sched_barrier(0);
    c[0] = 0;
#pragma unroll
    for (int k = 0; k < 16; ++k) c[k + 1] = c[k] + e[k][1];
    const u32 incl = wave_incl_scan(c[16]), p0 = la(wl) + incl - c[16];
    const u32 n2 = (u32)__builtin_amdgcn_readlane((int)incl, 63);
    bool raw = n2 > (u32)N2MAX;
    u32 d2 = 0, d3 = 0, np = 0;
    LAS u32* var = (LAS u32*)(wl + A2B + A3B);
    if (!raw) {
        WAVE_FENCE();
#pragma unroll
        for (int k = 0; k < 16; ++k) or_bytes<false>(p0 + c[k], vperm(0u, U[k], e[k][0]));
        WAVE_FENCE();
        d2 = (n2 + 15u) >> 4;
        const u32 n3 = pack_level2(tab, wl, n2, var + d2, wl + A2B, lane);
        raw = n3 > (u32)N3MAX;
        if (!raw) {
            u32 bits = 0;
            pack_level3(wl + A2B, n3, var + d2, (LAS u32*)wl, T, lane, bits, np);
            raw = bits > (u32)UBITS || np > (u32)NPATCH;
            d3 = (bits + 31u) >> 5;
            if (!raw && (u32)lane < np) var[d2 + d3 + (u32)lane] = ((const LAS u32*)wl)[lane];
            WAVE_FENCE();
        }
    }
    const u32 vd = d2 + d3 + np;
    u32 off = 0;
    if (lane == 0) off = __hip_atomic_fetch_add(alloc, raw ? 3072u : vd * 4u, __ATOMIC_RELAXED, __HIP_MEMORY_SCOPE_WORKGROUP);
    off = rfl(off) + CHDR;
    const int vo = CVAR + (int)off;
    if (lane == 0) __builtin_amdgcn_raw_buffer_store_b32(raw ? off << 17 : (T | (d3 << 8) | (np << 14) | ((off + d2 * 4u) << 17)), ch, 0, CVAR + (int)b * 4, AUX_SYS);
    if (raw) {
        asm volatile("s_waitcnt vmcnt(0)" ::: "memory");
#pragma unroll
        for (int g = 0; g < 8; ++g)
            st128<AUX_SYS>((u32x4){v[g][0], v[g][1], v[g][2], v[g][3]}, ch, lane * 16 + (g < 5 ? g : g - 5) * 1024, g < 5 ? fo : vo);
    } else {
        for (u32 o = (u32)lane; o < vd; o += 64u) __builtin_amdgcn_raw_buffer_store_b32(var[o], ch, (int)o * 4, vo, AUX_SYS);
    }
    WAVE_FENCE();
}
__device__ __forceinline__ bool chunk_done(LAS u32* cnt, int lane, u32& total) {
    u32 n = 0;
    if (lane == 0) n = __hip_atomic_fetch_add(cnt + 1, 1u, __ATOMIC_RELAXED, __HIP_MEMORY_SCOPE_WORKGROUP);
    if (rfl(n) != 7u) return false;
    total = rfl(*(volatile LAS u32*)cnt) + CHDR;
    return true;
}

struct Dest { rsrc_t r; u32 A, B, C, LH; };
__device__ __forceinline__ Dest dest_of(char* base, u32 kind, u32 b) {
    if (kind == 0u) return Dest{mk_rsrc(base + b * 4096u, 32768u + 4096u), 1024u, 32768u, 2048u, 256u};
    if (kind == 1u) return Dest{mk_rsrc(base + (b >> 2) * 32768u + (b & 1u) * 8192u + (b & 2u) * 512u, 16384u + 6144u + 1024u), 2048u, 4096u, 16384u, 256u};
    return Dest{mk_rsrc(base + (b & 3u) * (32u * 65536u) + (b >> 2) * 256u, 31u * 65536u + 256u), 65536u, 131072u, 16u * 65536u, 4u * 65536u};
}
template <int AUX>
__device__ __forceinline__ void store_group(const Dest& d, u32 loff, int g, u32x4 x) {
    st128<AUX>(x, d.r, (int)loff, (int)((g & 1 ? d.A : 0u) + (g & 2 ? d.B : 0u) + (g & 4 ? d.C : 0u)));
}

#define SCHED __builtin_amdgcn_sched_barrier(0)
__device__ __forceinline__ u32x2 get2(u32 pos) { const LAS u32* q = (const LAS u32*)(size_t)(pos & ~3u); return (u32x2){q[0], q[1]}; }
__device__ __forceinline__ void spread3(u32 w, u32 start, u32 n, u32x2 runs, u32 t8, u32 p) {
    if (n) {
        const u32 carry = (u32)__builtin_clz((w << 8) | start);
        const u64 x = ((u64)(t8 - runs[1]) << 32) | (t8 - runs[0] - carry);
        const u32 sh = 64u - 8u * n;
        or3((LAS u32*)(size_t)(p & ~3u), (x << sh) >> sh, (p & 3u) * 8u);
    }
}
__device__ __forceinline__ void decode_block(const u32x4 (&d)[5], u32 hdr, const LAS u32* ub, const LAS u32x2* tab, const LAS u32x2* tab3, LAS char* wl, int lane, u32x4 (&o)[8]) {
    const u32 T = hdr & 0xffu, d3 = (hdr >> 8) & 0x3fu, np = (hdr >> 14) & 0x1fu;
    auto ex = [&](u32 a, u32 b, u32 c) { return ((T - a) << 8) | ((T - b) << 16) | ((T - c) << 24); };
    const u32 t1 = ex(3, 4, 5), t2 = ex(6, 7, 2), t8 = (T - 8u) * 0x01010101u;
    const u32 w2 = *(ub - 1 - lane), p3i = la(ub) + (u32)lane - 3u;
    u32x2 e[16], e2[4], g[16];
    g[0] = get2(p3i);
    if (lane < A3B / 16) *(LAS u32x4*)(wl + A2B + lane * 16) = (u32x4){0, 0, 0, 0};
#pragma unroll
    for (int k = 0; k < 8; ++k) e[k] = tab[(d[4][k >> 2] >> (8 * (k & 3))) & 0xffu];
    SCHED;
    const u32 w3 = __builtin_amdgcn_alignbyte(g[0][1], g[0][0], p3i), x3 = (u32)lane < d3 * 4u ? w3 >> 24 : 0u;
    const u32x2 e3 = tab3[x3];
#pragma unroll
    for (int j = 0; j < 4; ++j) e2[j] = tab[(w2 >> (8 * j)) & 0xffu];
#pragma unroll
    for (int k = 8; k < 16; ++k) e[k] = tab[(d[4][k >> 2] >> (8 * (k & 3))) & 0xffu];
    SCHED;
    u32 c[17], f[5];
    c[0] = 0; f[0] = 0;
#pragma unroll
    for (int k = 0; k < 16; ++k) c[k + 1] = c[k] + e[k][1];
#pragma unroll
    for (int j = 0; j < 4; ++j) f[j + 1] = f[j] + e2[j][1];
    const u32 n3 = (u32)__builtin_popcount(x3);
    const u32 pk = c[16] | (f[4] << 11) | (n3 << 22);
    const u32 incl = wave_incl_scan(pk), ex0 = incl - pk, tot = (u32)__builtin_amdgcn_readlane((int)incl, 63);
    const u32 a3 = la(wl) + A2B, p1 = la(wl) + (ex0 & 0x7ffu), p2 = a3 + ((ex0 >> 11) & 0x1ffu);
    WAVE_FENCE();
    spread3(w3, lane < 3 ? 0x80000000u >> (8 * lane) : 0x80u, n3, e3, t8, a3 + ((ex0 >> 22) & 0x1ffu));
    for (u32 i0 = 64u, m = tot >> 22; i0 < d3 * 4u; i0 += 64u) {
        const u32 w = get_bytes(p3i + i0), x = i0 + (u32)lane < d3 * 4u ? w >> 24 : 0u, n = (u32)__builtin_popcount(x), in = wave_incl_scan(n);
        spread3(w, 0x80u, n, tab3[x], t8, a3 + ((m + in - n) & 0x1ffu));
        m += (u32)__builtin_amdgcn_readlane((int)in, 63);
    }
    WAVE_FENCE();
    if ((u32)lane < np) { const u32 pw = ub[d3 + (u32)lane]; *(LAS unsigned char*)(wl + A2B + ((pw >> 8) & 0x1ffu)) = (unsigned char)pw; }
    WAVE_FENCE();
#pragma unroll
    for (int j = 0; j < 4; ++j) { f[j] += p2; g[j] = get2(f[j]); }
    SCHED;
#pragma unroll
    for (int k = 0; k < 16; ++k) c[k] += p1;
    SCHED;
    u32x4 a2;
#pragma unroll
    for (int j = 0; j < 4; ++j) a2[j] = vperm(t2, __builtin_amdgcn_alignbyte(g[j][1], g[j][0], f[j]), e2[j][0]);
    *(LAS u32x4*)(wl + lane * 16) = a2;
    if ((tot & 0x7ffu) > 1024u) {
        const u32 cw = *(ub - 65 - lane);
        f[0] = 0;
#pragma unroll
        for (int j = 0; j < 4; ++j) { e2[j] = tab[(cw >> (8 * j)) & 0xffu]; f[j + 1] = f[j] + e2[j][1]; }
        const u32 p = a3 + (((tot >> 11) + wave_incl_scan(f[4]) - f[4]) & 0x1ffu);
#pragma unroll
        for (int j = 0; j < 4; ++j) a2[j] = vperm(t2, get_bytes(p + f[j]), e2[j][0]);
        *(LAS u32x4*)(wl + 1024 + lane * 16) = a2;
    }
    WAVE_FENCE();
#pragma unroll
    for (int k = 0; k < 16; ++k) g[k] = get2(c[k]);
    SCHED;
#pragma unroll
    for (int k = 0; k < 16; ++k) {
        const u32 x = vperm(t1, __builtin_amdgcn_alignbyte(g[k][1], g[k][0], c[k]), e[k][0]);
        const u32 smk = d[k >> 2][k & 3];
        const u32 r0 = vperm(x, smk, 0x05010400u), r1 = vperm(x, smk, 0x07030602u);
        o[k >> 1][2 * (k & 1)] = __builtin_amdgcn_alignbit(r0, r0, 1);
        o[k >> 1][2 * (k & 1) + 1] = __builtin_amdgcn_alignbit(r1, r1, 1);
    }
    WAVE_FENCE();
}

__device__ __forceinline__ char* frag_base(char* ws, bool isw, u32 s, u32 q, u32 rel, u32 call) {
    return ws + (isw ? OFF_WF : OFF_XF + (size_t)(call & 1u) * (32u << 20)) + (size_t)(s * 64u + (rel ? 6u + 14u * q + 2u * rel : 2u * q)) * 32768u;
}
__device__ __forceinline__ char* y_base(char* out, u32 i, u32 gp, u32 bj) { return out + (size_t)(i * 256u + bj * 128u) * 65536u + gp * 512u; }
__device__ __forceinline__ u32 gpanel(u32 r, u32 j) { return j < (u32)POWN ? r * 16u + j : ((j - POWN) & 7u) * 16u + POWN + ((j - POWN) >> 3); }
__device__ __forceinline__ u32 slot_in(bool isw, u32 s, u32 q, u32 dst) {
    return !isw ? s * 4u + q : (s < (u32)POWN ? 64u + (dst * POWN + s) * 4u + q : 64u + 28u * POWN + s * 4u + q);
}
__host__ __device__ __forceinline__ u32 batch_size(u32 bt) { return bt == 0u ? (u32)N0 : (bt == NB - 1u ? (u32)(NP - (NB - 1) * WB) : (u32)WB); }
struct InPos { bool isw; u32 s, q, bt; };
__host__ __device__ __forceinline__ InPos in_of(u32 c) {
    if (c < 4u * N0) { const u32 q = c / N0, pp = c - q * N0; return InPos{pp >= 16u, pp >= 16u ? pp - 16u : pp, q, 0u}; }
    const u32 bt = (c - 64u) / (4u * WB), r = c - 64u - bt * 4u * WB, q = r / batch_size(bt);
    return InPos{true, bt * WB + r - q * batch_size(bt), q, bt};
}

__device__ __forceinline__ void arrive(char* ws, u32 ci, u32 n, u32 call, int lane) {
    u32 old = 0;
    if (lane == 0) old = __hip_atomic_fetch_add((GAS u32*)(ws + OFF_RDY + ci * 128u), n, __ATOMIC_RELAXED, __HIP_MEMORY_SCOPE_SYSTEM);
    if (rfl(old) + n == call * ((ci & 7u) ? 56u : 32u) * batch_size(ci >> 3) && lane < 8)
        __hip_atomic_store((GAS u32*)(ws + OFF_FLAG + (u32)lane * 256u + ci * 4u), call, __ATOMIC_RELAXED, __HIP_MEMORY_SCOPE_SYSTEM);
}
__device__ __forceinline__ void load_x(const char* x, u32 s, u32 q, u32 b, int lane, u32x4 (&a)[8]) {
    const rsrc_t r = mk_rsrc(x + (size_t)(s * 256u + b * 32u) * 1024u + q * 256u, 31u * 1024u + 256u);
#pragma unroll
    for (int i = 0; i < 8; ++i) a[i] = __builtin_amdgcn_raw_buffer_load_b128(r, (lane & 15) * 1024 + (lane >> 4) * 16 + (i & 3) * 64, (i >> 2) * 16384, 0);
}

__device__ __forceinline__ bool give_up(GAS u32* tmo, u32 call, u64 t0) {
    if (rfl(__hip_atomic_load(tmo, __ATOMIC_RELAXED, __HIP_MEMORY_SCOPE_AGENT)) == call || __builtin_amdgcn_s_memrealtime() - t0 > DEADLINE) {
        __hip_atomic_store(tmo, call, __ATOMIC_RELAXED, __HIP_MEMORY_SCOPE_AGENT);
        return true;
    }
    return false;
}

#define STAMP_IT 0
#define STAMP_AT(i) do { } while (0)

__device__ __forceinline__ u32 lds_ld(LAS u32* p) { return rfl(*(volatile LAS u32*)p); }
__device__ __forceinline__ void dma_piece(rsrc_t r, u32 voff, u32 soff, u32 ldsaddr) {
    u32 keep;
    asm volatile("s_nop 4\n\ts_mov_b32 %0, m0\n\ts_mov_b32 m0, %1\n\ts_nop 0\n\tbuffer_load_dwordx4 %2, %3, %4 offen sc0 sc1 lds\n\ts_mov_b32 m0, %0"
                 : "=&s"(keep) : "s"(ldsaddr), "v"(voff), "s"(r), "s"(soff) : "memory");
}

__global__ void __launch_bounds__(512, 2) gemm_ar_fused(const Params p) {
    __shared__ __attribute__((aligned(16))) char smem[LDS_BYTES];
    LAS char* lds = (LAS char*)smem + BIG_OFF;
    LAS u32x2* tab = (LAS u32x2*)smem;
    LAS u32x2* tab3 = (LAS u32x2*)((LAS char*)smem + WL_OFF);
    const int tid = threadIdx.x, wid = (int)rfl((u32)tid >> 6), lane = tid & 63;
    const u32 rank = (u32)p.rank, bid = blockIdx.x;
    char* ws = p.ws[0];
    LAS char* wl = (LAS char*)smem + WL_OFF + wid * WLDS;
    LAS u32* ctl = (LAS u32*)((LAS char*)smem + CTL_OFF);
    GAS u32* tmo = (GAS u32*)(ws + OFF_TMO);
    STAMP_AT(0);
    GAS u32* callp = (GAS u32*)(ws + OFF_CALL + (bid * 8u + wid) * 64u);
    const u32 callv = __hip_atomic_load(callp, __ATOMIC_RELAXED, __HIP_MEMORY_SCOPE_AGENT);
    char* wsl = ws;
    auto setup = [&](bool unpack) {
        u32 t = (u32)tid;
        asm volatile("" : "+v"(t));
        if (t < 256u) build_table(tab, t, unpack);
        else { if (t < 256u + CTL_WORDS) ctl[t - 256u] = 0u; if (unpack) build_table3(tab3, t - 256u); }
        if (!unpack) {
            ((LAS u32*)lds)[t] = 0u;
#pragma unroll
            for (u32 k = 1; k < 8u; ++k) wsl = (t & 63u) == k ? p.ws[k] : wsl;
        }
        asm volatile("s_waitcnt lgkmcnt(0)" ::: "memory"); __builtin_amdgcn_s_barrier(); asm volatile("" ::: "memory");
    };

    const u32 role = bid >= (u32)NGEMM ? (p.role[((bid - NGEMM) >> 2) & 15u] >> ((bid & 3u) * 8u)) & 0xffu : 0u;
    if (role & 7u) {
        const u32 rel = role & 7u, pjj = role >> 3, src = (rank + rel) & 7u;
        char* pws = p.ws[rel];
        const u32 pn = pjj * PUSHB + (u32)wid - 1u - NLOAD;
        const bool push = wid > NLOAD && wid <= NLOAD + PUSHB && pn < 8u * PUSHC;
        u32x4 pa[8];
        if (push) load_x(p.x, pn >> 3, 0u, pn & 7u, lane, pa);
        setup(true);
        if (push) {
            const u32 call = rfl(callv) + 1u;
            const Dest dst = dest_of(frag_base(pws, false, pn >> 3, 0u, 8u - rel, call), 0u, pn & 7u);
#pragma unroll
            for (int g = 0; g < 8; ++g) asm volatile("" : "+v"(pa[g]));
#pragma unroll
            for (int g = 0; g < 8; ++g) { store_group<AUX_SYS>(dst, (u32)lane * 16u, g, pa[g]); asm volatile("s_waitcnt vmcnt(%0)" :: "n"(PUSHW - 1) : "memory"); }
            STAMP_AT(12);
            if (lane == 0) __hip_atomic_store(callp, call, __ATOMIC_RELAXED, __HIP_MEMORY_SCOPE_AGENT);
            asm volatile("s_waitcnt vmcnt(0)" ::: "memory");
            STAMP_AT(13);
            arrive(pws, 1u, 1u, call, lane);
        }
        LAS u32* vready = ctl; LAS u32* endk = ctl + 1; LAS u32* landed = ctl + 1 + wid; LAS u32* qavail = ctl + 4; LAS u32* prog = ctl + 5;
        LAS u32* vdone = ctl + 8; LAS u32* sdone = ctl + 16; LAS u32* ffree = ctl + 32;
        LAS u32* q = ctl + 64;
        const u32 l0 = la(lds), voff = (u32)lane * 16u;
        if (wid == 0) {
            const u32 call = rfl(callv) + 1u;
            if (lane == 0) __hip_atomic_store(callp, call, __ATOMIC_RELAXED, __HIP_MEMORY_SCOPE_AGENT);
            GAS u64* gran = (GAS u64*)(ws + OFF_GRAN) + (size_t)((call & 1u) * 8u + src) * NC;
            for (u32 k = 0;; ++k) {
                u32 c = PUSHC + pjj;
                if (k) {
                    while (lds_ld(prog) + LOOK < 8u * k) __builtin_amdgcn_s_sleep(2);
                    u32 tk = 0;
                    if (lane == 0) tk = __hip_atomic_fetch_add((GAS u32*)(ws + OFF_TICK + src * 128u), 1u, __ATOMIC_RELAXED, __HIP_MEMORY_SCOPE_AGENT);
                    c = rfl(tk) - (call - 1u) * (NC - PUSHC) + PULL_WG_PER_PEER + PUSHC;
                }
                u32 fl = 2u, vb = 0, kc = 2u;
                const char* from = pws; char* to = ws;
                if (c < (u32)NC) {
                    if (c >= (u32)NCIN) {
                        const u32 yc = c - NCIN;
                        from = pws + OFF_SY + (size_t)yc * CH;
                        to = y_base(p.out, (yc >> 1) & 15u, src * 16u + (yc >> 5), yc & 1u);
                    } else {
                        const InPos ip = in_of(c);
                        from = pws + OFF_SI + (size_t)((call & 1u) * NTASK + slot_in(ip.isw, ip.s, ip.q, rank)) * CH;
                        to = frag_base(ws, ip.isw, ip.s, ip.q, rel, call);
                        kc = (ip.isw ? 1u : 0u) | ((ip.bt * 8u + ip.q + 1u) << 2);
                    }
                    const u64 t0 = __builtin_amdgcn_s_memrealtime();
                    for (u32 spin = 0;; ++spin) {
                        const u64 g = __hip_atomic_load(gran + c, __ATOMIC_RELAXED, __HIP_MEMORY_SCOPE_SYSTEM);
                        if (rfl((u32)g) == call) { fl = c >= (u32)(NC - TAILC) ? 4u : 0u; vb = rfl((u32)(g >> 32)); break; }
                        if ((spin & 15u) == 15u && give_up(tmo, call, t0)) { fl = 1u; break; }
                        __builtin_amdgcn_s_sleep(1);
                    }
                    if (k == 0) STAMP_AT(1);
                }
                while (lds_ld(vdone + (k & (NV - 1))) < 8u * (k / NV)) __builtin_amdgcn_s_sleep(2);
                if (lane == 0) {
                    *(volatile LAS u32x4*)(q + (k & (NV - 1)) * 8u) = (u32x4){(u32)(u64)from, (u32)((u64)from >> 32), fl, vb};
                    *(volatile LAS u32x4*)(q + (k & (NV - 1)) * 8u + 4u) = (u32x4){(u32)(u64)to, (u32)((u64)to >> 32), kc, call};
                    *(volatile LAS u32*)qavail = k + 1u;
                }
                if (fl & 2u) { if (lane == 0) *(volatile LAS u32*)endk = k + 1u; break; }
                if (!(fl & 1u)) {
                    const u32 vl = vb > (u32)VSLOT ? (u32)CHDR : (vb + 15u) & ~15u;
                    const rsrc_t r = mk_rsrc(from, CVAR + vl);
                    for (u32 o = 0; o < vl; o += 1024u) dma_piece(r, voff + CVAR + o, 0u, l0 + VRING + (k & (NV - 1)) * VSLOT + o);
                    asm volatile("s_waitcnt vmcnt(0)" ::: "memory");
                }
                if (lane == 0) *(volatile LAS u32*)vready = k + 1u;
            }
        } else if (wid <= NLOAD) {
            __builtin_amdgcn_s_setprio(3);
            u32 issued = 0, pub = 0;
            auto drain = [&]() { asm volatile("s_waitcnt vmcnt(0)" ::: "memory"); pub = issued; if (lane == 0) *(volatile LAS u32*)landed = pub; };
            for (u32 k = 0;; ++k) {
                if (lds_ld(qavail) <= k) { drain(); while (lds_ld(qavail) <= k) __builtin_amdgcn_s_sleep(1); }
                const u32x4 e = *(volatile LAS u32x4*)(q + (k & (NV - 1)) * 8u);
                const u32 fl = rfl(e[2]);
                if (fl & 2u) break;
                const rsrc_t r = mk_rsrc((const void*)(((u64)e[1] << 32) | e[0]), (fl & 1u) ? 0u : (u32)CVAR);
                const bool wide = TAILC && (fl & 4u);
#pragma unroll 1
                for (u32 b = (u32)wid - 1u; b < 8u; b += NLOAD) {
                    const u32 n = 8u * k + b, sl = n & (NF - 1);
                    if (lane == 0 && wid == 1) *(volatile LAS u32*)prog = n + 1u;
                    if (lds_ld(ffree + sl) + NF <= n) { drain(); while (lds_ld(ffree + sl) + NF <= n) __builtin_amdgcn_s_sleep(1); }
#pragma unroll
                    for (u32 j = 0; j < 5u; ++j) {
                        dma_piece(r, voff, b * FIXB + j * 1024u, l0 + sl * FIXB + j * 1024u);
                        ++issued;
                        u32 done;
                        if (wide) { asm volatile("s_waitcnt vmcnt(%0)" :: "n"(2 * WINDOW - 1) : "memory"); done = issued - (2 * WINDOW - 1); }
                        else { asm volatile("s_waitcnt vmcnt(%0)" :: "n"(WINDOW - 1) : "memory"); done = issued - (WINDOW - 1); }
                        if ((wide || (j + 7u - WINDOW % 5u) % 5u == 0u) && (int)(done - pub) > 0) { pub = done; if (lane == 0) *(volatile LAS u32*)landed = pub; }
                    }
                }
            }
            drain();
            STAMP_AT(2);
        } else {
#define SEG(i)
            u32 owed = 0xffffffffu, call = 0;
            auto settle = [&]() {
                if (owed != 0xffffffffu) {
                    asm volatile("s_waitcnt vmcnt(0)" ::: "memory");
                    u32 old = 0;
                    if (lane == 0) old = __hip_atomic_fetch_add(sdone + (owed & 7u), 1u, __ATOMIC_RELAXED, __HIP_MEMORY_SCOPE_WORKGROUP);
                    if ((rfl(old) & 7u) == 7u) arrive(ws, owed >> 3, 8u, call, lane);
                    owed = 0xffffffffu;
                }
            };
            for (u32 n = (u32)wid - 1u - NLOAD;; n += NDEC) {
                const u32 k = n >> 3, b = n & 7u, sl = n & (NF - 1);
                bool end = false;
                for (;;) {
                    const u32x4 st = *(volatile LAS u32x4*)ctl;
                    if (rfl(st[0]) > k && rfl(NLOAD == 2 && (n & 1u) ? st[3] : st[2]) >= 5u * (n / NLOAD) + 5u) break;
                    if (rfl(st[1]) - 1u <= k) { end = true; break; }
                    settle();
                    __builtin_amdgcn_s_sleep(DEC_SLEEP);
                }
                if (end) break;
                asm volatile("" ::: "memory");
                u32x4 d[5];
                const LAS char* vs = lds + VRING + (k & (NV - 1)) * VSLOT;
                const u32 h = *(const LAS u32*)(vs + b * 4u);
                const u32x4 e0 = *(const LAS u32x4*)(q + (k & (NV - 1)) * 8u), e1 = *(const LAS u32x4*)(q + (k & (NV - 1)) * 8u + 4u);
                d[4] = *(const LAS u32x4*)(lds + sl * FIXB + 4096 + lane * 16);
#pragma unroll
                for (int j = 0; j < 4; ++j) d[j] = *(const LAS u32x4*)(lds + sl * FIXB + j * 1024 + lane * 16);
                asm volatile("s_waitcnt lgkmcnt(0)" ::: "memory");
                if (lane == 0) *(volatile LAS u32*)(ffree + sl) = n + 1u;
                asm volatile("" : "+v"(d[0]), "+v"(d[1]), "+v"(d[2]), "+v"(d[3]), "+v"(d[4]) :: "memory");
                const u32 hdr = rfl(h), off = (hdr >> 19) * 4u, kc = rfl(e1[2]);
                SEG(0);
                call = rfl(e1[3]);
                const bool far = rfl(e0[3]) > (u32)VSLOT;
                const LAS u32* var = (const LAS u32*)(vs + (off <= (u32)VSLOT ? off : 0u));
                u32x4 o[8];
                if (rfl(e0[2]) & 1u) {
#pragma unroll
                    for (int g = 0; g < 8; ++g) o[g] = (u32x4){0x7fc07fc0u, 0x7fc07fc0u, 0x7fc07fc0u, 0x7fc07fc0u};
                } else {
                    if (far) {
                        const rsrc_t r = mk_rsrc((const void*)(((u64)e0[1] << 32) | e0[0]), CH);
#pragma unroll
                        for (int j = 0; j < 3; ++j) o[5 + j] = __builtin_amdgcn_raw_buffer_load_b128(r, (int)(CVAR + off) - ((hdr & 0xffu) ? 512 : 0) + j * 1024 + lane * 16, 0, AUX_SYS);
                        if (lane < VARB / 16) *(LAS u32x4*)(wl + A2B + A3B + lane * 16) = o[5];
                        WAVE_FENCE();
                        var = (const LAS u32*)(wl + A2B + A3B + 512);
                    }
                    if (hdr & 0xffu) decode_block(d, hdr, var, tab, tab3, wl, lane, o);
                    else {
#pragma unroll
                        for (int g = 0; g < 5; ++g) o[g] = d[g];
                        if (!far) {
#pragma unroll
                            for (int j = 0; j < 3; ++j)
#pragma unroll
                                for (int i = 0; i < 4; ++i) o[5 + j][i] = *(const volatile LAS u32*)(var + j * 256 + lane * 4 + i);
                        }
                    }
                }
                const Dest dst = dest_of((char*)(((u64)e1[1] << 32) | e1[0]), kc & 3u, b);
                asm volatile("s_waitcnt lgkmcnt(0)" ::: "memory");
                if (lane == 0) __hip_atomic_fetch_add(vdone + (k & (NV - 1)), 1u, __ATOMIC_RELAXED, __HIP_MEMORY_SCOPE_WORKGROUP);
                SEG(1);
                settle();
                SEG(3);
                const u32 loff = (u32)(lane >> 4) * dst.LH + (u32)(lane & 15) * 16u;
                if ((kc & 3u) == 2u) {
#pragma unroll
                    for (int g = 0; g < 8; ++g) store_group<AUX_OUT>(dst, loff, g, o[g]);
                } else {
#pragma unroll
                    for (int g = 0; g < 8; ++g) store_group<AUX_DEV>(dst, loff, g, o[g]);
                    owed = (k & 7u) | ((kc >> 2) << 3);
                }
                SEG(2);
            }
            settle();
            STAMP_AT(3);
        }
        return;
    }

    {
        const bool packer = bid >= (u32)NGEMM;
        const u64 tb = __builtin_amdgcn_s_memrealtime();
        const u32 tend = packer ? (u32)NTASK : (u32)(NTASK - PACK_IN), b = (u32)wid;
        u32 it = 0;
        for (u32 task = packer ? (role >> 3) + NTASK - PACK_IN : bid; task < tend; task += packer ? (u32)NPACK : (u32)NGEMM, ++it) {
            if (task >= (u32)ENC_FIRST) while (__builtin_amdgcn_s_memrealtime() - tb < (u64)(ENC_DELAY + (packer ? 0u : task * ENC_PACE / 64u))) __builtin_amdgcn_s_sleep(4);
            if (it == STAMP_IT) STAMP_AT(12);
            const u32 te = task < 16u - PUSHC ? (task + PUSHC) << 4 : (p.task[task >> 1] >> ((task & 1u) * 16u)) & 0xffffu, c = te >> 4, k = te & 15u;
            const InPos ip = in_of(c);
            const u32 dd = (rank + k) & 7u;
            u32x4 a[8];
            if (ip.isw) {
                const rsrc_t r = mk_rsrc(p.w + (size_t)(ip.q * 128u + (b >> 1) * 32u) * 65536u + (gpanel(dd, ip.s) * 256u + (b & 1u) * 128u) * 2u, 31u * 65536u + 256u);
#pragma unroll
                for (int i = 0; i < 8; ++i) a[i] = __builtin_amdgcn_raw_buffer_load_b128(r, (lane >> 4) * 8 * 65536 + (lane & 15) * 16, i * 65536, AUX_IN);
            } else load_x(p.x, ip.s, ip.q, b, lane, a);
            if (it == 0) setup(false);
            u32 v[8][4];
            if (ip.isw) {
#pragma unroll
                for (int j = 0; j < 8; ++j)
#pragma unroll
                    for (int e = 0; e < 4; ++e) v[j][e] = vperm(a[2 * e + 1][j >> 1], a[2 * e][j >> 1], (j & 1) ? 0x07060302u : 0x05040100u);
            } else {
#pragma unroll
                for (int j = 0; j < 8; ++j)
#pragma unroll
                    for (int e = 0; e < 4; ++e) v[j][e] = a[j][e];
            }
            const bool own = (k & 7u) == 0u, send = k < 8u;
            const u32 call = rfl(callv) + 1u;
            LAS u32* cnt = (LAS u32*)lds + 4u * it;
            u32 total;
            if (send) {
                encode_block(v, mk_rsrc(ws + OFF_SI + (size_t)((call & 1u) * NTASK + slot_in(ip.isw, ip.s, ip.q, dd)) * CH, CH), b, cnt, tab, wl, lane);
                if (it == STAMP_IT) STAMP_AT(14);
                asm volatile("s_waitcnt vmcnt(0)" ::: "memory");
                if (chunk_done(cnt, lane, total) && (k ? (u32)lane == k : (u32)lane - 1u < 7u))
                    __hip_atomic_store((GAS u64*)(wsl + OFF_GRAN) + (size_t)((call & 1u) * 8u + rank) * NC + c,
                                       ((u64)total << 32) | call, __ATOMIC_RELAXED, __HIP_MEMORY_SCOPE_SYSTEM);
            }
            if (own) {
                const Dest dst = dest_of(frag_base(ws, ip.isw, ip.s, ip.q, 0u, call), ip.isw ? 1u : 0u, b);
#pragma unroll
                for (int g = 0; g < 8; ++g) store_group<AUX_DEV>(dst, (u32)lane * 16u, g, (u32x4){v[g][0], v[g][1], v[g][2], v[g][3]});
                asm volatile("s_waitcnt vmcnt(0)" ::: "memory");
                if (chunk_done(cnt + 2, lane, total)) arrive(ws, ip.bt * 8u, 8u, call, lane);
            }
            if (it == STAMP_IT) STAMP_AT(1);
        }
        if (lane == 0) __hip_atomic_store(callp, callv + 1u, __ATOMIC_RELAXED, __HIP_MEMORY_SCOPE_AGENT);
        if (packer) {
            STAMP_AT(4);
            if (wid >= 4) __builtin_amdgcn_s_setprio(1);
            const u32 call = rfl(callv) + 1u;
            GAS u32* tf = (GAS u32*)(ws + OFF_TILE + b * 4u);
            u32x4 a[8];
            u32 seen = 0, nanv = 0;
            auto fetch = [&](u32 yc) {
                if (rfl(seen) != call) {
                    const u64 t0 = __builtin_amdgcn_s_memrealtime();
                    while (rfl(__hip_atomic_load(tf + (yc >> 1) * 32u, __ATOMIC_RELAXED, __HIP_MEMORY_SCOPE_AGENT)) != call) {
                        if (give_up(tmo, call, t0)) { nanv = 0x7fc07fc0u; break; }
                        __builtin_amdgcn_s_sleep(POLL_SLEEP);
                    }
                }
                asm volatile("" ::: "memory");
                const Dest sy = dest_of(y_base(p.out, (yc >> 1) & 15u, rank * 16u + (yc >> 5), yc & 1u), 2u, b);
#pragma unroll
                for (int i = 0; i < 8; ++i) a[i] = __builtin_amdgcn_raw_buffer_load_b128(sy.r, (lane >> 4) * 4 * 65536 + (lane & 15) * 16, ((i & 3) + (i >> 2) * 16) * 65536, AUX_SYS);
                seen = __hip_atomic_load(tf + ((yc + NPACK) >> 1) * 32u, __ATOMIC_RELAXED, __HIP_MEMORY_SCOPE_AGENT);
            };
            auto publish = [&](u32 yc) {
                u32 total;
                if (chunk_done((LAS u32*)lds + 4u * it, lane, total) && (u32)lane - 1u < 7u)
                    __hip_atomic_store((GAS u64*)(wsl + OFF_GRAN) + (size_t)((call & 1u) * 8u + rank) * NC + NCIN + yc,
                                       ((u64)total << 32) | call, __ATOMIC_RELAXED, __HIP_MEMORY_SCOPE_SYSTEM);
                ++it;
            };
            u32 yc = role >> 3;
            fetch(yc);
            STAMP_AT(3);
            for (;; yc += NPACK) {
                u32 v[8][4];
#pragma unroll
                for (int j = 0; j < 8; ++j)
#pragma unroll
                    for (int e = 0; e < 4; ++e) v[j][e] = a[j][e] | nanv;
                if (yc + NPACK < (u32)NCY) fetch(yc + NPACK);
                encode_block(v, mk_rsrc(ws + OFF_SY + (size_t)yc * CH, CH), b, (LAS u32*)lds + 4u * it, tab, wl, lane);
                asm volatile("s_waitcnt vmcnt(0)" ::: "memory");
                publish(yc);
                if (yc + NPACK >= (u32)NCY) break;
            }
            STAMP_AT(2);
            return;
        }
        STAMP_AT(2);
    }

    const u32 call = rfl(callv) + 1u;
    const int wr = wid >> 2, wc = wid & 3;
    constexpr int HTB = 16384;
#define SA(b, h) (((b) * 2 + (h)) * HTB)
#define SB(b, h) ((4 + (b) * 2 + (h)) * HTB)
#define STAGE(bufoff, gbase) do { _Pragma("unroll") for (int _i = 0; _i < 2; ++_i) \
        __builtin_amdgcn_global_load_lds((const GAS u32*)((gbase) + voff + _i * 8192), (LAS u32*)(lds + (bufoff) + ldsw + _i * 8192), 16, 0, AUX_SYS); } while (0)
#define LDA(dst, b, h) do { _Pragma("unroll") for (int m = 0; m < 4; ++m) _Pragma("unroll") for (int k = 0; k < 2; ++k) dst[m][k] = *(const LAS bf16x8*)(lds + SA(b, h) + aoff + m * 2048 + k * 1024); } while (0)
#define LDB(dst, b, h) do { _Pragma("unroll") for (int n = 0; n < 2; ++n) _Pragma("unroll") for (int k = 0; k < 2; ++k) dst[n][k] = *(const LAS bf16x8*)(lds + SB(b, h) + boff + n * 2048 + k * 1024); } while (0)
#define MORD 1
#define MMA(ai, bj, At, Bt) do { __builtin_amdgcn_s_setprio(1); _Pragma("unroll") for (int _x = 0; _x < 16; ++_x) { \
        const int m = MORD == 0 ? _x >> 2 : MORD == 1 ? (_x >> 1) & 3 : MORD == 2 ? _x & 3 : MORD == 3 ? ((_x & 4) ? 3 - (_x & 3) : _x & 3) : _x >> 2, \
                  n = MORD == 0 ? (_x >> 1) & 1 : MORD == 1 ? (_x & 1) ^ ((_x >> 1) & 1) : MORD == 4 ? (_x & 1) ^ ((_x >> 1) & 1) : (_x >> 2) & 1, \
                  k = MORD == 0 ? _x & 1 : MORD == 4 ? (_x >> 1) & 1 : _x >> 3; \
        acc[ai][bj][m][n] = __builtin_amdgcn_mfma_f32_16x16x32_bf16(Bt[n][k], At[m][k], acc[ai][bj][m][n], 0, 0, 0); if (MORD) SCHED; } __builtin_amdgcn_s_setprio(0); } while (0)
#define MMA2(ai) do { MMA(ai, 0, At, B0); MMA(ai, 1, At, B1); } while (0)
#define KLOOP 2
#define WAIT_V(n) asm volatile("s_waitcnt vmcnt(" #n ")" ::: "memory")
#define WAIT_L(n) asm volatile("s_waitcnt lgkmcnt(" #n ")" ::: "memory")
#define BAR __builtin_amdgcn_s_barrier()
    const u32 voff = (u32)tid * 16u, ldsw = (u32)wid * 1024u;
    const int aoff = wr * 8192 + lane * 16, boff = wc * 4096 + lane * 16;
    const u32 idx = bid >> 3, hh = (bid >> 2) & 1u;
    u32 ti = (bid & 3u) * 4u + (idx < 4u * (WB / 2) ? idx & 3u : 2u * hh + (idx & 1u)), tj = idx < 4u * (WB / 2) ? 2u * (idx >> 2) + hh : WB - 1u;
    constexpr size_t kstep = 32768, hstep = 16384;
    const GAS char* xf = (const GAS char*)frag_base(ws, false, 0u, 0u, 0u, call);
    const GAS char* wf = (const GAS char*)(ws + OFF_WF);
    GAS u32* flags = (GAS u32*)(ws + OFF_FLAG + (bid & 7u) * 256u) + ((u32)lane & 7u);
    auto arrived = [&](u32 bt) { return (u32)__builtin_ctz(((u32)__builtin_amdgcn_ballot_w64(__hip_atomic_load(flags + bt * 8u, __ATOMIC_RELAXED, __HIP_MEMORY_SCOPE_SYSTEM) != call) & 31u) | 32u); };
#define STAGE_FIRST(pA, pB) do { STAGE(SB(0, 0), pB); STAGE(SA(0, 0), pA); STAGE(SB(0, 1), (pB) + hstep); STAGE(SA(0, 1), (pA) + hstep); \
        STAGE(SB(1, 0), (pB) + kstep); STAGE(SA(1, 0), (pA) + kstep); STAGE(SB(1, 1), (pB) + hstep + kstep); } while (0)
    u32 nwait = 0, bad = 0, staged = 0;
    for (u32 round = 0; tj < (u32)NP; ++round) {
        const GAS char* cA = wf + (size_t)tj * (2u << 20);
        const GAS char* cB = xf + (size_t)ti * (2u << 20);
        u32 ni = ti, nj = tj + WB;
        f32x4 acc[2][2][4][2];
#pragma unroll
        for (int a = 0; a < 2; ++a)
#pragma unroll
            for (int b = 0; b < 2; ++b)
#pragma unroll
                for (int m = 0; m < 4; ++m)
#pragma unroll
                    for (int n = 0; n < 2; ++n) acc[a][b][m][n] = (f32x4){0.f, 0.f, 0.f, 0.f};
        bf16x8 At[4][2], B0[2][2], B1[2][2];
        for (u32 q0 = 0; q0 < 5u;) {
            const u32 k0 = q0 ? q0 * 14u - 6u : 0u;
            if (!staged) {
                if (wid == 0) {
                    const u64 t0 = __builtin_amdgcn_s_memrealtime();
                    u32 q1;
                    for (;;) {
                        q1 = arrived(round);
                        if (q1 > q0) break;
                        if (give_up(tmo, call, t0)) { q1 = 5u | 8u; break; }
                        __builtin_amdgcn_s_sleep(POLL_SLEEP);
                    }
                    if (lane == 0) *(volatile LAS u32*)(ctl + (nwait & 1u)) = q1;
                    asm volatile("s_waitcnt lgkmcnt(0)" ::: "memory");
                }
                asm volatile("" ::: "memory"); BAR; asm volatile("" ::: "memory");
                const u32 te = lds_ld(ctl + (nwait & 1u));
                ++nwait; bad |= te >> 3;
                staged = te & 7u;
                STAGE_FIRST(cA + (size_t)k0 * kstep, cB + (size_t)k0 * kstep);
            }
            if (k0 == 0u && round < 4u) STAMP_AT(3 + round * 2);
            q0 = staged & 7u;
            const u32 k1 = q0 * 14u - 6u;
            if (wr == 1) BAR;
            bool lax = (staged >> 3) != 0u;
            if (lax) WAIT_V(22); else WAIT_V(6);
            BAR; BAR;
            for (u32 t = k0; t < k1; t += 2u) {
                const u32 t2 = t + 2u < k1 ? t + 2u : t;
                const GAS char* a1 = cA + (size_t)(t + 1u) * kstep;
                const GAS char* a2 = cA + (size_t)t2 * kstep; const GAS char* b2 = cB + (size_t)t2 * kstep;
                const GAS char* a3 = a2 + kstep; const GAS char* b3 = b2 + kstep;
                LDB(B0, 0, 0); LDB(B1, 0, 1); LDA(At, 0, 0); STAGE(SA(1, 1), a1 + hstep);
                if (lax) WAIT_V(24); else WAIT_V(8);
                WAIT_L(0); BAR; SCHED; MMA2(0); BAR; SCHED;
                LDA(At, 0, 1); STAGE(SB(0, 0), b2); STAGE(SB(0, 1), b2 + hstep); STAGE(SA(0, 0), a2);
                if (lax) WAIT_V(24); else WAIT_V(8);
                lax = false;
                WAIT_L(0); BAR; SCHED; MMA2(1); BAR; SCHED;
                LDB(B0, 1, 0); LDB(B1, 1, 1); LDA(At, 1, 0); STAGE(SA(0, 1), a2 + hstep);
                WAIT_V(8); WAIT_L(0); BAR; SCHED; MMA2(0); BAR; SCHED;
                LDA(At, 1, 1); STAGE(SB(1, 0), b3); STAGE(SB(1, 1), b3 + hstep); STAGE(SA(1, 0), a3);
                WAIT_V(8); WAIT_L(0); BAR; SCHED; MMA2(1); BAR; SCHED;
            }
            const bool last = q0 == 5u, draw = NLAST < WB && last && round == NB - 2u;
            if (wid == 0) {
                u32 tk = 0;
                if (draw && lane == 0) tk = __hip_atomic_fetch_add((GAS u32*)(ws + OFF_TICK + 8u * 128u), 1u, __ATOMIC_RELAXED, __HIP_MEMORY_SCOPE_AGENT) - (call - 1u) * NGEMM;
                const u32 q1 = arrived(last ? round + 1u : round);
                if (lane == 0) *(volatile LAS u32*)(ctl + (nwait & 1u)) = q1 | (tk << 8);
                asm volatile("s_waitcnt lgkmcnt(0)" ::: "memory");
            }
            WAIT_V(0);
            if (wr == 0) BAR;
            BAR;
            asm volatile("" ::: "memory");
            staged = lds_ld(ctl + (nwait & 1u));
            ++nwait;
            if (draw) {
                const u32 tk = staged >> 8, gr = tk / (4u * NLAST), r = tk - gr * 4u * NLAST;
                ni = (gr * 4u + (r & 3u)) & 15u; nj = gr < 4u ? (NB - 1u) * WB + (r >> 2) : (u32)NP;
            }
            staged &= 7u;
            if (staged <= (last ? 0u : q0) || (last && nj >= (u32)NP)) staged = 0u;
            if (staged) STAGE_FIRST(last ? wf + (size_t)nj * (2u << 20) : cA + (size_t)k1 * kstep, last ? xf + (size_t)ni * (2u << 20) : cB + (size_t)k1 * kstep);
        }
        if (round < 4u) STAMP_AT(4 + round * 2);
#pragma unroll
        for (int bj = 0; bj < 2; ++bj) {
            const Dest dst = dest_of(y_base(p.out, ti, gpanel(rank, tj), bj), 2u, (u32)wid);
            const u32 loff = (u32)(lane >> 4) * dst.LH + (u32)(lane & 15) * 16u;
            u32x4 v[8];
#pragma unroll
            for (int g = 0; g < 8; ++g)
#pragma unroll
                for (int q = 0; q < 4; ++q) {
                    typedef float f32x2 __attribute__((ext_vector_type(2)));
                    typedef __bf16 bf16x2 __attribute__((ext_vector_type(2)));
                    const f32x2 f = {acc[q >> 1][bj][(q & 1) * 2][g >> 2][g & 3], acc[q >> 1][bj][(q & 1) * 2 + 1][g >> 2][g & 3]};
                    v[g][q] = __builtin_bit_cast(u32, __builtin_convertvector(f, bf16x2));
                }
            if (bad) {
#pragma unroll
                for (int g = 0; g < 8; ++g) v[g] = (u32x4){0x7fc07fc0u, 0x7fc07fc0u, 0x7fc07fc0u, 0x7fc07fc0u};
            }
            if (tj < (u32)POWN) {
#pragma unroll
                for (int g = 0; g < 8; ++g) store_group<AUX_OWN>(dst, loff, g, v[g]);
            } else if (AUX_MID != AUX_OUT && staged) {
#pragma unroll
                for (int g = 0; g < 8; ++g) store_group<AUX_MID>(dst, loff, g, v[g]);
            } else {
#pragma unroll
                for (int g = 0; g < 8; ++g) store_group<AUX_OUT>(dst, loff, g, v[g]);
            }
        }
        if (tj < (u32)POWN) {
            asm volatile("s_waitcnt vmcnt(0)" ::: "memory");
            if (lane == 0) __hip_atomic_store((GAS u32*)(ws + OFF_TILE + (tj * 16u + ti) * 128u + (u32)wid * 4u), call, __ATOMIC_RELAXED, __HIP_MEMORY_SCOPE_AGENT);
        } else if (staged) staged |= 8u;
        ti = ni; tj = nj;
    }
    STAMP_AT(15);
}

static void build_tables(Params& p) {
    unsigned char* role = (unsigned char*)p.role;
    for (int i = 0; i < 64; ++i) role[i] = (unsigned char)(i < NPULL ? (1 + i % 7) | (i / 7) << 3 : (i - NPULL) << 3);
    {
        static_assert(NPULL == 56 && NPACK == 8, "roles");
        int cnt[2] = {0, 0};
        for (int j = 0; j < 8; ++j)
            for (int rel = 1; rel < 8; ++rel) {
                const int h = (((0xb2 >> p.rank) ^ (0xb2 >> ((p.rank + rel) & 7))) & 1), e = cnt[h]++;
                role[(e >> 2) * 8 + (h ^ (AFFINE == 2)) * 4 + ((e + (h ? e >> 2 : 0)) & 3)] = (unsigned char)(rel | j << 3);
            }
        for (int j = 0; j < 8; ++j) { const int e = 24 + j; role[(e >> 2) * 8 + (AFFINE == 2) * 4 + (e & 3)] = (unsigned char)(j << 3); }
    }
    unsigned short* t = (unsigned short*)p.task;
    int n = 0;
    for (u32 c = PUSHC; c < (u32)NCIN; ++c) {
        if (c == 16u) for (u32 e = 0; e < (u32)PUSHC; ++e) t[n++] = (unsigned short)(e << 4 | 8u);
        const InPos ip = in_of(c);
        if (ip.isw && ip.s < (u32)POWN) for (u32 k = 1; k <= 8; ++k) t[n++] = (unsigned short)(c << 4 | k);
        else t[n++] = (unsigned short)(c << 4);
    }
}

extern "C" void kernel_launch(int rank, int nranks, ncclComm_t comm, hipStream_t stream,
                              void* const* d_in, const int* in_sizes, int n_in, void* d_out, int out_size,
                              const int* in_dtypes, int out_dtype, void* const* d_out_all,
                              void* const* d_ws_all, size_t ws_bytes) {
    if (nranks != NR || (unsigned)rank >= (unsigned)NR || ws_bytes < WS_NEED || n_in != 2) { fprintf(stderr, "unsupported configuration\n"); return; }
    Params p;
    p.x = (const char*)d_in[0]; p.w = (const char*)d_in[1]; p.out = (char*)d_out; p.rank = rank; p.pad = 0;
    for (int i = 0; i < NR; ++i) p.ws[i] = (char*)d_ws_all[(rank + i) % NR];
    build_tables(p);
    hipLaunchKernelGGL(gemm_ar_fused, dim3(256), dim3(512), 0, stream, p);
    HIP_CHECK(hipGetLastError());
}
```

```cpp
#include <hip/hip_runtime.h>
#include <rccl/rccl.h>
#include <cstdio>

#define HIP_CHECK(expr) do { hipError_t _e = (expr); if (_e != hipSuccess) { \
    fprintf(stderr, "HIP error %s at %s:%d\n", hipGetErrorString(_e), __FILE__, __LINE__); \
    return; } } while (0)

typedef unsigned u32;
typedef unsigned long long u64;
typedef u32 u32x4 __attribute__((ext_vector_type(4)));
typedef u32 u32x2 __attribute__((ext_vector_type(2)));
typedef unsigned short u16x2 __attribute__((ext_vector_type(2)));
typedef short bf16x8 __attribute__((ext_vector_type(8)));
typedef float f32x4 __attribute__((ext_vector_type(4)));
#define LAS __attribute__((address_space(3)))
#define GAS __attribute__((address_space(1)))
typedef __amdgpu_buffer_rsrc_t rsrc_t;

constexpr int NR = 8;
#define PULL_WG_PER_PEER 8
#define WINDOW 7
#define PUSHC 5
#define PUSHW 1
#define PACK_IN 112
#define AFFINE 1
#define LOOK 5
#define RHO 6
#define ENC_FIRST (16 - PUSHC)
#define ENC_DELAY 150
#define ENC_PACE 256
#define POLL_SLEEP 8
#define WB 12
constexpr int NPULL = 7 * PULL_WG_PER_PEER, NGEMM = 16 * WB, NPACK = 256 - NGEMM - NPULL;
constexpr int POWN = 16 - RHO, NP = POWN + 8 * RHO;
constexpr int NB = (NP + WB - 1) / WB, N0 = 16 + WB;
constexpr int NCIN = 64 + 4 * NP, NCY = 32 * POWN, NC = NCIN + NCY;
constexpr int NTASK = 16 * 36;
constexpr int NDEC = 6;
constexpr u64 DEADLINE = 100000000ull;
constexpr int PUSHB = (8 * PUSHC + PULL_WG_PER_PEER - 1) / PULL_WG_PER_PEER;
constexpr int NLAST = NP - (NB - 1) * WB;
static_assert(LOOK <= 7 && NGEMM >= 192 && NPACK < 32 && NPACK >= 4 && (PACK_IN + NCY) / NPACK < 126 && NB <= 8 && POWN <= WB && PUSHB <= NDEC, "roles");

constexpr int FIXB = 5120, CVAR = 8 * FIXB, CHDR = 32, CH = 66560;
constexpr size_t OFF_CALL = 0;
constexpr size_t OFF_TMO = 131072;
constexpr size_t OFF_TICK = OFF_TMO + 256;
constexpr size_t OFF_RDY = OFF_TICK + 16 * 128;
constexpr size_t OFF_FLAG = OFF_RDY + 64 * 128;
constexpr size_t OFF_TILE = OFF_FLAG + 8 * 256;
constexpr size_t OFF_GRAN = 196608;
constexpr size_t OFF_DBG = 524288;
constexpr size_t OFF_XF = 1ull << 20;
constexpr size_t OFF_WF = OFF_XF + (64ull << 20);
constexpr size_t OFF_SY = OFF_WF + NP * (2ull << 20);
constexpr size_t OFF_SI = OFF_SY + (size_t)NCY * CH;
constexpr size_t WS_NEED = OFF_SI + 2ull * NTASK * CH;

constexpr int A2B = 2064, A3B = 528, VARB = 912, WLDS = A2B + A3B + VARB, N2MAX = 2032, N3MAX = 496, UBITS = 2016, NPATCH = 31;
constexpr int CTL_OFF = 2048, CTL_WORDS = 128, WL_OFF = 2560, BIG_OFF = 30720, LDS_BYTES = BIG_OFF + 131072;
static_assert(WL_OFF + 8 * WLDS <= BIG_OFF && LDS_BYTES <= 163840, "LDS map");
constexpr int NF = 16, NV = 4, VSLOT = 8192, VRING = NF * FIXB;
constexpr int AUX_SYS = 17, AUX_DEV = 16, AUX_OUT = 18, AUX_NT = 2;

struct Params {
    int rank; int pad;
    const char* x; const char* w; char* out;
    char* ws[NR];
    u32 role[16];
    u32 task[NTASK / 2];
};

__device__ __forceinline__ u32 rfl(u32 x) { return (u32)__builtin_amdgcn_readfirstlane((int)x); }
__device__ __forceinline__ rsrc_t mk_rsrc(const void* p, u32 bytes) {
    const u64 a = (u64)p; const u64 u = ((u64)rfl((u32)(a >> 32)) << 32) | rfl((u32)a);
    return __builtin_amdgcn_make_buffer_rsrc((void*)u, (short)0, (int)rfl(bytes), 0x00020000);
}
__device__ __forceinline__ u32 vperm(u32 a, u32 b, u32 s) { return __builtin_amdgcn_perm(a, b, s); }
__device__ __forceinline__ u32 wave_incl_scan(u32 x) {
    x += (u32)__builtin_amdgcn_update_dpp(0, (int)x, 0x111, 0xf, 0xf, false);
    x += (u32)__builtin_amdgcn_update_dpp(0, (int)x, 0x112, 0xf, 0xf, false);
    x += (u32)__builtin_amdgcn_update_dpp(0, (int)x, 0x114, 0xf, 0xf, false);
    x += (u32)__builtin_amdgcn_update_dpp(0, (int)x, 0x118, 0xf, 0xf, false);
    x += (u32)__builtin_amdgcn_update_dpp(0, (int)x, 0x142, 0xa, 0xf, false);
    x += (u32)__builtin_amdgcn_update_dpp(0, (int)x, 0x143, 0xc, 0xf, false);
    return x;
}
__device__ __forceinline__ u32 pk_sub(u32 a, u32 b) { return __builtin_bit_cast(u32, (u16x2)(__builtin_bit_cast(u16x2, a) - __builtin_bit_cast(u16x2, b))); }
template <int AUX>
__device__ __forceinline__ void st128(u32x4 x, rsrc_t r, int voff, int soff) {
    __builtin_amdgcn_raw_buffer_store_b128(x, r, voff, soff, AUX);
    asm volatile("s_nop 1" :: "v"(x) : "memory");
}
#define WAVE_FENCE() __builtin_amdgcn_fence(__ATOMIC_ACQ_REL, "wavefront")

__device__ __forceinline__ void build_table(LAS u32x2* tab, u32 i, bool unpack) {
    u32 n = 0, dep = 0, cmp = 0x0c0c0c0cu;
#pragma unroll
    for (u32 t = 0; t < 4; ++t) {
        const u32 c = (i >> (2 * t)) & 3u;
        dep |= (c ? 4u + c : n) << (8 * t);
        if (!c) { cmp = (cmp & ~(0xffu << (8 * n))) | (t << (8 * n)); ++n; }
    }
    tab[i] = (u32x2){unpack ? dep : cmp, n};
}
__device__ __forceinline__ void build_table3(LAS u32x2* tab3, u32 i) {
    u64 r = 0; u32 n = 0, run = 0;
#pragma unroll
    for (u32 t = 0; t < 8; ++t) { if ((i >> t) & 1u) { r |= (u64)run << (8 * n); ++n; run = 0; } else ++run; }
    tab3[i] = (u32x2){(u32)r, (u32)(r >> 32)};
}
__device__ __forceinline__ void or3(LAS u32* q, u64 x, u32 sh) {
    const u64 y = x << sh;
    __hip_atomic_fetch_or(q, (u32)y, __ATOMIC_RELAXED, __HIP_MEMORY_SCOPE_WORKGROUP);
    __hip_atomic_fetch_or(q + 1, (u32)(y >> 32), __ATOMIC_RELAXED, __HIP_MEMORY_SCOPE_WORKGROUP);
    __hip_atomic_fetch_or(q + 2, (u32)(((x >> 32) << sh) >> 32), __ATOMIC_RELAXED, __HIP_MEMORY_SCOPE_WORKGROUP);
}
__device__ __forceinline__ u32 code_byte(u32 hi, u32 lo, u32 x) {
    const u32 f = vperm(hi, lo, x);
    return __builtin_amdgcn_udot4((f ^ (f >> 2)) & 0x03030303u, 0x40100401u, 0u, false);
}
template <bool PRED>
__device__ __forceinline__ void or_bytes(u32 pos, u32 x) {
    const u64 y = (u64)x << ((pos & 3u) * 8u);
    LAS u32* q = (LAS u32*)(size_t)(pos & ~3u);
    if (!PRED || x) {
        __hip_atomic_fetch_or(q, (u32)y, __ATOMIC_RELAXED, __HIP_MEMORY_SCOPE_WORKGROUP);
        __hip_atomic_fetch_or(q + 1, (u32)(y >> 32), __ATOMIC_RELAXED, __HIP_MEMORY_SCOPE_WORKGROUP);
    }
}
__device__ __forceinline__ u32 get_bytes(u32 pos) {
    const LAS u32* q = (const LAS u32*)(size_t)(pos & ~3u);
    return __builtin_amdgcn_alignbyte(q[1], q[0], pos);
}
__device__ __forceinline__ u32 la(const LAS void* p) { return (u32)(size_t)p; }
__device__ __forceinline__ u32 pack_level2(const LAS u32x2* tab, const LAS char* src, u32 n, LAS u32* end, LAS char* dst, int lane) {
    u32 m = 0;
    for (u32 i0 = 0; i0 < n; i0 += 1024u) {
        const u32 i = i0 + (u32)lane * 16u, hi = 0x02010000u, lo = 0x00030000u;
        const u32x4 x = *(const LAS u32x4*)(src + (i < n ? i : 0u));
        u32 cw = code_byte(hi, lo, x[0]) | (code_byte(hi, lo, x[1]) << 8) | (code_byte(hi, lo, x[2]) << 16) | (code_byte(hi, lo, x[3]) << 24);
        const u32 valid = i < n ? n - i : 0u;
        if (valid < 16u) cw |= 0x55555555u << (2u * valid);
        u32x2 e[4]; u32 c[5]; c[0] = 0;
#pragma unroll
        for (int j = 0; j < 4; ++j) { e[j] = tab[(cw >> (8 * j)) & 0xffu]; c[j + 1] = c[j] + e[j][1]; }
        const u32 incl = wave_incl_scan(c[4]), p0 = la(dst) + m + incl - c[4];
        if (i < n) *(end - 1 - (i >> 4)) = cw;
        m += (u32)__builtin_amdgcn_readlane((int)incl, 63);
        if (m <= (u32)N3MAX) {
#pragma unroll
            for (int j = 0; j < 4; ++j) or_bytes<true>(p0 + c[j], vperm(0u, x[j], e[j][0]));
        }
    }
    WAVE_FENCE();
    return m;
}
__device__ __forceinline__ void pack_level3(const LAS char* src, u32 n, LAS u32* ub, LAS u32* pt, u32 T, int lane, u32& bits, u32& np) {
    for (u32 i0 = 0; i0 < n; i0 += 256u) {
        const u32 i = i0 + (u32)lane * 4u;
        const u32 x = *(const LAS u32*)(src + (i < n ? i : 0u));
        u32 cum = 0, pf = 0; u64 m = 0;
#pragma unroll
        for (u32 t = 0; t < 4; ++t) {
            const u32 r = ((x >> (8 * t)) - 8u) & 0xffu;
            const bool valid = i + t < n, big = r > 15u;
            cum += valid && !big ? r : 0u;
            m |= (u64)(valid ? 1u : 0u) << cum;
            cum += valid ? 1u : 0u;
            pf |= valid && big ? 1u << t : 0u;
        }
        const u32 incl = wave_incl_scan(cum), p = bits + incl - cum;
        bits += (u32)__builtin_amdgcn_readlane((int)incl, 63);
        if (bits <= (u32)UBITS && cum) or3(ub + (p >> 5), m, p & 31u);
        if (__builtin_amdgcn_ballot_w64(pf != 0u)) {
#pragma unroll
            for (u32 t = 0; t < 4; ++t) {
                const u64 bm = __builtin_amdgcn_ballot_w64((pf >> t) & 1u);
                const u32 at = np + __builtin_amdgcn_mbcnt_hi((u32)(bm >> 32), __builtin_amdgcn_mbcnt_lo((u32)bm, 0u));
                if ((pf >> t) & 1u) pt[at] = ((i + t) << 8) | ((T - (x >> (8 * t))) & 0xffu);
                np += (u32)__builtin_popcountll(bm);
            }
        }
    }
    WAVE_FENCE();
}
__device__ __forceinline__ void encode_block(const u32 (&v)[8][4], rsrc_t ch, u32 b, LAS u32* alloc, const LAS u32x2* tab, LAS char* wl, int lane) {
#pragma unroll
    for (int j = 0; j < 4; ++j) if (j < 3 || lane < (WLDS - 3072) / 16) *(LAS u32x4*)(wl + j * 1024 + lane * 16) = (u32x4){0, 0, 0, 0};
    u32 s = 0;
#pragma unroll
    for (int g = 0; g < 8; ++g) s = __builtin_amdgcn_udot4(__builtin_amdgcn_alignbit(v[g][0], v[g][0], 31), 0x01000100u, s, false);
    u32 T = ((u32)__builtin_amdgcn_readlane((int)wave_incl_scan(s), 63) >> 10) + 5u;
    T = T > 255u ? 255u : (T < 23u ? 23u : T);
    const u32 tt = (T * 256u + 255u) * 0x10001u;
    const int fo = (int)(b * FIXB);
    u32 U[16], l1[4] = {0, 0, 0, 0}, c[17];
    u32x2 e[16];
#pragma unroll
    for (int q = 0; q < 4; ++q) {
        u32x4 sm;
#pragma unroll
        for (int j = 0; j < 4; ++j) {
            const int k = 4 * q + j;
            const u32 a0 = v[k >> 1][(k & 1) * 2], a1 = v[k >> 1][(k & 1) * 2 + 1];
            const u32 r0 = __builtin_amdgcn_alignbit(a0, a0, 31), r1 = __builtin_amdgcn_alignbit(a1, a1, 31);
            sm[j] = vperm(r1, r0, 0x06040200u);
            U[k] = vperm(pk_sub(tt, r1), pk_sub(tt, r0), 0x07050301u);
            const u32 cb = code_byte(0x00000302u, 0x01000000u, U[k]);
            l1[q] |= cb << (8 * j);
            e[k] = tab[cb];
        }
        st128<AUX_SYS>(sm, ch, lane * 16 + q * 1024, fo);
    }
    st128<AUX_SYS>((u32x4){l1[0], l1[1], l1[2], l1[3]}, ch, lane * 16 + 4096, fo);
    __builtin_amdgcn_sched_barrier(0);
    c[0] = 0;
#pragma unroll
    for (int k = 0; k < 16; ++k) c[k + 1] = c[k] + e[k][1];
    const u32 incl = wave_incl_scan(c[16]), p0 = la(wl) + incl - c[16];
    const u32 n2 = (u32)__builtin_amdgcn_readlane((int)incl, 63);
    bool raw = n2 > (u32)N2MAX;
    u32 d2 = 0, d3 = 0, np = 0;
    LAS u32* var = (LAS u32*)(wl + A2B + A3B);
    if (!raw) {
        WAVE_FENCE();
#pragma unroll
        for (int k = 0; k < 16; ++k) or_bytes<false>(p0 + c[k], vperm(0u, U[k], e[k][0]));
        WAVE_FENCE();
        d2 = (n2 + 15u) >> 4;
        const u32 n3 = pack_level2(tab, wl, n2, var + d2, wl + A2B, lane);
        raw = n3 > (u32)N3MAX;
        if (!raw) {
            u32 bits = 0;
            pack_level3(wl + A2B, n3, var + d2, (LAS u32*)wl, T, lane, bits, np);
            raw = bits > (u32)UBITS || np > (u32)NPATCH;
            d3 = (bits + 31u) >> 5;
            if (!raw && (u32)lane < np) var[d2 + d3 + (u32)lane] = ((const LAS u32*)wl)[lane];
            WAVE_FENCE();
        }
    }
    const u32 vd = d2 + d3 + np;
    u32 off = 0;
    if (lane == 0) off = __hip_atomic_fetch_add(alloc, raw ? 3072u : vd * 4u, __ATOMIC_RELAXED, __HIP_MEMORY_SCOPE_WORKGROUP);
    off = rfl(off) + CHDR;
    const int vo = CVAR + (int)off;
    if (lane == 0) __builtin_amdgcn_raw_buffer_store_b32(raw ? off << 17 : (T | (d3 << 8) | (np << 14) | ((off + d2 * 4u) << 17)), ch, 0, CVAR + (int)b * 4, AUX_SYS);
    if (raw) {
        asm volatile("s_waitcnt vmcnt(0)" ::: "memory");
#pragma unroll
        for (int g = 0; g < 8; ++g)
            st128<AUX_SYS>((u32x4){v[g][0], v[g][1], v[g][2], v[g][3]}, ch, lane * 16 + (g < 5 ? g : g - 5) * 1024, g < 5 ? fo : vo);
    } else {
        for (u32 o = (u32)lane; o < vd; o += 64u) __builtin_amdgcn_raw_buffer_store_b32(var[o], ch, (int)o * 4, vo, AUX_SYS);
    }
    WAVE_FENCE();
}
__device__ __forceinline__ bool chunk_done(LAS u32* cnt, int lane, u32& total) {
    u32 n = 0;
    if (lane == 0) n = __hip_atomic_fetch_add(cnt + 1, 1u, __ATOMIC_RELAXED, __HIP_MEMORY_SCOPE_WORKGROUP);
    if (rfl(n) != 7u) return false;
    total = rfl(*(volatile LAS u32*)cnt) + CHDR;
    return true;
}

struct Dest { rsrc_t r; u32 A, B, C, LH; };
__device__ __forceinline__ Dest dest_of(char* base, u32 kind, u32 b) {
    if (kind == 0u) return Dest{mk_rsrc(base + b * 4096u, 32768u + 4096u), 1024u, 32768u, 2048u, 256u};
    if (kind == 1u) return Dest{mk_rsrc(base + (b >> 2) * 32768u + (b & 1u) * 8192u + (b & 2u) * 512u, 16384u + 6144u + 1024u), 2048u, 4096u, 16384u, 256u};
    return Dest{mk_rsrc(base + (b & 3u) * (32u * 65536u) + (b >> 2) * 256u, 31u * 65536u + 256u), 65536u, 131072u, 16u * 65536u, 4u * 65536u};
}
template <int AUX>
__device__ __forceinline__ void store_group(const Dest& d, u32 loff, int g, u32x4 x) {
    st128<AUX>(x, d.r, (int)loff, (int)((g & 1 ? d.A : 0u) + (g & 2 ? d.B : 0u) + (g & 4 ? d.C : 0u)));
}

#define SCHED __builtin_amdgcn_sched_barrier(0)
__device__ __forceinline__ u32x2 get2(u32 pos) { const LAS u32* q = (const LAS u32*)(size_t)(pos & ~3u); return (u32x2){q[0], q[1]}; }
__device__ __forceinline__ void spread3(u32 w, u32 start, u32 n, u32x2 runs, u32 t8, u32 p) {
    if (n) {
        const u32 carry = (u32)__builtin_clz((w << 8) | start);
        const u64 x = ((u64)(t8 - runs[1]) << 32) | (t8 - runs[0] - carry);
        const u32 sh = 64u - 8u * n;
        or3((LAS u32*)(size_t)(p & ~3u), (x << sh) >> sh, (p & 3u) * 8u);
    }
}
template <class Out>
__device__ __forceinline__ void decode_block(const u32x4 (&d)[5], u32 hdr, const LAS u32* ub, const LAS u32x2* tab, const LAS u32x2* tab3, LAS char* wl, int lane, Out out) {
    const u32 T = hdr & 0xffu, d3 = (hdr >> 8) & 0x3fu, np = (hdr >> 14) & 0x1fu;
    auto ex = [&](u32 a, u32 b, u32 c) { return ((T - a) << 8) | ((T - b) << 16) | ((T - c) << 24); };
    const u32 t1 = ex(3, 4, 5), t2 = ex(6, 7, 2), t8 = (T - 8u) * 0x01010101u;
    const u32 w2 = *(ub - 1 - lane), p3i = la(ub) + (u32)lane - 3u;
    u32x2 e[16], e2[4], g[16];
    g[0] = get2(p3i);
    if (lane < A3B / 16) *(LAS u32x4*)(wl + A2B + lane * 16) = (u32x4){0, 0, 0, 0};
#pragma unroll
    for (int k = 0; k < 8; ++k) e[k] = tab[(d[4][k >> 2] >> (8 * (k & 3))) & 0xffu];
    SCHED;
    const u32 w3 = __builtin_amdgcn_alignbyte(g[0][1], g[0][0], p3i), x3 = (u32)lane < d3 * 4u ? w3 >> 24 : 0u;
    const u32x2 e3 = tab3[x3];
#pragma unroll
    for (int j = 0; j < 4; ++j) e2[j] = tab[(w2 >> (8 * j)) & 0xffu];
#pragma unroll
    for (int k = 8; k < 16; ++k) e[k] = tab[(d[4][k >> 2] >> (8 * (k & 3))) & 0xffu];
    SCHED;
    u32 c[17], f[5];
    c[0] = 0; f[0] = 0;
#pragma unroll
    for (int k = 0; k < 16; ++k) c[k + 1] = c[k] + e[k][1];
#pragma unroll
    for (int j = 0; j < 4; ++j) f[j + 1] = f[j] + e2[j][1];
    const u32 n3 = (u32)__builtin_popcount(x3);
    const u32 pk = c[16] | (f[4] << 11) | (n3 << 22);
    const u32 incl = wave_incl_scan(pk), ex0 = incl - pk, tot = (u32)__builtin_amdgcn_readlane((int)incl, 63);
    const u32 a3 = la(wl) + A2B, p1 = la(wl) + (ex0 & 0x7ffu), p2 = a3 + ((ex0 >> 11) & 0x1ffu);
    WAVE_FENCE();
    spread3(w3, lane < 3 ? 0x80000000u >> (8 * lane) : 0x80u, n3, e3, t8, a3 + ((ex0 >> 22) & 0x1ffu));
    for (u32 i0 = 64u, m = tot >> 22; i0 < d3 * 4u; i0 += 64u) {
        const u32 w = get_bytes(p3i + i0), x = i0 + (u32)lane < d3 * 4u ? w >> 24 : 0u, n = (u32)__builtin_popcount(x), in = wave_incl_scan(n);
        spread3(w, 0x80u, n, tab3[x], t8, a3 + ((m + in - n) & 0x1ffu));
        m += (u32)__builtin_amdgcn_readlane((int)in, 63);
    }
    WAVE_FENCE();
    if ((u32)lane < np) { const u32 pw = ub[d3 + (u32)lane]; *(LAS unsigned char*)(wl + A2B + ((pw >> 8) & 0x1ffu)) = (unsigned char)pw; }
    WAVE_FENCE();
#pragma unroll
    for (int j = 0; j < 4; ++j) { f[j] += p2; g[j] = get2(f[j]); }
    SCHED;
#pragma unroll
    for (int k = 0; k < 16; ++k) c[k] += p1;
    SCHED;
    u32x4 a2;
#pragma unroll
    for (int j = 0; j < 4; ++j) a2[j] = vperm(t2, __builtin_amdgcn_alignbyte(g[j][1], g[j][0], f[j]), e2[j][0]);
    *(LAS u32x4*)(wl + lane * 16) = a2;
    if ((tot & 0x7ffu) > 1024u) {
        const u32 cw = *(ub - 65 - lane);
        f[0] = 0;
#pragma unroll
        for (int j = 0; j < 4; ++j) { e2[j] = tab[(cw >> (8 * j)) & 0xffu]; f[j + 1] = f[j] + e2[j][1]; }
        const u32 p = a3 + (((tot >> 11) + wave_incl_scan(f[4]) - f[4]) & 0x1ffu);
#pragma unroll
        for (int j = 0; j < 4; ++j) a2[j] = vperm(t2, get_bytes(p + f[j]), e2[j][0]);
        *(LAS u32x4*)(wl + 1024 + lane * 16) = a2;
    }
    WAVE_FENCE();
#pragma unroll
    for (int k = 0; k < 16; ++k) g[k] = get2(c[k]);
    SCHED;
    u32x4 o;
#pragma unroll
    for (int k = 0; k < 16; ++k) {
        const u32 x = vperm(t1, __builtin_amdgcn_alignbyte(g[k][1], g[k][0], c[k]), e[k][0]);
        const u32 smk = d[k >> 2][k & 3];
        const u32 r0 = vperm(x, smk, 0x05010400u), r1 = vperm(x, smk, 0x07030602u);
        o[2 * (k & 1)] = __builtin_amdgcn_alignbit(r0, r0, 1);
        o[2 * (k & 1) + 1] = __builtin_amdgcn_alignbit(r1, r1, 1);
        if (k & 1) out(k >> 1, o);
    }
    WAVE_FENCE();
}

__device__ __forceinline__ char* frag_base(char* ws, bool isw, u32 s, u32 q, u32 rel, u32 call) {
    return ws + (isw ? OFF_WF : OFF_XF + (size_t)(call & 1u) * (32u << 20)) + (size_t)(s * 64u + (rel ? 6u + 14u * q + 2u * rel : 2u * q)) * 32768u;
}
__device__ __forceinline__ char* y_base(char* out, u32 i, u32 gp, u32 bj) { return out + (size_t)(i * 256u + bj * 128u) * 65536u + gp * 512u; }
__device__ __forceinline__ u32 gpanel(u32 r, u32 j) { return j < (u32)POWN ? r * 16u + j : ((j - POWN) & 7u) * 16u + POWN + ((j - POWN) >> 3); }
__device__ __forceinline__ u32 slot_in(bool isw, u32 s, u32 q, u32 dst) {
    return !isw ? s * 4u + q : (s < (u32)POWN ? 64u + (dst * POWN + s) * 4u + q : 64u + 28u * POWN + s * 4u + q);
}
__host__ __device__ __forceinline__ u32 batch_size(u32 bt) { return bt == 0u ? (u32)N0 : (bt == NB - 1u ? (u32)(NP - (NB - 1) * WB) : (u32)WB); }
struct InPos { bool isw; u32 s, q, bt; };
__host__ __device__ __forceinline__ InPos in_of(u32 c) {
    if (c < 4u * N0) { const u32 q = c / N0, pp = c - q * N0; return InPos{pp >= 16u, pp >= 16u ? pp - 16u : pp, q, 0u}; }
    const u32 bt = (c - 64u) / (4u * WB), r = c - 64u - bt * 4u * WB, q = r / batch_size(bt);
    return InPos{true, bt * WB + r - q * batch_size(bt), q, bt};
}

__device__ __forceinline__ void arrive(char* ws, u32 ci, u32 n, u32 call, int lane) {
    u32 old = 0;
    if (lane == 0) old = __hip_atomic_fetch_add((GAS u32*)(ws + OFF_RDY + ci * 128u), n, __ATOMIC_RELAXED, __HIP_MEMORY_SCOPE_SYSTEM);
    if (rfl(old) + n == call * ((ci & 7u) ? 56u : 32u) * batch_size(ci >> 3) && lane < 8)
        __hip_atomic_store((GAS u32*)(ws + OFF_FLAG + (u32)lane * 256u + ci * 4u), call, __ATOMIC_RELAXED, __HIP_MEMORY_SCOPE_SYSTEM);
}
__device__ __forceinline__ void load_x(const char* x, u32 s, u32 q, u32 b, int lane, u32x4 (&a)[8]) {
    const rsrc_t r = mk_rsrc(x + (size_t)(s * 256u + b * 32u) * 1024u + q * 256u, 31u * 1024u + 256u);
#pragma unroll
    for (int i = 0; i < 8; ++i) a[i] = __builtin_amdgcn_raw_buffer_load_b128(r, (lane & 15) * 1024 + (lane >> 4) * 16 + (i & 3) * 64, (i >> 2) * 16384, 0);
}

__device__ __forceinline__ bool give_up(GAS u32* tmo, u32 call, u64 t0) {
    if (rfl(__hip_atomic_load(tmo, __ATOMIC_RELAXED, __HIP_MEMORY_SCOPE_AGENT)) == call || __builtin_amdgcn_s_memrealtime() - t0 > DEADLINE) {
        __hip_atomic_store(tmo, call, __ATOMIC_RELAXED, __HIP_MEMORY_SCOPE_AGENT);
        return true;
    }
    return false;
}

#define STAMP_IT 0
#define STAMP_AT(i) do { } while (0)

__device__ __forceinline__ u32 lds_ld(LAS u32* p) { return rfl(*(volatile LAS u32*)p); }
__device__ __forceinline__ void dma_piece(rsrc_t r, u32 voff, u32 soff, u32 ldsaddr) {
    u32 keep;
    asm volatile("s_nop 4\n\ts_mov_b32 %0, m0\n\ts_mov_b32 m0, %1\n\ts_nop 0\n\tbuffer_load_dwordx4 %2, %3, %4 offen sc0 sc1 lds\n\ts_mov_b32 m0, %0"
                 : "=&s"(keep) : "s"(ldsaddr), "v"(voff), "s"(r), "s"(soff) : "memory");
}

#define DMA_P(j, off) "s_waitcnt vmcnt(%8)\n\tbuffer_load_dwordx4 %2, %3, " off " sc0 sc1 lds\n\t.if %9 == " #j "\n\tds_max_u32 %6, %7\n\t.endif\n\t"
template <int W, int JP>
__device__ __forceinline__ void dma_block(rsrc_t r, u32 voff, u32 soff, u32 ldsaddr, u32 at, u32 val) {
    u32 keep;
    asm volatile("s_nop 2\n\ts_mov_b32 %0, m0\n\ts_mov_b32 m0, %1\n\t"
                 DMA_P(0, "%4 offen") DMA_P(1, "%4 offen offset:1024") DMA_P(2, "%4 offen offset:2048") DMA_P(3, "%4 offen offset:3072")
                 "s_add_u32 m0, m0, 0x1000\n\t" DMA_P(4, "%5 offen") "s_mov_b32 m0, %0"
                 : "=&s"(keep) : "s"(ldsaddr), "v"(voff), "s"(r), "s"(soff), "s"(soff + 4096u), "v"(at), "v"(val), "n"(W), "n"(JP) : "memory", "scc");
}

__global__ void __launch_bounds__(512, 2) gemm_ar_fused(const Params p) {
    __shared__ __attribute__((aligned(16))) char smem[LDS_BYTES];
    LAS char* lds = (LAS char*)smem + BIG_OFF;
    LAS u32x2* tab = (LAS u32x2*)smem;
    LAS u32x2* tab3 = (LAS u32x2*)((LAS char*)smem + WL_OFF);
    const int tid = threadIdx.x, wid = (int)rfl((u32)tid >> 6), lane = tid & 63;
    const u32 bid = blockIdx.x;
    u32 rank = (u32)p.rank, rolew = p.role[((bid - NGEMM) >> 2) & 15u];
    char* ws = p.ws[0]; const char* px = p.x;
    asm volatile("" : "+s"(rank), "+s"(rolew), "+s"(ws), "+s"(px));
    LAS char* wl = (LAS char*)smem + WL_OFF + wid * WLDS;
    LAS u32* ctl = (LAS u32*)((LAS char*)smem + CTL_OFF);
    GAS u32* tmo = (GAS u32*)(ws + OFF_TMO);
    STAMP_AT(0);
    const u32 role = bid >= (u32)NGEMM ? (rolew >> ((bid & 3u) * 8u)) & 0xffu : 0u;
    const u32 pn = (role & 7u) ? (role >> 3) * PUSHB + (u32)wid - 2u : (bid + PUSHC) * 8u + (u32)wid;
    const bool early = (role & 7u) ? wid >= 2 && wid < 2 + PUSHB && pn < 8u * PUSHC : bid < 16u - PUSHC;
    u32x4 a[8];
    if (early) load_x(px, pn >> 3, 0u, pn & 7u, lane, a);
    SCHED;
    GAS u32* callp = (GAS u32*)(ws + OFF_CALL + (bid * 8u + wid) * 64u);
    const u32 callv = __hip_atomic_load(callp, __ATOMIC_RELAXED, __HIP_MEMORY_SCOPE_AGENT);
    char* wsl = ws;
    auto setup = [&](bool unpack) {
        u32 t = (u32)tid;
        asm volatile("" : "+v"(t));
        if (t < 256u) build_table(tab, t, unpack);
        else { if (t < 256u + CTL_WORDS) ctl[t - 256u] = 0u; if (unpack) build_table3(tab3, t - 256u); }
        if (!unpack) {
            ((LAS u32*)lds)[t] = 0u;
#pragma unroll
            for (u32 k = 1; k < 8u; ++k) wsl = (t & 63u) == k ? p.ws[k] : wsl;
        }
        asm volatile("s_waitcnt lgkmcnt(0)" ::: "memory"); __builtin_amdgcn_s_barrier(); asm volatile("" ::: "memory");
    };

    if (role & 7u) {
        const u32 rel = role & 7u, pjj = role >> 3, src = (rank + rel) & 7u;
        char* pws = p.ws[rel];
        setup(true);
        if (early) {
            const u32 call = rfl(callv) + 1u;
            const Dest dst = dest_of(frag_base(pws, false, pn >> 3, 0u, 8u - rel, call), 0u, pn & 7u);
#pragma unroll
            for (int g = 0; g < 8; ++g) asm volatile("" : "+v"(a[g]));
#pragma unroll
            for (int g = 0; g < 8; ++g) { store_group<AUX_SYS>(dst, (u32)lane * 16u, g, a[g]); asm volatile("s_waitcnt vmcnt(%0)" :: "n"(PUSHW - 1) : "memory"); }
            STAMP_AT(12);
            if (lane == 0) __hip_atomic_store(callp, call, __ATOMIC_RELAXED, __HIP_MEMORY_SCOPE_AGENT);
            asm volatile("s_waitcnt vmcnt(0)" ::: "memory");
            STAMP_AT(13);
            arrive(pws, 1u, 1u, call, lane);
        }
        LAS u32* vready = ctl; LAS u32* endk = ctl + 1; LAS u32* landed = ctl + 2; LAS u32* prog = ctl + 5;
        LAS u32* vdone = ctl + 8; LAS u32* sdone = ctl + 16; LAS u32* ffree = ctl + 32;
        LAS u32* q = ctl + 64;
        const u32 l0 = la(lds), voff = (u32)lane * 16u;
        if (wid == 0) {
            const u32 call = rfl(callv) + 1u;
            if (lane == 0) __hip_atomic_store(callp, call, __ATOMIC_RELAXED, __HIP_MEMORY_SCOPE_AGENT);
            GAS u64* gran = (GAS u64*)(ws + OFF_GRAN) + (size_t)((call & 1u) * 8u + src) * NC;
            for (u32 k = 0;; ++k) {
                u32 c = PUSHC + pjj;
                if (k) {
                    while (lds_ld(prog) < k) __builtin_amdgcn_s_sleep(2);
                    u32 tk = 0;
                    if (lane == 0) tk = __hip_atomic_fetch_add((GAS u32*)(ws + OFF_TICK + src * 128u), 1u, __ATOMIC_RELAXED, __HIP_MEMORY_SCOPE_AGENT);
                    c = rfl(tk) - (call - 1u) * (NC - PUSHC) + PULL_WG_PER_PEER + PUSHC;
                }
                u32 fl = 2u, vb = 0, kc = 2u;
                const char* from = pws; char* to = ws;
                if (c < (u32)NC) {
                    if (c >= (u32)NCIN) {
                        const u32 yc = c - NCIN;
                        from = pws + OFF_SY + (size_t)yc * CH;
                        to = y_base(p.out, (yc >> 1) & 15u, src * 16u + (yc >> 5), yc & 1u);
                    } else {
                        const InPos ip = in_of(c);
                        from = pws + OFF_SI + (size_t)((call & 1u) * NTASK + slot_in(ip.isw, ip.s, ip.q, rank)) * CH;
                        to = frag_base(ws, ip.isw, ip.s, ip.q, rel, call);
                        kc = (ip.isw ? 1u : 0u) | ((ip.bt * 8u + ip.q + 1u) << 2);
                    }
                    const u64 t0 = __builtin_amdgcn_s_memrealtime();
                    for (u32 spin = 0;; ++spin) {
                        const u64 g = __hip_atomic_load(gran + c, __ATOMIC_RELAXED, __HIP_MEMORY_SCOPE_SYSTEM);
                        if (rfl((u32)g) == call) { fl = 0u; vb = rfl((u32)(g >> 32)); break; }
                        if ((spin & 15u) == 15u && give_up(tmo, call, t0)) { fl = 1u; break; }
                        __builtin_amdgcn_s_sleep(1);
                    }
                    if (k == 0) STAMP_AT(1);
                }
                while (lds_ld(vdone + (k & (NV - 1))) < 8u * (k / NV)) __builtin_amdgcn_s_sleep(2);
                if (lane == 0) {
                    *(volatile LAS u32x4*)(q + (k & (NV - 1)) * 8u + 4u) = (u32x4){(u32)(u64)to, (u32)((u64)to >> 32), kc, call};
                    *(volatile LAS u32x4*)(q + (k & (NV - 1)) * 8u) = (u32x4){(u32)(u64)from, (u32)((u64)from >> 32), fl | ((k + 1u) << 2), vb};
                }
                if (fl & 2u) { if (lane == 0) *(volatile LAS u32*)endk = k + 1u; break; }
                if (!(fl & 1u)) {
                    const u32 vl = vb > (u32)VSLOT ? (u32)CHDR : (vb + 15u) & ~15u;
                    const rsrc_t r = mk_rsrc(from, CVAR + vl);
                    for (u32 o = 0; o < vl; o += 1024u) { dma_piece(r, voff + CVAR + o, 0u, rfl(l0 + VRING + (k & (NV - 1)) * VSLOT + o)); if (k) asm volatile("s_waitcnt vmcnt(0)" ::: "memory"); }
                    asm volatile("s_waitcnt vmcnt(0)" ::: "memory");
                }
                if (lane == 0) *(volatile LAS u32*)vready = k + 1u;
            }
        } else if (wid == 1) {
            __builtin_amdgcn_s_setprio(3);
            constexpr u32 LAG = (WINDOW + 4) / 5, JP = (WINDOW + 4) % 5;
            const u32 la_landed = lane ? la(wl) + (u32)lane * 4u : la(landed);
            u32 n = 0;
            auto drain = [&]() { asm volatile("s_waitcnt vmcnt(0)" ::: "memory"); if (lane == 0) __hip_atomic_fetch_max(landed, n, __ATOMIC_RELAXED, __HIP_MEMORY_SCOPE_WORKGROUP); };
            u32x4 e = (u32x4){0, 0, 0, 0};
            u32 fr = 0;
            for (u32 k = 0;; ++k) {
                if (rfl(e[2]) >> 2 != k + 1u) {
                    e = *(volatile LAS u32x4*)(q + (k & (NV - 1)) * 8u);
                    if (rfl(e[2]) >> 2 != k + 1u) {
                        drain();
                        do { __builtin_amdgcn_s_sleep(1); e = *(volatile LAS u32x4*)(q + (k & (NV - 1)) * 8u); } while (rfl(e[2]) >> 2 != k + 1u);
                    }
                }
                const u32 fl = rfl(e[2]);
                if (fl & 2u) break;
                const rsrc_t r = mk_rsrc((const void*)(((u64)e[1] << 32) | e[0]), (fl & 1u) ? 0u : (u32)CVAR);
#pragma unroll
                for (u32 b = 0; b < 8u; ++b, ++n) {
                    const u32 sl = n & (NF - 1), m = n + 1u;
                    if (rfl(fr) + NF <= n && lds_ld(ffree + sl) + NF <= n) { drain(); while (lds_ld(ffree + sl) + NF <= n) __builtin_amdgcn_s_sleep(1); }
                    fr = *(volatile LAS u32*)(ffree + (m & (NF - 1)));
                    if (b == 7u) e = *(volatile LAS u32x4*)(q + ((k + 1u) & (NV - 1)) * 8u);
                    if (b == 7u - LOOK && lane == 0) *(volatile LAS u32*)prog = k + 1u;
                    dma_block<WINDOW - 1, JP>(r, voff, b * FIXB, rfl(l0 + sl * FIXB), la_landed, m > LAG ? m - LAG : 0u);
                }
            }
            drain();
            STAMP_AT(2);
        } else {
#define SEG(i)
            u32 owed = 0xffffffffu, call = 0;
            auto settle = [&]() {
                if (owed != 0xffffffffu) {
                    asm volatile("s_waitcnt vmcnt(0)" ::: "memory");
                    u32 old = 0;
                    if (lane == 0) old = __hip_atomic_fetch_add(sdone + (owed & 7u), 1u, __ATOMIC_RELAXED, __HIP_MEMORY_SCOPE_WORKGROUP);
                    if ((rfl(old) & 7u) == 7u) arrive(ws, owed >> 3, 8u, call, lane);
                    owed = 0xffffffffu;
                }
            };
            for (u32 n = (u32)wid - 2u;; n += NDEC) {
                const u32 k = n >> 3, b = n & 7u, sl = n & (NF - 1);
                bool end = false;
                for (;;) {
                    const u32x4 st = *(volatile LAS u32x4*)ctl;
                    if (rfl(st[0]) > k && rfl(st[2]) > n) break;
                    if (rfl(st[1]) - 1u <= k) { end = true; break; }
                    settle();
                    __builtin_amdgcn_s_sleep(1);
                }
                if (end) break;
                asm volatile("" ::: "memory");
                u32x4 d[5];
                const LAS char* vs = lds + VRING + (k & (NV - 1)) * VSLOT;
                const u32 h = *(const LAS u32*)(vs + b * 4u);
                const u32x4 e0 = *(const LAS u32x4*)(q + (k & (NV - 1)) * 8u), e1 = *(const LAS u32x4*)(q + (k & (NV - 1)) * 8u + 4u);
                d[4] = *(const LAS u32x4*)(lds + sl * FIXB + 4096 + lane * 16);
#pragma unroll
                for (int j = 0; j < 4; ++j) d[j] = *(const LAS u32x4*)(lds + sl * FIXB + j * 1024 + lane * 16);
                asm volatile("s_waitcnt lgkmcnt(0)" ::: "memory");
                if (lane == 0) *(volatile LAS u32*)(ffree + sl) = n + 1u;
                asm volatile("" : "+v"(d[0]), "+v"(d[1]), "+v"(d[2]), "+v"(d[3]), "+v"(d[4]) :: "memory");
                const u32 hdr = rfl(h), off = (hdr >> 19) * 4u, kc = rfl(e1[2]);
                SEG(0);
                call = rfl(e1[3]);
                const bool far = rfl(e0[3]) > (u32)VSLOT;
                const LAS u32* var = (const LAS u32*)(vs + (off <= (u32)VSLOT ? off : 0u));
                u32x4 o[8];
                if (rfl(e0[2]) & 1u) {
#pragma unroll
                    for (int g = 0; g < 8; ++g) o[g] = (u32x4){0x7fc07fc0u, 0x7fc07fc0u, 0x7fc07fc0u, 0x7fc07fc0u};
                } else {
                    if (far) {
                        const rsrc_t r = mk_rsrc((const void*)(((u64)e0[1] << 32) | e0[0]), CH);
#pragma unroll
                        for (int j = 0; j < 3; ++j) o[5 + j] = __builtin_amdgcn_raw_buffer_load_b128(r, (int)(CVAR + off) - ((hdr & 0xffu) ? 512 : 0) + j * 1024 + lane * 16, 0, AUX_SYS);
                        if (lane < VARB / 16) *(LAS u32x4*)(wl + A2B + A3B + lane * 16) = o[5];
                        WAVE_FENCE();
                        var = (const LAS u32*)(wl + A2B + A3B + 512);
                    }
                    if (hdr & 0xffu) decode_block(d, hdr, var, tab, tab3, wl, lane, [&](int g, u32x4 x) { o[g] = x; });
                    else {
#pragma unroll
                        for (int g = 0; g < 5; ++g) o[g] = d[g];
                        if (!far) {
#pragma unroll
                            for (int j = 0; j < 3; ++j)
#pragma unroll
                                for (int i = 0; i < 4; ++i) o[5 + j][i] = *(const volatile LAS u32*)(var + j * 256 + lane * 4 + i);
                        }
                    }
                }
                const Dest dst = dest_of((char*)(((u64)e1[1] << 32) | e1[0]), kc & 3u, b);
                asm volatile("s_waitcnt lgkmcnt(0)" ::: "memory");
                if (lane == 0) __hip_atomic_fetch_add(vdone + (k & (NV - 1)), 1u, __ATOMIC_RELAXED, __HIP_MEMORY_SCOPE_WORKGROUP);
                SEG(1);
                settle();
                SEG(3);
                const u32 loff = (u32)(lane >> 4) * dst.LH + (u32)(lane & 15) * 16u;
                if ((kc & 3u) == 2u) {
#pragma unroll
                    for (int g = 0; g < 8; ++g) store_group<AUX_OUT>(dst, loff, g, o[g]);
                } else {
#pragma unroll
                    for (int g = 0; g < 8; ++g) store_group<AUX_DEV>(dst, loff, g, o[g]);
                    owed = (k & 7u) | ((kc >> 2) << 3);
                }
                SEG(2);
            }
            settle();
            STAMP_AT(3);
        }
        return;
    }

    {
        const bool packer = bid >= (u32)NGEMM;
        const u64 tb = __builtin_amdgcn_s_memrealtime();
        const u32 tend = packer ? (u32)NTASK : (u32)(NTASK - PACK_IN), b = (u32)wid;
        u32 it = 0;
        for (u32 task = packer ? (role >> 3) + NTASK - PACK_IN : bid; task < tend; task += packer ? (u32)NPACK : (u32)NGEMM, ++it) {
            if (task >= (u32)ENC_FIRST) while (__builtin_amdgcn_s_memrealtime() - tb < (u64)(ENC_DELAY + (packer ? 0u : task * ENC_PACE / 64u))) __builtin_amdgcn_s_sleep(4);
            if (it == STAMP_IT) STAMP_AT(12);
            const u32 te = task < 16u - PUSHC ? (task + PUSHC) << 4 : (p.task[task >> 1] >> ((task & 1u) * 16u)) & 0xffffu, c = te >> 4, k = te & 15u;
            const InPos ip = in_of(c);
            const u32 dd = (rank + k) & 7u;
            if (ip.isw) {
                const rsrc_t r = mk_rsrc(p.w + (size_t)(ip.q * 128u + (b >> 1) * 32u) * 65536u + (gpanel(dd, ip.s) * 256u + (b & 1u) * 128u) * 2u, 31u * 65536u + 256u);
#pragma unroll
                for (int i = 0; i < 8; ++i) a[i] = __builtin_amdgcn_raw_buffer_load_b128(r, (lane >> 4) * 8 * 65536 + (lane & 15) * 16, i * 65536, AUX_NT);
            } else if (task >= 16u - PUSHC) load_x(px, ip.s, ip.q, b, lane, a);
            if (it == 0) setup(false);
            u32 v[8][4];
            if (ip.isw) {
#pragma unroll
                for (int j = 0; j < 8; ++j)
#pragma unroll
                    for (int e = 0; e < 4; ++e) v[j][e] = vperm(a[2 * e + 1][j >> 1], a[2 * e][j >> 1], (j & 1) ? 0x07060302u : 0x05040100u);
            } else {
#pragma unroll
                for (int j = 0; j < 8; ++j)
#pragma unroll
                    for (int e = 0; e < 4; ++e) v[j][e] = a[j][e];
            }
            const bool own = (k & 7u) == 0u, send = k < 8u;
            const u32 call = rfl(callv) + 1u;
            LAS u32* cnt = (LAS u32*)lds + 4u * it;
            u32 total;
            if (send) {
                encode_block(v, mk_rsrc(ws + OFF_SI + (size_t)((call & 1u) * NTASK + slot_in(ip.isw, ip.s, ip.q, dd)) * CH, CH), b, cnt, tab, wl, lane);
                if (it == STAMP_IT) STAMP_AT(14);
                asm volatile("s_waitcnt vmcnt(0)" ::: "memory");
                if (chunk_done(cnt, lane, total) && (k ? (u32)lane == k : (u32)lane - 1u < 7u))
                    __hip_atomic_store((GAS u64*)(wsl + OFF_GRAN) + (size_t)((call & 1u) * 8u + rank) * NC + c,
                                       ((u64)total << 32) | call, __ATOMIC_RELAXED, __HIP_MEMORY_SCOPE_SYSTEM);
            }
            if (own) {
                const Dest dst = dest_of(frag_base(ws, ip.isw, ip.s, ip.q, 0u, call), ip.isw ? 1u : 0u, b);
#pragma unroll
                for (int g = 0; g < 8; ++g) store_group<AUX_DEV>(dst, (u32)lane * 16u, g, (u32x4){v[g][0], v[g][1], v[g][2], v[g][3]});
                asm volatile("s_waitcnt vmcnt(0)" ::: "memory");
                if (chunk_done(cnt + 2, lane, total)) arrive(ws, ip.bt * 8u, 8u, call, lane);
            }
            if (it == STAMP_IT) STAMP_AT(1);
        }
        if (lane == 0) __hip_atomic_store(callp, callv + 1u, __ATOMIC_RELAXED, __HIP_MEMORY_SCOPE_AGENT);
        if (packer) {
            STAMP_AT(4);
            if (wid >= 4) __builtin_amdgcn_s_setprio(1);
            const u32 call = rfl(callv) + 1u;
            GAS u32* tf = (GAS u32*)(ws + OFF_TILE + b * 4u);
            u32 seen = 0, nanv = 0;
            auto fetch = [&](u32 yc) {
                if (rfl(seen) != call) {
                    const u64 t0 = __builtin_amdgcn_s_memrealtime();
                    while (rfl(__hip_atomic_load(tf + (yc >> 1) * 32u, __ATOMIC_RELAXED, __HIP_MEMORY_SCOPE_AGENT)) != call) {
                        if (give_up(tmo, call, t0)) { nanv = 0x7fc07fc0u; break; }
                        __builtin_amdgcn_s_sleep(POLL_SLEEP);
                    }
                }
                asm volatile("" ::: "memory");
                const Dest sy = dest_of(y_base(p.out, (yc >> 1) & 15u, rank * 16u + (yc >> 5), yc & 1u), 2u, b);
#pragma unroll
                for (int i = 0; i < 8; ++i) a[i] = __builtin_amdgcn_raw_buffer_load_b128(sy.r, (lane >> 4) * 4 * 65536 + (lane & 15) * 16, ((i & 3) + (i >> 2) * 16) * 65536, AUX_SYS);
                seen = __hip_atomic_load(tf + ((yc + NPACK) >> 1) * 32u, __ATOMIC_RELAXED, __HIP_MEMORY_SCOPE_AGENT);
            };
            auto publish = [&](u32 yc) {
                u32 total;
                if (chunk_done((LAS u32*)lds + 4u * it, lane, total) && (u32)lane - 1u < 7u)
                    __hip_atomic_store((GAS u64*)(wsl + OFF_GRAN) + (size_t)((call & 1u) * 8u + rank) * NC + NCIN + yc,
                                       ((u64)total << 32) | call, __ATOMIC_RELAXED, __HIP_MEMORY_SCOPE_SYSTEM);
                ++it;
            };
            u32 yc = role >> 3;
            fetch(yc);
            STAMP_AT(3);
            for (;; yc += NPACK) {
                u32 v[8][4];
#pragma unroll
                for (int j = 0; j < 8; ++j)
#pragma unroll
                    for (int e = 0; e < 4; ++e) v[j][e] = a[j][e] | nanv;
                if (yc + NPACK < (u32)NCY) fetch(yc + NPACK);
                encode_block(v, mk_rsrc(ws + OFF_SY + (size_t)yc * CH, CH), b, (LAS u32*)lds + 4u * it, tab, wl, lane);
                asm volatile("s_waitcnt vmcnt(0)" ::: "memory");
                publish(yc);
                if (yc + NPACK >= (u32)NCY) break;
            }
            STAMP_AT(2);
            return;
        }
        STAMP_AT(2);
    }

    const u32 call = rfl(callv) + 1u;
    const int wr = wid >> 2, wc = wid & 3;
    constexpr int HTB = 16384;
#define SA(b, h) (((b) * 2 + (h)) * HTB)
#define SB(b, h) ((4 + (b) * 2 + (h)) * HTB)
#define STAGE(bufoff, gbase) do { _Pragma("unroll") for (int _i = 0; _i < 2; ++_i) \
        __builtin_amdgcn_global_load_lds((const GAS u32*)((gbase) + voff + _i * 8192), (LAS u32*)(lds + (bufoff) + ldsw + _i * 8192), 16, 0, AUX_SYS); } while (0)
#define LDA(dst, b, h) do { _Pragma("unroll") for (int m = 0; m < 4; ++m) _Pragma("unroll") for (int k = 0; k < 2; ++k) dst[m][k] = *(const LAS bf16x8*)(lds + SA(b, h) + aoff + m * 2048 + k * 1024); } while (0)
#define LDB(dst, b, h) do { _Pragma("unroll") for (int n = 0; n < 2; ++n) _Pragma("unroll") for (int k = 0; k < 2; ++k) dst[n][k] = *(const LAS bf16x8*)(lds + SB(b, h) + boff + n * 2048 + k * 1024); } while (0)
#define MMA(ai, bj, At, Bt) do { _Pragma("unroll") for (int _x = 0; _x < 16; ++_x) { const int m = (_x >> 1) & 3, n = (_x ^ (_x >> 1)) & 1, k = _x >> 3; \
        acc[ai][bj][m][n] = __builtin_amdgcn_mfma_f32_16x16x32_bf16(Bt[n][k], At[m][k], acc[ai][bj][m][n], 0, 0, 0); SCHED; } } while (0)
#define MMA2(ai) do { __builtin_amdgcn_s_setprio(1); MMA(ai, 0, At, B0); MMA(ai, 1, At, B1); __builtin_amdgcn_s_setprio(0); } while (0)
#define WAIT_V(n) asm volatile("s_waitcnt vmcnt(" #n ")" ::: "memory")
#define WAIT_L(n) asm volatile("s_waitcnt lgkmcnt(" #n ")" ::: "memory")
#define BAR __builtin_amdgcn_s_barrier()
    const u32 voff = (u32)tid * 16u, ldsw = (u32)wid * 1024u;
    const int aoff = wr * 8192 + lane * 16, boff = wc * 4096 + lane * 16;
    const u32 idx = bid >> 3, hh = (bid >> 2) & 1u;
    u32 ti = (bid & 3u) * 4u + (idx < 4u * (WB / 2) ? idx & 3u : 2u * hh + (idx & 1u)), tj = idx < 4u * (WB / 2) ? 2u * (idx >> 2) + hh : WB - 1u;
    constexpr size_t kstep = 32768, hstep = 16384;
    const GAS char* xf = (const GAS char*)frag_base(ws, false, 0u, 0u, 0u, call);
    const GAS char* wf = (const GAS char*)(ws + OFF_WF);
    GAS u32* flags = (GAS u32*)(ws + OFF_FLAG + (bid & 7u) * 256u) + ((u32)lane & 7u);
    auto arrived = [&](u32 bt) { return (u32)__builtin_ctz(((u32)__builtin_amdgcn_ballot_w64(__hip_atomic_load(flags + bt * 8u, __ATOMIC_RELAXED, __HIP_MEMORY_SCOPE_SYSTEM) != call) & 31u) | 32u); };
#define STAGE_FIRST(pA, pB) do { STAGE(SB(0, 0), pB); STAGE(SA(0, 0), pA); STAGE(SB(0, 1), (pB) + hstep); STAGE(SA(0, 1), (pA) + hstep); \
        STAGE(SB(1, 0), (pB) + kstep); STAGE(SA(1, 0), (pA) + kstep); STAGE(SB(1, 1), (pB) + hstep + kstep); } while (0)
    u32 nwait = 0, bad = 0, staged = 0;
    for (u32 round = 0; tj < (u32)NP; ++round) {
        const GAS char* cA = wf + (size_t)tj * (2u << 20);
        const GAS char* cB = xf + (size_t)ti * (2u << 20);
        u32 ni = ti, nj = tj + WB;
        f32x4 acc[2][2][4][2];
#pragma unroll
        for (int a = 0; a < 2; ++a)
#pragma unroll
            for (int b = 0; b < 2; ++b)
#pragma unroll
                for (int m = 0; m < 4; ++m)
#pragma unroll
                    for (int n = 0; n < 2; ++n) acc[a][b][m][n] = (f32x4){0.f, 0.f, 0.f, 0.f};
        bf16x8 At[4][2], B0[2][2], B1[2][2];
        for (u32 q0 = 0; q0 < 5u;) {
            const u32 k0 = q0 ? q0 * 14u - 6u : 0u;
            if (!staged) {
                if (wid == 0) {
                    const u64 t0 = __builtin_amdgcn_s_memrealtime();
                    u32 q1;
                    for (;;) {
                        q1 = arrived(round);
                        if (q1 > q0) break;
                        if (give_up(tmo, call, t0)) { q1 = 5u | 8u; break; }
                        __builtin_amdgcn_s_sleep(POLL_SLEEP);
                    }
                    if (lane == 0) *(volatile LAS u32*)(ctl + (nwait & 1u)) = q1;
                    asm volatile("s_waitcnt lgkmcnt(0)" ::: "memory");
                }
                asm volatile("" ::: "memory"); BAR; asm volatile("" ::: "memory");
                const u32 te = lds_ld(ctl + (nwait & 1u));
                ++nwait; bad |= te >> 3;
                staged = te & 7u;
                STAGE_FIRST(cA + (size_t)k0 * kstep, cB + (size_t)k0 * kstep);
            }
            if (k0 == 0u && round < 4u) STAMP_AT(3 + round * 2);
            q0 = staged & 7u;
            const u32 k1 = q0 * 14u - 6u;
            if (wr == 1) BAR;
            bool lax = (staged >> 3) != 0u;
            if (lax) WAIT_V(22); else WAIT_V(6);
            BAR; BAR;
            for (u32 t = k0; t < k1; t += 2u) {
                const u32 t2 = t + 2u < k1 ? t + 2u : t;
                const GAS char* a1 = cA + (size_t)(t + 1u) * kstep;
                const GAS char* a2 = cA + (size_t)t2 * kstep; const GAS char* b2 = cB + (size_t)t2 * kstep;
                const GAS char* a3 = a2 + kstep; const GAS char* b3 = b2 + kstep;
                LDB(B0, 0, 0); LDB(B1, 0, 1); LDA(At, 0, 0); STAGE(SA(1, 1), a1 + hstep);
                if (lax) WAIT_V(24); else WAIT_V(8);
                WAIT_L(0); BAR; SCHED; MMA2(0); BAR; SCHED;
                LDA(At, 0, 1); STAGE(SB(0, 0), b2); STAGE(SB(0, 1), b2 + hstep); STAGE(SA(0, 0), a2);
                if (lax) WAIT_V(24); else WAIT_V(8);
                lax = false;
                WAIT_L(0); BAR; SCHED; MMA2(1); BAR; SCHED;
                LDB(B0, 1, 0); LDB(B1, 1, 1); LDA(At, 1, 0); STAGE(SA(0, 1), a2 + hstep);
                WAIT_V(8); WAIT_L(0); BAR; SCHED; MMA2(0); BAR; SCHED;
                LDA(At, 1, 1); STAGE(SB(1, 0), b3); STAGE(SB(1, 1), b3 + hstep); STAGE(SA(1, 0), a3);
                WAIT_V(8); WAIT_L(0); BAR; SCHED; MMA2(1); BAR; SCHED;
            }
            const bool last = q0 == 5u, draw = NLAST < WB && last && round == NB - 2u;
            if (wid == 0) {
                u32 tk = 0;
                if (draw && lane == 0) tk = __hip_atomic_fetch_add((GAS u32*)(ws + OFF_TICK + 8u * 128u), 1u, __ATOMIC_RELAXED, __HIP_MEMORY_SCOPE_AGENT) - (call - 1u) * NGEMM;
                const u32 q1 = arrived(last ? round + 1u : round);
                if (lane == 0) *(volatile LAS u32*)(ctl + (nwait & 1u)) = q1 | (tk << 8);
                asm volatile("s_waitcnt lgkmcnt(0)" ::: "memory");
            }
            WAIT_V(0);
            if (wr == 0) BAR;
            BAR;
            asm volatile("" ::: "memory");
            staged = lds_ld(ctl + (nwait & 1u));
            ++nwait;
            if (draw) {
                const u32 tk = staged >> 8, gr = tk / (4u * NLAST), r = tk - gr * 4u * NLAST;
                ni = (gr * 4u + (r & 3u)) & 15u; nj = gr < 4u ? (NB - 1u) * WB + (r >> 2) : (u32)NP;
            }
            staged &= 7u;
            if (staged <= (last ? 0u : q0) || (last && nj >= (u32)NP)) staged = 0u;
            if (staged) STAGE_FIRST(last ? wf + (size_t)nj * (2u << 20) : cA + (size_t)k1 * kstep, last ? xf + (size_t)ni * (2u << 20) : cB + (size_t)k1 * kstep);
        }
        if (round < 4u) STAMP_AT(4 + round * 2);
#pragma unroll
        for (int bj = 0; bj < 2; ++bj) {
            const Dest dst = dest_of(y_base(p.out, ti, gpanel(rank, tj), bj), 2u, (u32)wid);
            const u32 loff = (u32)(lane >> 4) * dst.LH + (u32)(lane & 15) * 16u;
            u32x4 v[8];
#pragma unroll
            for (int g = 0; g < 8; ++g)
#pragma unroll
                for (int q = 0; q < 4; ++q) {
                    typedef float f32x2 __attribute__((ext_vector_type(2)));
                    typedef __bf16 bf16x2 __attribute__((ext_vector_type(2)));
                    const f32x2 f = {acc[q >> 1][bj][(q & 1) * 2][g >> 2][g & 3], acc[q >> 1][bj][(q & 1) * 2 + 1][g >> 2][g & 3]};
                    v[g][q] = __builtin_bit_cast(u32, __builtin_convertvector(f, bf16x2));
                }
            if (bad) {
#pragma unroll
                for (int g = 0; g < 8; ++g) v[g] = (u32x4){0x7fc07fc0u, 0x7fc07fc0u, 0x7fc07fc0u, 0x7fc07fc0u};
            }
            if (tj < (u32)POWN) {
#pragma unroll
                for (int g = 0; g < 8; ++g) store_group<AUX_DEV>(dst, loff, g, v[g]);
            } else if (staged) {
#pragma unroll
                for (int g = 0; g < 8; ++g) store_group<AUX_NT>(dst, loff, g, v[g]);
            } else {
#pragma unroll
                for (int g = 0; g < 8; ++g) store_group<AUX_OUT>(dst, loff, g, v[g]);
            }
        }
        if (tj < (u32)POWN) {
            asm volatile("s_waitcnt vmcnt(0)" ::: "memory");
            if (lane == 0) __hip_atomic_store((GAS u32*)(ws + OFF_TILE + (tj * 16u + ti) * 128u + (u32)wid * 4u), call, __ATOMIC_RELAXED, __HIP_MEMORY_SCOPE_AGENT);
        } else if (staged) staged |= 8u;
        ti = ni; tj = nj;
    }
    STAMP_AT(15);
}

static void build_tables(Params& p) {
    unsigned char* role = (unsigned char*)p.role;
    for (int i = 0; i < 64; ++i) role[i] = (unsigned char)(i < NPULL ? (1 + i % 7) | (i / 7) << 3 : (i - NPULL) << 3);
    {
        constexpr int SL = (256 - NGEMM) / 2;
        static_assert(4 * PULL_WG_PER_PEER <= SL, "roles");
        int cnt[2] = {0, 0}, np = 0;
        for (int j = 0; j < PULL_WG_PER_PEER; ++j)
            for (int rel = 1; rel < 8; ++rel) {
                const int h = (((0xb2 >> p.rank) ^ (0xb2 >> ((p.rank + rel) & 7))) & 1), e = cnt[h]++;
                role[(e >> 2) * 8 + h * 4 + ((e + (h ? e >> 2 : 0)) & 3)] = (unsigned char)(rel | j << 3);
            }
        for (int h = 0; h < 2; ++h) for (int e = cnt[h]; e < SL; ++e) role[(e >> 2) * 8 + h * 4 + (e & 3)] = (unsigned char)(np++ << 3);
    }
    unsigned short* t = (unsigned short*)p.task;
    int n = 0;
    for (u32 c = PUSHC; c < (u32)NCIN; ++c) {
        if (c == 16u) for (u32 e = 0; e < (u32)PUSHC; ++e) t[n++] = (unsigned short)(e << 4 | 8u);
        const InPos ip = in_of(c);
        if (ip.isw && ip.s < (u32)POWN) for (u32 k = 1; k <= 8; ++k) t[n++] = (unsigned short)(c << 4 | k);
        else t[n++] = (unsigned short)(c << 4);
    }
}

extern "C" void kernel_launch(int rank, int nranks, ncclComm_t comm, hipStream_t stream,
                              void* const* d_in, const int* in_sizes, int n_in, void* d_out, int out_size,
                              const int* in_dtypes, int out_dtype, void* const* d_out_all,
                              void* const* d_ws_all, size_t ws_bytes) {
    if (nranks != NR || (unsigned)rank >= (unsigned)NR || ws_bytes < WS_NEED || n_in != 2) { fprintf(stderr, "unsupported configuration\n"); return; }
    Params p;
    p.x = (const char*)d_in[0]; p.w = (const char*)d_in[1]; p.out = (char*)d_out; p.rank = rank; p.pad = 0;
    for (int i = 0; i < NR; ++i) p.ws[i] = (char*)d_ws_all[(rank + i) % NR];
    build_tables(p);
    hipLaunchKernelGGL(gemm_ar_fused, dim3(256), dim3(512), 0, stream, p);
    HIP_CHECK(hipGetLastError());
}
```

```cpp
#include <hip/hip_runtime.h>
#include <rccl/rccl.h>
#include <cstdio>

#define HIP_CHECK(expr) do { hipError_t _e = (expr); if (_e != hipSuccess) { \
    fprintf(stderr, "HIP error %s at %s:%d\n", hipGetErrorString(_e), __FILE__, __LINE__); \
    return; } } while (0)

typedef unsigned u32;
typedef unsigned long long u64;
typedef u32 u32x4 __attribute__((ext_vector_type(4)));
typedef u32 u32x2 __attribute__((ext_vector_type(2)));
typedef unsigned short u16x2 __attribute__((ext_vector_type(2)));
typedef short bf16x8 __attribute__((ext_vector_type(8)));
typedef float f32x4 __attribute__((ext_vector_type(4)));
#define LAS __attribute__((address_space(3)))
#define GAS __attribute__((address_space(1)))
typedef __amdgpu_buffer_rsrc_t rsrc_t;

constexpr int NR = 8;
#define PULL_WG_PER_PEER 8
#define PAT 56565
#define LVAR 5
#define PUSHC 5
#define PUSHW 1
#define PACK_IN 112
#define AFFINE 1
#define PJOIN 1
#define DEC_SLEEP 2
#define LOOK 5
#define RHO 6
#define ENC_FIRST (16 - PUSHC)
#define ENC_DELAY 150
#define ENC_PACE 192
#define POLL_SLEEP 8
#define WB 12
constexpr int NPULL = 7 * PULL_WG_PER_PEER, NGEMM = 16 * WB, NPACK = 256 - NGEMM - NPULL;
constexpr int POWN = 16 - RHO, NP = POWN + 8 * RHO;
constexpr int NB = (NP + WB - 1) / WB, N0 = 16 + WB;
constexpr int NCIN = 64 + 4 * NP, NCY = 32 * POWN, NC = NCIN + NCY;
constexpr int NTASK = 16 * 36;
constexpr int NDEC = 6;
constexpr u64 DEADLINE = 100000000ull;
constexpr int PUSHB = (8 * PUSHC + PULL_WG_PER_PEER - 1) / PULL_WG_PER_PEER;
constexpr int NLAST = NP - (NB - 1) * WB;
static_assert(LOOK <= 7 && NGEMM >= 192 && NPACK < 32 && NPACK >= 4 && (PACK_IN + NCY) / NPACK < 126 && NB <= 8 && POWN <= WB && PUSHB <= NDEC, "roles");

constexpr int FIXB = 5120, CVAR = 8 * FIXB, CHDR = 32, CH = 66560;
constexpr size_t OFF_CALL = 0;
constexpr size_t OFF_TMO = 131072;
constexpr size_t OFF_TICK = OFF_TMO + 256;
constexpr size_t OFF_RDY = OFF_TICK + 16 * 128;
constexpr size_t OFF_FLAG = OFF_RDY + 64 * 128;
constexpr size_t OFF_TILE = OFF_FLAG + 8 * 256;
constexpr size_t OFF_GRAN = 196608;
constexpr size_t OFF_DBG = 524288;
constexpr size_t OFF_XF = 1ull << 20;
constexpr size_t OFF_WF = OFF_XF + (64ull << 20);
constexpr size_t OFF_SY = OFF_WF + NP * (2ull << 20);
constexpr size_t OFF_SI = OFF_SY + (size_t)NCY * CH;
constexpr size_t WS_NEED = OFF_SI + 2ull * NTASK * CH;

constexpr int A2B = 2064, A3B = 528, VARB = 912, WLDS = A2B + A3B + VARB, N2MAX = 2032, N3MAX = 496, UBITS = 2016, NPATCH = 31;
constexpr int CTL_OFF = 2048, CTL_WORDS = 128, WL_OFF = 2560, BIG_OFF = 30720, LDS_BYTES = BIG_OFF + 131072;
static_assert(WL_OFF + 8 * WLDS <= BIG_OFF && LDS_BYTES <= 163840, "LDS map");
constexpr int NF = 16, NV = 4, VSLOT = 8192, VRING = NF * FIXB;
constexpr int AUX_SYS = 17, AUX_DEV = 16, AUX_OUT = 18, AUX_NT = 2;

struct Params {
    int rank; int pad;
    const char* x; const char* w; char* out;
    char* ws[NR];
    u32 role[16];
    u32 task[NTASK / 2];
};

__device__ __forceinline__ u32 rfl(u32 x) { return (u32)__builtin_amdgcn_readfirstlane((int)x); }
__device__ __forceinline__ rsrc_t mk_rsrc(const void* p, u32 bytes) {
    const u64 a = (u64)p; const u64 u = ((u64)rfl((u32)(a >> 32)) << 32) | rfl((u32)a);
    return __builtin_amdgcn_make_buffer_rsrc((void*)u, (short)0, (int)rfl(bytes), 0x00020000);
}
__device__ __forceinline__ u32 vperm(u32 a, u32 b, u32 s) { return __builtin_amdgcn_perm(a, b, s); }
__device__ __forceinline__ u32 wave_incl_scan(u32 x) {
    x += (u32)__builtin_amdgcn_update_dpp(0, (int)x, 0x111, 0xf, 0xf, false);
    x += (u32)__builtin_amdgcn_update_dpp(0, (int)x, 0x112, 0xf, 0xf, false);
    x += (u32)__builtin_amdgcn_update_dpp(0, (int)x, 0x114, 0xf, 0xf, false);
    x += (u32)__builtin_amdgcn_update_dpp(0, (int)x, 0x118, 0xf, 0xf, false);
    x += (u32)__builtin_amdgcn_update_dpp(0, (int)x, 0x142, 0xa, 0xf, false);
    x += (u32)__builtin_amdgcn_update_dpp(0, (int)x, 0x143, 0xc, 0xf, false);
    return x;
}
__device__ __forceinline__ u32 pk_sub(u32 a, u32 b) { return __builtin_bit_cast(u32, (u16x2)(__builtin_bit_cast(u16x2, a) - __builtin_bit_cast(u16x2, b))); }
template <int AUX>
__device__ __forceinline__ void st128(u32x4 x, rsrc_t r, int voff, int soff) {
    __builtin_amdgcn_raw_buffer_store_b128(x, r, voff, soff, AUX);
    asm volatile("s_nop 1" :: "v"(x) : "memory");
}
#define WAVE_FENCE() __builtin_amdgcn_fence(__ATOMIC_ACQ_REL, "wavefront")

__device__ __forceinline__ void build_table(LAS u32x2* tab, u32 i, bool unpack) {
    u32 n = 0, dep = 0, cmp = 0x0c0c0c0cu;
#pragma unroll
    for (u32 t = 0; t < 4; ++t) {
        const u32 c = (i >> (2 * t)) & 3u;
        dep |= (c ? 4u + c : n) << (8 * t);
        if (!c) { cmp = (cmp & ~(0xffu << (8 * n))) | (t << (8 * n)); ++n; }
    }
    tab[i] = (u32x2){unpack ? dep : cmp, n};
}
__device__ __forceinline__ void build_table3(LAS u32x2* tab3, u32 i) {
    u64 r = 0; u32 n = 0, run = 0;
#pragma unroll
    for (u32 t = 0; t < 8; ++t) { if ((i >> t) & 1u) { r |= (u64)run << (8 * n); ++n; run = 0; } else ++run; }
    tab3[i] = (u32x2){(u32)r, (u32)(r >> 32)};
}
__device__ __forceinline__ void or3(LAS u32* q, u64 x, u32 sh) {
    const u64 y = x << sh;
    __hip_atomic_fetch_or(q, (u32)y, __ATOMIC_RELAXED, __HIP_MEMORY_SCOPE_WORKGROUP);
    __hip_atomic_fetch_or(q + 1, (u32)(y >> 32), __ATOMIC_RELAXED, __HIP_MEMORY_SCOPE_WORKGROUP);
    __hip_atomic_fetch_or(q + 2, (u32)(((x >> 32) << sh) >> 32), __ATOMIC_RELAXED, __HIP_MEMORY_SCOPE_WORKGROUP);
}
__device__ __forceinline__ u32 code_byte(u32 hi, u32 lo, u32 x) {
    const u32 f = vperm(hi, lo, x);
    return __builtin_amdgcn_udot4((f ^ (f >> 2)) & 0x03030303u, 0x40100401u, 0u, false);
}
template <bool PRED>
__device__ __forceinline__ void or_bytes(u32 pos, u32 x) {
    const u64 y = (u64)x << ((pos & 3u) * 8u);
    LAS u32* q = (LAS u32*)(size_t)(pos & ~3u);
    if (!PRED || x) {
        __hip_atomic_fetch_or(q, (u32)y, __ATOMIC_RELAXED, __HIP_MEMORY_SCOPE_WORKGROUP);
        __hip_atomic_fetch_or(q + 1, (u32)(y >> 32), __ATOMIC_RELAXED, __HIP_MEMORY_SCOPE_WORKGROUP);
    }
}
__device__ __forceinline__ u32 get_bytes(u32 pos) {
    const LAS u32* q = (const LAS u32*)(size_t)(pos & ~3u);
    return __builtin_amdgcn_alignbyte(q[1], q[0], pos);
}
__device__ __forceinline__ u32 la(const LAS void* p) { return (u32)(size_t)p; }
__device__ __forceinline__ u32 pack_level2(const LAS u32x2* tab, const LAS char* src, u32 n, LAS u32* end, LAS char* dst, int lane) {
    u32 m = 0;
    for (u32 i0 = 0; i0 < n; i0 += 1024u) {
        const u32 i = i0 + (u32)lane * 16u, hi = 0x02010000u, lo = 0x00030000u;
        const u32x4 x = *(const LAS u32x4*)(src + (i < n ? i : 0u));
        u32 cw = code_byte(hi, lo, x[0]) | (code_byte(hi, lo, x[1]) << 8) | (code_byte(hi, lo, x[2]) << 16) | (code_byte(hi, lo, x[3]) << 24);
        const u32 valid = i < n ? n - i : 0u;
        if (valid < 16u) cw |= 0x55555555u << (2u * valid);
        u32x2 e[4]; u32 c[5]; c[0] = 0;
#pragma unroll
        for (int j = 0; j < 4; ++j) { e[j] = tab[(cw >> (8 * j)) & 0xffu]; c[j + 1] = c[j] + e[j][1]; }
        const u32 incl = wave_incl_scan(c[4]), p0 = la(dst) + m + incl - c[4];
        if (i < n) *(end - 1 - (i >> 4)) = cw;
        m += (u32)__builtin_amdgcn_readlane((int)incl, 63);
        if (m <= (u32)N3MAX) {
#pragma unroll
            for (int j = 0; j < 4; ++j) or_bytes<true>(p0 + c[j], vperm(0u, x[j], e[j][0]));
        }
    }
    WAVE_FENCE();
    return m;
}
__device__ __forceinline__ void pack_level3(const LAS char* src, u32 n, LAS u32* ub, LAS u32* pt, u32 T, int lane, u32& bits, u32& np) {
    for (u32 i0 = 0; i0 < n; i0 += 256u) {
        const u32 i = i0 + (u32)lane * 4u;
        const u32 x = *(const LAS u32*)(src + (i < n ? i : 0u));
        u32 cum = 0, pf = 0; u64 m = 0;
#pragma unroll
        for (u32 t = 0; t < 4; ++t) {
            const u32 r = ((x >> (8 * t)) - 8u) & 0xffu;
            const bool valid = i + t < n, big = r > 15u;
            cum += valid && !big ? r : 0u;
            m |= (u64)(valid ? 1u : 0u) << cum;
            cum += valid ? 1u : 0u;
            pf |= valid && big ? 1u << t : 0u;
        }
        const u32 incl = wave_incl_scan(cum), p = bits + incl - cum;
        bits += (u32)__builtin_amdgcn_readlane((int)incl, 63);
        if (bits <= (u32)UBITS && cum) or3(ub + (p >> 5), m, p & 31u);
        if (__builtin_amdgcn_ballot_w64(pf != 0u)) {
#pragma unroll
            for (u32 t = 0; t < 4; ++t) {
                const u64 bm = __builtin_amdgcn_ballot_w64((pf >> t) & 1u);
                const u32 at = np + __builtin_amdgcn_mbcnt_hi((u32)(bm >> 32), __builtin_amdgcn_mbcnt_lo((u32)bm, 0u));
                if ((pf >> t) & 1u) pt[at] = ((i + t) << 8) | ((T - (x >> (8 * t))) & 0xffu);
                np += (u32)__builtin_popcountll(bm);
            }
        }
    }
    WAVE_FENCE();
}
__device__ __forceinline__ void encode_block(const u32 (&v)[8][4], rsrc_t ch, u32 b, LAS u32* alloc, const LAS u32x2* tab, LAS char* wl, int lane) {
#pragma unroll
    for (int j = 0; j < 4; ++j) if (j < 3 || lane < (WLDS - 3072) / 16) *(LAS u32x4*)(wl + j * 1024 + lane * 16) = (u32x4){0, 0, 0, 0};
    u32 s = 0;
#pragma unroll
    for (int g = 0; g < 8; ++g) s = __builtin_amdgcn_udot4(__builtin_amdgcn_alignbit(v[g][0], v[g][0], 31), 0x01000100u, s, false);
    u32 T = ((u32)__builtin_amdgcn_readlane((int)wave_incl_scan(s), 63) >> 10) + 5u;
    T = T > 255u ? 255u : (T < 23u ? 23u : T);
    const u32 tt = (T * 256u + 255u) * 0x10001u;
    const int fo = (int)(b * FIXB);
    u32 U[16], l1[4] = {0, 0, 0, 0}, c[17];
    u32x2 e[16];
#pragma unroll
    for (int q = 0; q < 4; ++q) {
        u32x4 sm;
#pragma unroll
        for (int j = 0; j < 4; ++j) {
            const int k = 4 * q + j;
            const u32 a0 = v[k >> 1][(k & 1) * 2], a1 = v[k >> 1][(k & 1) * 2 + 1];
            const u32 r0 = __builtin_amdgcn_alignbit(a0, a0, 31), r1 = __builtin_amdgcn_alignbit(a1, a1, 31);
            sm[j] = vperm(r1, r0, 0x06040200u);
            U[k] = vperm(pk_sub(tt, r1), pk_sub(tt, r0), 0x07050301u);
            const u32 cb = code_byte(0x00000302u, 0x01000000u, U[k]);
            l1[q] |= cb << (8 * j);
            e[k] = tab[cb];
        }
        st128<AUX_SYS>(sm, ch, lane * 16 + q * 1024, fo);
    }
    st128<AUX_SYS>((u32x4){l1[0], l1[1], l1[2], l1[3]}, ch, lane * 16 + 4096, fo);
    __builtin_amdgcn_sched_barrier(0);
    c[0] = 0;
#pragma unroll
    for (int k = 0; k < 16; ++k) c[k + 1] = c[k] + e[k][1];
    const u32 incl = wave_incl_scan(c[16]), p0 = la(wl) + incl - c[16];
    const u32 n2 = (u32)__builtin_amdgcn_readlane((int)incl, 63);
    bool raw = n2 > (u32)N2MAX;
    u32 d2 = 0, d3 = 0, np = 0;
    LAS u32* var = (LAS u32*)(wl + A2B + A3B);
    if (!raw) {
        WAVE_FENCE();
#pragma unroll
        for (int k = 0; k < 16; ++k) or_bytes<false>(p0 + c[k], vperm(0u, U[k], e[k][0]));
        WAVE_FENCE();
        d2 = (n2 + 15u) >> 4;
        const u32 n3 = pack_level2(tab, wl, n2, var + d2, wl + A2B, lane);
        raw = n3 > (u32)N3MAX;
        if (!raw) {
            u32 bits = 0;
            pack_level3(wl + A2B, n3, var + d2, (LAS u32*)wl, T, lane, bits, np);
            raw = bits > (u32)UBITS || np > (u32)NPATCH;
            d3 = (bits + 31u) >> 5;
            if (!raw && (u32)lane < np) var[d2 + d3 + (u32)lane] = ((const LAS u32*)wl)[lane];
            WAVE_FENCE();
        }
    }
    const u32 vd = d2 + d3 + np;
    u32 off = 0;
    if (lane == 0) off = __hip_atomic_fetch_add(alloc, raw ? 3072u : vd * 4u, __ATOMIC_RELAXED, __HIP_MEMORY_SCOPE_WORKGROUP);
    off = rfl(off) + CHDR;
    const int vo = CVAR + (int)off;
    if (lane == 0) __builtin_amdgcn_raw_buffer_store_b32(raw ? off << 17 : (T | (d3 << 8) | (np << 14) | ((off + d2 * 4u) << 17)), ch, 0, CVAR + (int)b * 4, AUX_SYS);
    if (raw) {
        asm volatile("s_waitcnt vmcnt(0)" ::: "memory");
#pragma unroll
        for (int g = 0; g < 8; ++g)
            st128<AUX_SYS>((u32x4){v[g][0], v[g][1], v[g][2], v[g][3]}, ch, lane * 16 + (g < 5 ? g : g - 5) * 1024, g < 5 ? fo : vo);
    } else {
        for (u32 o = (u32)lane; o < vd; o += 64u) __builtin_amdgcn_raw_buffer_store_b32(var[o], ch, (int)o * 4, vo, AUX_SYS);
    }
    WAVE_FENCE();
}
__device__ __forceinline__ bool chunk_done(LAS u32* cnt, int lane, u32& total) {
    u32 n = 0;
    if (lane == 0) n = __hip_atomic_fetch_add(cnt + 1, 1u, __ATOMIC_RELAXED, __HIP_MEMORY_SCOPE_WORKGROUP);
    if (rfl(n) != 7u) return false;
    total = rfl(*(volatile LAS u32*)cnt) + CHDR;
    return true;
}

struct Dest { rsrc_t r; u32 A, B, C, LH; };
__device__ __forceinline__ Dest dest_of(char* base, u32 kind, u32 b) {
    if (kind == 0u) return Dest{mk_rsrc(base + b * 4096u, 32768u + 4096u), 1024u, 32768u, 2048u, 256u};
    if (kind == 1u) return Dest{mk_rsrc(base + (b >> 2) * 32768u + (b & 1u) * 8192u + (b & 2u) * 512u, 16384u + 6144u + 1024u), 2048u, 4096u, 16384u, 256u};
    return Dest{mk_rsrc(base + (b & 3u) * (32u * 65536u) + (b >> 2) * 256u, 31u * 65536u + 256u), 65536u, 131072u, 16u * 65536u, 4u * 65536u};
}
template <int AUX>
__device__ __forceinline__ void store_group(const Dest& d, u32 loff, int g, u32x4 x) {
    st128<AUX>(x, d.r, (int)loff, (int)((g & 1 ? d.A : 0u) + (g & 2 ? d.B : 0u) + (g & 4 ? d.C : 0u)));
}

#define SCHED __builtin_amdgcn_sched_barrier(0)
__device__ __forceinline__ u32x2 get2(u32 pos) { const LAS u32* q = (const LAS u32*)(size_t)(pos & ~3u); return (u32x2){q[0], q[1]}; }
__device__ __forceinline__ void spread3(u32 w, u32 start, u32 n, u32x2 runs, u32 t8, u32 p) {
    if (n) {
        const u32 carry = (u32)__builtin_clz((w << 8) | start);
        const u64 x = ((u64)(t8 - runs[1]) << 32) | (t8 - runs[0] - carry);
        const u32 sh = 64u - 8u * n;
        or3((LAS u32*)(size_t)(p & ~3u), (x << sh) >> sh, (p & 3u) * 8u);
    }
}
template <class Out>
__device__ __forceinline__ void decode_block(const u32x4 (&d)[5], u32 hdr, const LAS u32* ub, const LAS u32x2* tab, const LAS u32x2* tab3, LAS char* wl, int lane, Out out) {
    const u32 T = hdr & 0xffu, d3 = (hdr >> 8) & 0x3fu, np = (hdr >> 14) & 0x1fu;
    auto ex = [&](u32 a, u32 b, u32 c) { return ((T - a) << 8) | ((T - b) << 16) | ((T - c) << 24); };
    const u32 t1 = ex(3, 4, 5), t2 = ex(6, 7, 2), t8 = (T - 8u) * 0x01010101u;
    const u32 w2 = *(ub - 1 - lane), p3i = la(ub) + (u32)lane - 3u;
    u32x2 e[16], e2[4], g[16];
    g[0] = get2(p3i);
    if (lane < A3B / 16) *(LAS u32x4*)(wl + A2B + lane * 16) = (u32x4){0, 0, 0, 0};
#pragma unroll
    for (int k = 0; k < 8; ++k) e[k] = tab[(d[4][k >> 2] >> (8 * (k & 3))) & 0xffu];
    SCHED;
    const u32 w3 = __builtin_amdgcn_alignbyte(g[0][1], g[0][0], p3i), x3 = (u32)lane < d3 * 4u ? w3 >> 24 : 0u;
    const u32x2 e3 = tab3[x3];
#pragma unroll
    for (int j = 0; j < 4; ++j) e2[j] = tab[(w2 >> (8 * j)) & 0xffu];
#pragma unroll
    for (int k = 8; k < 16; ++k) e[k] = tab[(d[4][k >> 2] >> (8 * (k & 3))) & 0xffu];
    SCHED;
    u32 c[17], f[5];
    c[0] = 0; f[0] = 0;
#pragma unroll
    for (int k = 0; k < 16; ++k) c[k + 1] = c[k] + e[k][1];
#pragma unroll
    for (int j = 0; j < 4; ++j) f[j + 1] = f[j] + e2[j][1];
    const u32 n3 = (u32)__builtin_popcount(x3);
    const u32 pk = c[16] | (f[4] << 11) | (n3 << 22);
    const u32 incl = wave_incl_scan(pk), ex0 = incl - pk, tot = (u32)__builtin_amdgcn_readlane((int)incl, 63);
    const u32 a3 = la(wl) + A2B, p1 = la(wl) + (ex0 & 0x7ffu), p2 = a3 + ((ex0 >> 11) & 0x1ffu);
    WAVE_FENCE();
    spread3(w3, lane < 3 ? 0x80000000u >> (8 * lane) : 0x80u, n3, e3, t8, a3 + ((ex0 >> 22) & 0x1ffu));
    for (u32 i0 = 64u, m = tot >> 22; i0 < d3 * 4u; i0 += 64u) {
        const u32 w = get_bytes(p3i + i0), x = i0 + (u32)lane < d3 * 4u ? w >> 24 : 0u, n = (u32)__builtin_popcount(x), in = wave_incl_scan(n);
        spread3(w, 0x80u, n, tab3[x], t8, a3 + ((m + in - n) & 0x1ffu));
        m += (u32)__builtin_amdgcn_readlane((int)in, 63);
    }
    WAVE_FENCE();
    if ((u32)lane < np) { const u32 pw = ub[d3 + (u32)lane]; *(LAS unsigned char*)(wl + A2B + ((pw >> 8) & 0x1ffu)) = (unsigned char)pw; }
    WAVE_FENCE();
#pragma unroll
    for (int j = 0; j < 4; ++j) { f[j] += p2; g[j] = get2(f[j]); }
    SCHED;
#pragma unroll
    for (int k = 0; k < 16; ++k) c[k] += p1;
    SCHED;
    u32x4 a2;
#pragma unroll
    for (int j = 0; j < 4; ++j) a2[j] = vperm(t2, __builtin_amdgcn_alignbyte(g[j][1], g[j][0], f[j]), e2[j][0]);
    *(LAS u32x4*)(wl + lane * 16) = a2;
    if ((tot & 0x7ffu) > 1024u) {
        const u32 cw = *(ub - 65 - lane);
        f[0] = 0;
#pragma unroll
        for (int j = 0; j < 4; ++j) { e2[j] = tab[(cw >> (8 * j)) & 0xffu]; f[j + 1] = f[j] + e2[j][1]; }
        const u32 p = a3 + (((tot >> 11) + wave_incl_scan(f[4]) - f[4]) & 0x1ffu);
#pragma unroll
        for (int j = 0; j < 4; ++j) a2[j] = vperm(t2, get_bytes(p + f[j]), e2[j][0]);
        *(LAS u32x4*)(wl + 1024 + lane * 16) = a2;
    }
    WAVE_FENCE();
#pragma unroll
    for (int k = 0; k < 16; ++k) g[k] = get2(c[k]);
    SCHED;
    u32x4 o;
#pragma unroll
    for (int k = 0; k < 16; ++k) {
        const u32 x = vperm(t1, __builtin_amdgcn_alignbyte(g[k][1], g[k][0], c[k]), e[k][0]);
        const u32 smk = d[k >> 2][k & 3];
        const u32 r0 = vperm(x, smk, 0x05010400u), r1 = vperm(x, smk, 0x07030602u);
        o[2 * (k & 1)] = __builtin_amdgcn_alignbit(r0, r0, 1);
        o[2 * (k & 1) + 1] = __builtin_amdgcn_alignbit(r1, r1, 1);
        if (k & 1) out(k >> 1, o);
    }
    WAVE_FENCE();
}

__device__ __forceinline__ char* frag_base(char* ws, bool isw, u32 s, u32 q, u32 rel, u32 call) {
    return ws + (isw ? OFF_WF : OFF_XF + (size_t)(call & 1u) * (32u << 20)) + (size_t)(s * 64u + (rel ? 6u + 14u * q + 2u * rel : 2u * q)) * 32768u;
}
__device__ __forceinline__ char* y_base(char* out, u32 i, u32 gp, u32 bj) { return out + (size_t)(i * 256u + bj * 128u) * 65536u + gp * 512u; }
__device__ __forceinline__ u32 gpanel(u32 r, u32 j) { return j < (u32)POWN ? r * 16u + j : ((j - POWN) & 7u) * 16u + POWN + ((j - POWN) >> 3); }
__device__ __forceinline__ u32 slot_in(bool isw, u32 s, u32 q, u32 dst) {
    return !isw ? s * 4u + q : (s < (u32)POWN ? 64u + (dst * POWN + s) * 4u + q : 64u + 28u * POWN + s * 4u + q);
}
__host__ __device__ __forceinline__ u32 batch_size(u32 bt) { return bt == 0u ? (u32)N0 : (bt == NB - 1u ? (u32)(NP - (NB - 1) * WB) : (u32)WB); }
struct InPos { bool isw; u32 s, q, bt; };
__host__ __device__ __forceinline__ InPos in_of(u32 c) {
    if (c < 4u * N0) { const u32 q = c / N0, pp = c - q * N0; return InPos{pp >= 16u, pp >= 16u ? pp - 16u : pp, q, 0u}; }
    const u32 bt = (c - 64u) / (4u * WB), r = c - 64u - bt * 4u * WB, q = r / batch_size(bt);
    return InPos{true, bt * WB + r - q * batch_size(bt), q, bt};
}

__device__ __forceinline__ void arrive(char* ws, u32 ci, u32 n, u32 call, int lane) {
    u32 old = 0;
    if (lane == 0) old = __hip_atomic_fetch_add((GAS u32*)(ws + OFF_RDY + ci * 128u), n, __ATOMIC_RELAXED, __HIP_MEMORY_SCOPE_SYSTEM);
    if (rfl(old) + n == call * ((ci & 7u) ? 56u : 32u) * batch_size(ci >> 3) && lane < 8)
        __hip_atomic_store((GAS u32*)(ws + OFF_FLAG + (u32)lane * 256u + ci * 4u), call, __ATOMIC_RELAXED, __HIP_MEMORY_SCOPE_SYSTEM);
}
__device__ __forceinline__ void load_x(const char* x, u32 s, u32 q, u32 b, int lane, u32x4 (&a)[8]) {
    const rsrc_t r = mk_rsrc(x + (size_t)(s * 256u + b * 32u) * 1024u + q * 256u, 31u * 1024u + 256u);
#pragma unroll
    for (int i = 0; i < 8; ++i) a[i] = __builtin_amdgcn_raw_buffer_load_b128(r, (lane & 15) * 1024 + (lane >> 4) * 16 + (i & 3) * 64, (i >> 2) * 16384, 0);
}

__device__ __forceinline__ bool give_up(GAS u32* tmo, u32 call, u64 t0) {
    if (rfl(__hip_atomic_load(tmo, __ATOMIC_RELAXED, __HIP_MEMORY_SCOPE_AGENT)) == call || __builtin_amdgcn_s_memrealtime() - t0 > DEADLINE) {
        __hip_atomic_store(tmo, call, __ATOMIC_RELAXED, __HIP_MEMORY_SCOPE_AGENT);
        return true;
    }
    return false;
}

#define STAMP_IT 0
#define STAMP_AT(i) do { } while (0)

__device__ __forceinline__ u32 lds_ld(LAS u32* p) { return rfl(*(volatile LAS u32*)p); }
__device__ __forceinline__ void dma_piece(rsrc_t r, u32 voff, u32 soff, u32 ldsaddr) {
    u32 keep;
    asm volatile("s_nop 4\n\ts_mov_b32 %0, m0\n\ts_mov_b32 m0, %1\n\ts_nop 0\n\tbuffer_load_dwordx4 %2, %3, %4 offen sc0 sc1 lds\n\ts_mov_b32 m0, %0"
                 : "=&s"(keep) : "s"(ldsaddr), "v"(voff), "s"(r), "s"(soff) : "memory");
}

#define DMA_P(j, w, off) "s_waitcnt vmcnt(" w ")\n\tbuffer_load_dwordx4 %2, %3, " off " sc0 sc1 lds\n\t.if %8 == " #j "\n\tds_max_u32 %6, %7\n\t.endif\n\t"
__device__ __forceinline__ void dma_block(rsrc_t r, u32 voff, u32 soff, u32 ldsaddr, u32 at, u32 val) {
    constexpr int W0 = PAT / 10000, W1 = PAT / 1000 % 10, W2 = PAT / 100 % 10, W3 = PAT / 10 % 10, W4 = PAT % 10;
    constexpr int JP = W0 <= 5 ? 0 : (W1 <= 6 ? 1 : (W2 <= 7 ? 2 : (W3 <= 8 ? 3 : 4)));
    u32 keep;
    asm volatile("s_nop 2\n\ts_mov_b32 %0, m0\n\ts_mov_b32 m0, %1\n\t"
                 DMA_P(0, "%9", "%4 offen") DMA_P(1, "%10", "%4 offen offset:1024") DMA_P(2, "%11", "%4 offen offset:2048") DMA_P(3, "%12", "%4 offen offset:3072")
                 "s_add_u32 m0, m0, 0x1000\n\t" DMA_P(4, "%13", "%5 offen") "s_mov_b32 m0, %0"
                 : "=&s"(keep) : "s"(ldsaddr), "v"(voff), "s"(r), "s"(soff), "s"(soff + 4096u), "v"(at), "v"(val), "n"(JP),
                   "n"(W0), "n"(W1), "n"(W2), "n"(W3), "n"(W4) : "memory", "scc");
}

__global__ void __launch_bounds__(512, 2) gemm_ar_fused(const Params p) {
    __shared__ __attribute__((aligned(16))) char smem[LDS_BYTES];
    LAS char* lds = (LAS char*)smem + BIG_OFF;
    LAS u32x2* tab = (LAS u32x2*)smem;
    LAS u32x2* tab3 = (LAS u32x2*)((LAS char*)smem + WL_OFF);
    const int tid = threadIdx.x, wid = (int)rfl((u32)tid >> 6), lane = tid & 63;
    const u32 bid = blockIdx.x;
    u32 rank = (u32)p.rank, rolew = p.role[((bid - NGEMM) >> 2) & 15u];
    char* ws = p.ws[0]; const char* px = p.x;
    asm volatile("" : "+s"(rank), "+s"(rolew), "+s"(ws), "+s"(px));
    LAS char* wl = (LAS char*)smem + WL_OFF + wid * WLDS;
    LAS u32* ctl = (LAS u32*)((LAS char*)smem + CTL_OFF);
    GAS u32* tmo = (GAS u32*)(ws + OFF_TMO);
    STAMP_AT(0);
    const u32 role = bid >= (u32)NGEMM ? (rolew >> ((bid & 3u) * 8u)) & 0xffu : 0u;
    const u32 pn = (role & 7u) ? (role >> 3) * PUSHB + (u32)wid - 2u : (bid + PUSHC) * 8u + (u32)wid;
    const bool early = (role & 7u) ? wid >= 2 && wid < 2 + PUSHB && pn < 8u * PUSHC : bid < 16u - PUSHC;
    u32x4 a[8];
    if (early) load_x(px, pn >> 3, 0u, pn & 7u, lane, a);
    SCHED;
    GAS u32* callp = (GAS u32*)(ws + OFF_CALL + (bid * 8u + wid) * 64u);
    const u32 callv = __hip_atomic_load(callp, __ATOMIC_RELAXED, __HIP_MEMORY_SCOPE_AGENT);
    char* wsl = ws;
    auto setup = [&](bool unpack) {
        u32 t = (u32)tid;
        asm volatile("" : "+v"(t));
        if (t < 256u) build_table(tab, t, unpack);
        else { if (t < 256u + CTL_WORDS) ctl[t - 256u] = 0u; if (unpack) build_table3(tab3, t - 256u); }
        if (!unpack) {
            ((LAS u32*)lds)[t] = 0u;
#pragma unroll
            for (u32 k = 1; k < 8u; ++k) wsl = (t & 63u) == k ? p.ws[k] : wsl;
        }
        asm volatile("s_waitcnt lgkmcnt(0)" ::: "memory"); __builtin_amdgcn_s_barrier(); asm volatile("" ::: "memory");
    };

    if (role & 7u) {
        const u32 rel = role & 7u, pjj = role >> 3, src = (rank + rel) & 7u;
        char* pws = p.ws[rel];
        setup(true);
        if (early) {
            const u32 call = rfl(callv) + 1u;
            const Dest dst = dest_of(frag_base(pws, false, pn >> 3, 0u, 8u - rel, call), 0u, pn & 7u);
#pragma unroll
            for (int g = 0; g < 8; ++g) asm volatile("" : "+v"(a[g]));
#pragma unroll
            for (int g = 0; g < 8; ++g) { store_group<AUX_SYS>(dst, (u32)lane * 16u, g, a[g]); asm volatile("s_waitcnt vmcnt(%0)" :: "n"(PUSHW - 1) : "memory"); }
            STAMP_AT(12);
            if (lane == 0) __hip_atomic_store(callp, call, __ATOMIC_RELAXED, __HIP_MEMORY_SCOPE_AGENT);
            asm volatile("s_waitcnt vmcnt(0)" ::: "memory");
            STAMP_AT(13);
            arrive(pws, 1u, 1u, call, lane);
        }
        LAS u32* vready = ctl; LAS u32* endk = ctl + 1; LAS u32* landed = ctl + 2; LAS u32* prog = ctl + 5;
        LAS u32* vdone = ctl + 8; LAS u32* sdone = ctl + 16; LAS u32* ffree = ctl + 32;
        LAS u32* q = ctl + 64;
        const u32 l0 = la(lds), voff = (u32)lane * 16u;
        if (wid == 0) {
            const u32 call = rfl(callv) + 1u;
            if (lane == 0) __hip_atomic_store(callp, call, __ATOMIC_RELAXED, __HIP_MEMORY_SCOPE_AGENT);
            GAS u64* gran = (GAS u64*)(ws + OFF_GRAN) + (size_t)((call & 1u) * 8u + src) * NC;
            for (u32 k = 0;; ++k) {
                u32 c = PUSHC + pjj;
                if (k) {
                    while (lds_ld(prog) < k) __builtin_amdgcn_s_sleep(2);
                    u32 tk = 0;
                    if (lane == 0) tk = __hip_atomic_fetch_add((GAS u32*)(ws + OFF_TICK + src * 128u), 1u, __ATOMIC_RELAXED, __HIP_MEMORY_SCOPE_AGENT);
                    c = rfl(tk) - (call - 1u) * (NC - PUSHC) + PULL_WG_PER_PEER + PUSHC;
                }
                u32 fl = 2u, vb = 0, kc = 2u;
                const char* from = pws; char* to = ws;
                if (c < (u32)NC) {
                    if (c >= (u32)NCIN) {
                        const u32 yc = c - NCIN;
                        from = pws + OFF_SY + (size_t)yc * CH;
                        to = y_base(p.out, (yc >> 1) & 15u, src * 16u + (yc >> 5), yc & 1u);
                    } else {
                        const InPos ip = in_of(c);
                        from = pws + OFF_SI + (size_t)((call & 1u) * NTASK + slot_in(ip.isw, ip.s, ip.q, rank)) * CH;
                        to = frag_base(ws, ip.isw, ip.s, ip.q, rel, call);
                        kc = (ip.isw ? 1u : 0u) | ((ip.bt * 8u + ip.q + 1u) << 2);
                    }
                    const u64 t0 = __builtin_amdgcn_s_memrealtime();
                    for (u32 spin = 0;; ++spin) {
                        const u64 g = __hip_atomic_load(gran + c, __ATOMIC_RELAXED, __HIP_MEMORY_SCOPE_SYSTEM);
                        if (rfl((u32)g) == call) { fl = 0u; vb = rfl((u32)(g >> 32)); break; }
                        if ((spin & 15u) == 15u && give_up(tmo, call, t0)) { fl = 1u; break; }
                        __builtin_amdgcn_s_sleep(1);
                    }
                    if (k == 0) STAMP_AT(1);
                }
                while (lds_ld(vdone + (k & (NV - 1))) < 8u * (k / NV)) __builtin_amdgcn_s_sleep(2);
                if (lane == 0) {
                    *(volatile LAS u32x4*)(q + (k & (NV - 1)) * 8u + 4u) = (u32x4){(u32)(u64)to, (u32)((u64)to >> 32), kc, call};
                    *(volatile LAS u32x4*)(q + (k & (NV - 1)) * 8u) = (u32x4){(u32)(u64)from, (u32)((u64)from >> 32), fl | ((k + 1u) << 2), vb};
                }
                if (fl & 2u) { if (lane == 0) *(volatile LAS u32*)endk = k + 1u; break; }
                if (!(fl & 1u)) {
                    const u32 vl = vb > (u32)VSLOT ? (u32)CHDR : (vb + 15u) & ~15u;
                    const rsrc_t r = mk_rsrc(from, CVAR + vl);
                    for (u32 o = 0; o < vl; o += 1024u) { dma_piece(r, voff + CVAR + o, 0u, rfl(l0 + VRING + (k & (NV - 1)) * VSLOT + o)); if (k) asm volatile("s_waitcnt vmcnt(0)" ::: "memory"); }
                    asm volatile("s_waitcnt vmcnt(0)" ::: "memory");
                }
                if (lane == 0) *(volatile LAS u32*)vready = k + 1u;
            }
        } else if (wid == 1) {
            __builtin_amdgcn_s_setprio(3);
            const u32 la_landed = lane ? la(wl) + (u32)lane * 4u : la(landed);
            u32 n = 0;
            auto drain = [&]() { asm volatile("s_waitcnt vmcnt(0)" ::: "memory"); if (lane == 0) __hip_atomic_fetch_max(landed, n, __ATOMIC_RELAXED, __HIP_MEMORY_SCOPE_WORKGROUP); };
            u32x4 e = (u32x4){0, 0, 0, 0};
            u32 fr = 0;
            for (u32 k = 0;; ++k) {
                if (rfl(e[2]) >> 2 != k + 1u) {
                    e = *(volatile LAS u32x4*)(q + (k & (NV - 1)) * 8u);
                    if (rfl(e[2]) >> 2 != k + 1u) {
                        drain();
                        do { __builtin_amdgcn_s_sleep(1); e = *(volatile LAS u32x4*)(q + (k & (NV - 1)) * 8u); } while (rfl(e[2]) >> 2 != k + 1u);
                    }
                }
                const u32 fl = rfl(e[2]);
                if (fl & 2u) break;
                const rsrc_t r = mk_rsrc((const void*)(((u64)e[1] << 32) | e[0]), (fl & 1u) ? 0u : (u32)CVAR);
#pragma unroll
                for (u32 b = 0; b < 8u; ++b, ++n) {
                    const u32 sl = n & (NF - 1), m = n + 1u;
                    if (rfl(fr) + NF <= n && lds_ld(ffree + sl) + NF <= n) { drain(); while (lds_ld(ffree + sl) + NF <= n) __builtin_amdgcn_s_sleep(1); }
                    fr = *(volatile LAS u32*)(ffree + (m & (NF - 1)));
                    if (b == 7u) e = *(volatile LAS u32x4*)(q + ((k + 1u) & (NV - 1)) * 8u);
                    if (b == (LVAR ? pjj % LVAR : 7u - LOOK) && lane == 0) *(volatile LAS u32*)prog = k + 1u;
                    dma_block(r, voff, b * FIXB, rfl(l0 + sl * FIXB), la_landed, n ? n - 1u : 0u);
                }
            }
            drain();
            STAMP_AT(2);
        } else {
#define SEG(i)
            u32 owed = 0xffffffffu, call = 0;
            auto settle = [&]() {
                if (owed != 0xffffffffu) {
                    asm volatile("s_waitcnt vmcnt(0)" ::: "memory");
                    u32 old = 0;
                    if (lane == 0) old = __hip_atomic_fetch_add(sdone + (owed & 7u), 1u, __ATOMIC_RELAXED, __HIP_MEMORY_SCOPE_WORKGROUP);
                    if ((rfl(old) & 7u) == 7u) arrive(ws, owed >> 3, 8u, call, lane);
                    owed = 0xffffffffu;
                }
            };
            for (u32 n = (u32)wid - 2u;; n += NDEC) {
                const u32 k = n >> 3, b = n & 7u, sl = n & (NF - 1);
                bool end = false;
                for (;;) {
                    const u32x4 st = *(volatile LAS u32x4*)ctl;
                    if (rfl(st[0]) > k && rfl(st[2]) > n) break;
                    if (rfl(st[1]) - 1u <= k) { end = true; break; }
                    settle();
                    __builtin_amdgcn_s_sleep(DEC_SLEEP);
                }
                if (end) break;
                asm volatile("" ::: "memory");
                u32x4 d[5];
                const LAS char* vs = lds + VRING + (k & (NV - 1)) * VSLOT;
                const u32 h = *(const LAS u32*)(vs + b * 4u);
                const u32x4 e0 = *(const LAS u32x4*)(q + (k & (NV - 1)) * 8u), e1 = *(const LAS u32x4*)(q + (k & (NV - 1)) * 8u + 4u);
                d[4] = *(const LAS u32x4*)(lds + sl * FIXB + 4096 + lane * 16);
#pragma unroll
                for (int j = 0; j < 4; ++j) d[j] = *(const LAS u32x4*)(lds + sl * FIXB + j * 1024 + lane * 16);
                asm volatile("s_waitcnt lgkmcnt(0)" ::: "memory");
                if (lane == 0) *(volatile LAS u32*)(ffree + sl) = n + 1u;
                asm volatile("" : "+v"(d[0]), "+v"(d[1]), "+v"(d[2]), "+v"(d[3]), "+v"(d[4]) :: "memory");
                const u32 hdr = rfl(h), off = (hdr >> 19) * 4u, kc = rfl(e1[2]);
                SEG(0);
                call = rfl(e1[3]);
                const bool far = rfl(e0[3]) > (u32)VSLOT;
                const LAS u32* var = (const LAS u32*)(vs + (off <= (u32)VSLOT ? off : 0u));
                u32x4 o[8];
                if (rfl(e0[2]) & 1u) {
#pragma unroll
                    for (int g = 0; g < 8; ++g) o[g] = (u32x4){0x7fc07fc0u, 0x7fc07fc0u, 0x7fc07fc0u, 0x7fc07fc0u};
                } else {
                    if (far) {
                        const rsrc_t r = mk_rsrc((const void*)(((u64)e0[1] << 32) | e0[0]), CH);
#pragma unroll
                        for (int j = 0; j < 3; ++j) o[5 + j] = __builtin_amdgcn_raw_buffer_load_b128(r, (int)(CVAR + off) - ((hdr & 0xffu) ? 512 : 0) + j * 1024 + lane * 16, 0, AUX_SYS);
                        if (lane < VARB / 16) *(LAS u32x4*)(wl + A2B + A3B + lane * 16) = o[5];
                        WAVE_FENCE();
                        var = (const LAS u32*)(wl + A2B + A3B + 512);
                    }
                    if (hdr & 0xffu) decode_block(d, hdr, var, tab, tab3, wl, lane, [&](int g, u32x4 x) { o[g] = x; });
                    else {
#pragma unroll
                        for (int g = 0; g < 5; ++g) o[g] = d[g];
                        if (!far) {
#pragma unroll
                            for (int j = 0; j < 3; ++j)
#pragma unroll
                                for (int i = 0; i < 4; ++i) o[5 + j][i] = *(const volatile LAS u32*)(var + j * 256 + lane * 4 + i);
                        }
                    }
                }
                const Dest dst = dest_of((char*)(((u64)e1[1] << 32) | e1[0]), kc & 3u, b);
                asm volatile("s_waitcnt lgkmcnt(0)" ::: "memory");
                if (lane == 0) __hip_atomic_fetch_add(vdone + (k & (NV - 1)), 1u, __ATOMIC_RELAXED, __HIP_MEMORY_SCOPE_WORKGROUP);
                SEG(1);
                settle();
                SEG(3);
                const u32 loff = (u32)(lane >> 4) * dst.LH + (u32)(lane & 15) * 16u;
                if ((kc & 3u) == 2u) {
#pragma unroll
                    for (int g = 0; g < 8; ++g) store_group<AUX_OUT>(dst, loff, g, o[g]);
                } else {
#pragma unroll
                    for (int g = 0; g < 8; ++g) store_group<AUX_DEV>(dst, loff, g, o[g]);
                    owed = (k & 7u) | ((kc >> 2) << 3);
                }
                SEG(2);
            }
            settle();
            STAMP_AT(3);
        }
        return;
    }

    {
        const bool packer = bid >= (u32)NGEMM;
        const u64 tb = __builtin_amdgcn_s_memrealtime();
        const u32 tend = packer ? (u32)NTASK : (u32)(NTASK - PACK_IN), b = (u32)wid;
        u32 it = 0;
        for (u32 task = packer ? (role >> 3) + NTASK - PACK_IN : bid; task < tend; task += packer ? (u32)NPACK : (u32)NGEMM, ++it) {
            if (task >= (u32)ENC_FIRST) while (__builtin_amdgcn_s_memrealtime() - tb < (u64)(ENC_DELAY + (packer ? 0u : task * ENC_PACE / 64u))) __builtin_amdgcn_s_sleep(4);
            if (it == STAMP_IT) STAMP_AT(12);
            const u32 te = task < 16u - PUSHC ? (task + PUSHC) << 4 : (p.task[task >> 1] >> ((task & 1u) * 16u)) & 0xffffu, c = te >> 4, k = te & 15u;
            const InPos ip = in_of(c);
            const u32 dd = (rank + k) & 7u;
            if (ip.isw) {
                const rsrc_t r = mk_rsrc(p.w + (size_t)(ip.q * 128u + (b >> 1) * 32u) * 65536u + (gpanel(dd, ip.s) * 256u + (b & 1u) * 128u) * 2u, 31u * 65536u + 256u);
#pragma unroll
                for (int i = 0; i < 8; ++i) a[i] = __builtin_amdgcn_raw_buffer_load_b128(r, (lane >> 4) * 8 * 65536 + (lane & 15) * 16, i * 65536, AUX_NT);
            } else if (task >= 16u - PUSHC) load_x(px, ip.s, ip.q, b, lane, a);
            if (it == 0) setup(false);
            u32 v[8][4];
            if (ip.isw) {
#pragma unroll
                for (int j = 0; j < 8; ++j)
#pragma unroll
                    for (int e = 0; e < 4; ++e) v[j][e] = vperm(a[2 * e + 1][j >> 1], a[2 * e][j >> 1], (j & 1) ? 0x07060302u : 0x05040100u);
            } else {
#pragma unroll
                for (int j = 0; j < 8; ++j)
#pragma unroll
                    for (int e = 0; e < 4; ++e) v[j][e] = a[j][e];
            }
            const bool own = (k & 7u) == 0u, send = k < 8u;
            const u32 call = rfl(callv) + 1u;
            LAS u32* cnt = (LAS u32*)lds + 4u * it;
            u32 total;
            if (send) {
                encode_block(v, mk_rsrc(ws + OFF_SI + (size_t)((call & 1u) * NTASK + slot_in(ip.isw, ip.s, ip.q, dd)) * CH, CH), b, cnt, tab, wl, lane);
                if (it == STAMP_IT) STAMP_AT(14);
                asm volatile("s_waitcnt vmcnt(0)" ::: "memory");
                if (chunk_done(cnt, lane, total) && (k ? (u32)lane == k : (u32)lane - 1u < 7u))
                    __hip_atomic_store((GAS u64*)(wsl + OFF_GRAN) + (size_t)((call & 1u) * 8u + rank) * NC + c,
                                       ((u64)total << 32) | call, __ATOMIC_RELAXED, __HIP_MEMORY_SCOPE_SYSTEM);
            }
            if (own) {
                const Dest dst = dest_of(frag_base(ws, ip.isw, ip.s, ip.q, 0u, call), ip.isw ? 1u : 0u, b);
#pragma unroll
                for (int g = 0; g < 8; ++g) store_group<AUX_DEV>(dst, (u32)lane * 16u, g, (u32x4){v[g][0], v[g][1], v[g][2], v[g][3]});
                asm volatile("s_waitcnt vmcnt(0)" ::: "memory");
                if (chunk_done(cnt + 2, lane, total)) arrive(ws, ip.bt * 8u, 8u, call, lane);
            }
            if (it == STAMP_IT) STAMP_AT(1);
        }
        if (lane == 0) __hip_atomic_store(callp, callv + 1u, __ATOMIC_RELAXED, __HIP_MEMORY_SCOPE_AGENT);
        if (packer) {
            STAMP_AT(4);
            if (wid >= 4) __builtin_amdgcn_s_setprio(1);
            const u32 call = rfl(callv) + 1u;
            GAS u32* tf = (GAS u32*)(ws + OFF_TILE + b * 4u);
            u32 seen = 0, nanv = 0;
            auto fetch = [&](u32 yc) {
                if (rfl(seen) != call) {
                    const u64 t0 = __builtin_amdgcn_s_memrealtime();
                    while (rfl(__hip_atomic_load(tf + (yc >> 1) * 32u, __ATOMIC_RELAXED, __HIP_MEMORY_SCOPE_AGENT)) != call) {
                        if (give_up(tmo, call, t0)) { nanv = 0x7fc07fc0u; break; }
                        __builtin_amdgcn_s_sleep(POLL_SLEEP);
                    }
                }
                asm volatile("" ::: "memory");
                const Dest sy = dest_of(y_base(p.out, (yc >> 1) & 15u, rank * 16u + (yc >> 5), yc & 1u), 2u, b);
#pragma unroll
                for (int i = 0; i < 8; ++i) a[i] = __builtin_amdgcn_raw_buffer_load_b128(sy.r, (lane >> 4) * 4 * 65536 + (lane & 15) * 16, ((i & 3) + (i >> 2) * 16) * 65536, AUX_SYS);
                seen = __hip_atomic_load(tf + ((yc + NPACK) >> 1) * 32u, __ATOMIC_RELAXED, __HIP_MEMORY_SCOPE_AGENT);
            };
            auto publish = [&](u32 yc) {
                u32 total;
                if (chunk_done((LAS u32*)lds + 4u * it, lane, total) && (u32)lane - 1u < 7u)
                    __hip_atomic_store((GAS u64*)(wsl + OFF_GRAN) + (size_t)((call & 1u) * 8u + rank) * NC + NCIN + yc,
                                       ((u64)total << 32) | call, __ATOMIC_RELAXED, __HIP_MEMORY_SCOPE_SYSTEM);
                ++it;
            };
            u32 yc = role >> 3;
            fetch(yc);
            STAMP_AT(3);
            for (;; yc += NPACK) {
                u32 v[8][4];
#pragma unroll
                for (int j = 0; j < 8; ++j)
#pragma unroll
                    for (int e = 0; e < 4; ++e) v[j][e] = a[j][e] | nanv;
                if (yc + NPACK < (u32)NCY) fetch(yc + NPACK);
                encode_block(v, mk_rsrc(ws + OFF_SY + (size_t)yc * CH, CH), b, (LAS u32*)lds + 4u * it, tab, wl, lane);
                asm volatile("s_waitcnt vmcnt(0)" ::: "memory");
                publish(yc);
                if (yc + NPACK >= (u32)NCY) break;
            }
            __builtin_amdgcn_s_setprio(0);
            if (!PJOIN || NLAST == WB) { STAMP_AT(2); return; }
        }
        STAMP_AT(2);
    }

    const u32 call = rfl(callv) + 1u;
    const int wr = wid >> 2, wc = wid & 3;
    constexpr int HTB = 16384;
#define SA(b, h) (((b) * 2 + (h)) * HTB)
#define SB(b, h) ((4 + (b) * 2 + (h)) * HTB)
#define STAGE(bufoff, gbase) do { _Pragma("unroll") for (int _i = 0; _i < 2; ++_i) \
        __builtin_amdgcn_global_load_lds((const GAS u32*)((gbase) + voff + _i * 8192), (LAS u32*)(lds + (bufoff) + ldsw + _i * 8192), 16, 0, AUX_SYS); } while (0)
#define LDA(dst, b, h) do { _Pragma("unroll") for (int m = 0; m < 4; ++m) _Pragma("unroll") for (int k = 0; k < 2; ++k) dst[m][k] = *(const LAS bf16x8*)(lds + SA(b, h) + aoff + m * 2048 + k * 1024); } while (0)
#define LDB(dst, b, h) do { _Pragma("unroll") for (int n = 0; n < 2; ++n) _Pragma("unroll") for (int k = 0; k < 2; ++k) dst[n][k] = *(const LAS bf16x8*)(lds + SB(b, h) + boff + n * 2048 + k * 1024); } while (0)
#define MMA(ai, bj, At, Bt) do { _Pragma("unroll") for (int _x = 0; _x < 16; ++_x) { const int m = (_x >> 1) & 3, n = (_x ^ (_x >> 1)) & 1, k = _x >> 3; \
        acc[ai][bj][m][n] = __builtin_amdgcn_mfma_f32_16x16x32_bf16(Bt[n][k], At[m][k], acc[ai][bj][m][n], 0, 0, 0); SCHED; } } while (0)
#define MMA2(ai) do { __builtin_amdgcn_s_setprio(1); MMA(ai, 0, At, B0); MMA(ai, 1, At, B1); __builtin_amdgcn_s_setprio(0); } while (0)
#define WAIT_V(n) asm volatile("s_waitcnt vmcnt(" #n ")" ::: "memory")
#define WAIT_L(n) asm volatile("s_waitcnt lgkmcnt(" #n ")" ::: "memory")
#define BAR __builtin_amdgcn_s_barrier()
    const u32 voff = (u32)tid * 16u, ldsw = (u32)wid * 1024u;
    const int aoff = wr * 8192 + lane * 16, boff = wc * 4096 + lane * 16;
    const u32 idx = bid >> 3, hh = (bid >> 2) & 1u;
    u32 ti = (bid & 3u) * 4u + (idx < 4u * (WB / 2) ? idx & 3u : 2u * hh + (idx & 1u)), tj = idx < 4u * (WB / 2) ? 2u * (idx >> 2) + hh : WB - 1u;
    constexpr size_t kstep = 32768, hstep = 16384;
    const GAS char* xf = (const GAS char*)frag_base(ws, false, 0u, 0u, 0u, call);
    const GAS char* wf = (const GAS char*)(ws + OFF_WF);
    GAS u32* flags = (GAS u32*)(ws + OFF_FLAG + (bid & 7u) * 256u) + ((u32)lane & 7u);
    auto arrived = [&](u32 bt) { return (u32)__builtin_ctz(((u32)__builtin_amdgcn_ballot_w64(__hip_atomic_load(flags + bt * 8u, __ATOMIC_RELAXED, __HIP_MEMORY_SCOPE_SYSTEM) != call) & 31u) | 32u); };
#define STAGE_FIRST(pA, pB) do { STAGE(SB(0, 0), pB); STAGE(SA(0, 0), pA); STAGE(SB(0, 1), (pB) + hstep); STAGE(SA(0, 1), (pA) + hstep); \
        STAGE(SB(1, 0), (pB) + kstep); STAGE(SA(1, 0), (pA) + kstep); STAGE(SB(1, 1), (pB) + hstep + kstep); } while (0)
    u32 nwait = 0, bad = 0, staged = 0;
    auto draw_ticket = [&]() { return __hip_atomic_fetch_add((GAS u32*)(ws + OFF_TICK + 8u * 128u), 1u, __ATOMIC_RELAXED, __HIP_MEMORY_SCOPE_AGENT) - (call - 1u) * (NGEMM + (PJOIN ? NPACK : 0)); };
    auto last_tile = [&](u32 tk, u32& i, u32& j) { const u32 gr = tk / (4u * NLAST), r = tk - gr * 4u * NLAST; i = (gr * 4u + (r & 3u)) & 15u; j = gr < 4u ? (NB - 1u) * WB + (r >> 2) : (u32)NP; };
    u32 round = 0;
    if (bid >= (u32)NGEMM) {
        if (tid == 0) *(volatile LAS u32*)(ctl + 2) = draw_ticket();
        asm volatile("s_waitcnt lgkmcnt(0)" ::: "memory"); BAR; asm volatile("" ::: "memory");
        last_tile(lds_ld(ctl + 2), ti, tj);
        round = NB - 1u;
    }
    for (; tj < (u32)NP; ++round) {
        const GAS char* cA = wf + (size_t)tj * (2u << 20);
        const GAS char* cB = xf + (size_t)ti * (2u << 20);
        u32 ni = ti, nj = tj + WB;
        f32x4 acc[2][2][4][2];
#pragma unroll
        for (int a = 0; a < 2; ++a)
#pragma unroll
            for (int b = 0; b < 2; ++b)
#pragma unroll
                for (int m = 0; m < 4; ++m)
#pragma unroll
                    for (int n = 0; n < 2; ++n) acc[a][b][m][n] = (f32x4){0.f, 0.f, 0.f, 0.f};
        bf16x8 At[4][2], B0[2][2], B1[2][2];
        for (u32 q0 = 0; q0 < 5u;) {
            const u32 k0 = q0 ? q0 * 14u - 6u : 0u;
            if (!staged) {
                if (wid == 0) {
                    const u64 t0 = __builtin_amdgcn_s_memrealtime();
                    u32 q1;
                    for (u32 spin = 0;; ++spin) {
                        q1 = arrived(round);
                        if (q1 > q0) break;
                        if ((spin & 7u) == 7u && give_up(tmo, call, t0)) { q1 = 5u | 8u; break; }
                        __builtin_amdgcn_s_sleep(POLL_SLEEP);
                    }
                    if (lane == 0) *(volatile LAS u32*)(ctl + (nwait & 1u)) = q1;
                    asm volatile("s_waitcnt lgkmcnt(0)" ::: "memory");
                }
                asm volatile("" ::: "memory"); BAR; asm volatile("" ::: "memory");
                const u32 te = lds_ld(ctl + (nwait & 1u));
                ++nwait; bad |= te >> 3;
                staged = te & 7u;
                STAGE_FIRST(cA + (size_t)k0 * kstep, cB + (size_t)k0 * kstep);
            }
            if (k0 == 0u && round < 4u) STAMP_AT(3 + round * 2);
            q0 = staged & 7u;
            const u32 k1 = q0 * 14u - 6u;
            if (wr == 1) BAR;
            bool lax = (staged >> 3) != 0u;
            if (lax) WAIT_V(22); else WAIT_V(6);
            BAR; BAR;
            for (u32 t = k0; t < k1; t += 2u) {
                const u32 t2 = t + 2u < k1 ? t + 2u : t;
                const GAS char* a1 = cA + (size_t)(t + 1u) * kstep;
                const GAS char* a2 = cA + (size_t)t2 * kstep; const GAS char* b2 = cB + (size_t)t2 * kstep;
                const GAS char* a3 = a2 + kstep; const GAS char* b3 = b2 + kstep;
                LDB(B0, 0, 0); LDB(B1, 0, 1); LDA(At, 0, 0); STAGE(SA(1, 1), a1 + hstep);
                if (lax) WAIT_V(24); else WAIT_V(8);
                WAIT_L(0); BAR; SCHED; MMA2(0); BAR; SCHED;
                LDA(At, 0, 1); STAGE(SB(0, 0), b2); STAGE(SB(0, 1), b2 + hstep); STAGE(SA(0, 0), a2);
                if (lax) WAIT_V(24); else WAIT_V(8);
                lax = false;
                WAIT_L(0); BAR; SCHED; MMA2(1); BAR; SCHED;
                LDB(B0, 1, 0); LDB(B1, 1, 1); LDA(At, 1, 0); STAGE(SA(0, 1), a2 + hstep);
                WAIT_V(8); WAIT_L(0); BAR; SCHED; MMA2(0); BAR; SCHED;
                LDA(At, 1, 1); STAGE(SB(1, 0), b3); STAGE(SB(1, 1), b3 + hstep); STAGE(SA(1, 0), a3);
                WAIT_V(8); WAIT_L(0); BAR; SCHED; MMA2(1); BAR; SCHED;
            }
            const bool last = q0 == 5u, draw = NLAST < WB && last && round == NB - 2u;
            if (wid == 0) {
                u32 tk = 0;
                if (draw && lane == 0) tk = draw_ticket();
                const u32 q1 = arrived(last ? round + 1u : round);
                if (lane == 0) *(volatile LAS u32*)(ctl + (nwait & 1u)) = q1 | (tk << 8);
                asm volatile("s_waitcnt lgkmcnt(0)" ::: "memory");
            }
            WAIT_V(0);
            if (wr == 0) BAR;
            BAR;
            asm volatile("" ::: "memory");
            staged = lds_ld(ctl + (nwait & 1u));
            ++nwait;
            if (draw) last_tile(staged >> 8, ni, nj);
            staged &= 7u;
            if (staged <= (last ? 0u : q0) || (last && nj >= (u32)NP)) staged = 0u;
            if (staged) STAGE_FIRST(last ? wf + (size_t)nj * (2u << 20) : cA + (size_t)k1 * kstep, last ? xf + (size_t)ni * (2u << 20) : cB + (size_t)k1 * kstep);
        }
        if (round < 4u) STAMP_AT(4 + round * 2);
#pragma unroll
        for (int bj = 0; bj < 2; ++bj) {
            const Dest dst = dest_of(y_base(p.out, ti, gpanel(rank, tj), bj), 2u, (u32)wid);
            const u32 loff = (u32)(lane >> 4) * dst.LH + (u32)(lane & 15) * 16u;
            u32x4 v[8];
#pragma unroll
            for (int g = 0; g < 8; ++g)
#pragma unroll
                for (int q = 0; q < 4; ++q) {
                    typedef float f32x2 __attribute__((ext_vector_type(2)));
                    typedef __bf16 bf16x2 __attribute__((ext_vector_type(2)));
                    const f32x2 f = {acc[q >> 1][bj][(q & 1) * 2][g >> 2][g & 3], acc[q >> 1][bj][(q & 1) * 2 + 1][g >> 2][g & 3]};
                    v[g][q] = __builtin_bit_cast(u32, __builtin_convertvector(f, bf16x2));
                }
            if (bad) {
#pragma unroll
                for (int g = 0; g < 8; ++g) v[g] = (u32x4){0x7fc07fc0u, 0x7fc07fc0u, 0x7fc07fc0u, 0x7fc07fc0u};
            }
            if (tj < (u32)POWN) {
#pragma unroll
                for (int g = 0; g < 8; ++g) store_group<AUX_DEV>(dst, loff, g, v[g]);
            } else if (staged) {
#pragma unroll
                for (int g = 0; g < 8; ++g) store_group<AUX_NT>(dst, loff, g, v[g]);
            } else {
#pragma unroll
                for (int g = 0; g < 8; ++g) store_group<AUX_OUT>(dst, loff, g, v[g]);
            }
        }
        if (tj < (u32)POWN) {
            asm volatile("s_waitcnt vmcnt(0)" ::: "memory");
            if (lane == 0) __hip_atomic_store((GAS u32*)(ws + OFF_TILE + (tj * 16u + ti) * 128u + (u32)wid * 4u), call, __ATOMIC_RELAXED, __HIP_MEMORY_SCOPE_AGENT);
        } else if (staged) staged |= 8u;
        ti = ni; tj = nj;
    }
    STAMP_AT(15);
}

static void build_tables(Params& p) {
    unsigned char* role = (unsigned char*)p.role;
    for (int i = 0; i < 64; ++i) role[i] = (unsigned char)(i < NPULL ? (1 + i % 7) | (i / 7) << 3 : (i - NPULL) << 3);
    {
        constexpr int SL = (256 - NGEMM) / 2;
        static_assert(4 * PULL_WG_PER_PEER <= SL, "roles");
        int cnt[2] = {0, 0}, np = 0;
        for (int j = 0; j < PULL_WG_PER_PEER; ++j)
            for (int rel = 1; rel < 8; ++rel) {
                const int h = (((0xb2 >> p.rank) ^ (0xb2 >> ((p.rank + rel) & 7))) & 1), e = cnt[h]++;
                role[(e >> 2) * 8 + h * 4 + ((e + (h ? e >> 2 : 0)) & 3)] = (unsigned char)(rel | j << 3);
            }
        for (int h = 0; h < 2; ++h) for (int e = cnt[h]; e < SL; ++e) role[(e >> 2) * 8 + h * 4 + (e & 3)] = (unsigned char)(np++ << 3);
    }
    unsigned short* t = (unsigned short*)p.task;
    int n = 0;
    for (u32 c = PUSHC; c < (u32)NCIN; ++c) {
        if (c == 16u) for (u32 e = 0; e < (u32)PUSHC; ++e) t[n++] = (unsigned short)(e << 4 | 8u);
        const InPos ip = in_of(c);
        if (ip.isw && ip.s < (u32)POWN) for (u32 k = 1; k <= 8; ++k) t[n++] = (unsigned short)(c << 4 | k);
        else t[n++] = (unsigned short)(c << 4);
    }
}

extern "C" void kernel_launch(int rank, int nranks, ncclComm_t comm, hipStream_t stream,
                              void* const* d_in, const int* in_sizes, int n_in, void* d_out, int out_size,
                              const int* in_dtypes, int out_dtype, void* const* d_out_all,
                              void* const* d_ws_all, size_t ws_bytes) {
    if (nranks != NR || (unsigned)rank >= (unsigned)NR || ws_bytes < WS_NEED || n_in != 2) { fprintf(stderr, "unsupported configuration\n"); return; }
    Params p;
    p.x = (const char*)d_in[0]; p.w = (const char*)d_in[1]; p.out = (char*)d_out; p.rank = rank; p.pad = 0;
    for (int i = 0; i < NR; ++i) p.ws[i] = (char*)d_ws_all[(rank + i) % NR];
    build_tables(p);
    hipLaunchKernelGGL(gemm_ar_fused, dim3(256), dim3(512), 0, stream, p);
    HIP_CHECK(hipGetLastError());
}
```

```cpp
#include <hip/hip_runtime.h>
#include <rccl/rccl.h>
#include <cstdio>

#define HIP_CHECK(expr) do { hipError_t _e = (expr); if (_e != hipSuccess) { \
    fprintf(stderr, "HIP error %s at %s:%d\n", hipGetErrorString(_e), __FILE__, __LINE__); \
    return; } } while (0)

typedef unsigned u32;
typedef unsigned long long u64;
typedef u32 u32x4 __attribute__((ext_vector_type(4)));
typedef u32 u32x2 __attribute__((ext_vector_type(2)));
typedef unsigned short u16x2 __attribute__((ext_vector_type(2)));
typedef short bf16x8 __attribute__((ext_vector_type(8)));
typedef float f32x4 __attribute__((ext_vector_type(4)));
#define LAS __attribute__((address_space(3)))
#define GAS __attribute__((address_space(1)))
typedef __amdgpu_buffer_rsrc_t rsrc_t;

constexpr int NR = 8;
#define PULL_WG_PER_PEER 8
#define PAT 56565
#define LVAR 5
#define PUSHC 5
#define PUSHW 1
#define PACK_IN 112
#define AFFINE 1
#define PJOIN 1
#define DEC_SLEEP 2
#define RHO 6
#define ENC_DELAY 150
#define ENC_PACE 192
#define POLL_SLEEP 8
#define WB 12
constexpr int NPULL = 7 * PULL_WG_PER_PEER, NGEMM = 16 * WB, NPACK = 256 - NGEMM - NPULL;
constexpr int POWN = 16 - RHO, NP = POWN + 8 * RHO;
constexpr int NB = (NP + WB - 1) / WB, N0 = 16 + WB;
constexpr int NCIN = 64 + 4 * NP, NCY = 32 * POWN, NC = NCIN + NCY;
constexpr int NTASK = 16 * 36;
constexpr int NDEC = 6;
constexpr u64 DEADLINE = 100000000ull;
constexpr int PUSHB = (8 * PUSHC + PULL_WG_PER_PEER - 1) / PULL_WG_PER_PEER;
constexpr int NLAST = NP - (NB - 1) * WB;
static_assert(NGEMM >= 192 && NPACK < 32 && NPACK >= 4 && (PACK_IN + NCY) / NPACK < 126 && NB <= 8 && POWN <= WB && PUSHB <= NDEC, "roles");

constexpr int FIXB = 5120, CVAR = 8 * FIXB, CHDR = 32, CH = 66560;
constexpr size_t OFF_CALL = 0;
constexpr size_t OFF_TMO = 131072;
constexpr size_t OFF_TICK = OFF_TMO + 256;
constexpr size_t OFF_RDY = OFF_TICK + 16 * 128;
constexpr size_t OFF_FLAG = OFF_RDY + 64 * 128;
constexpr size_t OFF_TILE = OFF_FLAG + 8 * 256;
constexpr size_t OFF_GRAN = 196608;
constexpr size_t OFF_DBG = 524288;
constexpr size_t OFF_XF = 1ull << 20;
constexpr size_t OFF_WF = OFF_XF + (64ull << 20);
constexpr size_t OFF_SY = OFF_WF + NP * (2ull << 20);
constexpr size_t OFF_SI = OFF_SY + (size_t)NCY * CH;
constexpr size_t WS_NEED = OFF_SI + 2ull * NTASK * CH;

constexpr int A2B = 2064, A3B = 528, VARB = 912, WLDS = A2B + A3B + VARB, N2MAX = 2032, N3MAX = 496, UBITS = 2016, NPATCH = 31;
constexpr int CTL_OFF = 2048, CTL_WORDS = 128, WL_OFF = 2560, BIG_OFF = 30720, LDS_BYTES = BIG_OFF + 131072;
static_assert(WL_OFF + 8 * WLDS <= BIG_OFF && LDS_BYTES <= 163840, "LDS map");
constexpr int NF = 16, NV = 4, VSLOT = 8192, VRING = NF * FIXB;
constexpr int AUX_SYS = 17, AUX_DEV = 16, AUX_OUT = 18, AUX_NT = 2;

struct Params {
    int rank; int pad;
    const char* x; const char* w; char* out;
    char* ws[NR];
    u32 role[16];
    u32 task[NTASK / 2];
};

__device__ __forceinline__ u32 rfl(u32 x) { return (u32)__builtin_amdgcn_readfirstlane((int)x); }
__device__ __forceinline__ rsrc_t mk_rsrc(const void* p, u32 bytes) {
    const u64 a = (u64)p; const u64 u = ((u64)rfl((u32)(a >> 32)) << 32) | rfl((u32)a);
    return __builtin_amdgcn_make_buffer_rsrc((void*)u, (short)0, (int)rfl(bytes), 0x00020000);
}
__device__ __forceinline__ u32 vperm(u32 a, u32 b, u32 s) { return __builtin_amdgcn_perm(a, b, s); }
__device__ __forceinline__ u32 wave_incl_scan(u32 x) {
    x += (u32)__builtin_amdgcn_update_dpp(0, (int)x, 0x111, 0xf, 0xf, false);
    x += (u32)__builtin_amdgcn_update_dpp(0, (int)x, 0x112, 0xf, 0xf, false);
    x += (u32)__builtin_amdgcn_update_dpp(0, (int)x, 0x114, 0xf, 0xf, false);
    x += (u32)__builtin_amdgcn_update_dpp(0, (int)x, 0x118, 0xf, 0xf, false);
    x += (u32)__builtin_amdgcn_update_dpp(0, (int)x, 0x142, 0xa, 0xf, false);
    x += (u32)__builtin_amdgcn_update_dpp(0, (int)x, 0x143, 0xc, 0xf, false);
    return x;
}
__device__ __forceinline__ u32 pk_sub(u32 a, u32 b) { return __builtin_bit_cast(u32, (u16x2)(__builtin_bit_cast(u16x2, a) - __builtin_bit_cast(u16x2, b))); }
template <int AUX>
__device__ __forceinline__ void st128(u32x4 x, rsrc_t r, int voff, int soff) {
    __builtin_amdgcn_raw_buffer_store_b128(x, r, voff, soff, AUX);
    asm volatile("s_nop 1" :: "v"(x) : "memory");
}
#define WAVE_FENCE() __builtin_amdgcn_fence(__ATOMIC_ACQ_REL, "wavefront")

__device__ __forceinline__ void build_table(LAS u32x2* tab, u32 i, bool unpack) {
    u32 n = 0, dep = 0, cmp = 0x0c0c0c0cu;
#pragma unroll
    for (u32 t = 0; t < 4; ++t) {
        const u32 c = (i >> (2 * t)) & 3u;
        dep |= (c ? 4u + c : n) << (8 * t);
        if (!c) { cmp = (cmp & ~(0xffu << (8 * n))) | (t << (8 * n)); ++n; }
    }
    tab[i] = (u32x2){unpack ? dep : cmp, n};
}
__device__ __forceinline__ void build_table3(LAS u32x2* tab3, u32 i) {
    u64 r = 0; u32 n = 0, run = 0;
#pragma unroll
    for (u32 t = 0; t < 8; ++t) { if ((i >> t) & 1u) { r |= (u64)run << (8 * n); ++n; run = 0; } else ++run; }
    tab3[i] = (u32x2){(u32)r, (u32)(r >> 32)};
}
__device__ __forceinline__ void or3(LAS u32* q, u64 x, u32 sh) {
    const u64 y = x << sh;
    __hip_atomic_fetch_or(q, (u32)y, __ATOMIC_RELAXED, __HIP_MEMORY_SCOPE_WORKGROUP);
    __hip_atomic_fetch_or(q + 1, (u32)(y >> 32), __ATOMIC_RELAXED, __HIP_MEMORY_SCOPE_WORKGROUP);
    __hip_atomic_fetch_or(q + 2, (u32)(((x >> 32) << sh) >> 32), __ATOMIC_RELAXED, __HIP_MEMORY_SCOPE_WORKGROUP);
}
__device__ __forceinline__ u32 code_byte(u32 hi, u32 lo, u32 x) {
    const u32 f = vperm(hi, lo, x);
    return __builtin_amdgcn_udot4((f ^ (f >> 2)) & 0x03030303u, 0x40100401u, 0u, false);
}
template <bool PRED>
__device__ __forceinline__ void or_bytes(u32 pos, u32 x) {
    const u64 y = (u64)x << ((pos & 3u) * 8u);
    LAS u32* q = (LAS u32*)(size_t)(pos & ~3u);
    if (!PRED || x) {
        __hip_atomic_fetch_or(q, (u32)y, __ATOMIC_RELAXED, __HIP_MEMORY_SCOPE_WORKGROUP);
        __hip_atomic_fetch_or(q + 1, (u32)(y >> 32), __ATOMIC_RELAXED, __HIP_MEMORY_SCOPE_WORKGROUP);
    }
}
__device__ __forceinline__ u32 get_bytes(u32 pos) {
    const LAS u32* q = (const LAS u32*)(size_t)(pos & ~3u);
    return __builtin_amdgcn_alignbyte(q[1], q[0], pos);
}
__device__ __forceinline__ u32 la(const LAS void* p) { return (u32)(size_t)p; }
__device__ __forceinline__ u32 pack_level2(const LAS u32x2* tab, const LAS char* src, u32 n, LAS u32* end, LAS char* dst, int lane) {
    u32 m = 0;
    for (u32 i0 = 0; i0 < n; i0 += 1024u) {
        const u32 i = i0 + (u32)lane * 16u, hi = 0x02010000u, lo = 0x00030000u;
        const u32x4 x = *(const LAS u32x4*)(src + (i < n ? i : 0u));
        u32 cw = code_byte(hi, lo, x[0]) | (code_byte(hi, lo, x[1]) << 8) | (code_byte(hi, lo, x[2]) << 16) | (code_byte(hi, lo, x[3]) << 24);
        const u32 valid = i < n ? n - i : 0u;
        if (valid < 16u) cw |= 0x55555555u << (2u * valid);
        u32x2 e[4]; u32 c[5]; c[0] = 0;
#pragma unroll
        for (int j = 0; j < 4; ++j) { e[j] = tab[(cw >> (8 * j)) & 0xffu]; c[j + 1] = c[j] + e[j][1]; }
        const u32 incl = wave_incl_scan(c[4]), p0 = la(dst) + m + incl - c[4];
        if (i < n) *(end - 1 - (i >> 4)) = cw;
        m += (u32)__builtin_amdgcn_readlane((int)incl, 63);
        if (m <= (u32)N3MAX) {
#pragma unroll
            for (int j = 0; j < 4; ++j) or_bytes<true>(p0 + c[j], vperm(0u, x[j], e[j][0]));
        }
    }
    WAVE_FENCE();
    return m;
}
__device__ __forceinline__ void pack_level3(const LAS char* src, u32 n, LAS u32* ub, LAS u32* pt, u32 T, int lane, u32& bits, u32& np) {
    for (u32 i0 = 0; i0 < n; i0 += 256u) {
        const u32 i = i0 + (u32)lane * 4u;
        const u32 x = *(const LAS u32*)(src + (i < n ? i : 0u));
        u32 cum = 0, pf = 0; u64 m = 0;
#pragma unroll
        for (u32 t = 0; t < 4; ++t) {
            const u32 r = ((x >> (8 * t)) - 8u) & 0xffu;
            const bool valid = i + t < n, big = r > 15u;
            cum += valid && !big ? r : 0u;
            m |= (u64)(valid ? 1u : 0u) << cum;
            cum += valid ? 1u : 0u;
            pf |= valid && big ? 1u << t : 0u;
        }
        const u32 incl = wave_incl_scan(cum), p = bits + incl - cum;
        bits += (u32)__builtin_amdgcn_readlane((int)incl, 63);
        if (bits <= (u32)UBITS && cum) or3(ub + (p >> 5), m, p & 31u);
        if (__builtin_amdgcn_ballot_w64(pf != 0u)) {
#pragma unroll
            for (u32 t = 0; t < 4; ++t) {
                const u64 bm = __builtin_amdgcn_ballot_w64((pf >> t) & 1u);
                const u32 at = np + __builtin_amdgcn_mbcnt_hi((u32)(bm >> 32), __builtin_amdgcn_mbcnt_lo((u32)bm, 0u));
                if ((pf >> t) & 1u) pt[at] = ((i + t) << 8) | ((T - (x >> (8 * t))) & 0xffu);
                np += (u32)__builtin_popcountll(bm);
            }
        }
    }
    WAVE_FENCE();
}
__device__ __forceinline__ void encode_block(const u32 (&v)[8][4], rsrc_t ch, u32 b, LAS u32* alloc, const LAS u32x2* tab, LAS char* wl, int lane) {
#pragma unroll
    for (int j = 0; j < 4; ++j) if (j < 3 || lane < (WLDS - 3072) / 16) *(LAS u32x4*)(wl + j * 1024 + lane * 16) = (u32x4){0, 0, 0, 0};
    u32 s = 0;
#pragma unroll
    for (int g = 0; g < 8; ++g) s = __builtin_amdgcn_udot4(__builtin_amdgcn_alignbit(v[g][0], v[g][0], 31), 0x01000100u, s, false);
    u32 T = ((u32)__builtin_amdgcn_readlane((int)wave_incl_scan(s), 63) >> 10) + 5u;
    T = T > 255u ? 255u : (T < 23u ? 23u : T);
    const u32 tt = (T * 256u + 255u) * 0x10001u;
    const int fo = (int)(b * FIXB);
    u32 U[16], l1[4] = {0, 0, 0, 0}, c[17];
    u32x2 e[16];
#pragma unroll
    for (int q = 0; q < 4; ++q) {
        u32x4 sm;
#pragma unroll
        for (int j = 0; j < 4; ++j) {
            const int k = 4 * q + j;
            const u32 a0 = v[k >> 1][(k & 1) * 2], a1 = v[k >> 1][(k & 1) * 2 + 1];
            const u32 r0 = __builtin_amdgcn_alignbit(a0, a0, 31), r1 = __builtin_amdgcn_alignbit(a1, a1, 31);
            sm[j] = vperm(r1, r0, 0x06040200u);
            U[k] = vperm(pk_sub(tt, r1), pk_sub(tt, r0), 0x07050301u);
            const u32 cb = code_byte(0x00000302u, 0x01000000u, U[k]);
            l1[q] |= cb << (8 * j);
            e[k] = tab[cb];
        }
        st128<AUX_SYS>(sm, ch, lane * 16 + q * 1024, fo);
    }
    st128<AUX_SYS>((u32x4){l1[0], l1[1], l1[2], l1[3]}, ch, lane * 16 + 4096, fo);
    __builtin_amdgcn_sched_barrier(0);
    c[0] = 0;
#pragma unroll
    for (int k = 0; k < 16; ++k) c[k + 1] = c[k] + e[k][1];
    const u32 incl = wave_incl_scan(c[16]), p0 = la(wl) + incl - c[16];
    const u32 n2 = (u32)__builtin_amdgcn_readlane((int)incl, 63);
    bool raw = n2 > (u32)N2MAX;
    u32 d2 = 0, d3 = 0, np = 0;
    LAS u32* var = (LAS u32*)(wl + A2B + A3B);
    if (!raw) {
        WAVE_FENCE();
#pragma unroll
        for (int k = 0; k < 16; ++k) or_bytes<false>(p0 + c[k], vperm(0u, U[k], e[k][0]));
        WAVE_FENCE();
        d2 = (n2 + 15u) >> 4;
        const u32 n3 = pack_level2(tab, wl, n2, var + d2, wl + A2B, lane);
        raw = n3 > (u32)N3MAX;
        if (!raw) {
            u32 bits = 0;
            pack_level3(wl + A2B, n3, var + d2, (LAS u32*)wl, T, lane, bits, np);
            raw = bits > (u32)UBITS || np > (u32)NPATCH;
            d3 = (bits + 31u) >> 5;
            if (!raw && (u32)lane < np) var[d2 + d3 + (u32)lane] = ((const LAS u32*)wl)[lane];
            WAVE_FENCE();
        }
    }
    const u32 vd = d2 + d3 + np;
    u32 off = 0;
    if (lane == 0) off = __hip_atomic_fetch_add(alloc, raw ? 3072u : vd * 4u, __ATOMIC_RELAXED, __HIP_MEMORY_SCOPE_WORKGROUP);
    off = rfl(off) + CHDR;
    const int vo = CVAR + (int)off;
    if (lane == 0) __builtin_amdgcn_raw_buffer_store_b32(raw ? off << 17 : (T | (d3 << 8) | (np << 14) | ((off + d2 * 4u) << 17)), ch, 0, CVAR + (int)b * 4, AUX_SYS);
    if (raw) {
        asm volatile("s_waitcnt vmcnt(0)" ::: "memory");
#pragma unroll
        for (int g = 0; g < 8; ++g)
            st128<AUX_SYS>((u32x4){v[g][0], v[g][1], v[g][2], v[g][3]}, ch, lane * 16 + (g < 5 ? g : g - 5) * 1024, g < 5 ? fo : vo);
    } else {
        for (u32 o = (u32)lane; o < vd; o += 64u) __builtin_amdgcn_raw_buffer_store_b32(var[o], ch, (int)o * 4, vo, AUX_SYS);
    }
    WAVE_FENCE();
}
__device__ __forceinline__ bool chunk_done(LAS u32* cnt, int lane, u32& total) {
    u32 n = 0;
    if (lane == 0) n = __hip_atomic_fetch_add(cnt + 1, 1u, __ATOMIC_RELAXED, __HIP_MEMORY_SCOPE_WORKGROUP);
    if (rfl(n) != 7u) return false;
    total = rfl(*(volatile LAS u32*)cnt) + CHDR;
    return true;
}

struct Dest { rsrc_t r; u32 A, B, C, LH; };
__device__ __forceinline__ Dest dest_of(char* base, u32 kind, u32 b) {
    if (kind == 0u) return Dest{mk_rsrc(base + b * 4096u, 32768u + 4096u), 1024u, 32768u, 2048u, 256u};
    if (kind == 1u) return Dest{mk_rsrc(base + (b >> 2) * 32768u + (b & 1u) * 8192u + (b & 2u) * 512u, 16384u + 6144u + 1024u), 2048u, 4096u, 16384u, 256u};
    return Dest{mk_rsrc(base + (b & 3u) * (32u * 65536u) + (b >> 2) * 256u, 31u * 65536u + 256u), 65536u, 131072u, 16u * 65536u, 4u * 65536u};
}
template <int AUX>
__device__ __forceinline__ void store_group(const Dest& d, u32 loff, int g, u32x4 x) {
    st128<AUX>(x, d.r, (int)loff, (int)((g & 1 ? d.A : 0u) + (g & 2 ? d.B : 0u) + (g & 4 ? d.C : 0u)));
}

#define SCHED __builtin_amdgcn_sched_barrier(0)
__device__ __forceinline__ u32x2 get2(u32 pos) { const LAS u32* q = (const LAS u32*)(size_t)(pos & ~3u); return (u32x2){q[0], q[1]}; }
__device__ __forceinline__ void spread3(u32 w, u32 start, u32 n, u32x2 runs, u32 t8, u32 p) {
    if (n) {
        const u32 carry = (u32)__builtin_clz((w << 8) | start);
        const u64 x = ((u64)(t8 - runs[1]) << 32) | (t8 - runs[0] - carry);
        const u32 sh = 64u - 8u * n;
        or3((LAS u32*)(size_t)(p & ~3u), (x << sh) >> sh, (p & 3u) * 8u);
    }
}
template <class Out>
__device__ __forceinline__ void decode_block(const u32x4 (&d)[5], u32 hdr, const LAS u32* ub, const LAS u32x2* tab, const LAS u32x2* tab3, LAS char* wl, int lane, Out out) {
    const u32 T = hdr & 0xffu, d3 = (hdr >> 8) & 0x3fu, np = (hdr >> 14) & 0x1fu;
    auto ex = [&](u32 a, u32 b, u32 c) { return ((T - a) << 8) | ((T - b) << 16) | ((T - c) << 24); };
    const u32 t1 = ex(3, 4, 5), t2 = ex(6, 7, 2), t8 = (T - 8u) * 0x01010101u;
    const u32 w2 = *(ub - 1 - lane), p3i = la(ub) + (u32)lane - 3u;
    u32x2 e[16], e2[4], g[16];
    g[0] = get2(p3i);
    if (lane < A3B / 16) *(LAS u32x4*)(wl + A2B + lane * 16) = (u32x4){0, 0, 0, 0};
#pragma unroll
    for (int k = 0; k < 8; ++k) e[k] = tab[(d[4][k >> 2] >> (8 * (k & 3))) & 0xffu];
    SCHED;
    const u32 w3 = __builtin_amdgcn_alignbyte(g[0][1], g[0][0], p3i), x3 = (u32)lane < d3 * 4u ? w3 >> 24 : 0u;
    const u32x2 e3 = tab3[x3];
#pragma unroll
    for (int j = 0; j < 4; ++j) e2[j] = tab[(w2 >> (8 * j)) & 0xffu];
#pragma unroll
    for (int k = 8; k < 16; ++k) e[k] = tab[(d[4][k >> 2] >> (8 * (k & 3))) & 0xffu];
    SCHED;
    u32 c[17], f[5];
    c[0] = 0; f[0] = 0;
#pragma unroll
    for (int k = 0; k < 16; ++k) c[k + 1] = c[k] + e[k][1];
#pragma unroll
    for (int j = 0; j < 4; ++j) f[j + 1] = f[j] + e2[j][1];
    const u32 n3 = (u32)__builtin_popcount(x3);
    const u32 pk = c[16] | (f[4] << 11) | (n3 << 22);
    const u32 incl = wave_incl_scan(pk), ex0 = incl - pk, tot = (u32)__builtin_amdgcn_readlane((int)incl, 63);
    const u32 a3 = la(wl) + A2B, p1 = la(wl) + (ex0 & 0x7ffu), p2 = a3 + ((ex0 >> 11) & 0x1ffu);
    WAVE_FENCE();
    spread3(w3, lane < 3 ? 0x80000000u >> (8 * lane) : 0x80u, n3, e3, t8, a3 + ((ex0 >> 22) & 0x1ffu));
    for (u32 i0 = 64u, m = tot >> 22; i0 < d3 * 4u; i0 += 64u) {
        const u32 w = get_bytes(p3i + i0), x = i0 + (u32)lane < d3 * 4u ? w >> 24 : 0u, n = (u32)__builtin_popcount(x), in = wave_incl_scan(n);
        spread3(w, 0x80u, n, tab3[x], t8, a3 + ((m + in - n) & 0x1ffu));
        m += (u32)__builtin_amdgcn_readlane((int)in, 63);
    }
    WAVE_FENCE();
    if ((u32)lane < np) { const u32 pw = ub[d3 + (u32)lane]; *(LAS unsigned char*)(wl + A2B + ((pw >> 8) & 0x1ffu)) = (unsigned char)pw; }
    WAVE_FENCE();
#pragma unroll
    for (int j = 0; j < 4; ++j) { f[j] += p2; g[j] = get2(f[j]); }
    SCHED;
#pragma unroll
    for (int k = 0; k < 16; ++k) c[k] += p1;
    SCHED;
    u32x4 a2;
#pragma unroll
    for (int j = 0; j < 4; ++j) a2[j] = vperm(t2, __builtin_amdgcn_alignbyte(g[j][1], g[j][0], f[j]), e2[j][0]);
    *(LAS u32x4*)(wl + lane * 16) = a2;
    if ((tot & 0x7ffu) > 1024u) {
        const u32 cw = *(ub - 65 - lane);
        f[0] = 0;
#pragma unroll
        for (int j = 0; j < 4; ++j) { e2[j] = tab[(cw >> (8 * j)) & 0xffu]; f[j + 1] = f[j] + e2[j][1]; }
        const u32 p = a3 + (((tot >> 11) + wave_incl_scan(f[4]) - f[4]) & 0x1ffu);
#pragma unroll
        for (int j = 0; j < 4; ++j) a2[j] = vperm(t2, get_bytes(p + f[j]), e2[j][0]);
        *(LAS u32x4*)(wl + 1024 + lane * 16) = a2;
    }
    WAVE_FENCE();
#pragma unroll
    for (int k = 0; k < 16; ++k) g[k] = get2(c[k]);
    SCHED;
    u32x4 o;
#pragma unroll
    for (int k = 0; k < 16; ++k) {
        const u32 x = vperm(t1, __builtin_amdgcn_alignbyte(g[k][1], g[k][0], c[k]), e[k][0]);
        const u32 smk = d[k >> 2][k & 3];
        const u32 r0 = vperm(x, smk, 0x05010400u), r1 = vperm(x, smk, 0x07030602u);
        o[2 * (k & 1)] = __builtin_amdgcn_alignbit(r0, r0, 1);
        o[2 * (k & 1) + 1] = __builtin_amdgcn_alignbit(r1, r1, 1);
        if (k & 1) out(k >> 1, o);
    }
    WAVE_FENCE();
}

__device__ __forceinline__ char* frag_base(char* ws, bool isw, u32 s, u32 q, u32 rel, u32 call) {
    return ws + (isw ? OFF_WF : OFF_XF + (size_t)(call & 1u) * (32u << 20)) + (size_t)(s * 64u + (rel ? 6u + 14u * q + 2u * rel : 2u * q)) * 32768u;
}
__device__ __forceinline__ char* y_base(char* out, u32 i, u32 gp, u32 bj) { return out + (size_t)(i * 256u + bj * 128u) * 65536u + gp * 512u; }
__device__ __forceinline__ u32 gpanel(u32 r, u32 j) { return j < (u32)POWN ? r * 16u + j : ((j - POWN) & 7u) * 16u + POWN + ((j - POWN) >> 3); }
__device__ __forceinline__ u32 slot_in(bool isw, u32 s, u32 q, u32 dst) {
    return !isw ? s * 4u + q : (s < (u32)POWN ? 64u + (dst * POWN + s) * 4u + q : 64u + 28u * POWN + s * 4u + q);
}
__host__ __device__ __forceinline__ u32 batch_size(u32 bt) { return bt == 0u ? (u32)N0 : (bt == NB - 1u ? (u32)(NP - (NB - 1) * WB) : (u32)WB); }
struct InPos { bool isw; u32 s, q, bt; };
__host__ __device__ __forceinline__ InPos in_of(u32 c) {
    if (c < 4u * N0) { const u32 q = c / N0, pp = c - q * N0; return InPos{pp >= 16u, pp >= 16u ? pp - 16u : pp, q, 0u}; }
    const u32 bt = (c - 64u) / (4u * WB), r = c - 64u - bt * 4u * WB, q = r / batch_size(bt);
    return InPos{true, bt * WB + r - q * batch_size(bt), q, bt};
}

__device__ __forceinline__ void arrive(char* ws, u32 ci, u32 n, u32 call, int lane) {
    u32 old = 0;
    if (lane == 0) old = __hip_atomic_fetch_add((GAS u32*)(ws + OFF_RDY + ci * 128u), n, __ATOMIC_RELAXED, __HIP_MEMORY_SCOPE_SYSTEM);
    if (rfl(old) + n == call * ((ci & 7u) ? 56u : 32u) * batch_size(ci >> 3) && lane < 8)
        __hip_atomic_store((GAS u32*)(ws + OFF_FLAG + (u32)lane * 256u + ci * 4u), call, __ATOMIC_RELAXED, __HIP_MEMORY_SCOPE_SYSTEM);
}
__device__ __forceinline__ void load_x(const char* x, u32 s, u32 q, u32 b, int lane, u32x4 (&a)[8]) {
    const rsrc_t r = mk_rsrc(x + (size_t)(s * 256u + b * 32u) * 1024u + q * 256u, 31u * 1024u + 256u);
#pragma unroll
    for (int i = 0; i < 8; ++i) a[i] = __builtin_amdgcn_raw_buffer_load_b128(r, (lane & 15) * 1024 + (lane >> 4) * 16 + (i & 3) * 64, (i >> 2) * 16384, 0);
}

__device__ __forceinline__ bool give_up(GAS u32* tmo, u32 call, u64 t0) {
    if (rfl(__hip_atomic_load(tmo, __ATOMIC_RELAXED, __HIP_MEMORY_SCOPE_AGENT)) == call || __builtin_amdgcn_s_memrealtime() - t0 > DEADLINE) {
        __hip_atomic_store(tmo, call, __ATOMIC_RELAXED, __HIP_MEMORY_SCOPE_AGENT);
        return true;
    }
    return false;
}

#define STAMP_IT 0
#define STAMP_AT(i) do { } while (0)

__device__ __forceinline__ u32 lds_ld(LAS u32* p) { return rfl(*(volatile LAS u32*)p); }
__device__ __forceinline__ void dma_piece(rsrc_t r, u32 voff, u32 soff, u32 ldsaddr) {
    u32 keep;
    asm volatile("s_nop 4\n\ts_mov_b32 %0, m0\n\ts_mov_b32 m0, %1\n\ts_nop 0\n\tbuffer_load_dwordx4 %2, %3, %4 offen sc0 sc1 lds\n\ts_mov_b32 m0, %0"
                 : "=&s"(keep) : "s"(ldsaddr), "v"(voff), "s"(r), "s"(soff) : "memory");
}

#define DMA_P(j, w, off) "s_waitcnt vmcnt(" w ")\n\tbuffer_load_dwordx4 %2, %3, " off " sc0 sc1 lds\n\t.if %8 == " #j "\n\tds_max_u32 %6, %7\n\t.endif\n\t"
__device__ __forceinline__ void dma_block(rsrc_t r, u32 voff, u32 soff, u32 ldsaddr, u32 at, u32 val) {
    constexpr int W0 = PAT / 10000, W1 = PAT / 1000 % 10, W2 = PAT / 100 % 10, W3 = PAT / 10 % 10, W4 = PAT % 10;
    constexpr int JP = W0 <= 5 ? 0 : (W1 <= 6 ? 1 : (W2 <= 7 ? 2 : (W3 <= 8 ? 3 : 4)));
    u32 keep;
    asm volatile("s_nop 2\n\ts_mov_b32 %0, m0\n\ts_mov_b32 m0, %1\n\t"
                 DMA_P(0, "%9", "%4 offen") DMA_P(1, "%10", "%4 offen offset:1024") DMA_P(2, "%11", "%4 offen offset:2048") DMA_P(3, "%12", "%4 offen offset:3072")
                 "s_add_u32 m0, m0, 0x1000\n\t" DMA_P(4, "%13", "%5 offen") "s_mov_b32 m0, %0"
                 : "=&s"(keep) : "s"(ldsaddr), "v"(voff), "s"(r), "s"(soff), "s"(soff + 4096u), "v"(at), "v"(val), "n"(JP),
                   "n"(W0), "n"(W1), "n"(W2), "n"(W3), "n"(W4) : "memory", "scc");
}

__global__ void __launch_bounds__(512, 2) gemm_ar_fused(const Params p) {
    __shared__ __attribute__((aligned(16))) char smem[LDS_BYTES];
    LAS char* lds = (LAS char*)smem + BIG_OFF;
    LAS u32x2* tab = (LAS u32x2*)smem;
    LAS u32x2* tab3 = (LAS u32x2*)((LAS char*)smem + WL_OFF);
    const int tid = threadIdx.x, wid = (int)rfl((u32)tid >> 6), lane = tid & 63;
    const u32 bid = blockIdx.x;
    u32 rank = (u32)p.rank, rolew = p.role[((bid - NGEMM) >> 2) & 15u];
    char* ws = p.ws[0]; const char* px = p.x;
    asm volatile("" : "+s"(rank), "+s"(rolew), "+s"(ws), "+s"(px));
    LAS char* wl = (LAS char*)smem + WL_OFF + wid * WLDS;
    LAS u32* ctl = (LAS u32*)((LAS char*)smem + CTL_OFF);
    GAS u32* tmo = (GAS u32*)(ws + OFF_TMO);
    STAMP_AT(0);
    const u32 role = bid >= (u32)NGEMM ? (rolew >> ((bid & 3u) * 8u)) & 0xffu : 0u;
    const u32 pn = (role & 7u) ? (role >> 3) * PUSHB + (u32)wid - 2u : (bid + PUSHC) * 8u + (u32)wid;
    const bool early = (role & 7u) ? wid >= 2 && wid < 2 + PUSHB && pn < 8u * PUSHC : bid < 16u - PUSHC;
    u32x4 a[8];
    if (early) load_x(px, pn >> 3, 0u, pn & 7u, lane, a);
    SCHED;
    GAS u32* callp = (GAS u32*)(ws + OFF_CALL + (bid * 8u + wid) * 64u);
    const u32 callv = __hip_atomic_load(callp, __ATOMIC_RELAXED, __HIP_MEMORY_SCOPE_AGENT);
    char* wsl = ws;
    auto setup = [&](bool unpack) {
        u32 t = (u32)tid;
        asm volatile("" : "+v"(t));
        if (t < 256u) build_table(tab, t, unpack);
        else { if (t < 256u + CTL_WORDS) ctl[t - 256u] = 0u; if (unpack) build_table3(tab3, t - 256u); }
        if (!unpack) {
            ((LAS u32*)lds)[t] = 0u;
#pragma unroll
            for (u32 k = 1; k < 8u; ++k) wsl = (t & 63u) == k ? p.ws[k] : wsl;
        }
        asm volatile("s_waitcnt lgkmcnt(0)" ::: "memory"); __builtin_amdgcn_s_barrier(); asm volatile("" ::: "memory");
    };

    if (role & 7u) {
        const u32 rel = role & 7u, pjj = role >> 3, src = (rank + rel) & 7u;
        char* pws = p.ws[rel];
        setup(true);
        if (early) {
            const u32 call = rfl(callv) + 1u;
            const Dest dst = dest_of(frag_base(pws, false, pn >> 3, 0u, 8u - rel, call), 0u, pn & 7u);
#pragma unroll
            for (int g = 0; g < 8; ++g) asm volatile("" : "+v"(a[g]));
#pragma unroll
            for (int g = 0; g < 8; ++g) { store_group<AUX_SYS>(dst, (u32)lane * 16u, g, a[g]); asm volatile("s_waitcnt vmcnt(%0)" :: "n"(PUSHW - 1) : "memory"); }
            STAMP_AT(12);
            if (lane == 0) __hip_atomic_store(callp, call, __ATOMIC_RELAXED, __HIP_MEMORY_SCOPE_AGENT);
            asm volatile("s_waitcnt vmcnt(0)" ::: "memory");
            STAMP_AT(13);
            arrive(pws, 1u, 1u, call, lane);
        }
        LAS u32* vready = ctl; LAS u32* endk = ctl + 1; LAS u32* landed = ctl + 2; LAS u32* prog = ctl + 5;
        LAS u32* vdone = ctl + 8; LAS u32* sdone = ctl + 16; LAS u32* ffree = ctl + 32;
        LAS u32* q = ctl + 64;
        const u32 l0 = la(lds), voff = (u32)lane * 16u;
        if (wid == 0) {
            const u32 call = rfl(callv) + 1u;
            if (lane == 0) __hip_atomic_store(callp, call, __ATOMIC_RELAXED, __HIP_MEMORY_SCOPE_AGENT);
            GAS u64* gran = (GAS u64*)(ws + OFF_GRAN) + (size_t)((call & 1u) * 8u + src) * NC;
            for (u32 k = 0;; ++k) {
                u32 c = PUSHC + pjj;
                if (k) {
                    while (lds_ld(prog) < k) __builtin_amdgcn_s_sleep(2);
                    u32 tk = 0;
                    if (lane == 0) tk = __hip_atomic_fetch_add((GAS u32*)(ws + OFF_TICK + src * 128u), 1u, __ATOMIC_RELAXED, __HIP_MEMORY_SCOPE_AGENT);
                    c = rfl(tk) - (call - 1u) * (NC - PUSHC) + PULL_WG_PER_PEER + PUSHC;
                }
                u32 fl = 2u, vb = 0, kc = 2u;
                const char* from = pws; char* to = ws;
                if (c < (u32)NC) {
                    if (c >= (u32)NCIN) {
                        const u32 yc = c - NCIN;
                        from = pws + OFF_SY + (size_t)yc * CH;
                        to = y_base(p.out, (yc >> 1) & 15u, src * 16u + (yc >> 5), yc & 1u);
                    } else {
                        const InPos ip = in_of(c);
                        from = pws + OFF_SI + (size_t)((call & 1u) * NTASK + slot_in(ip.isw, ip.s, ip.q, rank)) * CH;
                        to = frag_base(ws, ip.isw, ip.s, ip.q, rel, call);
                        kc = (ip.isw ? 1u : 0u) | ((ip.bt * 8u + ip.q + 1u) << 2);
                    }
                    const u64 t0 = __builtin_amdgcn_s_memrealtime();
                    for (u32 spin = 0;; ++spin) {
                        const u64 g = __hip_atomic_load(gran + c, __ATOMIC_RELAXED, __HIP_MEMORY_SCOPE_SYSTEM);
                        if (rfl((u32)g) == call) { fl = 0u; vb = rfl((u32)(g >> 32)); break; }
                        if ((spin & 15u) == 15u && give_up(tmo, call, t0)) { fl = 1u; break; }
                        __builtin_amdgcn_s_sleep(1);
                    }
                    if (k == 0) STAMP_AT(1);
                }
                while (lds_ld(vdone + (k & (NV - 1))) < 8u * (k / NV)) __builtin_amdgcn_s_sleep(2);
                if (lane == 0) {
                    *(volatile LAS u32x4*)(q + (k & (NV - 1)) * 8u + 4u) = (u32x4){(u32)(u64)to, (u32)((u64)to >> 32), kc, call};
                    *(volatile LAS u32x4*)(q + (k & (NV - 1)) * 8u) = (u32x4){(u32)(u64)from, (u32)((u64)from >> 32), fl | ((k + 1u) << 2), vb};
                }
                if (fl & 2u) { if (lane == 0) *(volatile LAS u32*)endk = k + 1u; break; }
                if (!(fl & 1u)) {
                    const u32 vl = vb > (u32)VSLOT ? (u32)CHDR : (vb + 15u) & ~15u;
                    const rsrc_t r = mk_rsrc(from, CVAR + vl);
                    for (u32 o = 0; o < vl; o += 1024u) { dma_piece(r, voff + CVAR + o, 0u, rfl(l0 + VRING + (k & (NV - 1)) * VSLOT + o)); if (k) asm volatile("s_waitcnt vmcnt(0)" ::: "memory"); }
                    asm volatile("s_waitcnt vmcnt(0)" ::: "memory");
                }
                if (lane == 0) *(volatile LAS u32*)vready = k + 1u;
            }
        } else if (wid == 1) {
            __builtin_amdgcn_s_setprio(3);
            const u32 la_landed = lane ? la(wl) + (u32)lane * 4u : la(landed);
            u32 n = 0;
            auto drain = [&]() { asm volatile("s_waitcnt vmcnt(0)" ::: "memory"); if (lane == 0) __hip_atomic_fetch_max(landed, n, __ATOMIC_RELAXED, __HIP_MEMORY_SCOPE_WORKGROUP); };
            u32x4 e = (u32x4){0, 0, 0, 0};
            u32 fr = 0;
            for (u32 k = 0;; ++k) {
                if (rfl(e[2]) >> 2 != k + 1u) {
                    e = *(volatile LAS u32x4*)(q + (k & (NV - 1)) * 8u);
                    if (rfl(e[2]) >> 2 != k + 1u) {
                        drain();
                        do { __builtin_amdgcn_s_sleep(1); e = *(volatile LAS u32x4*)(q + (k & (NV - 1)) * 8u); } while (rfl(e[2]) >> 2 != k + 1u);
                    }
                }
                const u32 fl = rfl(e[2]);
                if (fl & 2u) break;
                const rsrc_t r = mk_rsrc((const void*)(((u64)e[1] << 32) | e[0]), (fl & 1u) ? 0u : (u32)CVAR);
#pragma unroll
                for (u32 b = 0; b < 8u; ++b, ++n) {
                    const u32 sl = n & (NF - 1), m = n + 1u;
                    if (rfl(fr) + NF <= n && lds_ld(ffree + sl) + NF <= n) { drain(); while (lds_ld(ffree + sl) + NF <= n) __builtin_amdgcn_s_sleep(1); }
                    fr = *(volatile LAS u32*)(ffree + (m & (NF - 1)));
                    if (b == 7u) e = *(volatile LAS u32x4*)(q + ((k + 1u) & (NV - 1)) * 8u);
                    if (b == pjj % LVAR && lane == 0) *(volatile LAS u32*)prog = k + 1u;
                    dma_block(r, voff, b * FIXB, rfl(l0 + sl * FIXB), la_landed, n ? n - 1u : 0u);
                }
            }
            drain();
            STAMP_AT(2);
        } else {
#define SEG(i)
            u32 owed = 0xffffffffu, call = 0;
            auto settle = [&]() {
                if (owed != 0xffffffffu) {
                    asm volatile("s_waitcnt vmcnt(0)" ::: "memory");
                    u32 old = 0;
                    if (lane == 0) old = __hip_atomic_fetch_add(sdone + (owed & 7u), 1u, __ATOMIC_RELAXED, __HIP_MEMORY_SCOPE_WORKGROUP);
                    if ((rfl(old) & 7u) == 7u) arrive(ws, owed >> 3, 8u, call, lane);
                    owed = 0xffffffffu;
                }
            };
            for (u32 n = (u32)wid - 2u;; n += NDEC) {
                const u32 k = n >> 3, b = n & 7u, sl = n & (NF - 1);
                bool end = false;
                for (;;) {
                    const u32x4 st = *(volatile LAS u32x4*)ctl;
                    if (rfl(st[0]) > k && rfl(st[2]) > n) break;
                    if (rfl(st[1]) - 1u <= k) { end = true; break; }
                    settle();
                    __builtin_amdgcn_s_sleep(DEC_SLEEP);
                }
                if (end) break;
                asm volatile("" ::: "memory");
                u32x4 d[5];
                const LAS char* vs = lds + VRING + (k & (NV - 1)) * VSLOT;
                const u32 h = *(const LAS u32*)(vs + b * 4u);
                const u32x4 e0 = *(const LAS u32x4*)(q + (k & (NV - 1)) * 8u), e1 = *(const LAS u32x4*)(q + (k & (NV - 1)) * 8u + 4u);
                d[4] = *(const LAS u32x4*)(lds + sl * FIXB + 4096 + lane * 16);
#pragma unroll
                for (int j = 0; j < 4; ++j) d[j] = *(const LAS u32x4*)(lds + sl * FIXB + j * 1024 + lane * 16);
                asm volatile("s_waitcnt lgkmcnt(0)" ::: "memory");
                if (lane == 0) *(volatile LAS u32*)(ffree + sl) = n + 1u;
                asm volatile("" : "+v"(d[0]), "+v"(d[1]), "+v"(d[2]), "+v"(d[3]), "+v"(d[4]) :: "memory");
                const u32 hdr = rfl(h), off = (hdr >> 19) * 4u, kc = rfl(e1[2]);
                SEG(0);
                call = rfl(e1[3]);
                const bool far = rfl(e0[3]) > (u32)VSLOT;
                const LAS u32* var = (const LAS u32*)(vs + (off <= (u32)VSLOT ? off : 0u));
                u32x4 o[8];
                if (rfl(e0[2]) & 1u) {
#pragma unroll
                    for (int g = 0; g < 8; ++g) o[g] = (u32x4){0x7fc07fc0u, 0x7fc07fc0u, 0x7fc07fc0u, 0x7fc07fc0u};
                } else {
                    if (far) {
                        const rsrc_t r = mk_rsrc((const void*)(((u64)e0[1] << 32) | e0[0]), CH);
#pragma unroll
                        for (int j = 0; j < 3; ++j) o[5 + j] = __builtin_amdgcn_raw_buffer_load_b128(r, (int)(CVAR + off) - ((hdr & 0xffu) ? 512 : 0) + j * 1024 + lane * 16, 0, AUX_SYS);
                        if (lane < VARB / 16) *(LAS u32x4*)(wl + A2B + A3B + lane * 16) = o[5];
                        WAVE_FENCE();
                        var = (const LAS u32*)(wl + A2B + A3B + 512);
                    }
                    if (hdr & 0xffu) decode_block(d, hdr, var, tab, tab3, wl, lane, [&](int g, u32x4 x) { o[g] = x; });
                    else {
#pragma unroll
                        for (int g = 0; g < 5; ++g) o[g] = d[g];
                        if (!far) {
#pragma unroll
                            for (int j = 0; j < 3; ++j)
#pragma unroll
                                for (int i = 0; i < 4; ++i) o[5 + j][i] = *(const volatile LAS u32*)(var + j * 256 + lane * 4 + i);
                        }
                    }
                }
                const Dest dst = dest_of((char*)(((u64)e1[1] << 32) | e1[0]), kc & 3u, b);
                asm volatile("s_waitcnt lgkmcnt(0)" ::: "memory");
                if (lane == 0) __hip_atomic_fetch_add(vdone + (k & (NV - 1)), 1u, __ATOMIC_RELAXED, __HIP_MEMORY_SCOPE_WORKGROUP);
                SEG(1);
                settle();
                SEG(3);
                const u32 loff = (u32)(lane >> 4) * dst.LH + (u32)(lane & 15) * 16u;
                if ((kc & 3u) == 2u) {
#pragma unroll
                    for (int g = 0; g < 8; ++g) store_group<AUX_OUT>(dst, loff, g, o[g]);
                } else {
#pragma unroll
                    for (int g = 0; g < 8; ++g) store_group<AUX_DEV>(dst, loff, g, o[g]);
                    owed = (k & 7u) | ((kc >> 2) << 3);
                }
                SEG(2);
            }
            settle();
            STAMP_AT(3);
        }
        return;
    }

    {
        const bool packer = bid >= (u32)NGEMM;
        const u64 tb = __builtin_amdgcn_s_memrealtime();
        const u32 tend = packer ? (u32)NTASK : (u32)(NTASK - PACK_IN), b = (u32)wid;
        u32 it = 0;
        for (u32 task = packer ? (role >> 3) + NTASK - PACK_IN : bid; task < tend; task += packer ? (u32)NPACK : (u32)NGEMM, ++it) {
            if (task >= 16u - PUSHC) while (__builtin_amdgcn_s_memrealtime() - tb < (u64)(ENC_DELAY + (packer ? 0u : task * ENC_PACE / 64u))) __builtin_amdgcn_s_sleep(4);
            if (it == STAMP_IT) STAMP_AT(12);
            const u32 te = task < 16u - PUSHC ? (task + PUSHC) << 4 : (p.task[task >> 1] >> ((task & 1u) * 16u)) & 0xffffu, c = te >> 4, k = te & 15u;
            const InPos ip = in_of(c);
            const u32 dd = (rank + k) & 7u;
            if (ip.isw) {
                const rsrc_t r = mk_rsrc(p.w + (size_t)(ip.q * 128u + (b >> 1) * 32u) * 65536u + (gpanel(dd, ip.s) * 256u + (b & 1u) * 128u) * 2u, 31u * 65536u + 256u);
#pragma unroll
                for (int i = 0; i < 8; ++i) a[i] = __builtin_amdgcn_raw_buffer_load_b128(r, (lane >> 4) * 8 * 65536 + (lane & 15) * 16, i * 65536, AUX_NT);
            } else if (task >= 16u - PUSHC) load_x(px, ip.s, ip.q, b, lane, a);
            if (it == 0) setup(false);
            u32 v[8][4];
            if (ip.isw) {
#pragma unroll
                for (int j = 0; j < 8; ++j)
#pragma unroll
                    for (int e = 0; e < 4; ++e) v[j][e] = vperm(a[2 * e + 1][j >> 1], a[2 * e][j >> 1], (j & 1) ? 0x07060302u : 0x05040100u);
            } else {
#pragma unroll
                for (int j = 0; j < 8; ++j)
#pragma unroll
                    for (int e = 0; e < 4; ++e) v[j][e] = a[j][e];
            }
            const bool own = (k & 7u) == 0u, send = k < 8u;
            const u32 call = rfl(callv) + 1u;
            LAS u32* cnt = (LAS u32*)lds + 4u * it;
            u32 total;
            if (send) {
                encode_block(v, mk_rsrc(ws + OFF_SI + (size_t)((call & 1u) * NTASK + slot_in(ip.isw, ip.s, ip.q, dd)) * CH, CH), b, cnt, tab, wl, lane);
                if (it == STAMP_IT) STAMP_AT(14);
                asm volatile("s_waitcnt vmcnt(0)" ::: "memory");
                if (chunk_done(cnt, lane, total) && (k ? (u32)lane == k : (u32)lane - 1u < 7u))
                    __hip_atomic_store((GAS u64*)(wsl + OFF_GRAN) + (size_t)((call & 1u) * 8u + rank) * NC + c,
                                       ((u64)total << 32) | call, __ATOMIC_RELAXED, __HIP_MEMORY_SCOPE_SYSTEM);
            }
            if (own) {
                const Dest dst = dest_of(frag_base(ws, ip.isw, ip.s, ip.q, 0u, call), ip.isw ? 1u : 0u, b);
#pragma unroll
                for (int g = 0; g < 8; ++g) store_group<AUX_DEV>(dst, (u32)lane * 16u, g, (u32x4){v[g][0], v[g][1], v[g][2], v[g][3]});
                asm volatile("s_waitcnt vmcnt(0)" ::: "memory");
                if (chunk_done(cnt + 2, lane, total)) arrive(ws, ip.bt * 8u, 8u, call, lane);
            }
            if (it == STAMP_IT) STAMP_AT(1);
        }
        if (lane == 0) __hip_atomic_store(callp, callv + 1u, __ATOMIC_RELAXED, __HIP_MEMORY_SCOPE_AGENT);
        if (packer) {
            STAMP_AT(4);
            if (wid >= 4) __builtin_amdgcn_s_setprio(1);
            const u32 call = rfl(callv) + 1u;
            GAS u32* tf = (GAS u32*)(ws + OFF_TILE + b * 4u);
            u32 seen = 0, nanv = 0;
            auto fetch = [&](u32 yc) {
                if (rfl(seen) != call) {
                    const u64 t0 = __builtin_amdgcn_s_memrealtime();
                    while (rfl(__hip_atomic_load(tf + (yc >> 1) * 32u, __ATOMIC_RELAXED, __HIP_MEMORY_SCOPE_AGENT)) != call) {
                        if (give_up(tmo, call, t0)) { nanv = 0x7fc07fc0u; break; }
                        __builtin_amdgcn_s_sleep(POLL_SLEEP);
                    }
                }
                asm volatile("" ::: "memory");
                const Dest sy = dest_of(y_base(p.out, (yc >> 1) & 15u, rank * 16u + (yc >> 5), yc & 1u), 2u, b);
#pragma unroll
                for (int i = 0; i < 8; ++i) a[i] = __builtin_amdgcn_raw_buffer_load_b128(sy.r, (lane >> 4) * 4 * 65536 + (lane & 15) * 16, ((i & 3) + (i >> 2) * 16) * 65536, AUX_SYS);
                seen = __hip_atomic_load(tf + ((yc + NPACK) >> 1) * 32u, __ATOMIC_RELAXED, __HIP_MEMORY_SCOPE_AGENT);
            };
            auto publish = [&](u32 yc) {
                u32 total;
                if (chunk_done((LAS u32*)lds + 4u * it, lane, total) && (u32)lane - 1u < 7u)
                    __hip_atomic_store((GAS u64*)(wsl + OFF_GRAN) + (size_t)((call & 1u) * 8u + rank) * NC + NCIN + yc,
                                       ((u64)total << 32) | call, __ATOMIC_RELAXED, __HIP_MEMORY_SCOPE_SYSTEM);
                ++it;
            };
            u32 yc = role >> 3;
            fetch(yc);
            STAMP_AT(3);
            for (;; yc += NPACK) {
                u32 v[8][4];
#pragma unroll
                for (int j = 0; j < 8; ++j)
#pragma unroll
                    for (int e = 0; e < 4; ++e) v[j][e] = a[j][e] | nanv;
                if (yc + NPACK < (u32)NCY) fetch(yc + NPACK);
                encode_block(v, mk_rsrc(ws + OFF_SY + (size_t)yc * CH, CH), b, (LAS u32*)lds + 4u * it, tab, wl, lane);
                asm volatile("s_waitcnt vmcnt(0)" ::: "memory");
                publish(yc);
                if (yc + NPACK >= (u32)NCY) break;
            }
            __builtin_amdgcn_s_setprio(0);
            if (!PJOIN || NLAST == WB) { STAMP_AT(2); return; }
        }
        STAMP_AT(2);
    }

    const u32 call = rfl(callv) + 1u;
    const int wr = wid >> 2, wc = wid & 3;
    constexpr int HTB = 16384;
#define SA(b, h) (((b) * 2 + (h)) * HTB)
#define SB(b, h) ((4 + (b) * 2 + (h)) * HTB)
#define STAGE(bufoff, gbase) do { _Pragma("unroll") for (int _i = 0; _i < 2; ++_i) \
        __builtin_amdgcn_global_load_lds((const GAS u32*)((gbase) + voff + _i * 8192), (LAS u32*)(lds + (bufoff) + ldsw + _i * 8192), 16, 0, AUX_SYS); } while (0)
#define LDA(dst, b, h) do { _Pragma("unroll") for (int m = 0; m < 4; ++m) _Pragma("unroll") for (int k = 0; k < 2; ++k) dst[m][k] = *(const LAS bf16x8*)(lds + SA(b, h) + aoff + m * 2048 + k * 1024); } while (0)
#define LDB(dst, b, h) do { _Pragma("unroll") for (int n = 0; n < 2; ++n) _Pragma("unroll") for (int k = 0; k < 2; ++k) dst[n][k] = *(const LAS bf16x8*)(lds + SB(b, h) + boff + n * 2048 + k * 1024); } while (0)
#define MMA(ai, bj, At, Bt) do { _Pragma("unroll") for (int _x = 0; _x < 16; ++_x) { const int m = (_x >> 1) & 3, n = (_x ^ (_x >> 1)) & 1, k = _x >> 3; \
        acc[ai][bj][m][n] = __builtin_amdgcn_mfma_f32_16x16x32_bf16(Bt[n][k], At[m][k], acc[ai][bj][m][n], 0, 0, 0); SCHED; } } while (0)
#define MMA2(ai) do { __builtin_amdgcn_s_setprio(1); MMA(ai, 0, At, B0); MMA(ai, 1, At, B1); __builtin_amdgcn_s_setprio(0); } while (0)
#define WAIT_V(n) asm volatile("s_waitcnt vmcnt(" #n ")" ::: "memory")
#define WAIT_L(n) asm volatile("s_waitcnt lgkmcnt(" #n ")" ::: "memory")
#define BAR __builtin_amdgcn_s_barrier()
    const u32 voff = (u32)tid * 16u, ldsw = (u32)wid * 1024u;
    const int aoff = wr * 8192 + lane * 16, boff = wc * 4096 + lane * 16;
    const u32 idx = bid >> 3, hh = (bid >> 2) & 1u;
    u32 ti = (bid & 3u) * 4u + (idx < 4u * (WB / 2) ? idx & 3u : 2u * hh + (idx & 1u)), tj = idx < 4u * (WB / 2) ? 2u * (idx >> 2) + hh : WB - 1u;
    constexpr size_t kstep = 32768, hstep = 16384;
    const GAS char* xf = (const GAS char*)frag_base(ws, false, 0u, 0u, 0u, call);
    const GAS char* wf = (const GAS char*)(ws + OFF_WF);
    GAS u32* flags = (GAS u32*)(ws + OFF_FLAG + (bid & 7u) * 256u) + ((u32)lane & 7u);
    auto arrived = [&](u32 bt) { return (u32)__builtin_ctz(((u32)__builtin_amdgcn_ballot_w64(__hip_atomic_load(flags + bt * 8u, __ATOMIC_RELAXED, __HIP_MEMORY_SCOPE_SYSTEM) != call) & 31u) | 32u); };
#define STAGE_FIRST(pA, pB) do { STAGE(SB(0, 0), pB); STAGE(SA(0, 0), pA); STAGE(SB(0, 1), (pB) + hstep); STAGE(SA(0, 1), (pA) + hstep); \
        STAGE(SB(1, 0), (pB) + kstep); STAGE(SA(1, 0), (pA) + kstep); STAGE(SB(1, 1), (pB) + hstep + kstep); } while (0)
    u32 nwait = 0, bad = 0, staged = 0;
    auto draw_ticket = [&]() { return __hip_atomic_fetch_add((GAS u32*)(ws + OFF_TICK + 8u * 128u), 1u, __ATOMIC_RELAXED, __HIP_MEMORY_SCOPE_AGENT) - (call - 1u) * (NGEMM + (PJOIN ? NPACK : 0)); };
    auto last_tile = [&](u32 tk, u32& i, u32& j) { const u32 gr = tk / (4u * NLAST), r = tk - gr * 4u * NLAST; i = (gr * 4u + (r & 3u)) & 15u; j = gr < 4u ? (NB - 1u) * WB + (r >> 2) : (u32)NP; };
    u32 round = 0;
    if (bid >= (u32)NGEMM) {
        if (tid == 0) *(volatile LAS u32*)(ctl + 2) = draw_ticket();
        asm volatile("s_waitcnt lgkmcnt(0)" ::: "memory"); BAR; asm volatile("" ::: "memory");
        last_tile(lds_ld(ctl + 2), ti, tj);
        round = NB - 1u;
    }
    for (; tj < (u32)NP; ++round) {
        const GAS char* cA = wf + (size_t)tj * (2u << 20);
        const GAS char* cB = xf + (size_t)ti * (2u << 20);
        u32 ni = ti, nj = tj + WB;
        f32x4 acc[2][2][4][2];
#pragma unroll
        for (int a = 0; a < 2; ++a)
#pragma unroll
            for (int b = 0; b < 2; ++b)
#pragma unroll
                for (int m = 0; m < 4; ++m)
#pragma unroll
                    for (int n = 0; n < 2; ++n) acc[a][b][m][n] = (f32x4){0.f, 0.f, 0.f, 0.f};
        bf16x8 At[4][2], B0[2][2], B1[2][2];
        for (u32 q0 = 0; q0 < 5u;) {
            const u32 k0 = q0 ? q0 * 14u - 6u : 0u;
            if (!staged) {
                if (wid == 0) {
                    const u64 t0 = __builtin_amdgcn_s_memrealtime();
                    u32 q1;
                    for (u32 spin = 0;; ++spin) {
                        q1 = arrived(round);
                        if (q1 > q0) break;
                        if ((spin & 7u) == 7u && give_up(tmo, call, t0)) { q1 = 5u | 8u; break; }
                        __builtin_amdgcn_s_sleep(POLL_SLEEP);
                    }
                    if (lane == 0) *(volatile LAS u32*)(ctl + (nwait & 1u)) = q1;
                    asm volatile("s_waitcnt lgkmcnt(0)" ::: "memory");
                }
                asm volatile("" ::: "memory"); BAR; asm volatile("" ::: "memory");
                const u32 te = lds_ld(ctl + (nwait & 1u));
                ++nwait; bad |= te >> 3;
                staged = te & 7u;
                STAGE_FIRST(cA + (size_t)k0 * kstep, cB + (size_t)k0 * kstep);
            }
            if (k0 == 0u && round < 4u) STAMP_AT(3 + round * 2);
            q0 = staged & 7u;
            const u32 k1 = q0 * 14u - 6u;
            if (wr == 1) BAR;
            bool lax = (staged >> 3) != 0u;
            if (lax) WAIT_V(22); else WAIT_V(6);
            BAR; BAR;
            for (u32 t = k0; t < k1; t += 2u) {
                const u32 t2 = t + 2u < k1 ? t + 2u : t;
                const GAS char* a1 = cA + (size_t)(t + 1u) * kstep;
                const GAS char* a2 = cA + (size_t)t2 * kstep; const GAS char* b2 = cB + (size_t)t2 * kstep;
                const GAS char* a3 = a2 + kstep; const GAS char* b3 = b2 + kstep;
                LDB(B0, 0, 0); LDB(B1, 0, 1); LDA(At, 0, 0); STAGE(SA(1, 1), a1 + hstep);
                if (lax) WAIT_V(24); else WAIT_V(8);
                WAIT_L(0); BAR; SCHED; MMA2(0); BAR; SCHED;
                LDA(At, 0, 1); STAGE(SB(0, 0), b2); STAGE(SB(0, 1), b2 + hstep); STAGE(SA(0, 0), a2);
                if (lax) WAIT_V(24); else WAIT_V(8);
                lax = false;
                WAIT_L(0); BAR; SCHED; MMA2(1); BAR; SCHED;
                LDB(B0, 1, 0); LDB(B1, 1, 1); LDA(At, 1, 0); STAGE(SA(0, 1), a2 + hstep);
                WAIT_V(8); WAIT_L(0); BAR; SCHED; MMA2(0); BAR; SCHED;
                LDA(At, 1, 1); STAGE(SB(1, 0), b3); STAGE(SB(1, 1), b3 + hstep); STAGE(SA(1, 0), a3);
                WAIT_V(8); WAIT_L(0); BAR; SCHED; MMA2(1); BAR; SCHED;
            }
            const bool last = q0 == 5u, draw = NLAST < WB && last && round == NB - 2u;
            if (wid == 0) {
                u32 tk = 0;
                if (draw && lane == 0) tk = draw_ticket();
                const u32 q1 = arrived(last ? round + 1u : round);
                if (lane == 0) *(volatile LAS u32*)(ctl + (nwait & 1u)) = q1 | (tk << 8);
                asm volatile("s_waitcnt lgkmcnt(0)" ::: "memory");
            }
            WAIT_V(0);
            if (wr == 0) BAR;
            BAR;
            asm volatile("" ::: "memory");
            staged = lds_ld(ctl + (nwait & 1u));
            ++nwait;
            if (draw) last_tile(staged >> 8, ni, nj);
            staged &= 7u;
            if (staged <= (last ? 0u : q0) || (last && nj >= (u32)NP)) staged = 0u;
            if (staged) STAGE_FIRST(last ? wf + (size_t)nj * (2u << 20) : cA + (size_t)k1 * kstep, last ? xf + (size_t)ni * (2u << 20) : cB + (size_t)k1 * kstep);
        }
        if (round < 4u) STAMP_AT(4 + round * 2);
#pragma unroll
        for (int bj = 0; bj < 2; ++bj) {
            const Dest dst = dest_of(y_base(p.out, ti, gpanel(rank, tj), bj), 2u, (u32)wid);
            const u32 loff = (u32)(lane >> 4) * dst.LH + (u32)(lane & 15) * 16u;
            u32x4 v[8];
#pragma unroll
            for (int g = 0; g < 8; ++g)
#pragma unroll
                for (int q = 0; q < 4; ++q) {
                    typedef float f32x2 __attribute__((ext_vector_type(2)));
                    typedef __bf16 bf16x2 __attribute__((ext_vector_type(2)));
                    const f32x2 f = {acc[q >> 1][bj][(q & 1) * 2][g >> 2][g & 3], acc[q >> 1][bj][(q & 1) * 2 + 1][g >> 2][g & 3]};
                    v[g][q] = __builtin_bit_cast(u32, __builtin_convertvector(f, bf16x2));
                }
            if (bad) {
#pragma unroll
                for (int g = 0; g < 8; ++g) v[g] = (u32x4){0x7fc07fc0u, 0x7fc07fc0u, 0x7fc07fc0u, 0x7fc07fc0u};
            }
            if (tj < (u32)POWN) {
#pragma unroll
                for (int g = 0; g < 8; ++g) store_group<AUX_DEV>(dst, loff, g, v[g]);
            } else if (staged) {
#pragma unroll
                for (int g = 0; g < 8; ++g) store_group<AUX_NT>(dst, loff, g, v[g]);
            } else {
#pragma unroll
                for (int g = 0; g < 8; ++g) store_group<AUX_OUT>(dst, loff, g, v[g]);
            }
        }
        if (tj < (u32)POWN) {
            asm volatile("s_waitcnt vmcnt(0)" ::: "memory");
            if (lane == 0) __hip_atomic_store((GAS u32*)(ws + OFF_TILE + (tj * 16u + ti) * 128u + (u32)wid * 4u), call, __ATOMIC_RELAXED, __HIP_MEMORY_SCOPE_AGENT);
        } else if (staged) staged |= 8u;
        ti = ni; tj = nj;
    }
    STAMP_AT(15);
}

static void build_tables(Params& p) {
    unsigned char* role = (unsigned char*)p.role;
    for (int i = 0; i < 64; ++i) role[i] = (unsigned char)(i < NPULL ? (1 + i % 7) | (i / 7) << 3 : (i - NPULL) << 3);
    {
        constexpr int SL = (256 - NGEMM) / 2;
        static_assert(4 * PULL_WG_PER_PEER <= SL, "roles");
        int cnt[2] = {0, 0}, np = 0;
        for (int j = 0; j < PULL_WG_PER_PEER; ++j)
            for (int rel = 1; rel < 8; ++rel) {
                const int h = (((0xb2 >> p.rank) ^ (0xb2 >> ((p.rank + rel) & 7))) & 1), e = cnt[h]++;
                role[(e >> 2) * 8 + h * 4 + ((e + (h ? e >> 2 : 0)) & 3)] = (unsigned char)(rel | j << 3);
            }
        for (int h = 0; h < 2; ++h) for (int e = cnt[h]; e < SL; ++e) role[(e >> 2) * 8 + h * 4 + (e & 3)] = (unsigned char)(np++ << 3);
    }
    unsigned short* t = (unsigned short*)p.task;
    int n = 0;
    for (u32 c = PUSHC; c < (u32)NCIN; ++c) {
        if (c == 16u) for (u32 e = 0; e < (u32)PUSHC; ++e) t[n++] = (unsigned short)(e << 4 | 8u);
        const InPos ip = in_of(c);
        if (ip.isw && ip.s < (u32)POWN) for (u32 k = 1; k <= 8; ++k) t[n++] = (unsigned short)(c << 4 | k);
        else t[n++] = (unsigned short)(c << 4);
    }
}

extern "C" void kernel_launch(int rank, int nranks, ncclComm_t comm, hipStream_t stream,
                              void* const* d_in, const int* in_sizes, int n_in, void* d_out, int out_size,
                              const int* in_dtypes, int out_dtype, void* const* d_out_all,
                              void* const* d_ws_all, size_t ws_bytes) {
    if (nranks != NR || (unsigned)rank >= (unsigned)NR || ws_bytes < WS_NEED || n_in != 2) { fprintf(stderr, "unsupported configuration\n"); return; }
    Params p;
    p.x = (const char*)d_in[0]; p.w = (const char*)d_in[1]; p.out = (char*)d_out; p.rank = rank; p.pad = 0;
    for (int i = 0; i < NR; ++i) p.ws[i] = (char*)d_ws_all[(rank + i) % NR];
    build_tables(p);
    hipLaunchKernelGGL(gemm_ar_fused, dim3(256), dim3(512), 0, stream, p);
    HIP_CHECK(hipGetLastError());
}
```

```cpp
#include <hip/hip_runtime.h>
#include <rccl/rccl.h>
#include <cstdio>

#define HIP_CHECK(expr) do { hipError_t _e = (expr); if (_e != hipSuccess) { \
    fprintf(stderr, "HIP error %s at %s:%d\n", hipGetErrorString(_e), __FILE__, __LINE__); \
    return; } } while (0)

typedef unsigned u32;
typedef unsigned long long u64;
typedef u32 u32x4 __attribute__((ext_vector_type(4)));
typedef u32 u32x2 __attribute__((ext_vector_type(2)));
typedef unsigned short u16x2 __attribute__((ext_vector_type(2)));
typedef short bf16x8 __attribute__((ext_vector_type(8)));
typedef float f32x4 __attribute__((ext_vector_type(4)));
#define LAS __attribute__((address_space(3)))
#define GAS __attribute__((address_space(1)))
typedef __amdgpu_buffer_rsrc_t rsrc_t;

constexpr int NR = 8;
#define PULL_WG_PER_PEER 8
#define PAT 56565
#define LVAR 5
#define PUSHC 5
#define PUSHW 1
#define PACK_IN 112
#define AFFINE 1
#define TOUCH 3
#define TOUCH_AT 100
#define PJOIN 1
#define DEC_SLEEP 2
#define RHO 6
#define ENC_DELAY 150
#define ENC_PACE 192
#define POLL_SLEEP 8
#define WB 12
constexpr int NPULL = 7 * PULL_WG_PER_PEER, NGEMM = 16 * WB, NPACK = 256 - NGEMM - NPULL;
constexpr int POWN = 16 - RHO, NP = POWN + 8 * RHO;
constexpr int NB = (NP + WB - 1) / WB, N0 = 16 + WB;
constexpr int NCIN = 64 + 4 * NP, NCY = 32 * POWN, NC = NCIN + NCY;
constexpr int NTASK = 16 * 36;
constexpr int NDEC = 6;
constexpr u64 DEADLINE = 100000000ull;
constexpr int PUSHB = (8 * PUSHC + PULL_WG_PER_PEER - 1) / PULL_WG_PER_PEER;
constexpr int NLAST = NP - (NB - 1) * WB;
static_assert(NGEMM >= 192 && NPACK < 32 && NPACK >= 4 && (PACK_IN + NCY) / NPACK < 126 && NB <= 8 && POWN <= WB && PUSHB <= NDEC, "roles");

constexpr int FIXB = 5120, CVAR = 8 * FIXB, CHDR = 32, CH = 66560;
constexpr size_t OFF_CALL = 0;
constexpr size_t OFF_TMO = 131072;
constexpr size_t OFF_TICK = OFF_TMO + 256;
constexpr size_t OFF_RDY = OFF_TICK + 16 * 128;
constexpr size_t OFF_FLAG = OFF_RDY + 64 * 128;
constexpr size_t OFF_TILE = OFF_FLAG + 8 * 256;
constexpr size_t OFF_GRAN = 196608;
constexpr size_t OFF_DBG = 524288;
constexpr size_t OFF_XF = 1ull << 20;
constexpr size_t OFF_WF = OFF_XF + (64ull << 20);
constexpr size_t OFF_SY = OFF_WF + NP * (2ull << 20);
constexpr size_t OFF_SI = OFF_SY + (size_t)NCY * CH;
constexpr size_t WS_NEED = OFF_SI + 2ull * NTASK * CH;

constexpr int A2B = 2064, A3B = 528, VARB = 912, WLDS = A2B + A3B + VARB, N2MAX = 2032, N3MAX = 496, UBITS = 2016, NPATCH = 31;
constexpr int CTL_OFF = 2048, CTL_WORDS = 128, WL_OFF = 2560, BIG_OFF = 30720, LDS_BYTES = BIG_OFF + 131072;
static_assert(WL_OFF + 8 * WLDS <= BIG_OFF && LDS_BYTES <= 163840, "LDS map");
constexpr int NF = 16, NV = 4, VSLOT = 8192, VRING = NF * FIXB;
constexpr int AUX_SYS = 17, AUX_DEV = 16, AUX_OUT = 18, AUX_NT = 2;

struct Params {
    int rank; int pad;
    const char* x; const char* w; char* out;
    char* ws[NR];
    u32 role[16];
    u32 task[NTASK / 2];
};

__device__ __forceinline__ u32 rfl(u32 x) { return (u32)__builtin_amdgcn_readfirstlane((int)x); }
__device__ __forceinline__ rsrc_t mk_rsrc(const void* p, u32 bytes) {
    const u64 a = (u64)p; const u64 u = ((u64)rfl((u32)(a >> 32)) << 32) | rfl((u32)a);
    return __builtin_amdgcn_make_buffer_rsrc((void*)u, (short)0, (int)rfl(bytes), 0x00020000);
}
__device__ __forceinline__ u32 vperm(u32 a, u32 b, u32 s) { return __builtin_amdgcn_perm(a, b, s); }
__device__ __forceinline__ u32 wave_incl_scan(u32 x) {
    x += (u32)__builtin_amdgcn_update_dpp(0, (int)x, 0x111, 0xf, 0xf, false);
    x += (u32)__builtin_amdgcn_update_dpp(0, (int)x, 0x112, 0xf, 0xf, false);
    x += (u32)__builtin_amdgcn_update_dpp(0, (int)x, 0x114, 0xf, 0xf, false);
    x += (u32)__builtin_amdgcn_update_dpp(0, (int)x, 0x118, 0xf, 0xf, false);
    x += (u32)__builtin_amdgcn_update_dpp(0, (int)x, 0x142, 0xa, 0xf, false);
    x += (u32)__builtin_amdgcn_update_dpp(0, (int)x, 0x143, 0xc, 0xf, false);
    return x;
}
__device__ __forceinline__ u32 pk_sub(u32 a, u32 b) { return __builtin_bit_cast(u32, (u16x2)(__builtin_bit_cast(u16x2, a) - __builtin_bit_cast(u16x2, b))); }
template <int AUX>
__device__ __forceinline__ void st128(u32x4 x, rsrc_t r, int voff, int soff) {
    __builtin_amdgcn_raw_buffer_store_b128(x, r, voff, soff, AUX);
    asm volatile("s_nop 1" :: "v"(x) : "memory");
}
#define WAVE_FENCE() __builtin_amdgcn_fence(__ATOMIC_ACQ_REL, "wavefront")

__device__ __forceinline__ void build_table(LAS u32x2* tab, u32 i, bool unpack) {
    u32 n = 0, dep = 0, cmp = 0x0c0c0c0cu;
#pragma unroll
    for (u32 t = 0; t < 4; ++t) {
        const u32 c = (i >> (2 * t)) & 3u;
        dep |= (c ? 4u + c : n) << (8 * t);
        if (!c) { cmp = (cmp & ~(0xffu << (8 * n))) | (t << (8 * n)); ++n; }
    }
    tab[i] = (u32x2){unpack ? dep : cmp, n};
}
__device__ __forceinline__ void build_table3(LAS u32x2* tab3, u32 i) {
    u64 r = 0; u32 n = 0, run = 0;
#pragma unroll
    for (u32 t = 0; t < 8; ++t) { if ((i >> t) & 1u) { r |= (u64)run << (8 * n); ++n; run = 0; } else ++run; }
    tab3[i] = (u32x2){(u32)r, (u32)(r >> 32)};
}
__device__ __forceinline__ void or3(LAS u32* q, u64 x, u32 sh) {
    const u64 y = x << sh;
    __hip_atomic_fetch_or(q, (u32)y, __ATOMIC_RELAXED, __HIP_MEMORY_SCOPE_WORKGROUP);
    __hip_atomic_fetch_or(q + 1, (u32)(y >> 32), __ATOMIC_RELAXED, __HIP_MEMORY_SCOPE_WORKGROUP);
    __hip_atomic_fetch_or(q + 2, (u32)(((x >> 32) << sh) >> 32), __ATOMIC_RELAXED, __HIP_MEMORY_SCOPE_WORKGROUP);
}
__device__ __forceinline__ u32 code_byte(u32 hi, u32 lo, u32 x) {
    const u32 f = vperm(hi, lo, x);
    return __builtin_amdgcn_udot4((f ^ (f >> 2)) & 0x03030303u, 0x40100401u, 0u, false);
}
template <bool PRED>
__device__ __forceinline__ void or_bytes(u32 pos, u32 x) {
    const u64 y = (u64)x << ((pos & 3u) * 8u);
    LAS u32* q = (LAS u32*)(size_t)(pos & ~3u);
    if (!PRED || x) {
        __hip_atomic_fetch_or(q, (u32)y, __ATOMIC_RELAXED, __HIP_MEMORY_SCOPE_WORKGROUP);
        __hip_atomic_fetch_or(q + 1, (u32)(y >> 32), __ATOMIC_RELAXED, __HIP_MEMORY_SCOPE_WORKGROUP);
    }
}
__device__ __forceinline__ u32 get_bytes(u32 pos) {
    const LAS u32* q = (const LAS u32*)(size_t)(pos & ~3u);
    return __builtin_amdgcn_alignbyte(q[1], q[0], pos);
}
__device__ __forceinline__ u32 la(const LAS void* p) { return (u32)(size_t)p; }
__device__ __forceinline__ u32 pack_level2(const LAS u32x2* tab, const LAS char* src, u32 n, LAS u32* end, LAS char* dst, int lane) {
    u32 m = 0;
    for (u32 i0 = 0; i0 < n; i0 += 1024u) {
        const u32 i = i0 + (u32)lane * 16u, hi = 0x02010000u, lo = 0x00030000u;
        const u32x4 x = *(const LAS u32x4*)(src + (i < n ? i : 0u));
        u32 cw = code_byte(hi, lo, x[0]) | (code_byte(hi, lo, x[1]) << 8) | (code_byte(hi, lo, x[2]) << 16) | (code_byte(hi, lo, x[3]) << 24);
        const u32 valid = i < n ? n - i : 0u;
        if (valid < 16u) cw |= 0x55555555u << (2u * valid);
        u32x2 e[4]; u32 c[5]; c[0] = 0;
#pragma unroll
        for (int j = 0; j < 4; ++j) { e[j] = tab[(cw >> (8 * j)) & 0xffu]; c[j + 1] = c[j] + e[j][1]; }
        const u32 incl = wave_incl_scan(c[4]), p0 = la(dst) + m + incl - c[4];
        if (i < n) *(end - 1 - (i >> 4)) = cw;
        m += (u32)__builtin_amdgcn_readlane((int)incl, 63);
        if (m <= (u32)N3MAX) {
#pragma unroll
            for (int j = 0; j < 4; ++j) or_bytes<true>(p0 + c[j], vperm(0u, x[j], e[j][0]));
        }
    }
    WAVE_FENCE();
    return m;
}
__device__ __forceinline__ void pack_level3(const LAS char* src, u32 n, LAS u32* ub, LAS u32* pt, u32 T, int lane, u32& bits, u32& np) {
    for (u32 i0 = 0; i0 < n; i0 += 256u) {
        const u32 i = i0 + (u32)lane * 4u;
        const u32 x = *(const LAS u32*)(src + (i < n ? i : 0u));
        u32 cum = 0, pf = 0; u64 m = 0;
#pragma unroll
        for (u32 t = 0; t < 4; ++t) {
            const u32 r = ((x >> (8 * t)) - 8u) & 0xffu;
            const bool valid = i + t < n, big = r > 15u;
            cum += valid && !big ? r : 0u;
            m |= (u64)(valid ? 1u : 0u) << cum;
            cum += valid ? 1u : 0u;
            pf |= valid && big ? 1u << t : 0u;
        }
        const u32 incl = wave_incl_scan(cum), p = bits + incl - cum;
        bits += (u32)__builtin_amdgcn_readlane((int)incl, 63);
        if (bits <= (u32)UBITS && cum) or3(ub + (p >> 5), m, p & 31u);
        if (__builtin_amdgcn_ballot_w64(pf != 0u)) {
#pragma unroll
            for (u32 t = 0; t < 4; ++t) {
                const u64 bm = __builtin_amdgcn_ballot_w64((pf >> t) & 1u);
                const u32 at = np + __builtin_amdgcn_mbcnt_hi((u32)(bm >> 32), __builtin_amdgcn_mbcnt_lo((u32)bm, 0u));
                if ((pf >> t) & 1u) pt[at] = ((i + t) << 8) | ((T - (x >> (8 * t))) & 0xffu);
                np += (u32)__builtin_popcountll(bm);
            }
        }
    }
    WAVE_FENCE();
}
__device__ __forceinline__ void encode_block(const u32 (&v)[8][4], rsrc_t ch, u32 b, LAS u32* alloc, const LAS u32x2* tab, LAS char* wl, int lane) {
#pragma unroll
    for (int j = 0; j < 4; ++j) if (j < 3 || lane < (WLDS - 3072) / 16) *(LAS u32x4*)(wl + j * 1024 + lane * 16) = (u32x4){0, 0, 0, 0};
    u32 s = 0;
#pragma unroll
    for (int g = 0; g < 8; ++g) s = __builtin_amdgcn_udot4(__builtin_amdgcn_alignbit(v[g][0], v[g][0], 31), 0x01000100u, s, false);
    u32 T = ((u32)__builtin_amdgcn_readlane((int)wave_incl_scan(s), 63) >> 10) + 5u;
    T = T > 255u ? 255u : (T < 23u ? 23u : T);
    const u32 tt = (T * 256u + 255u) * 0x10001u;
    const int fo = (int)(b * FIXB);
    u32 U[16], l1[4] = {0, 0, 0, 0}, c[17];
    u32x2 e[16];
#pragma unroll
    for (int q = 0; q < 4; ++q) {
        u32x4 sm;
#pragma unroll
        for (int j = 0; j < 4; ++j) {
            const int k = 4 * q + j;
            const u32 a0 = v[k >> 1][(k & 1) * 2], a1 = v[k >> 1][(k & 1) * 2 + 1];
            const u32 r0 = __builtin_amdgcn_alignbit(a0, a0, 31), r1 = __builtin_amdgcn_alignbit(a1, a1, 31);
            sm[j] = vperm(r1, r0, 0x06040200u);
            U[k] = vperm(pk_sub(tt, r1), pk_sub(tt, r0), 0x07050301u);
            const u32 cb = code_byte(0x00000302u, 0x01000000u, U[k]);
            l1[q] |= cb << (8 * j);
            e[k] = tab[cb];
        }
        st128<AUX_SYS>(sm, ch, lane * 16 + q * 1024, fo);
    }
    st128<AUX_SYS>((u32x4){l1[0], l1[1], l1[2], l1[3]}, ch, lane * 16 + 4096, fo);
    __builtin_amdgcn_sched_barrier(0);
    c[0] = 0;
#pragma unroll
    for (int k = 0; k < 16; ++k) c[k + 1] = c[k] + e[k][1];
    const u32 incl = wave_incl_scan(c[16]), p0 = la(wl) + incl - c[16];
    const u32 n2 = (u32)__builtin_amdgcn_readlane((int)incl, 63);
    bool raw = n2 > (u32)N2MAX;
    u32 d2 = 0, d3 = 0, np = 0;
    LAS u32* var = (LAS u32*)(wl + A2B + A3B);
    if (!raw) {
        WAVE_FENCE();
#pragma unroll
        for (int k = 0; k < 16; ++k) or_bytes<false>(p0 + c[k], vperm(0u, U[k], e[k][0]));
        WAVE_FENCE();
        d2 = (n2 + 15u) >> 4;
        const u32 n3 = pack_level2(tab, wl, n2, var + d2, wl + A2B, lane);
        raw = n3 > (u32)N3MAX;
        if (!raw) {
            u32 bits = 0;
            pack_level3(wl + A2B, n3, var + d2, (LAS u32*)wl, T, lane, bits, np);
            raw = bits > (u32)UBITS || np > (u32)NPATCH;
            d3 = (bits + 31u) >> 5;
            if (!raw && (u32)lane < np) var[d2 + d3 + (u32)lane] = ((const LAS u32*)wl)[lane];
            WAVE_FENCE();
        }
    }
    const u32 vd = d2 + d3 + np;
    u32 off = 0;
    if (lane == 0) off = __hip_atomic_fetch_add(alloc, raw ? 3072u : vd * 4u, __ATOMIC_RELAXED, __HIP_MEMORY_SCOPE_WORKGROUP);
    off = rfl(off) + CHDR;
    const int vo = CVAR + (int)off;
    if (lane == 0) __builtin_amdgcn_raw_buffer_store_b32(raw ? off << 17 : (T | (d3 << 8) | (np << 14) | ((off + d2 * 4u) << 17)), ch, 0, CVAR + (int)b * 4, AUX_SYS);
    if (raw) {
        asm volatile("s_waitcnt vmcnt(0)" ::: "memory");
#pragma unroll
        for (int g = 0; g < 8; ++g)
            st128<AUX_SYS>((u32x4){v[g][0], v[g][1], v[g][2], v[g][3]}, ch, lane * 16 + (g < 5 ? g : g - 5) * 1024, g < 5 ? fo : vo);
    } else {
        for (u32 o = (u32)lane; o < vd; o += 64u) __builtin_amdgcn_raw_buffer_store_b32(var[o], ch, (int)o * 4, vo, AUX_SYS);
    }
    WAVE_FENCE();
}
__device__ __forceinline__ bool chunk_done(LAS u32* cnt, int lane, u32& total) {
    u32 n = 0;
    if (lane == 0) n = __hip_atomic_fetch_add(cnt + 1, 1u, __ATOMIC_RELAXED, __HIP_MEMORY_SCOPE_WORKGROUP);
    if (rfl(n) != 7u) return false;
    total = rfl(*(volatile LAS u32*)cnt) + CHDR;
    return true;
}

struct Dest { rsrc_t r; u32 A, B, C, LH; };
__device__ __forceinline__ Dest dest_of(char* base, u32 kind, u32 b) {
    if (kind == 0u) return Dest{mk_rsrc(base + b * 4096u, 32768u + 4096u), 1024u, 32768u, 2048u, 256u};
    if (kind == 1u) return Dest{mk_rsrc(base + (b >> 2) * 32768u + (b & 1u) * 8192u + (b & 2u) * 512u, 16384u + 6144u + 1024u), 2048u, 4096u, 16384u, 256u};
    return Dest{mk_rsrc(base + (b & 3u) * (32u * 65536u) + (b >> 2) * 256u, 31u * 65536u + 256u), 65536u, 131072u, 16u * 65536u, 4u * 65536u};
}
template <int AUX>
__device__ __forceinline__ void store_group(const Dest& d, u32 loff, int g, u32x4 x) {
    st128<AUX>(x, d.r, (int)loff, (int)((g & 1 ? d.A : 0u) + (g & 2 ? d.B : 0u) + (g & 4 ? d.C : 0u)));
}

#define SCHED __builtin_amdgcn_sched_barrier(0)
__device__ __forceinline__ u32x2 get2(u32 pos) { const LAS u32* q = (const LAS u32*)(size_t)(pos & ~3u); return (u32x2){q[0], q[1]}; }
__device__ __forceinline__ void spread3(u32 w, u32 start, u32 n, u32x2 runs, u32 t8, u32 p) {
    if (n) {
        const u32 carry = (u32)__builtin_clz((w << 8) | start);
        const u64 x = ((u64)(t8 - runs[1]) << 32) | (t8 - runs[0] - carry);
        const u32 sh = 64u - 8u * n;
        or3((LAS u32*)(size_t)(p & ~3u), (x << sh) >> sh, (p & 3u) * 8u);
    }
}
template <class Out>
__device__ __forceinline__ void decode_block(const u32x4 (&d)[5], u32 hdr, const LAS u32* ub, const LAS u32x2* tab, const LAS u32x2* tab3, LAS char* wl, int lane, Out out) {
    const u32 T = hdr & 0xffu, d3 = (hdr >> 8) & 0x3fu, np = (hdr >> 14) & 0x1fu;
    auto ex = [&](u32 a, u32 b, u32 c) { return ((T - a) << 8) | ((T - b) << 16) | ((T - c) << 24); };
    const u32 t1 = ex(3, 4, 5), t2 = ex(6, 7, 2), t8 = (T - 8u) * 0x01010101u;
    const u32 w2 = *(ub - 1 - lane), p3i = la(ub) + (u32)lane - 3u;
    u32x2 e[16], e2[4], g[16];
    g[0] = get2(p3i);
    if (lane < A3B / 16) *(LAS u32x4*)(wl + A2B + lane * 16) = (u32x4){0, 0, 0, 0};
#pragma unroll
    for (int k = 0; k < 8; ++k) e[k] = tab[(d[4][k >> 2] >> (8 * (k & 3))) & 0xffu];
    SCHED;
    const u32 w3 = __builtin_amdgcn_alignbyte(g[0][1], g[0][0], p3i), x3 = (u32)lane < d3 * 4u ? w3 >> 24 : 0u;
    const u32x2 e3 = tab3[x3];
#pragma unroll
    for (int j = 0; j < 4; ++j) e2[j] = tab[(w2 >> (8 * j)) & 0xffu];
#pragma unroll
    for (int k = 8; k < 16; ++k) e[k] = tab[(d[4][k >> 2] >> (8 * (k & 3))) & 0xffu];
    SCHED;
    u32 c[17], f[5];
    c[0] = 0; f[0] = 0;
#pragma unroll
    for (int k = 0; k < 16; ++k) c[k + 1] = c[k] + e[k][1];
#pragma unroll
    for (int j = 0; j < 4; ++j) f[j + 1] = f[j] + e2[j][1];
    const u32 n3 = (u32)__builtin_popcount(x3);
    const u32 pk = c[16] | (f[4] << 11) | (n3 << 22);
    const u32 incl = wave_incl_scan(pk), ex0 = incl - pk, tot = (u32)__builtin_amdgcn_readlane((int)incl, 63);
    const u32 a3 = la(wl) + A2B, p1 = la(wl) + (ex0 & 0x7ffu), p2 = a3 + ((ex0 >> 11) & 0x1ffu);
    WAVE_FENCE();
    spread3(w3, lane < 3 ? 0x80000000u >> (8 * lane) : 0x80u, n3, e3, t8, a3 + ((ex0 >> 22) & 0x1ffu));
    for (u32 i0 = 64u, m = tot >> 22; i0 < d3 * 4u; i0 += 64u) {
        const u32 w = get_bytes(p3i + i0), x = i0 + (u32)lane < d3 * 4u ? w >> 24 : 0u, n = (u32)__builtin_popcount(x), in = wave_incl_scan(n);
        spread3(w, 0x80u, n, tab3[x], t8, a3 + ((m + in - n) & 0x1ffu));
        m += (u32)__builtin_amdgcn_readlane((int)in, 63);
    }
    WAVE_FENCE();
    if ((u32)lane < np) { const u32 pw = ub[d3 + (u32)lane]; *(LAS unsigned char*)(wl + A2B + ((pw >> 8) & 0x1ffu)) = (unsigned char)pw; }
    WAVE_FENCE();
#pragma unroll
    for (int j = 0; j < 4; ++j) { f[j] += p2; g[j] = get2(f[j]); }
    SCHED;
#pragma unroll
    for (int k = 0; k < 16; ++k) c[k] += p1;
    SCHED;
    u32x4 a2;
#pragma unroll
    for (int j = 0; j < 4; ++j) a2[j] = vperm(t2, __builtin_amdgcn_alignbyte(g[j][1], g[j][0], f[j]), e2[j][0]);
    *(LAS u32x4*)(wl + lane * 16) = a2;
    if ((tot & 0x7ffu) > 1024u) {
        const u32 cw = *(ub - 65 - lane);
        f[0] = 0;
#pragma unroll
        for (int j = 0; j < 4; ++j) { e2[j] = tab[(cw >> (8 * j)) & 0xffu]; f[j + 1] = f[j] + e2[j][1]; }
        const u32 p = a3 + (((tot >> 11) + wave_incl_scan(f[4]) - f[4]) & 0x1ffu);
#pragma unroll
        for (int j = 0; j < 4; ++j) a2[j] = vperm(t2, get_bytes(p + f[j]), e2[j][0]);
        *(LAS u32x4*)(wl + 1024 + lane * 16) = a2;
    }
    WAVE_FENCE();
#pragma unroll
    for (int k = 0; k < 16; ++k) g[k] = get2(c[k]);
    SCHED;
    u32x4 o;
#pragma unroll
    for (int k = 0; k < 16; ++k) {
        const u32 x = vperm(t1, __builtin_amdgcn_alignbyte(g[k][1], g[k][0], c[k]), e[k][0]);
        const u32 smk = d[k >> 2][k & 3];
        const u32 r0 = vperm(x, smk, 0x05010400u), r1 = vperm(x, smk, 0x07030602u);
        o[2 * (k & 1)] = __builtin_amdgcn_alignbit(r0, r0, 1);
        o[2 * (k & 1) + 1] = __builtin_amdgcn_alignbit(r1, r1, 1);
        if (k & 1) out(k >> 1, o);
    }
    WAVE_FENCE();
}

__device__ __forceinline__ char* frag_base(char* ws, bool isw, u32 s, u32 q, u32 rel, u32 call) {
    return ws + (isw ? OFF_WF : OFF_XF + (size_t)(call & 1u) * (32u << 20)) + (size_t)(s * 64u + (rel ? 6u + 14u * q + 2u * rel : 2u * q)) * 32768u;
}
__device__ __forceinline__ char* y_base(char* out, u32 i, u32 gp, u32 bj) { return out + (size_t)(i * 256u + bj * 128u) * 65536u + gp * 512u; }
__device__ __forceinline__ u32 gpanel(u32 r, u32 j) { return j < (u32)POWN ? r * 16u + j : ((j - POWN) & 7u) * 16u + POWN + ((j - POWN) >> 3); }
__device__ __forceinline__ u32 slot_in(bool isw, u32 s, u32 q, u32 dst) {
    return !isw ? s * 4u + q : (s < (u32)POWN ? 64u + (dst * POWN + s) * 4u + q : 64u + 28u * POWN + s * 4u + q);
}
__host__ __device__ __forceinline__ u32 batch_size(u32 bt) { return bt == 0u ? (u32)N0 : (bt == NB - 1u ? (u32)(NP - (NB - 1) * WB) : (u32)WB); }
struct InPos { bool isw; u32 s, q, bt; };
__host__ __device__ __forceinline__ InPos in_of(u32 c) {
    if (c < 4u * N0) { const u32 q = c / N0, pp = c - q * N0; return InPos{pp >= 16u, pp >= 16u ? pp - 16u : pp, q, 0u}; }
    const u32 bt = (c - 64u) / (4u * WB), r = c - 64u - bt * 4u * WB, q = r / batch_size(bt);
    return InPos{true, bt * WB + r - q * batch_size(bt), q, bt};
}

__device__ __forceinline__ void arrive(char* ws, u32 ci, u32 n, u32 call, int lane) {
    u32 old = 0;
    if (lane == 0) old = __hip_atomic_fetch_add((GAS u32*)(ws + OFF_RDY + ci * 128u), n, __ATOMIC_RELAXED, __HIP_MEMORY_SCOPE_SYSTEM);
    if (rfl(old) + n == call * ((ci & 7u) ? 56u : 32u) * batch_size(ci >> 3) && lane < 8)
        __hip_atomic_store((GAS u32*)(ws + OFF_FLAG + (u32)lane * 256u + ci * 4u), call, __ATOMIC_RELAXED, __HIP_MEMORY_SCOPE_SYSTEM);
}
__device__ __forceinline__ void load_x(const char* x, u32 s, u32 q, u32 b, int lane, u32x4 (&a)[8]) {
    const rsrc_t r = mk_rsrc(x + (size_t)(s * 256u + b * 32u) * 1024u + q * 256u, 31u * 1024u + 256u);
#pragma unroll
    for (int i = 0; i < 8; ++i) a[i] = __builtin_amdgcn_raw_buffer_load_b128(r, (lane & 15) * 1024 + (lane >> 4) * 16 + (i & 3) * 64, (i >> 2) * 16384, 0);
}

__device__ __forceinline__ bool give_up(GAS u32* tmo, u32 call, u64 t0) {
    if (rfl(__hip_atomic_load(tmo, __ATOMIC_RELAXED, __HIP_MEMORY_SCOPE_AGENT)) == call || __builtin_amdgcn_s_memrealtime() - t0 > DEADLINE) {
        __hip_atomic_store(tmo, call, __ATOMIC_RELAXED, __HIP_MEMORY_SCOPE_AGENT);
        return true;
    }
    return false;
}

#define STAMP_IT 0
#define STAMP_AT(i) do { } while (0)

__device__ __forceinline__ u32 lds_ld(LAS u32* p) { return rfl(*(volatile LAS u32*)p); }
__device__ __forceinline__ void dma_piece(rsrc_t r, u32 voff, u32 soff, u32 ldsaddr) {
    u32 keep;
    asm volatile("s_nop 4\n\ts_mov_b32 %0, m0\n\ts_mov_b32 m0, %1\n\ts_nop 0\n\tbuffer_load_dwordx4 %2, %3, %4 offen sc0 sc1 lds\n\ts_mov_b32 m0, %0"
                 : "=&s"(keep) : "s"(ldsaddr), "v"(voff), "s"(r), "s"(soff) : "memory");
}

#define DMA_P(j, w, off) "s_waitcnt vmcnt(" w ")\n\tbuffer_load_dwordx4 %2, %3, " off " sc0 sc1 lds\n\t.if %8 == " #j "\n\tds_max_u32 %6, %7\n\t.endif\n\t"
__device__ __forceinline__ void dma_block(rsrc_t r, u32 voff, u32 soff, u32 ldsaddr, u32 at, u32 val) {
    constexpr int W0 = PAT / 10000, W1 = PAT / 1000 % 10, W2 = PAT / 100 % 10, W3 = PAT / 10 % 10, W4 = PAT % 10;
    constexpr int JP = W0 <= 5 ? 0 : (W1 <= 6 ? 1 : (W2 <= 7 ? 2 : (W3 <= 8 ? 3 : 4)));
    u32 keep;
    asm volatile("s_nop 2\n\ts_mov_b32 %0, m0\n\ts_mov_b32 m0, %1\n\t"
                 DMA_P(0, "%9", "%4 offen") DMA_P(1, "%10", "%4 offen offset:1024") DMA_P(2, "%11", "%4 offen offset:2048") DMA_P(3, "%12", "%4 offen offset:3072")
                 "s_add_u32 m0, m0, 0x1000\n\t" DMA_P(4, "%13", "%5 offen") "s_mov_b32 m0, %0"
                 : "=&s"(keep) : "s"(ldsaddr), "v"(voff), "s"(r), "s"(soff), "s"(soff + 4096u), "v"(at), "v"(val), "n"(JP),
                   "n"(W0), "n"(W1), "n"(W2), "n"(W3), "n"(W4) : "memory", "scc");
}

__global__ void __launch_bounds__(512, 2) gemm_ar_fused(const Params p) {
    __shared__ __attribute__((aligned(16))) char smem[LDS_BYTES];
    LAS char* lds = (LAS char*)smem + BIG_OFF;
    LAS u32x2* tab = (LAS u32x2*)smem;
    LAS u32x2* tab3 = (LAS u32x2*)((LAS char*)smem + WL_OFF);
    const int tid = threadIdx.x, wid = (int)rfl((u32)tid >> 6), lane = tid & 63;
    const u32 bid = blockIdx.x;
    u32 rank = (u32)p.rank, rolew = p.role[((bid - NGEMM) >> 2) & 15u];
    char* ws = p.ws[0]; const char* px = p.x;
    asm volatile("" : "+s"(rank), "+s"(rolew), "+s"(ws), "+s"(px));
    LAS char* wl = (LAS char*)smem + WL_OFF + wid * WLDS;
    LAS u32* ctl = (LAS u32*)((LAS char*)smem + CTL_OFF);
    GAS u32* tmo = (GAS u32*)(ws + OFF_TMO);
    STAMP_AT(0);
    const u32 role = bid >= (u32)NGEMM ? (rolew >> ((bid & 3u) * 8u)) & 0xffu : 0u;
    const u32 pn = (role & 7u) ? (role >> 3) * PUSHB + (u32)wid - 2u : (bid + PUSHC) * 8u + (u32)wid;
    const bool early = (role & 7u) ? wid >= 2 && wid < 2 + PUSHB && pn < 8u * PUSHC : bid < 16u - PUSHC;
    u32x4 a[8];
    if (early) load_x(px, pn >> 3, 0u, pn & 7u, lane, a);
    SCHED;
    GAS u32* callp = (GAS u32*)(ws + OFF_CALL + (bid * 8u + wid) * 64u);
    const u32 callv = __hip_atomic_load(callp, __ATOMIC_RELAXED, __HIP_MEMORY_SCOPE_AGENT);
    char* wsl = ws;
    auto setup = [&](bool unpack) {
        u32 t = (u32)tid;
        asm volatile("" : "+v"(t));
        if (t < 256u) build_table(tab, t, unpack);
        else { if (t < 256u + CTL_WORDS) ctl[t - 256u] = 0u; if (unpack) build_table3(tab3, t - 256u); }
        if (!unpack) {
            ((LAS u32*)lds)[t] = 0u;
#pragma unroll
            for (u32 k = 1; k < 8u; ++k) wsl = (t & 63u) == k ? p.ws[k] : wsl;
        }
        asm volatile("s_waitcnt lgkmcnt(0)" ::: "memory"); __builtin_amdgcn_s_barrier(); asm volatile("" ::: "memory");
    };

    if (role & 7u) {
        const u32 rel = role & 7u, pjj = role >> 3, src = (rank + rel) & 7u;
        char* pws = p.ws[rel];
        setup(true);
        if (early) {
            const u32 call = rfl(callv) + 1u;
            const Dest dst = dest_of(frag_base(pws, false, pn >> 3, 0u, 8u - rel, call), 0u, pn & 7u);
#pragma unroll
            for (int g = 0; g < 8; ++g) asm volatile("" : "+v"(a[g]));
#pragma unroll
            for (int g = 0; g < 8; ++g) { store_group<AUX_SYS>(dst, (u32)lane * 16u, g, a[g]); asm volatile("s_waitcnt vmcnt(%0)" :: "n"(PUSHW - 1) : "memory"); }
            STAMP_AT(12);
            if (lane == 0) __hip_atomic_store(callp, call, __ATOMIC_RELAXED, __HIP_MEMORY_SCOPE_AGENT);
            asm volatile("s_waitcnt vmcnt(0)" ::: "memory");
            STAMP_AT(13);
            arrive(pws, 1u, 1u, call, lane);
        }
        LAS u32* vready = ctl; LAS u32* endk = ctl + 1; LAS u32* landed = ctl + 2; LAS u32* prog = ctl + 5;
        LAS u32* vdone = ctl + 8; LAS u32* sdone = ctl + 16; LAS u32* ffree = ctl + 32;
        LAS u32* q = ctl + 64;
        const u32 l0 = la(lds), voff = (u32)lane * 16u;
        if (wid == 0) {
            const u32 call = rfl(callv) + 1u;
            if (lane == 0) __hip_atomic_store(callp, call, __ATOMIC_RELAXED, __HIP_MEMORY_SCOPE_AGENT);
            GAS u64* gran = (GAS u64*)(ws + OFF_GRAN) + (size_t)((call & 1u) * 8u + src) * NC;
            for (u32 k = 0;; ++k) {
                u32 c = PUSHC + pjj;
                if (k) {
                    while (lds_ld(prog) < k) __builtin_amdgcn_s_sleep(2);
                    u32 tk = 0;
                    if (lane == 0) tk = __hip_atomic_fetch_add((GAS u32*)(ws + OFF_TICK + src * 128u), 1u, __ATOMIC_RELAXED, __HIP_MEMORY_SCOPE_AGENT);
                    c = rfl(tk) - (call - 1u) * (NC - PUSHC) + PULL_WG_PER_PEER + PUSHC;
                }
                u32 fl = 2u, vb = 0, kc = 2u;
                const char* from = pws; char* to = ws;
                if (c < (u32)NC) {
                    if (c >= (u32)NCIN) {
                        const u32 yc = c - NCIN;
                        from = pws + OFF_SY + (size_t)yc * CH;
                        to = y_base(p.out, (yc >> 1) & 15u, src * 16u + (yc >> 5), yc & 1u);
                    } else {
                        const InPos ip = in_of(c);
                        from = pws + OFF_SI + (size_t)((call & 1u) * NTASK + slot_in(ip.isw, ip.s, ip.q, rank)) * CH;
                        to = frag_base(ws, ip.isw, ip.s, ip.q, rel, call);
                        kc = (ip.isw ? 1u : 0u) | ((ip.bt * 8u + ip.q + 1u) << 2);
                    }
                    if ((TOUCH & 1) && k == 0u) {
                        const u32 warm = __hip_atomic_load((const GAS u32*)((lane & 1) ? to : from), __ATOMIC_RELAXED, __HIP_MEMORY_SCOPE_SYSTEM);
                        asm volatile("" :: "v"(warm));
                    }
                    const u64 t0 = __builtin_amdgcn_s_memrealtime();
                    for (u32 spin = 0;; ++spin) {
                        const u64 g = __hip_atomic_load(gran + c, __ATOMIC_RELAXED, __HIP_MEMORY_SCOPE_SYSTEM);
                        if (rfl((u32)g) == call) { fl = 0u; vb = rfl((u32)(g >> 32)); break; }
                        if ((spin & 15u) == 15u && give_up(tmo, call, t0)) { fl = 1u; break; }
                        __builtin_amdgcn_s_sleep(1);
                    }
                    if (k == 0) STAMP_AT(1);
                }
                while (lds_ld(vdone + (k & (NV - 1))) < 8u * (k / NV)) __builtin_amdgcn_s_sleep(2);
                if (lane == 0) {
                    *(volatile LAS u32x4*)(q + (k & (NV - 1)) * 8u + 4u) = (u32x4){(u32)(u64)to, (u32)((u64)to >> 32), kc, call};
                    *(volatile LAS u32x4*)(q + (k & (NV - 1)) * 8u) = (u32x4){(u32)(u64)from, (u32)((u64)from >> 32), fl | ((k + 1u) << 2), vb};
                }
                if (fl & 2u) { if (lane == 0) *(volatile LAS u32*)endk = k + 1u; break; }
                if (!(fl & 1u)) {
                    const u32 vl = vb > (u32)VSLOT ? (u32)CHDR : (vb + 15u) & ~15u;
                    const rsrc_t r = mk_rsrc(from, CVAR + vl);
                    for (u32 o = 0; o < vl; o += 1024u) { dma_piece(r, voff + CVAR + o, 0u, rfl(l0 + VRING + (k & (NV - 1)) * VSLOT + o)); if (k) asm volatile("s_waitcnt vmcnt(0)" ::: "memory"); }
                    asm volatile("s_waitcnt vmcnt(0)" ::: "memory");
                }
                if (lane == 0) *(volatile LAS u32*)vready = k + 1u;
            }
        } else if (wid == 1) {
            __builtin_amdgcn_s_setprio(3);
            const u32 la_landed = lane ? la(wl) + (u32)lane * 4u : la(landed);
            u32 n = 0;
            auto drain = [&]() { asm volatile("s_waitcnt vmcnt(0)" ::: "memory"); if (lane == 0) __hip_atomic_fetch_max(landed, n, __ATOMIC_RELAXED, __HIP_MEMORY_SCOPE_WORKGROUP); };
            u32x4 e = (u32x4){0, 0, 0, 0};
            u32 fr = 0;
            for (u32 k = 0;; ++k) {
                if (rfl(e[2]) >> 2 != k + 1u) {
                    e = *(volatile LAS u32x4*)(q + (k & (NV - 1)) * 8u);
                    if (rfl(e[2]) >> 2 != k + 1u) {
                        drain();
                        do { __builtin_amdgcn_s_sleep(1); e = *(volatile LAS u32x4*)(q + (k & (NV - 1)) * 8u); } while (rfl(e[2]) >> 2 != k + 1u);
                    }
                }
                const u32 fl = rfl(e[2]);
                if (fl & 2u) break;
                const rsrc_t r = mk_rsrc((const void*)(((u64)e[1] << 32) | e[0]), (fl & 1u) ? 0u : (u32)CVAR);
#pragma unroll
                for (u32 b = 0; b < 8u; ++b, ++n) {
                    const u32 sl = n & (NF - 1), m = n + 1u;
                    if (rfl(fr) + NF <= n && lds_ld(ffree + sl) + NF <= n) { drain(); while (lds_ld(ffree + sl) + NF <= n) __builtin_amdgcn_s_sleep(1); }
                    fr = *(volatile LAS u32*)(ffree + (m & (NF - 1)));
                    if (b == 7u) e = *(volatile LAS u32x4*)(q + ((k + 1u) & (NV - 1)) * 8u);
                    if (b == pjj % LVAR && lane == 0) *(volatile LAS u32*)prog = k + 1u;
                    dma_block(r, voff, b * FIXB, rfl(l0 + sl * FIXB), la_landed, n ? n - 1u : 0u);
                }
            }
            drain();
            STAMP_AT(2);
        } else {
#define SEG(i)
            u32 owed = 0xffffffffu, call = 0;
            auto settle = [&]() {
                if (owed != 0xffffffffu) {
                    asm volatile("s_waitcnt vmcnt(0)" ::: "memory");
                    u32 old = 0;
                    if (lane == 0) old = __hip_atomic_fetch_add(sdone + (owed & 7u), 1u, __ATOMIC_RELAXED, __HIP_MEMORY_SCOPE_WORKGROUP);
                    if ((rfl(old) & 7u) == 7u) arrive(ws, owed >> 3, 8u, call, lane);
                    owed = 0xffffffffu;
                }
            };
            for (u32 n = (u32)wid - 2u;; n += NDEC) {
                const u32 k = n >> 3, b = n & 7u, sl = n & (NF - 1);
                bool end = false;
                for (;;) {
                    const u32x4 st = *(volatile LAS u32x4*)ctl;
                    if (rfl(st[0]) > k && rfl(st[2]) > n) break;
                    if (rfl(st[1]) - 1u <= k) { end = true; break; }
                    settle();
                    __builtin_amdgcn_s_sleep(DEC_SLEEP);
                }
                if (end) break;
                asm volatile("" ::: "memory");
                u32x4 d[5];
                const LAS char* vs = lds + VRING + (k & (NV - 1)) * VSLOT;
                const u32 h = *(const LAS u32*)(vs + b * 4u);
                const u32x4 e0 = *(const LAS u32x4*)(q + (k & (NV - 1)) * 8u), e1 = *(const LAS u32x4*)(q + (k & (NV - 1)) * 8u + 4u);
                d[4] = *(const LAS u32x4*)(lds + sl * FIXB + 4096 + lane * 16);
#pragma unroll
                for (int j = 0; j < 4; ++j) d[j] = *(const LAS u32x4*)(lds + sl * FIXB + j * 1024 + lane * 16);
                asm volatile("s_waitcnt lgkmcnt(0)" ::: "memory");
                if (lane == 0) *(volatile LAS u32*)(ffree + sl) = n + 1u;
                asm volatile("" : "+v"(d[0]), "+v"(d[1]), "+v"(d[2]), "+v"(d[3]), "+v"(d[4]) :: "memory");
                const u32 hdr = rfl(h), off = (hdr >> 19) * 4u, kc = rfl(e1[2]);
                SEG(0);
                call = rfl(e1[3]);
                const bool far = rfl(e0[3]) > (u32)VSLOT;
                const LAS u32* var = (const LAS u32*)(vs + (off <= (u32)VSLOT ? off : 0u));
                u32x4 o[8];
                if (rfl(e0[2]) & 1u) {
#pragma unroll
                    for (int g = 0; g < 8; ++g) o[g] = (u32x4){0x7fc07fc0u, 0x7fc07fc0u, 0x7fc07fc0u, 0x7fc07fc0u};
                } else {
                    if (far) {
                        const rsrc_t r = mk_rsrc((const void*)(((u64)e0[1] << 32) | e0[0]), CH);
#pragma unroll
                        for (int j = 0; j < 3; ++j) o[5 + j] = __builtin_amdgcn_raw_buffer_load_b128(r, (int)(CVAR + off) - ((hdr & 0xffu) ? 512 : 0) + j * 1024 + lane * 16, 0, AUX_SYS);
                        if (lane < VARB / 16) *(LAS u32x4*)(wl + A2B + A3B + lane * 16) = o[5];
                        WAVE_FENCE();
                        var = (const LAS u32*)(wl + A2B + A3B + 512);
                    }
                    if (hdr & 0xffu) decode_block(d, hdr, var, tab, tab3, wl, lane, [&](int g, u32x4 x) { o[g] = x; });
                    else {
#pragma unroll
                        for (int g = 0; g < 5; ++g) o[g] = d[g];
                        if (!far) {
#pragma unroll
                            for (int j = 0; j < 3; ++j)
#pragma unroll
                                for (int i = 0; i < 4; ++i) o[5 + j][i] = *(const volatile LAS u32*)(var + j * 256 + lane * 4 + i);
                        }
                    }
                }
                const Dest dst = dest_of((char*)(((u64)e1[1] << 32) | e1[0]), kc & 3u, b);
                asm volatile("s_waitcnt lgkmcnt(0)" ::: "memory");
                if (lane == 0) __hip_atomic_fetch_add(vdone + (k & (NV - 1)), 1u, __ATOMIC_RELAXED, __HIP_MEMORY_SCOPE_WORKGROUP);
                SEG(1);
                settle();
                SEG(3);
                const u32 loff = (u32)(lane >> 4) * dst.LH + (u32)(lane & 15) * 16u;
                if ((kc & 3u) == 2u) {
#pragma unroll
                    for (int g = 0; g < 8; ++g) store_group<AUX_OUT>(dst, loff, g, o[g]);
                } else {
#pragma unroll
                    for (int g = 0; g < 8; ++g) store_group<AUX_DEV>(dst, loff, g, o[g]);
                    owed = (k & 7u) | ((kc >> 2) << 3);
                }
                SEG(2);
            }
            settle();
            STAMP_AT(3);
        }
        return;
    }

    {
        const bool packer = bid >= (u32)NGEMM;
        const u64 tb = __builtin_amdgcn_s_memrealtime();
        if ((TOUCH & 2) && bid - 120u < 8u && wid == 0) {
            const char* q = ws;
#pragma unroll
            for (u32 k = 1; k < 8u; ++k) q = (u32)lane == k ? p.ws[k] : q;
            while (__builtin_amdgcn_s_memrealtime() - tb < TOUCH_AT) __builtin_amdgcn_s_sleep(2);
            const u32 warm = __hip_atomic_load((const GAS u32*)(q + OFF_GRAN), __ATOMIC_RELAXED, __HIP_MEMORY_SCOPE_SYSTEM);
            asm volatile("" :: "v"(warm));
        }
        const u32 tend = packer ? (u32)NTASK : (u32)(NTASK - PACK_IN), b = (u32)wid;
        u32 it = 0;
        for (u32 task = packer ? (role >> 3) + NTASK - PACK_IN : bid; task < tend; task += packer ? (u32)NPACK : (u32)NGEMM, ++it) {
            if (task >= 16u - PUSHC) while (__builtin_amdgcn_s_memrealtime() - tb < (u64)(ENC_DELAY + (packer ? 0u : task * ENC_PACE / 64u))) __builtin_amdgcn_s_sleep(4);
            if (it == STAMP_IT) STAMP_AT(12);
            const u32 te = task < 16u - PUSHC ? (task + PUSHC) << 4 : (p.task[task >> 1] >> ((task & 1u) * 16u)) & 0xffffu, c = te >> 4, k = te & 15u;
            const InPos ip = in_of(c);
            const u32 dd = (rank + k) & 7u;
            if (ip.isw) {
                const rsrc_t r = mk_rsrc(p.w + (size_t)(ip.q * 128u + (b >> 1) * 32u) * 65536u + (gpanel(dd, ip.s) * 256u + (b & 1u) * 128u) * 2u, 31u * 65536u + 256u);
#pragma unroll
                for (int i = 0; i < 8; ++i) a[i] = __builtin_amdgcn_raw_buffer_load_b128(r, (lane >> 4) * 8 * 65536 + (lane & 15) * 16, i * 65536, AUX_NT);
            } else if (task >= 16u - PUSHC) load_x(px, ip.s, ip.q, b, lane, a);
            if (it == 0) setup(false);
            u32 v[8][4];
            if (ip.isw) {
#pragma unroll
                for (int j = 0; j < 8; ++j)
#pragma unroll
                    for (int e = 0; e < 4; ++e) v[j][e] = vperm(a[2 * e + 1][j >> 1], a[2 * e][j >> 1], (j & 1) ? 0x07060302u : 0x05040100u);
            } else {
#pragma unroll
                for (int j = 0; j < 8; ++j)
#pragma unroll
                    for (int e = 0; e < 4; ++e) v[j][e] = a[j][e];
            }
            const bool own = (k & 7u) == 0u, send = k < 8u;
            const u32 call = rfl(callv) + 1u;
            LAS u32* cnt = (LAS u32*)lds + 4u * it;
            u32 total;
            if (send) {
                encode_block(v, mk_rsrc(ws + OFF_SI + (size_t)((call & 1u) * NTASK + slot_in(ip.isw, ip.s, ip.q, dd)) * CH, CH), b, cnt, tab, wl, lane);
                if (it == STAMP_IT) STAMP_AT(14);
                asm volatile("s_waitcnt vmcnt(0)" ::: "memory");
                if (chunk_done(cnt, lane, total) && (k ? (u32)lane == k : (u32)lane - 1u < 7u))
                    __hip_atomic_store((GAS u64*)(wsl + OFF_GRAN) + (size_t)((call & 1u) * 8u + rank) * NC + c,
                                       ((u64)total << 32) | call, __ATOMIC_RELAXED, __HIP_MEMORY_SCOPE_SYSTEM);
            }
            if (own) {
                const Dest dst = dest_of(frag_base(ws, ip.isw, ip.s, ip.q, 0u, call), ip.isw ? 1u : 0u, b);
#pragma unroll
                for (int g = 0; g < 8; ++g) store_group<AUX_DEV>(dst, (u32)lane * 16u, g, (u32x4){v[g][0], v[g][1], v[g][2], v[g][3]});
                asm volatile("s_waitcnt vmcnt(0)" ::: "memory");
                if (chunk_done(cnt + 2, lane, total)) arrive(ws, ip.bt * 8u, 8u, call, lane);
            }
            if (it == STAMP_IT) STAMP_AT(1);
        }
        if (lane == 0) __hip_atomic_store(callp, callv + 1u, __ATOMIC_RELAXED, __HIP_MEMORY_SCOPE_AGENT);
        if (packer) {
            STAMP_AT(4);
            if (wid >= 4) __builtin_amdgcn_s_setprio(1);
            const u32 call = rfl(callv) + 1u;
            GAS u32* tf = (GAS u32*)(ws + OFF_TILE + b * 4u);
            u32 seen = 0, nanv = 0;
            auto fetch = [&](u32 yc) {
                if (rfl(seen) != call) {
                    const u64 t0 = __builtin_amdgcn_s_memrealtime();
                    while (rfl(__hip_atomic_load(tf + (yc >> 1) * 32u, __ATOMIC_RELAXED, __HIP_MEMORY_SCOPE_AGENT)) != call) {
                        if (give_up(tmo, call, t0)) { nanv = 0x7fc07fc0u; break; }
                        __builtin_amdgcn_s_sleep(POLL_SLEEP);
                    }
                }
                asm volatile("" ::: "memory");
                const Dest sy = dest_of(y_base(p.out, (yc >> 1) & 15u, rank * 16u + (yc >> 5), yc & 1u), 2u, b);
#pragma unroll
                for (int i = 0; i < 8; ++i) a[i] = __builtin_amdgcn_raw_buffer_load_b128(sy.r, (lane >> 4) * 4 * 65536 + (lane & 15) * 16, ((i & 3) + (i >> 2) * 16) * 65536, AUX_SYS);
                seen = __hip_atomic_load(tf + ((yc + NPACK) >> 1) * 32u, __ATOMIC_RELAXED, __HIP_MEMORY_SCOPE_AGENT);
            };
            auto publish = [&](u32 yc) {
                u32 total;
                if (chunk_done((LAS u32*)lds + 4u * it, lane, total) && (u32)lane - 1u < 7u)
                    __hip_atomic_store((GAS u64*)(wsl + OFF_GRAN) + (size_t)((call & 1u) * 8u + rank) * NC + NCIN + yc,
                                       ((u64)total << 32) | call, __ATOMIC_RELAXED, __HIP_MEMORY_SCOPE_SYSTEM);
                ++it;
            };
            u32 yc = role >> 3;
            fetch(yc);
            STAMP_AT(3);
            for (;; yc += NPACK) {
                u32 v[8][4];
#pragma unroll
                for (int j = 0; j < 8; ++j)
#pragma unroll
                    for (int e = 0; e < 4; ++e) v[j][e] = a[j][e] | nanv;
                if (yc + NPACK < (u32)NCY) fetch(yc + NPACK);
                encode_block(v, mk_rsrc(ws + OFF_SY + (size_t)yc * CH, CH), b, (LAS u32*)lds + 4u * it, tab, wl, lane);
                asm volatile("s_waitcnt vmcnt(0)" ::: "memory");
                publish(yc);
                if (yc + NPACK >= (u32)NCY) break;
            }
            __builtin_amdgcn_s_setprio(0);
            if (!PJOIN || NLAST == WB) { STAMP_AT(2); return; }
        }
        STAMP_AT(2);
    }

    const u32 call = rfl(callv) + 1u;
    const int wr = wid >> 2, wc = wid & 3;
    constexpr int HTB = 16384;
#define SA(b, h) (((b) * 2 + (h)) * HTB)
#define SB(b, h) ((4 + (b) * 2 + (h)) * HTB)
#define STAGE(bufoff, gbase) do { _Pragma("unroll") for (int _i = 0; _i < 2; ++_i) \
        __builtin_amdgcn_global_load_lds((const GAS u32*)((gbase) + voff + _i * 8192), (LAS u32*)(lds + (bufoff) + ldsw + _i * 8192), 16, 0, AUX_SYS); } while (0)
#define LDA(dst, b, h) do { _Pragma("unroll") for (int m = 0; m < 4; ++m) _Pragma("unroll") for (int k = 0; k < 2; ++k) dst[m][k] = *(const LAS bf16x8*)(lds + SA(b, h) + aoff + m * 2048 + k * 1024); } while (0)
#define LDB(dst, b, h) do { _Pragma("unroll") for (int n = 0; n < 2; ++n) _Pragma("unroll") for (int k = 0; k < 2; ++k) dst[n][k] = *(const LAS bf16x8*)(lds + SB(b, h) + boff + n * 2048 + k * 1024); } while (0)
#define MMA(ai, bj, At, Bt) do { _Pragma("unroll") for (int _x = 0; _x < 16; ++_x) { const int m = (_x >> 1) & 3, n = (_x ^ (_x >> 1)) & 1, k = _x >> 3; \
        acc[ai][bj][m][n] = __builtin_amdgcn_mfma_f32_16x16x32_bf16(Bt[n][k], At[m][k], acc[ai][bj][m][n], 0, 0, 0); SCHED; } } while (0)
#define MMA2(ai) do { __builtin_amdgcn_s_setprio(1); MMA(ai, 0, At, B0); MMA(ai, 1, At, B1); __builtin_amdgcn_s_setprio(0); } while (0)
#define WAIT_V(n) asm volatile("s_waitcnt vmcnt(" #n ")" ::: "memory")
#define WAIT_L(n) asm volatile("s_waitcnt lgkmcnt(" #n ")" ::: "memory")
#define BAR __builtin_amdgcn_s_barrier()
    const u32 voff = (u32)tid * 16u, ldsw = (u32)wid * 1024u;
    const int aoff = wr * 8192 + lane * 16, boff = wc * 4096 + lane * 16;
    const u32 idx = bid >> 3, hh = (bid >> 2) & 1u;
    u32 ti = (bid & 3u) * 4u + (idx < 4u * (WB / 2) ? idx & 3u : 2u * hh + (idx & 1u)), tj = idx < 4u * (WB / 2) ? 2u * (idx >> 2) + hh : WB - 1u;
    constexpr size_t kstep = 32768, hstep = 16384;
    const GAS char* xf = (const GAS char*)frag_base(ws, false, 0u, 0u, 0u, call);
    const GAS char* wf = (const GAS char*)(ws + OFF_WF);
    GAS u32* flags = (GAS u32*)(ws + OFF_FLAG + (bid & 7u) * 256u) + ((u32)lane & 7u);
    auto arrived = [&](u32 bt) { return (u32)__builtin_ctz(((u32)__builtin_amdgcn_ballot_w64(__hip_atomic_load(flags + bt * 8u, __ATOMIC_RELAXED, __HIP_MEMORY_SCOPE_SYSTEM) != call) & 31u) | 32u); };
#define STAGE_FIRST(pA, pB) do { STAGE(SB(0, 0), pB); STAGE(SA(0, 0), pA); STAGE(SB(0, 1), (pB) + hstep); STAGE(SA(0, 1), (pA) + hstep); \
        STAGE(SB(1, 0), (pB) + kstep); STAGE(SA(1, 0), (pA) + kstep); STAGE(SB(1, 1), (pB) + hstep + kstep); } while (0)
    u32 nwait = 0, bad = 0, staged = 0;
    auto draw_ticket = [&]() { return __hip_atomic_fetch_add((GAS u32*)(ws + OFF_TICK + 8u * 128u), 1u, __ATOMIC_RELAXED, __HIP_MEMORY_SCOPE_AGENT) - (call - 1u) * (NGEMM + (PJOIN ? NPACK : 0)); };
    auto last_tile = [&](u32 tk, u32& i, u32& j) { const u32 gr = tk / (4u * NLAST), r = tk - gr * 4u * NLAST; i = (gr * 4u + (r & 3u)) & 15u; j = gr < 4u ? (NB - 1u) * WB + (r >> 2) : (u32)NP; };
    u32 round = 0;
    if (bid >= (u32)NGEMM) {
        if (tid == 0) *(volatile LAS u32*)(ctl + 2) = draw_ticket();
        asm volatile("s_waitcnt lgkmcnt(0)" ::: "memory"); BAR; asm volatile("" ::: "memory");
        last_tile(lds_ld(ctl + 2), ti, tj);
        round = NB - 1u;
    }
    for (; tj < (u32)NP; ++round) {
        const GAS char* cA = wf + (size_t)tj * (2u << 20);
        const GAS char* cB = xf + (size_t)ti * (2u << 20);
        u32 ni = ti, nj = tj + WB;
        f32x4 acc[2][2][4][2];
#pragma unroll
        for (int a = 0; a < 2; ++a)
#pragma unroll
            for (int b = 0; b < 2; ++b)
#pragma unroll
                for (int m = 0; m < 4; ++m)
#pragma unroll
                    for (int n = 0; n < 2; ++n) acc[a][b][m][n] = (f32x4){0.f, 0.f, 0.f, 0.f};
        bf16x8 At[4][2], B0[2][2], B1[2][2];
        for (u32 q0 = 0; q0 < 5u;) {
            const u32 k0 = q0 ? q0 * 14u - 6u : 0u;
            if (!staged) {
                if (wid == 0) {
                    const u64 t0 = __builtin_amdgcn_s_memrealtime();
                    u32 q1;
                    for (u32 spin = 0;; ++spin) {
                        q1 = arrived(round);
                        if (q1 > q0) break;
                        if ((spin & 7u) == 7u && give_up(tmo, call, t0)) { q1 = 5u | 8u; break; }
                        __builtin_amdgcn_s_sleep(POLL_SLEEP);
                    }
                    if (lane == 0) *(volatile LAS u32*)(ctl + (nwait & 1u)) = q1;
                    asm volatile("s_waitcnt lgkmcnt(0)" ::: "memory");
                }
                asm volatile("" ::: "memory"); BAR; asm volatile("" ::: "memory");
                const u32 te = lds_ld(ctl + (nwait & 1u));
                ++nwait; bad |= te >> 3;
                staged = te & 7u;
                STAGE_FIRST(cA + (size_t)k0 * kstep, cB + (size_t)k0 * kstep);
            }
            if (k0 == 0u && round < 4u) STAMP_AT(3 + round * 2);
            q0 = staged & 7u;
            const u32 k1 = q0 * 14u - 6u;
            if (wr == 1) BAR;
            bool lax = (staged >> 3) != 0u;
            if (lax) WAIT_V(22); else WAIT_V(6);
            BAR; BAR;
            for (u32 t = k0; t < k1; t += 2u) {
                const u32 t2 = t + 2u < k1 ? t + 2u : t;
                const GAS char* a1 = cA + (size_t)(t + 1u) * kstep;
                const GAS char* a2 = cA + (size_t)t2 * kstep; const GAS char* b2 = cB + (size_t)t2 * kstep;
                const GAS char* a3 = a2 + kstep; const GAS char* b3 = b2 + kstep;
                LDB(B0, 0, 0); LDB(B1, 0, 1); LDA(At, 0, 0); STAGE(SA(1, 1), a1 + hstep);
                if (lax) WAIT_V(24); else WAIT_V(8);
                WAIT_L(0); BAR; SCHED; MMA2(0); BAR; SCHED;
                LDA(At, 0, 1); STAGE(SB(0, 0), b2); STAGE(SB(0, 1), b2 + hstep); STAGE(SA(0, 0), a2);
                if (lax) WAIT_V(24); else WAIT_V(8);
                lax = false;
                WAIT_L(0); BAR; SCHED; MMA2(1); BAR; SCHED;
                LDB(B0, 1, 0); LDB(B1, 1, 1); LDA(At, 1, 0); STAGE(SA(0, 1), a2 + hstep);
                WAIT_V(8); WAIT_L(0); BAR; SCHED; MMA2(0); BAR; SCHED;
                LDA(At, 1, 1); STAGE(SB(1, 0), b3); STAGE(SB(1, 1), b3 + hstep); STAGE(SA(1, 0), a3);
                WAIT_V(8); WAIT_L(0); BAR; SCHED; MMA2(1); BAR; SCHED;
            }
            const bool last = q0 == 5u, draw = NLAST < WB && last && round == NB - 2u;
            if (wid == 0) {
                u32 tk = 0;
                if (draw && lane == 0) tk = draw_ticket();
                const u32 q1 = arrived(last ? round + 1u : round);
                if (lane == 0) *(volatile LAS u32*)(ctl + (nwait & 1u)) = q1 | (tk << 8);
                asm volatile("s_waitcnt lgkmcnt(0)" ::: "memory");
            }
            WAIT_V(0);
            if (wr == 0) BAR;
            BAR;
            asm volatile("" ::: "memory");
            staged = lds_ld(ctl + (nwait & 1u));
            ++nwait;
            if (draw) last_tile(staged >> 8, ni, nj);
            staged &= 7u;
            if (staged <= (last ? 0u : q0) || (last && nj >= (u32)NP)) staged = 0u;
            if (staged) STAGE_FIRST(last ? wf + (size_t)nj * (2u << 20) : cA + (size_t)k1 * kstep, last ? xf + (size_t)ni * (2u << 20) : cB + (size_t)k1 * kstep);
        }
        if (round < 4u) STAMP_AT(4 + round * 2);
#pragma unroll
        for (int bj = 0; bj < 2; ++bj) {
            const Dest dst = dest_of(y_base(p.out, ti, gpanel(rank, tj), bj), 2u, (u32)wid);
            const u32 loff = (u32)(lane >> 4) * dst.LH + (u32)(lane & 15) * 16u;
            u32x4 v[8];
#pragma unroll
            for (int g = 0; g < 8; ++g)
#pragma unroll
                for (int q = 0; q < 4; ++q) {
                    typedef float f32x2 __attribute__((ext_vector_type(2)));
                    typedef __bf16 bf16x2 __attribute__((ext_vector_type(2)));
                    const f32x2 f = {acc[q >> 1][bj][(q & 1) * 2][g >> 2][g & 3], acc[q >> 1][bj][(q & 1) * 2 + 1][g >> 2][g & 3]};
                    v[g][q] = __builtin_bit_cast(u32, __builtin_convertvector(f, bf16x2));
                }
            if (bad) {
#pragma unroll
                for (int g = 0; g < 8; ++g) v[g] = (u32x4){0x7fc07fc0u, 0x7fc07fc0u, 0x7fc07fc0u, 0x7fc07fc0u};
            }
            if (tj < (u32)POWN) {
#pragma unroll
                for (int g = 0; g < 8; ++g) store_group<AUX_DEV>(dst, loff, g, v[g]);
            } else if (staged) {
#pragma unroll
                for (int g = 0; g < 8; ++g) store_group<AUX_NT>(dst, loff, g, v[g]);
            } else {
#pragma unroll
                for (int g = 0; g < 8; ++g) store_group<AUX_OUT>(dst, loff, g, v[g]);
            }
        }
        if (tj < (u32)POWN) {
            asm volatile("s_waitcnt vmcnt(0)" ::: "memory");
            if (lane == 0) __hip_atomic_store((GAS u32*)(ws + OFF_TILE + (tj * 16u + ti) * 128u + (u32)wid * 4u), call, __ATOMIC_RELAXED, __HIP_MEMORY_SCOPE_AGENT);
        } else if (staged) staged |= 8u;
        ti = ni; tj = nj;
    }
    STAMP_AT(15);
}

static void build_tables(Params& p) {
    unsigned char* role = (unsigned char*)p.role;
    for (int i = 0; i < 64; ++i) role[i] = (unsigned char)(i < NPULL ? (1 + i % 7) | (i / 7) << 3 : (i - NPULL) << 3);
    {
        constexpr int SL = (256 - NGEMM) / 2;
        static_assert(4 * PULL_WG_PER_PEER <= SL, "roles");
        int cnt[2] = {0, 0}, np = 0;
        for (int j = 0; j < PULL_WG_PER_PEER; ++j)
            for (int rel = 1; rel < 8; ++rel) {
                const int h = (((0xb2 >> p.rank) ^ (0xb2 >> ((p.rank + rel) & 7))) & 1), e = cnt[h]++;
                role[(e >> 2) * 8 + h * 4 + ((e + (h ? e >> 2 : 0)) & 3)] = (unsigned char)(rel | j << 3);
            }
        for (int h = 0; h < 2; ++h) for (int e = cnt[h]; e < SL; ++e) role[(e >> 2) * 8 + h * 4 + (e & 3)] = (unsigned char)(np++ << 3);
    }
    unsigned short* t = (unsigned short*)p.task;
    int n = 0;
    for (u32 c = PUSHC; c < (u32)NCIN; ++c) {
        if (c == 16u) for (u32 e = 0; e < (u32)PUSHC; ++e) t[n++] = (unsigned short)(e << 4 | 8u);
        const InPos ip = in_of(c);
        if (ip.isw && ip.s < (u32)POWN) for (u32 k = 1; k <= 8; ++k) t[n++] = (unsigned short)(c << 4 | k);
        else t[n++] = (unsigned short)(c << 4);
    }
}

extern "C" void kernel_launch(int rank, int nranks, ncclComm_t comm, hipStream_t stream,
                              void* const* d_in, const int* in_sizes, int n_in, void* d_out, int out_size,
                              const int* in_dtypes, int out_dtype, void* const* d_out_all,
                              void* const* d_ws_all, size_t ws_bytes) {
    if (nranks != NR || (unsigned)rank >= (unsigned)NR || ws_bytes < WS_NEED || n_in != 2) { fprintf(stderr, "unsupported configuration\n"); return; }
    Params p;
    p.x = (const char*)d_in[0]; p.w = (const char*)d_in[1]; p.out = (char*)d_out; p.rank = rank; p.pad = 0;
    for (int i = 0; i < NR; ++i) p.ws[i] = (char*)d_ws_all[(rank + i) % NR];
    build_tables(p);
    hipLaunchKernelGGL(gemm_ar_fused, dim3(256), dim3(512), 0, stream, p);
    HIP_CHECK(hipGetLastError());
}
```

```cpp
#include <hip/hip_runtime.h>
#include <rccl/rccl.h>
#include <cstdio>

#define HIP_CHECK(expr) do { hipError_t _e = (expr); if (_e != hipSuccess) { \
    fprintf(stderr, "HIP error %s at %s:%d\n", hipGetErrorString(_e), __FILE__, __LINE__); \
    return; } } while (0)

typedef unsigned u32;
typedef unsigned long long u64;
typedef u32 u32x4 __attribute__((ext_vector_type(4)));
typedef u32 u32x2 __attribute__((ext_vector_type(2)));
typedef unsigned short u16x2 __attribute__((ext_vector_type(2)));
typedef short bf16x8 __attribute__((ext_vector_type(8)));
typedef float f32x4 __attribute__((ext_vector_type(4)));
#define LAS __attribute__((address_space(3)))
#define GAS __attribute__((address_space(1)))
typedef __amdgpu_buffer_rsrc_t rsrc_t;

constexpr int NR = 8;
#define PULL_WG_PER_PEER 8
#define PAT 56565
#define LVAR 5
#define PUSHC 5
#define PUSHW 1
#define PACK_IN 112
#define AFFINE 1
#define TOUCH 7
#define TOUCH_AT 100
#define CODEPF 1
#define PJOIN 1
#define DEC_SLEEP 2
#define RHO 6
#define ENC_DELAY 150
#define ENC_PACE 192
#define POLL_SLEEP 8
#define WB 12
constexpr int NPULL = 7 * PULL_WG_PER_PEER, NGEMM = 16 * WB, NPACK = 256 - NGEMM - NPULL;
constexpr int POWN = 16 - RHO, NP = POWN + 8 * RHO;
constexpr int NB = (NP + WB - 1) / WB, N0 = 16 + WB;
constexpr int NCIN = 64 + 4 * NP, NCY = 32 * POWN, NC = NCIN + NCY;
constexpr int NTASK = 16 * 36;
constexpr int NDEC = 6;
constexpr u64 DEADLINE = 100000000ull;
constexpr int PUSHB = (8 * PUSHC + PULL_WG_PER_PEER - 1) / PULL_WG_PER_PEER;
constexpr int NLAST = NP - (NB - 1) * WB;
static_assert(NGEMM >= 192 && NPACK < 32 && NPACK >= 4 && (PACK_IN + NCY) / NPACK < 126 && NB <= 8 && POWN <= WB && PUSHB <= NDEC, "roles");

constexpr int FIXB = 5120, CVAR = 8 * FIXB, CHDR = 32, CH = 66560;
constexpr size_t OFF_CALL = 0;
constexpr size_t OFF_TMO = 131072;
constexpr size_t OFF_TICK = OFF_TMO + 256;
constexpr size_t OFF_RDY = OFF_TICK + 16 * 128;
constexpr size_t OFF_FLAG = OFF_RDY + 64 * 128;
constexpr size_t OFF_TILE = OFF_FLAG + 8 * 256;
constexpr size_t OFF_GRAN = 196608;
constexpr size_t OFF_DBG = 524288;
constexpr size_t OFF_XF = 1ull << 20;
constexpr size_t OFF_WF = OFF_XF + (64ull << 20);
constexpr size_t OFF_SY = OFF_WF + NP * (2ull << 20);
constexpr size_t OFF_SI = OFF_SY + (size_t)NCY * CH;
constexpr size_t WS_NEED = OFF_SI + 2ull * NTASK * CH;

constexpr int A2B = 2064, A3B = 528, VARB = 912, WLDS = A2B + A3B + VARB, N2MAX = 2032, N3MAX = 496, UBITS = 2016, NPATCH = 31;
constexpr int CTL_OFF = 2048, CTL_WORDS = 128, WL_OFF = 2560, BIG_OFF = 30720, LDS_BYTES = BIG_OFF + 131072;
static_assert(WL_OFF + 8 * WLDS <= BIG_OFF && LDS_BYTES <= 163840, "LDS map");
constexpr int NF = 16, NV = 4, VSLOT = 8192, VRING = NF * FIXB;
constexpr int AUX_SYS = 17, AUX_DEV = 16, AUX_OUT = 18, AUX_NT = 2;

struct Params {
    int rank; int pad;
    const char* x; const char* w; char* out;
    char* ws[NR];
    u32 role[16];
    u32 task[NTASK / 2];
};

__device__ __forceinline__ u32 rfl(u32 x) { return (u32)__builtin_amdgcn_readfirstlane((int)x); }
__device__ __forceinline__ rsrc_t mk_rsrc(const void* p, u32 bytes) {
    const u64 a = (u64)p; const u64 u = ((u64)rfl((u32)(a >> 32)) << 32) | rfl((u32)a);
    return __builtin_amdgcn_make_buffer_rsrc((void*)u, (short)0, (int)rfl(bytes), 0x00020000);
}
__device__ __forceinline__ u32 vperm(u32 a, u32 b, u32 s) { return __builtin_amdgcn_perm(a, b, s); }
__device__ __forceinline__ u32 wave_incl_scan(u32 x) {
    x += (u32)__builtin_amdgcn_update_dpp(0, (int)x, 0x111, 0xf, 0xf, false);
    x += (u32)__builtin_amdgcn_update_dpp(0, (int)x, 0x112, 0xf, 0xf, false);
    x += (u32)__builtin_amdgcn_update_dpp(0, (int)x, 0x114, 0xf, 0xf, false);
    x += (u32)__builtin_amdgcn_update_dpp(0, (int)x, 0x118, 0xf, 0xf, false);
    x += (u32)__builtin_amdgcn_update_dpp(0, (int)x, 0x142, 0xa, 0xf, false);
    x += (u32)__builtin_amdgcn_update_dpp(0, (int)x, 0x143, 0xc, 0xf, false);
    return x;
}
__device__ __forceinline__ u32 pk_sub(u32 a, u32 b) { return __builtin_bit_cast(u32, (u16x2)(__builtin_bit_cast(u16x2, a) - __builtin_bit_cast(u16x2, b))); }
template <int AUX>
__device__ __forceinline__ void st128(u32x4 x, rsrc_t r, int voff, int soff) {
    __builtin_amdgcn_raw_buffer_store_b128(x, r, voff, soff, AUX);
    asm volatile("s_nop 1" :: "v"(x) : "memory");
}
#define WAVE_FENCE() __builtin_amdgcn_fence(__ATOMIC_ACQ_REL, "wavefront")

struct TabEntry { u32 dep, cmp, n, r0, r1; };
__device__ __forceinline__ TabEntry table_entry(u32 i) {
    u32 n = 0, dep = 0, cmp = 0x0c0c0c0cu;
#pragma unroll
    for (u32 t = 0; t < 4; ++t) {
        const u32 c = (i >> (2 * t)) & 3u;
        dep |= (c ? 4u + c : n) << (8 * t);
        cmp ^= c ? 0u : (0x0cu ^ t) << (8 * n);
        n += c ? 0u : 1u;
    }
    u64 r = 0;
#pragma unroll
    for (u32 t = 0; t < 7; ++t) r += (u64)(((i >> t) & 1u) ^ ((i >> t) ? 1u : 0u)) << (8u * (u32)__builtin_popcount(i & ((1u << t) - 1u)));
    return TabEntry{dep, cmp, n, (u32)r, (u32)(r >> 32)};
}
__device__ __forceinline__ void or3(LAS u32* q, u64 x, u32 sh) {
    const u64 y = x << sh;
    __hip_atomic_fetch_or(q, (u32)y, __ATOMIC_RELAXED, __HIP_MEMORY_SCOPE_WORKGROUP);
    __hip_atomic_fetch_or(q + 1, (u32)(y >> 32), __ATOMIC_RELAXED, __HIP_MEMORY_SCOPE_WORKGROUP);
    __hip_atomic_fetch_or(q + 2, (u32)(((x >> 32) << sh) >> 32), __ATOMIC_RELAXED, __HIP_MEMORY_SCOPE_WORKGROUP);
}
__device__ __forceinline__ u32 code_byte(u32 hi, u32 lo, u32 x) {
    const u32 f = vperm(hi, lo, x);
    return __builtin_amdgcn_udot4((f ^ (f >> 2)) & 0x03030303u, 0x40100401u, 0u, false);
}
template <bool PRED>
__device__ __forceinline__ void or_bytes(u32 pos, u32 x) {
    const u64 y = (u64)x << ((pos & 3u) * 8u);
    LAS u32* q = (LAS u32*)(size_t)(pos & ~3u);
    if (!PRED || x) {
        __hip_atomic_fetch_or(q, (u32)y, __ATOMIC_RELAXED, __HIP_MEMORY_SCOPE_WORKGROUP);
        __hip_atomic_fetch_or(q + 1, (u32)(y >> 32), __ATOMIC_RELAXED, __HIP_MEMORY_SCOPE_WORKGROUP);
    }
}
__device__ __forceinline__ u32 get_bytes(u32 pos) {
    const LAS u32* q = (const LAS u32*)(size_t)(pos & ~3u);
    return __builtin_amdgcn_alignbyte(q[1], q[0], pos);
}
__device__ __forceinline__ u32 la(const LAS void* p) { return (u32)(size_t)p; }
__device__ __forceinline__ u32 pack_level2(const LAS u32x2* tab, const LAS char* src, u32 n, LAS u32* end, LAS char* dst, int lane) {
    u32 m = 0;
    for (u32 i0 = 0; i0 < n; i0 += 1024u) {
        const u32 i = i0 + (u32)lane * 16u, hi = 0x02010000u, lo = 0x00030000u;
        const u32x4 x = *(const LAS u32x4*)(src + (i < n ? i : 0u));
        u32 cw = code_byte(hi, lo, x[0]) | (code_byte(hi, lo, x[1]) << 8) | (code_byte(hi, lo, x[2]) << 16) | (code_byte(hi, lo, x[3]) << 24);
        const u32 valid = i < n ? n - i : 0u;
        if (valid < 16u) cw |= 0x55555555u << (2u * valid);
        u32x2 e[4]; u32 c[5]; c[0] = 0;
#pragma unroll
        for (int j = 0; j < 4; ++j) { e[j] = tab[(cw >> (8 * j)) & 0xffu]; c[j + 1] = c[j] + e[j][1]; }
        const u32 incl = wave_incl_scan(c[4]), p0 = la(dst) + m + incl - c[4];
        if (i < n) *(end - 1 - (i >> 4)) = cw;
        m += (u32)__builtin_amdgcn_readlane((int)incl, 63);
        if (m <= (u32)N3MAX) {
#pragma unroll
            for (int j = 0; j < 4; ++j) or_bytes<true>(p0 + c[j], vperm(0u, x[j], e[j][0]));
        }
    }
    WAVE_FENCE();
    return m;
}
__device__ __forceinline__ void pack_level3(const LAS char* src, u32 n, LAS u32* ub, LAS u32* pt, u32 T, int lane, u32& bits, u32& np) {
    for (u32 i0 = 0; i0 < n; i0 += 256u) {
        const u32 i = i0 + (u32)lane * 4u;
        const u32 x = *(const LAS u32*)(src + (i < n ? i : 0u));
        u32 cum = 0, pf = 0; u64 m = 0;
#pragma unroll
        for (u32 t = 0; t < 4; ++t) {
            const u32 r = ((x >> (8 * t)) - 8u) & 0xffu;
            const bool valid = i + t < n, big = r > 15u;
            cum += valid && !big ? r : 0u;
            m |= (u64)(valid ? 1u : 0u) << cum;
            cum += valid ? 1u : 0u;
            pf |= valid && big ? 1u << t : 0u;
        }
        const u32 incl = wave_incl_scan(cum), p = bits + incl - cum;
        bits += (u32)__builtin_amdgcn_readlane((int)incl, 63);
        if (bits <= (u32)UBITS && cum) or3(ub + (p >> 5), m, p & 31u);
        if (__builtin_amdgcn_ballot_w64(pf != 0u)) {
#pragma unroll
            for (u32 t = 0; t < 4; ++t) {
                const u64 bm = __builtin_amdgcn_ballot_w64((pf >> t) & 1u);
                const u32 at = np + __builtin_amdgcn_mbcnt_hi((u32)(bm >> 32), __builtin_amdgcn_mbcnt_lo((u32)bm, 0u));
                if ((pf >> t) & 1u) pt[at] = ((i + t) << 8) | ((T - (x >> (8 * t))) & 0xffu);
                np += (u32)__builtin_popcountll(bm);
            }
        }
    }
    WAVE_FENCE();
}
__device__ __forceinline__ void encode_block(const u32 (&v)[8][4], rsrc_t ch, u32 b, LAS u32* alloc, const LAS u32x2* tab, LAS char* wl, int lane) {
#pragma unroll
    for (int j = 0; j < 4; ++j) if (j < 3 || lane < (WLDS - 3072) / 16) *(LAS u32x4*)(wl + j * 1024 + lane * 16) = (u32x4){0, 0, 0, 0};
    u32 s = 0;
#pragma unroll
    for (int g = 0; g < 8; ++g) s = __builtin_amdgcn_udot4(__builtin_amdgcn_alignbit(v[g][0], v[g][0], 31), 0x01000100u, s, false);
    u32 T = ((u32)__builtin_amdgcn_readlane((int)wave_incl_scan(s), 63) >> 10) + 5u;
    T = T > 255u ? 255u : (T < 23u ? 23u : T);
    const u32 tt = (T * 256u + 255u) * 0x10001u;
    const int fo = (int)(b * FIXB);
    u32 U[16], l1[4] = {0, 0, 0, 0}, c[17];
    u32x2 e[16];
#pragma unroll
    for (int q = 0; q < 4; ++q) {
        u32x4 sm;
#pragma unroll
        for (int j = 0; j < 4; ++j) {
            const int k = 4 * q + j;
            const u32 a0 = v[k >> 1][(k & 1) * 2], a1 = v[k >> 1][(k & 1) * 2 + 1];
            const u32 r0 = __builtin_amdgcn_alignbit(a0, a0, 31), r1 = __builtin_amdgcn_alignbit(a1, a1, 31);
            sm[j] = vperm(r1, r0, 0x06040200u);
            U[k] = vperm(pk_sub(tt, r1), pk_sub(tt, r0), 0x07050301u);
            const u32 cb = code_byte(0x00000302u, 0x01000000u, U[k]);
            l1[q] |= cb << (8 * j);
            e[k] = tab[cb];
        }
        st128<AUX_SYS>(sm, ch, lane * 16 + q * 1024, fo);
    }
    st128<AUX_SYS>((u32x4){l1[0], l1[1], l1[2], l1[3]}, ch, lane * 16 + 4096, fo);
    __builtin_amdgcn_sched_barrier(0);
    c[0] = 0;
#pragma unroll
    for (int k = 0; k < 16; ++k) c[k + 1] = c[k] + e[k][1];
    const u32 incl = wave_incl_scan(c[16]), p0 = la(wl) + incl - c[16];
    const u32 n2 = (u32)__builtin_amdgcn_readlane((int)incl, 63);
    bool raw = n2 > (u32)N2MAX;
    u32 d2 = 0, d3 = 0, np = 0;
    LAS u32* var = (LAS u32*)(wl + A2B + A3B);
    if (!raw) {
        WAVE_FENCE();
#pragma unroll
        for (int k = 0; k < 16; ++k) or_bytes<false>(p0 + c[k], vperm(0u, U[k], e[k][0]));
        WAVE_FENCE();
        d2 = (n2 + 15u) >> 4;
        const u32 n3 = pack_level2(tab, wl, n2, var + d2, wl + A2B, lane);
        raw = n3 > (u32)N3MAX;
        if (!raw) {
            u32 bits = 0;
            pack_level3(wl + A2B, n3, var + d2, (LAS u32*)wl, T, lane, bits, np);
            raw = bits > (u32)UBITS || np > (u32)NPATCH;
            d3 = (bits + 31u) >> 5;
            if (!raw && (u32)lane < np) var[d2 + d3 + (u32)lane] = ((const LAS u32*)wl)[lane];
            WAVE_FENCE();
        }
    }
    const u32 vd = d2 + d3 + np;
    u32 off = 0;
    if (lane == 0) off = __hip_atomic_fetch_add(alloc, raw ? 3072u : vd * 4u, __ATOMIC_RELAXED, __HIP_MEMORY_SCOPE_WORKGROUP);
    off = rfl(off) + CHDR;
    const int vo = CVAR + (int)off;
    if (lane == 0) __builtin_amdgcn_raw_buffer_store_b32(raw ? off << 17 : (T | (d3 << 8) | (np << 14) | ((off + d2 * 4u) << 17)), ch, 0, CVAR + (int)b * 4, AUX_SYS);
    if (raw) {
        asm volatile("s_waitcnt vmcnt(0)" ::: "memory");
#pragma unroll
        for (int g = 0; g < 8; ++g)
            st128<AUX_SYS>((u32x4){v[g][0], v[g][1], v[g][2], v[g][3]}, ch, lane * 16 + (g < 5 ? g : g - 5) * 1024, g < 5 ? fo : vo);
    } else {
        for (u32 o = (u32)lane; o < vd; o += 64u) __builtin_amdgcn_raw_buffer_store_b32(var[o], ch, (int)o * 4, vo, AUX_SYS);
    }
    WAVE_FENCE();
}
__device__ __forceinline__ bool chunk_done(LAS u32* cnt, int lane, u32& total) {
    u32 n = 0;
    if (lane == 0) n = __hip_atomic_fetch_add(cnt + 1, 1u, __ATOMIC_RELAXED, __HIP_MEMORY_SCOPE_WORKGROUP);
    if (rfl(n) != 7u) return false;
    total = rfl(*(volatile LAS u32*)cnt) + CHDR;
    return true;
}

struct Dest { rsrc_t r; u32 A, B, C, LH; };
__device__ __forceinline__ Dest dest_of(char* base, u32 kind, u32 b) {
    if (kind == 0u) return Dest{mk_rsrc(base + b * 4096u, 32768u + 4096u), 1024u, 32768u, 2048u, 256u};
    if (kind == 1u) return Dest{mk_rsrc(base + (b >> 2) * 32768u + (b & 1u) * 8192u + (b & 2u) * 512u, 16384u + 6144u + 1024u), 2048u, 4096u, 16384u, 256u};
    return Dest{mk_rsrc(base + (b & 3u) * (32u * 65536u) + (b >> 2) * 256u, 31u * 65536u + 256u), 65536u, 131072u, 16u * 65536u, 4u * 65536u};
}
template <int AUX>
__device__ __forceinline__ void store_group(const Dest& d, u32 loff, int g, u32x4 x) {
    st128<AUX>(x, d.r, (int)loff, (int)((g & 1 ? d.A : 0u) + (g & 2 ? d.B : 0u) + (g & 4 ? d.C : 0u)));
}

#define SCHED __builtin_amdgcn_sched_barrier(0)
__device__ __forceinline__ u32x2 get2(u32 pos) { const LAS u32* q = (const LAS u32*)(size_t)(pos & ~3u); return (u32x2){q[0], q[1]}; }
__device__ __forceinline__ void spread3(u32 w, u32 start, u32 n, u32x2 runs, u32 t8, u32 p) {
    if (n) {
        const u32 carry = (u32)__builtin_clz((w << 8) | start);
        const u64 x = ((u64)(t8 - runs[1]) << 32) | (t8 - runs[0] - carry);
        const u32 sh = 64u - 8u * n;
        or3((LAS u32*)(size_t)(p & ~3u), (x << sh) >> sh, (p & 3u) * 8u);
    }
}
template <class Out>
__device__ __forceinline__ void decode_block(const u32x4 (&d)[5], u32 hdr, const LAS u32* ub, const LAS u32x2* tab, const LAS u32x2* tab3, LAS char* wl, int lane, Out out) {
    const u32 T = hdr & 0xffu, d3 = (hdr >> 8) & 0x3fu, np = (hdr >> 14) & 0x1fu;
    auto ex = [&](u32 a, u32 b, u32 c) { return ((T - a) << 8) | ((T - b) << 16) | ((T - c) << 24); };
    const u32 t1 = ex(3, 4, 5), t2 = ex(6, 7, 2), t8 = (T - 8u) * 0x01010101u;
    const u32 w2 = *(ub - 1 - lane), p3i = la(ub) + (u32)lane - 3u;
    u32x2 e[16], e2[4], g[16];
    g[0] = get2(p3i);
    if (lane < A3B / 16) *(LAS u32x4*)(wl + A2B + lane * 16) = (u32x4){0, 0, 0, 0};
#pragma unroll
    for (int k = 0; k < 8; ++k) e[k] = tab[(d[4][k >> 2] >> (8 * (k & 3))) & 0xffu];
    SCHED;
    const u32 w3 = __builtin_amdgcn_alignbyte(g[0][1], g[0][0], p3i), x3 = (u32)lane < d3 * 4u ? w3 >> 24 : 0u;
    const u32x2 e3 = tab3[x3];
#pragma unroll
    for (int j = 0; j < 4; ++j) e2[j] = tab[(w2 >> (8 * j)) & 0xffu];
#pragma unroll
    for (int k = 8; k < 16; ++k) e[k] = tab[(d[4][k >> 2] >> (8 * (k & 3))) & 0xffu];
    SCHED;
    u32 c[17], f[5];
    c[0] = 0; f[0] = 0;
#pragma unroll
    for (int k = 0; k < 16; ++k) c[k + 1] = c[k] + e[k][1];
#pragma unroll
    for (int j = 0; j < 4; ++j) f[j + 1] = f[j] + e2[j][1];
    const u32 n3 = (u32)__builtin_popcount(x3);
    const u32 pk = c[16] | (f[4] << 11) | (n3 << 22);
    const u32 incl = wave_incl_scan(pk), ex0 = incl - pk, tot = (u32)__builtin_amdgcn_readlane((int)incl, 63);
    const u32 a3 = la(wl) + A2B, p1 = la(wl) + (ex0 & 0x7ffu), p2 = a3 + ((ex0 >> 11) & 0x1ffu);
    WAVE_FENCE();
    spread3(w3, lane < 3 ? 0x80000000u >> (8 * lane) : 0x80u, n3, e3, t8, a3 + ((ex0 >> 22) & 0x1ffu));
    for (u32 i0 = 64u, m = tot >> 22; i0 < d3 * 4u; i0 += 64u) {
        const u32 w = get_bytes(p3i + i0), x = i0 + (u32)lane < d3 * 4u ? w >> 24 : 0u, n = (u32)__builtin_popcount(x), in = wave_incl_scan(n);
        spread3(w, 0x80u, n, tab3[x], t8, a3 + ((m + in - n) & 0x1ffu));
        m += (u32)__builtin_amdgcn_readlane((int)in, 63);
    }
    WAVE_FENCE();
    if ((u32)lane < np) { const u32 pw = ub[d3 + (u32)lane]; *(LAS unsigned char*)(wl + A2B + ((pw >> 8) & 0x1ffu)) = (unsigned char)pw; }
    WAVE_FENCE();
#pragma unroll
    for (int j = 0; j < 4; ++j) { f[j] += p2; g[j] = get2(f[j]); }
    SCHED;
#pragma unroll
    for (int k = 0; k < 16; ++k) c[k] += p1;
    SCHED;
    u32x4 a2;
#pragma unroll
    for (int j = 0; j < 4; ++j) a2[j] = vperm(t2, __builtin_amdgcn_alignbyte(g[j][1], g[j][0], f[j]), e2[j][0]);
    *(LAS u32x4*)(wl + lane * 16) = a2;
    if ((tot & 0x7ffu) > 1024u) {
        const u32 cw = *(ub - 65 - lane);
        f[0] = 0;
#pragma unroll
        for (int j = 0; j < 4; ++j) { e2[j] = tab[(cw >> (8 * j)) & 0xffu]; f[j + 1] = f[j] + e2[j][1]; }
        const u32 p = a3 + (((tot >> 11) + wave_incl_scan(f[4]) - f[4]) & 0x1ffu);
#pragma unroll
        for (int j = 0; j < 4; ++j) a2[j] = vperm(t2, get_bytes(p + f[j]), e2[j][0]);
        *(LAS u32x4*)(wl + 1024 + lane * 16) = a2;
    }
    WAVE_FENCE();
#pragma unroll
    for (int k = 0; k < 16; ++k) g[k] = get2(c[k]);
    SCHED;
    u32x4 o;
#pragma unroll
    for (int k = 0; k < 16; ++k) {
        const u32 x = vperm(t1, __builtin_amdgcn_alignbyte(g[k][1], g[k][0], c[k]), e[k][0]);
        const u32 smk = d[k >> 2][k & 3];
        const u32 r0 = vperm(x, smk, 0x05010400u), r1 = vperm(x, smk, 0x07030602u);
        o[2 * (k & 1)] = __builtin_amdgcn_alignbit(r0, r0, 1);
        o[2 * (k & 1) + 1] = __builtin_amdgcn_alignbit(r1, r1, 1);
        if (k & 1) out(k >> 1, o);
    }
    WAVE_FENCE();
}

__device__ __forceinline__ char* frag_base(char* ws, bool isw, u32 s, u32 q, u32 rel, u32 call) {
    return ws + (isw ? OFF_WF : OFF_XF + (size_t)(call & 1u) * (32u << 20)) + (size_t)(s * 64u + (rel ? 6u + 14u * q + 2u * rel : 2u * q)) * 32768u;
}
__device__ __forceinline__ char* y_base(char* out, u32 i, u32 gp, u32 bj) { return out + (size_t)(i * 256u + bj * 128u) * 65536u + gp * 512u; }
__device__ __forceinline__ u32 gpanel(u32 r, u32 j) { return j < (u32)POWN ? r * 16u + j : ((j - POWN) & 7u) * 16u + POWN + ((j - POWN) >> 3); }
__device__ __forceinline__ u32 slot_in(bool isw, u32 s, u32 q, u32 dst) {
    return !isw ? s * 4u + q : (s < (u32)POWN ? 64u + (dst * POWN + s) * 4u + q : 64u + 28u * POWN + s * 4u + q);
}
__host__ __device__ __forceinline__ u32 batch_size(u32 bt) { return bt == 0u ? (u32)N0 : (bt == NB - 1u ? (u32)(NP - (NB - 1) * WB) : (u32)WB); }
struct InPos { bool isw; u32 s, q, bt; };
__host__ __device__ __forceinline__ InPos in_of(u32 c) {
    if (c < 4u * N0) { const u32 q = c / N0, pp = c - q * N0; return InPos{pp >= 16u, pp >= 16u ? pp - 16u : pp, q, 0u}; }
    const u32 bt = (c - 64u) / (4u * WB), r = c - 64u - bt * 4u * WB, q = r / batch_size(bt);
    return InPos{true, bt * WB + r - q * batch_size(bt), q, bt};
}

__device__ __forceinline__ void arrive(char* ws, u32 ci, u32 n, u32 call, int lane) {
    u32 old = 0;
    if (lane == 0) old = __hip_atomic_fetch_add((GAS u32*)(ws + OFF_RDY + ci * 128u), n, __ATOMIC_RELAXED, __HIP_MEMORY_SCOPE_SYSTEM);
    if (rfl(old) + n == call * ((ci & 7u) ? 56u : 32u) * batch_size(ci >> 3) && lane < 8)
        __hip_atomic_store((GAS u32*)(ws + OFF_FLAG + (u32)lane * 256u + ci * 4u), call, __ATOMIC_RELAXED, __HIP_MEMORY_SCOPE_SYSTEM);
}
__device__ __forceinline__ void load_x(const char* x, u32 s, u32 q, u32 b, int lane, u32x4 (&a)[8], bool on = true) {
    const rsrc_t r = mk_rsrc(x + (size_t)((s & 15u) * 256u + b * 32u) * 1024u + q * 256u, on ? 31u * 1024u + 256u : 0u);
#pragma unroll
    for (int i = 0; i < 8; ++i) a[i] = __builtin_amdgcn_raw_buffer_load_b128(r, (lane >> 2) * 1024 + ((lane & 3) ^ ((lane >> 5) << 1)) * 16 + (i & 3) * 64, (i >> 2) * 16384, 0);
}

__device__ __forceinline__ bool give_up(GAS u32* tmo, u32 call, u64 t0) {
    if (rfl(__hip_atomic_load(tmo, __ATOMIC_RELAXED, __HIP_MEMORY_SCOPE_AGENT)) == call || __builtin_amdgcn_s_memrealtime() - t0 > DEADLINE) {
        __hip_atomic_store(tmo, call, __ATOMIC_RELAXED, __HIP_MEMORY_SCOPE_AGENT);
        return true;
    }
    return false;
}

#define STAMP_IT 0
#define STAMP_AT(i) do { } while (0)

__device__ __forceinline__ u32 lds_ld(LAS u32* p) { return rfl(*(volatile LAS u32*)p); }
__device__ __forceinline__ void dma_piece(rsrc_t r, u32 voff, u32 soff, u32 ldsaddr) {
    u32 keep;
    asm volatile("s_nop 4\n\ts_mov_b32 %0, m0\n\ts_mov_b32 m0, %1\n\ts_nop 0\n\tbuffer_load_dwordx4 %2, %3, %4 offen sc0 sc1 lds\n\ts_mov_b32 m0, %0"
                 : "=&s"(keep) : "s"(ldsaddr), "v"(voff), "s"(r), "s"(soff) : "memory");
}

#define DMA_P(j, w, off) "s_waitcnt vmcnt(" w ")\n\tbuffer_load_dwordx4 %2, %3, " off " sc0 sc1 lds\n\t.if %8 == " #j "\n\tds_max_u32 %6, %7\n\t.endif\n\t"
__device__ __forceinline__ void dma_block(rsrc_t r, u32 voff, u32 soff, u32 ldsaddr, u32 at, u32 val) {
    constexpr int W0 = PAT / 10000, W1 = PAT / 1000 % 10, W2 = PAT / 100 % 10, W3 = PAT / 10 % 10, W4 = PAT % 10;
    constexpr int JP = W0 <= 5 ? 0 : (W1 <= 6 ? 1 : (W2 <= 7 ? 2 : (W3 <= 8 ? 3 : 4)));
    u32 keep;
    asm volatile("s_nop 2\n\ts_mov_b32 %0, m0\n\ts_mov_b32 m0, %1\n\t"
                 DMA_P(0, "%9", "%4 offen") DMA_P(1, "%10", "%4 offen offset:1024") DMA_P(2, "%11", "%4 offen offset:2048") DMA_P(3, "%12", "%4 offen offset:3072")
                 "s_add_u32 m0, m0, 0x1000\n\t" DMA_P(4, "%13", "%5 offen") "s_mov_b32 m0, %0"
                 : "=&s"(keep) : "s"(ldsaddr), "v"(voff), "s"(r), "s"(soff), "s"(soff + 4096u), "v"(at), "v"(val), "n"(JP),
                   "n"(W0), "n"(W1), "n"(W2), "n"(W3), "n"(W4) : "memory", "scc");
}

__global__ void __launch_bounds__(512, 2) gemm_ar_fused(const Params p) {
    __shared__ __attribute__((aligned(16))) char smem[LDS_BYTES];
    LAS char* lds = (LAS char*)smem + BIG_OFF;
    LAS u32x2* tab = (LAS u32x2*)smem;
    LAS u32x2* tab3 = (LAS u32x2*)((LAS char*)smem + WL_OFF);
    const int tid = threadIdx.x, wid = (int)rfl((u32)tid >> 6), lane = tid & 63;
    const u32 bid = blockIdx.x;
    if (CODEPF && bid - 16u < 64u) {
        u64 pc; u32 left;
        asm volatile("s_getpc_b64 %0\n.Lcode_here:\n\ts_mov_b32 %1, .Lcode_far-.Lcode_here" : "=s"(pc), "=s"(left));
        const u32 o = ((bid - 16u) >> 3) * 8192u + (u32)tid * 16u;
        if (o + 16u <= left) { const u32x4 t = *(const GAS u32x4*)(pc + o); asm volatile("" :: "v"(t)); }
    }
    u32 rank = (u32)p.rank, rolew = p.role[((bid - NGEMM) >> 2) & 15u];
    char* ws = p.ws[0]; const char* px = p.x;
    SCHED;
    TabEntry te = table_entry((u32)tid & 255u);
    asm volatile("" : "+s"(rank), "+s"(rolew), "+s"(ws), "+s"(px), "+v"(te.dep), "+v"(te.cmp), "+v"(te.n), "+v"(te.r0), "+v"(te.r1));
    LAS char* wl = (LAS char*)smem + WL_OFF + wid * WLDS;
    LAS u32* ctl = (LAS u32*)((LAS char*)smem + CTL_OFF);
    GAS u32* tmo = (GAS u32*)(ws + OFF_TMO);
    STAMP_AT(0);
    const u32 role = bid >= (u32)NGEMM ? (rolew >> ((bid & 3u) * 8u)) & 0xffu : 0u;
    const u32 pn = (role & 7u) ? (role >> 3) * PUSHB + (u32)wid - 2u : (bid + PUSHC) * 8u + (u32)wid;
    const bool early = (role & 7u) ? wid >= 2 && wid < 2 + PUSHB && pn < 8u * PUSHC : bid < 16u - PUSHC;
    GAS u32* callp = (GAS u32*)(ws + OFF_CALL + (bid * 8u + wid) * 64u);
    const u32 callv = __hip_atomic_load(callp, __ATOMIC_RELAXED, __HIP_MEMORY_SCOPE_AGENT);
    u32x4 a[8];
    load_x(px, pn >> 3, 0u, pn & 7u, lane, a, early);
    SCHED;
    char* wsl = ws;
    if (tid < 256) tab[tid] = (u32x2){(role & 7u) ? te.dep : te.cmp, te.n};
    else { if (tid < 256 + CTL_WORDS) ctl[tid - 256] = 0u; if (role & 7u) tab3[tid - 256] = (u32x2){te.r0, te.r1}; }
    if (!(role & 7u)) ((LAS u32*)lds)[tid] = 0u;
#pragma unroll
    for (int k = 1; k < 8; ++k) wsl = lane == k ? p.ws[k] : wsl;
    asm volatile("s_waitcnt lgkmcnt(0)" : "+v"(wsl) :: "memory"); __builtin_amdgcn_s_barrier(); asm volatile("" ::: "memory");

    if (role & 7u) {
        const u32 rel = role & 7u, pjj = role >> 3, src = (rank + rel) & 7u;
        char* pws = p.ws[rel];
        if ((TOUCH & 4) && wid == 1) { const u32 warm = __hip_atomic_load((const GAS u32*)(pws + OFF_XF + (size_t)(lane & 1) * (32u << 20) + (size_t)((pjj * PUSHB) >> 3) * (2u << 20)), __ATOMIC_RELAXED, __HIP_MEMORY_SCOPE_SYSTEM); asm volatile("" :: "v"(warm)); }
        if (early) {
            const u32 call = rfl(callv) + 1u;
            const Dest dst = dest_of(frag_base(pws, false, pn >> 3, 0u, 8u - rel, call), 0u, pn & 7u);
#pragma unroll
            for (int g = 0; g < 8; ++g) { store_group<AUX_SYS>(dst, (u32)lane * 16u, g, a[g]); asm volatile("s_waitcnt vmcnt(%0)" :: "n"(PUSHW - 1) : "memory"); }
            STAMP_AT(12);
            if (lane == 0) __hip_atomic_store(callp, call, __ATOMIC_RELAXED, __HIP_MEMORY_SCOPE_AGENT);
            asm volatile("s_waitcnt vmcnt(0)" ::: "memory");
            STAMP_AT(13);
            arrive(pws, 1u, 1u, call, lane);
        }
        LAS u32* vready = ctl; LAS u32* endk = ctl + 1; LAS u32* landed = ctl + 2; LAS u32* prog = ctl + 5;
        LAS u32* vdone = ctl + 8; LAS u32* sdone = ctl + 16; LAS u32* ffree = ctl + 32;
        LAS u32* q = ctl + 64;
        const u32 l0 = la(lds), voff = (u32)lane * 16u;
        if (wid == 0) {
            const u32 call = rfl(callv) + 1u;
            if (lane == 0) __hip_atomic_store(callp, call, __ATOMIC_RELAXED, __HIP_MEMORY_SCOPE_AGENT);
            GAS u64* gran = (GAS u64*)(ws + OFF_GRAN) + (size_t)((call & 1u) * 8u + src) * NC;
            for (u32 k = 0;; ++k) {
                u32 c = PUSHC + pjj;
                if (k) {
                    while (lds_ld(prog) < k) __builtin_amdgcn_s_sleep(2);
                    u32 tk = 0;
                    if (lane == 0) tk = __hip_atomic_fetch_add((GAS u32*)(ws + OFF_TICK + src * 128u), 1u, __ATOMIC_RELAXED, __HIP_MEMORY_SCOPE_AGENT);
                    c = rfl(tk) - (call - 1u) * (NC - PUSHC) + PULL_WG_PER_PEER + PUSHC;
                }
                u32 fl = 2u, vb = 0, kc = 2u;
                const char* from = pws; char* to = ws;
                if (c < (u32)NC) {
                    if (c >= (u32)NCIN) {
                        const u32 yc = c - NCIN;
                        from = pws + OFF_SY + (size_t)yc * CH;
                        to = y_base(p.out, (yc >> 1) & 15u, src * 16u + (yc >> 5), yc & 1u);
                    } else {
                        const InPos ip = in_of(c);
                        from = pws + OFF_SI + (size_t)((call & 1u) * NTASK + slot_in(ip.isw, ip.s, ip.q, rank)) * CH;
                        to = frag_base(ws, ip.isw, ip.s, ip.q, rel, call);
                        kc = (ip.isw ? 1u : 0u) | ((ip.bt * 8u + ip.q + 1u) << 2);
                    }
                    if ((TOUCH & 1) && k == 0u) {
                        const u32 warm = __hip_atomic_load((const GAS u32*)((lane & 1) ? to : from), __ATOMIC_RELAXED, __HIP_MEMORY_SCOPE_SYSTEM);
                        asm volatile("" :: "v"(warm));
                    }
                    const u64 t0 = __builtin_amdgcn_s_memrealtime();
                    for (u32 spin = 0;; ++spin) {
                        const u64 g = __hip_atomic_load(gran + c, __ATOMIC_RELAXED, __HIP_MEMORY_SCOPE_SYSTEM);
                        if (rfl((u32)g) == call) { fl = 0u; vb = rfl((u32)(g >> 32)); break; }
                        if ((spin & 15u) == 15u && give_up(tmo, call, t0)) { fl = 1u; break; }
                        __builtin_amdgcn_s_sleep(1);
                    }
                    if (k == 0) STAMP_AT(1);
                }
                while (lds_ld(vdone + (k & (NV - 1))) < 8u * (k / NV)) __builtin_amdgcn_s_sleep(2);
                if (lane == 0) {
                    *(volatile LAS u32x4*)(q + (k & (NV - 1)) * 8u + 4u) = (u32x4){(u32)(u64)to, (u32)((u64)to >> 32), kc, call};
                    *(volatile LAS u32x4*)(q + (k & (NV - 1)) * 8u) = (u32x4){(u32)(u64)from, (u32)((u64)from >> 32), fl | ((k + 1u) << 2), vb};
                }
                if (fl & 2u) { if (lane == 0) *(volatile LAS u32*)endk = k + 1u; break; }
                if (!(fl & 1u)) {
                    const u32 vl = vb > (u32)VSLOT ? (u32)CHDR : (vb + 15u) & ~15u;
                    const rsrc_t r = mk_rsrc(from, CVAR + vl);
                    for (u32 o = 0; o < vl; o += 1024u) { dma_piece(r, voff + CVAR + o, 0u, rfl(l0 + VRING + (k & (NV - 1)) * VSLOT + o)); if (k) asm volatile("s_waitcnt vmcnt(0)" ::: "memory"); }
                    asm volatile("s_waitcnt vmcnt(0)" ::: "memory");
                }
                if (lane == 0) *(volatile LAS u32*)vready = k + 1u;
            }
        } else if (wid == 1) {
            __builtin_amdgcn_s_setprio(3);
            const u32 la_landed = lane ? la(wl) + (u32)lane * 4u : la(landed);
            u32 n = 0;
            auto drain = [&]() { asm volatile("s_waitcnt vmcnt(0)" ::: "memory"); if (lane == 0) __hip_atomic_fetch_max(landed, n, __ATOMIC_RELAXED, __HIP_MEMORY_SCOPE_WORKGROUP); };
            u32x4 e = (u32x4){0, 0, 0, 0};
            u32 fr = 0;
            for (u32 k = 0;; ++k) {
                if (rfl(e[2]) >> 2 != k + 1u) {
                    e = *(volatile LAS u32x4*)(q + (k & (NV - 1)) * 8u);
                    if (rfl(e[2]) >> 2 != k + 1u) {
                        drain();
                        do { __builtin_amdgcn_s_sleep(1); e = *(volatile LAS u32x4*)(q + (k & (NV - 1)) * 8u); } while (rfl(e[2]) >> 2 != k + 1u);
                    }
                }
                const u32 fl = rfl(e[2]);
                if (fl & 2u) break;
                const rsrc_t r = mk_rsrc((const void*)(((u64)e[1] << 32) | e[0]), (fl & 1u) ? 0u : (u32)CVAR);
#pragma unroll
                for (u32 b = 0; b < 8u; ++b, ++n) {
                    const u32 sl = n & (NF - 1), m = n + 1u;
                    if (rfl(fr) + NF <= n && lds_ld(ffree + sl) + NF <= n) { drain(); while (lds_ld(ffree + sl) + NF <= n) __builtin_amdgcn_s_sleep(1); }
                    fr = *(volatile LAS u32*)(ffree + (m & (NF - 1)));
                    if (b == 7u) e = *(volatile LAS u32x4*)(q + ((k + 1u) & (NV - 1)) * 8u);
                    if (b == pjj % LVAR && lane == 0) *(volatile LAS u32*)prog = k + 1u;
                    dma_block(r, voff, b * FIXB, rfl(l0 + sl * FIXB), la_landed, n ? n - 1u : 0u);
                }
            }
            drain();
            STAMP_AT(2);
        } else {
#define SEG(i)
            u32 owed = 0xffffffffu, call = 0;
            auto settle = [&]() {
                if (owed != 0xffffffffu) {
                    asm volatile("s_waitcnt vmcnt(0)" ::: "memory");
                    u32 old = 0;
                    if (lane == 0) old = __hip_atomic_fetch_add(sdone + (owed & 7u), 1u, __ATOMIC_RELAXED, __HIP_MEMORY_SCOPE_WORKGROUP);
                    if ((rfl(old) & 7u) == 7u) arrive(ws, owed >> 3, 8u, call, lane);
                    owed = 0xffffffffu;
                }
            };
            for (u32 n = (u32)wid - 2u;; n += NDEC) {
                const u32 k = n >> 3, b = n & 7u, sl = n & (NF - 1);
                bool end = false;
                for (;;) {
                    const u32x4 st = *(volatile LAS u32x4*)ctl;
                    if (rfl(st[0]) > k && rfl(st[2]) > n) break;
                    if (rfl(st[1]) - 1u <= k) { end = true; break; }
                    settle();
                    __builtin_amdgcn_s_sleep(DEC_SLEEP);
                }
                if (end) break;
                asm volatile("" ::: "memory");
                u32x4 d[5];
                const LAS char* vs = lds + VRING + (k & (NV - 1)) * VSLOT;
                const u32 h = *(const LAS u32*)(vs + b * 4u);
                const u32x4 e0 = *(const LAS u32x4*)(q + (k & (NV - 1)) * 8u), e1 = *(const LAS u32x4*)(q + (k & (NV - 1)) * 8u + 4u);
                d[4] = *(const LAS u32x4*)(lds + sl * FIXB + 4096 + lane * 16);
#pragma unroll
                for (int j = 0; j < 4; ++j) d[j] = *(const LAS u32x4*)(lds + sl * FIXB + j * 1024 + lane * 16);
                asm volatile("s_waitcnt lgkmcnt(0)" ::: "memory");
                if (lane == 0) *(volatile LAS u32*)(ffree + sl) = n + 1u;
                asm volatile("" : "+v"(d[0]), "+v"(d[1]), "+v"(d[2]), "+v"(d[3]), "+v"(d[4]) :: "memory");
                const u32 hdr = rfl(h), off = (hdr >> 19) * 4u, kc = rfl(e1[2]);
                SEG(0);
                call = rfl(e1[3]);
                const bool far = rfl(e0[3]) > (u32)VSLOT;
                const LAS u32* var = (const LAS u32*)(vs + (off <= (u32)VSLOT ? off : 0u));
                u32x4 o[8];
                if (rfl(e0[2]) & 1u) {
#pragma unroll
                    for (int g = 0; g < 8; ++g) o[g] = (u32x4){0x7fc07fc0u, 0x7fc07fc0u, 0x7fc07fc0u, 0x7fc07fc0u};
                } else {
                    if (far) {
                        const rsrc_t r = mk_rsrc((const void*)(((u64)e0[1] << 32) | e0[0]), CH);
#pragma unroll
                        for (int j = 0; j < 3; ++j) o[5 + j] = __builtin_amdgcn_raw_buffer_load_b128(r, (int)(CVAR + off) - ((hdr & 0xffu) ? 512 : 0) + j * 1024 + lane * 16, 0, AUX_SYS);
                        if (lane < VARB / 16) *(LAS u32x4*)(wl + A2B + A3B + lane * 16) = o[5];
                        WAVE_FENCE();
                        var = (const LAS u32*)(wl + A2B + A3B + 512);
                    }
                    if (hdr & 0xffu) decode_block(d, hdr, var, tab, tab3, wl, lane, [&](int g, u32x4 x) { o[g] = x; });
                    else {
#pragma unroll
                        for (int g = 0; g < 5; ++g) o[g] = d[g];
                        if (!far) {
#pragma unroll
                            for (int j = 0; j < 3; ++j)
#pragma unroll
                                for (int i = 0; i < 4; ++i) o[5 + j][i] = *(const volatile LAS u32*)(var + j * 256 + lane * 4 + i);
                        }
                    }
                }
                const Dest dst = dest_of((char*)(((u64)e1[1] << 32) | e1[0]), kc & 3u, b);
                asm volatile("s_waitcnt lgkmcnt(0)" ::: "memory");
                if (lane == 0) __hip_atomic_fetch_add(vdone + (k & (NV - 1)), 1u, __ATOMIC_RELAXED, __HIP_MEMORY_SCOPE_WORKGROUP);
                SEG(1);
                settle();
                SEG(3);
                const u32 loff = (u32)(lane >> 4) * dst.LH + (u32)(lane & 15) * 16u;
                if ((kc & 3u) == 2u) {
#pragma unroll
                    for (int g = 0; g < 8; ++g) store_group<AUX_OUT>(dst, loff, g, o[g]);
                } else {
#pragma unroll
                    for (int g = 0; g < 8; ++g) store_group<AUX_DEV>(dst, loff, g, o[g]);
                    owed = (k & 7u) | ((kc >> 2) << 3);
                }
                SEG(2);
            }
            settle();
            STAMP_AT(3);
        }
        return;
    }

    {
        const bool packer = bid >= (u32)NGEMM;
        const u64 tb = __builtin_amdgcn_s_memrealtime();
        if ((TOUCH & 2) && bid - 120u < 8u && wid == 0) {
            while (__builtin_amdgcn_s_memrealtime() - tb < TOUCH_AT) __builtin_amdgcn_s_sleep(2);
            const u32 warm = __hip_atomic_load((const GAS u32*)(wsl + OFF_GRAN), __ATOMIC_RELAXED, __HIP_MEMORY_SCOPE_SYSTEM);
            asm volatile("" :: "v"(warm));
        }
        const u32 tend = packer ? (u32)NTASK : (u32)(NTASK - PACK_IN), b = (u32)wid;
        u32 it = 0;
        for (u32 task = packer ? (role >> 3) + NTASK - PACK_IN : bid; task < tend; task += packer ? (u32)NPACK : (u32)NGEMM, ++it) {
            if (task >= 16u - PUSHC) while (__builtin_amdgcn_s_memrealtime() - tb < (u64)(ENC_DELAY + (packer ? 0u : task * ENC_PACE / 64u))) __builtin_amdgcn_s_sleep(4);
            if (it == STAMP_IT) STAMP_AT(12);
            const u32 tw = task < 16u - PUSHC ? (task + PUSHC) << 4 : (p.task[task >> 1] >> ((task & 1u) * 16u)) & 0xffffu, c = tw >> 4, k = tw & 15u;
            const InPos ip = in_of(c);
            const u32 dd = (rank + k) & 7u;
            if (ip.isw) {
                const rsrc_t r = mk_rsrc(p.w + (size_t)(ip.q * 128u + (b >> 1) * 32u) * 65536u + (gpanel(dd, ip.s) * 256u + (b & 1u) * 128u) * 2u, 31u * 65536u + 256u);
#pragma unroll
                for (int i = 0; i < 8; ++i) a[i] = __builtin_amdgcn_raw_buffer_load_b128(r, (lane >> 4) * 8 * 65536 + (lane & 15) * 16, i * 65536, AUX_NT);
            } else if (task >= 16u - PUSHC) load_x(px, ip.s, ip.q, b, lane, a);
            u32 v[8][4];
            if (ip.isw) {
#pragma unroll
                for (int j = 0; j < 8; ++j)
#pragma unroll
                    for (int e = 0; e < 4; ++e) v[j][e] = vperm(a[2 * e + 1][j >> 1], a[2 * e][j >> 1], (j & 1) ? 0x07060302u : 0x05040100u);
            } else {
#pragma unroll
                for (int j = 0; j < 8; ++j)
#pragma unroll
                    for (int e = 0; e < 4; ++e) v[j][e] = a[j][e];
            }
            const bool own = (k & 7u) == 0u, send = k < 8u;
            const u32 call = rfl(callv) + 1u;
            LAS u32* cnt = (LAS u32*)lds + 4u * it;
            u32 total;
            if (send) {
                encode_block(v, mk_rsrc(ws + OFF_SI + (size_t)((call & 1u) * NTASK + slot_in(ip.isw, ip.s, ip.q, dd)) * CH, CH), b, cnt, tab, wl, lane);
                if (it == STAMP_IT) STAMP_AT(14);
                asm volatile("s_waitcnt vmcnt(0)" ::: "memory");
                if (chunk_done(cnt, lane, total) && (k ? (u32)lane == k : (u32)lane - 1u < 7u))
                    __hip_atomic_store((GAS u64*)(wsl + OFF_GRAN) + (size_t)((call & 1u) * 8u + rank) * NC + c,
                                       ((u64)total << 32) | call, __ATOMIC_RELAXED, __HIP_MEMORY_SCOPE_SYSTEM);
            }
            if (own) {
                const Dest dst = dest_of(frag_base(ws, ip.isw, ip.s, ip.q, 0u, call), ip.isw ? 1u : 0u, b);
#pragma unroll
                for (int g = 0; g < 8; ++g) store_group<AUX_DEV>(dst, (u32)lane * 16u, g, (u32x4){v[g][0], v[g][1], v[g][2], v[g][3]});
                asm volatile("s_waitcnt vmcnt(0)" ::: "memory");
                if (chunk_done(cnt + 2, lane, total)) arrive(ws, ip.bt * 8u, 8u, call, lane);
            }
            if (it == STAMP_IT) STAMP_AT(1);
        }
        if (lane == 0) __hip_atomic_store(callp, callv + 1u, __ATOMIC_RELAXED, __HIP_MEMORY_SCOPE_AGENT);
        if (packer) {
            STAMP_AT(4);
            if (wid >= 4) __builtin_amdgcn_s_setprio(1);
            const u32 call = rfl(callv) + 1u;
            GAS u32* tf = (GAS u32*)(ws + OFF_TILE + b * 4u);
            u32 seen = 0, nanv = 0;
            auto fetch = [&](u32 yc) {
                if (rfl(seen) != call) {
                    const u64 t0 = __builtin_amdgcn_s_memrealtime();
                    while (rfl(__hip_atomic_load(tf + (yc >> 1) * 32u, __ATOMIC_RELAXED, __HIP_MEMORY_SCOPE_AGENT)) != call) {
                        if (give_up(tmo, call, t0)) { nanv = 0x7fc07fc0u; break; }
                        __builtin_amdgcn_s_sleep(POLL_SLEEP);
                    }
                }
                asm volatile("" ::: "memory");
                const Dest sy = dest_of(y_base(p.out, (yc >> 1) & 15u, rank * 16u + (yc >> 5), yc & 1u), 2u, b);
#pragma unroll
                for (int i = 0; i < 8; ++i) a[i] = __builtin_amdgcn_raw_buffer_load_b128(sy.r, (lane >> 4) * 4 * 65536 + (lane & 15) * 16, ((i & 3) + (i >> 2) * 16) * 65536, AUX_SYS);
                seen = __hip_atomic_load(tf + ((yc + NPACK) >> 1) * 32u, __ATOMIC_RELAXED, __HIP_MEMORY_SCOPE_AGENT);
            };
            auto publish = [&](u32 yc) {
                u32 total;
                if (chunk_done((LAS u32*)lds + 4u * it, lane, total) && (u32)lane - 1u < 7u)
                    __hip_atomic_store((GAS u64*)(wsl + OFF_GRAN) + (size_t)((call & 1u) * 8u + rank) * NC + NCIN + yc,
                                       ((u64)total << 32) | call, __ATOMIC_RELAXED, __HIP_MEMORY_SCOPE_SYSTEM);
                ++it;
            };
            u32 yc = role >> 3;
            fetch(yc);
            STAMP_AT(3);
            for (;; yc += NPACK) {
                u32 v[8][4];
#pragma unroll
                for (int j = 0; j < 8; ++j)
#pragma unroll
                    for (int e = 0; e < 4; ++e) v[j][e] = a[j][e] | nanv;
                if (yc + NPACK < (u32)NCY) fetch(yc + NPACK);
                encode_block(v, mk_rsrc(ws + OFF_SY + (size_t)yc * CH, CH), b, (LAS u32*)lds + 4u * it, tab, wl, lane);
                asm volatile("s_waitcnt vmcnt(0)" ::: "memory");
                publish(yc);
                if (yc + NPACK >= (u32)NCY) break;
            }
            __builtin_amdgcn_s_setprio(0);
            if (!PJOIN || NLAST == WB) { STAMP_AT(2); return; }
        }
        STAMP_AT(2);
    }

    const u32 call = rfl(callv) + 1u;
    const int wr = wid >> 2, wc = wid & 3;
    constexpr int HTB = 16384;
#define SA(b, h) (((b) * 2 + (h)) * HTB)
#define SB(b, h) ((4 + (b) * 2 + (h)) * HTB)
#define STAGE(bufoff, gbase) do { _Pragma("unroll") for (int _i = 0; _i < 2; ++_i) \
        __builtin_amdgcn_global_load_lds((const GAS u32*)((gbase) + voff + _i * 8192), (LAS u32*)(lds + (bufoff) + ldsw + _i * 8192), 16, 0, AUX_SYS); } while (0)
#define LDA(dst, b, h) do { _Pragma("unroll") for (int m = 0; m < 4; ++m) _Pragma("unroll") for (int k = 0; k < 2; ++k) dst[m][k] = *(const LAS bf16x8*)(lds + SA(b, h) + aoff + m * 2048 + k * 1024); } while (0)
#define LDB(dst, b, h) do { _Pragma("unroll") for (int n = 0; n < 2; ++n) _Pragma("unroll") for (int k = 0; k < 2; ++k) dst[n][k] = *(const LAS bf16x8*)(lds + SB(b, h) + boff + n * 2048 + k * 1024); } while (0)
#define MMA(ai, bj, At, Bt) do { _Pragma("unroll") for (int _x = 0; _x < 16; ++_x) { const int m = (_x >> 1) & 3, n = (_x ^ (_x >> 1)) & 1, k = _x >> 3; \
        acc[ai][bj][m][n] = __builtin_amdgcn_mfma_f32_16x16x32_bf16(Bt[n][k], At[m][k], acc[ai][bj][m][n], 0, 0, 0); SCHED; } } while (0)
#define MMA2(ai) do { __builtin_amdgcn_s_setprio(1); MMA(ai, 0, At, B0); MMA(ai, 1, At, B1); __builtin_amdgcn_s_setprio(0); } while (0)
#define WAIT_V(n) asm volatile("s_waitcnt vmcnt(" #n ")" ::: "memory")
#define WAIT_L(n) asm volatile("s_waitcnt lgkmcnt(" #n ")" ::: "memory")
#define BAR __builtin_amdgcn_s_barrier()
    const u32 voff = (u32)tid * 16u, ldsw = (u32)wid * 1024u;
    const int aoff = wr * 8192 + lane * 16, boff = wc * 4096 + ((lane & 15) * 4 + ((lane >> 4) ^ (((lane >> 3) & 1) << 1))) * 16;
    const u32 idx = bid >> 3, hh = (bid >> 2) & 1u;
    u32 ti = (bid & 3u) * 4u + (idx < 4u * (WB / 2) ? idx & 3u : 2u * hh + (idx & 1u)), tj = idx < 4u * (WB / 2) ? 2u * (idx >> 2) + hh : WB - 1u;
    constexpr size_t kstep = 32768, hstep = 16384;
    const GAS char* xf = (const GAS char*)frag_base(ws, false, 0u, 0u, 0u, call);
    const GAS char* wf = (const GAS char*)(ws + OFF_WF);
    GAS u32* flags = (GAS u32*)(ws + OFF_FLAG + (bid & 7u) * 256u) + ((u32)lane & 7u);
    auto arrived = [&](u32 bt) { return (u32)__builtin_ctz(((u32)__builtin_amdgcn_ballot_w64(__hip_atomic_load(flags + bt * 8u, __ATOMIC_RELAXED, __HIP_MEMORY_SCOPE_SYSTEM) != call) & 31u) | 32u); };
#define STAGE_FIRST(pA, pB) do { STAGE(SB(0, 0), pB); STAGE(SA(0, 0), pA); STAGE(SB(0, 1), (pB) + hstep); STAGE(SA(0, 1), (pA) + hstep); \
        STAGE(SB(1, 0), (pB) + kstep); STAGE(SA(1, 0), (pA) + kstep); STAGE(SB(1, 1), (pB) + hstep + kstep); } while (0)
    u32 nwait = 0, bad = 0, staged = 0;
    auto draw_ticket = [&]() { return __hip_atomic_fetch_add((GAS u32*)(ws + OFF_TICK + 8u * 128u), 1u, __ATOMIC_RELAXED, __HIP_MEMORY_SCOPE_AGENT) - (call - 1u) * (NGEMM + (PJOIN ? NPACK : 0)); };
    auto last_tile = [&](u32 tk, u32& i, u32& j) { const u32 gr = tk / (4u * NLAST), r = tk - gr * 4u * NLAST; i = (gr * 4u + (r & 3u)) & 15u; j = gr < 4u ? (NB - 1u) * WB + (r >> 2) : (u32)NP; };
    u32 round = 0;
    if (bid >= (u32)NGEMM) {
        if (tid == 0) *(volatile LAS u32*)(ctl + 2) = draw_ticket();
        asm volatile("s_waitcnt lgkmcnt(0)" ::: "memory"); BAR; asm volatile("" ::: "memory");
        last_tile(lds_ld(ctl + 2), ti, tj);
        round = NB - 1u;
    }
    for (; tj < (u32)NP; ++round) {
        const GAS char* cA = wf + (size_t)tj * (2u << 20);
        const GAS char* cB = xf + (size_t)ti * (2u << 20);
        u32 ni = ti, nj = tj + WB;
        f32x4 acc[2][2][4][2];
#pragma unroll
        for (int a = 0; a < 2; ++a)
#pragma unroll
            for (int b = 0; b < 2; ++b)
#pragma unroll
                for (int m = 0; m < 4; ++m)
#pragma unroll
                    for (int n = 0; n < 2; ++n) acc[a][b][m][n] = (f32x4){0.f, 0.f, 0.f, 0.f};
        bf16x8 At[4][2], B0[2][2], B1[2][2];
        for (u32 q0 = 0; q0 < 5u;) {
            const u32 k0 = q0 ? q0 * 14u - 6u : 0u;
            if (!staged) {
                if (wid == 0) {
                    const u64 t0 = __builtin_amdgcn_s_memrealtime();
                    u32 q1;
                    for (u32 spin = 0;; ++spin) {
                        q1 = arrived(round);
                        if (q1 > q0) break;
                        if ((spin & 7u) == 7u && give_up(tmo, call, t0)) { q1 = 5u | 8u; break; }
                        __builtin_amdgcn_s_sleep(POLL_SLEEP);
                    }
                    if (lane == 0) *(volatile LAS u32*)(ctl + (nwait & 1u)) = q1;
                    asm volatile("s_waitcnt lgkmcnt(0)" ::: "memory");
                }
                asm volatile("" ::: "memory"); BAR; asm volatile("" ::: "memory");
                const u32 te = lds_ld(ctl + (nwait & 1u));
                ++nwait; bad |= te >> 3;
                staged = te & 7u;
                STAGE_FIRST(cA + (size_t)k0 * kstep, cB + (size_t)k0 * kstep);
            }
            if (k0 == 0u && round < 4u) STAMP_AT(3 + round * 2);
            q0 = staged & 7u;
            const u32 k1 = q0 * 14u - 6u;
            if (wr == 1) BAR;
            bool lax = (staged >> 3) != 0u;
            if (lax) WAIT_V(22); else WAIT_V(6);
            BAR; BAR;
            for (u32 t = k0; t < k1; t += 2u) {
                const u32 t2 = t + 2u < k1 ? t + 2u : t;
                const GAS char* a1 = cA + (size_t)(t + 1u) * kstep;
                const GAS char* a2 = cA + (size_t)t2 * kstep; const GAS char* b2 = cB + (size_t)t2 * kstep;
                const GAS char* a3 = a2 + kstep; const GAS char* b3 = b2 + kstep;
                LDB(B0, 0, 0); LDB(B1, 0, 1); LDA(At, 0, 0); STAGE(SA(1, 1), a1 + hstep);
                if (lax) WAIT_V(24); else WAIT_V(8);
                WAIT_L(0); BAR; SCHED; MMA2(0); BAR; SCHED;
                LDA(At, 0, 1); STAGE(SB(0, 0), b2); STAGE(SB(0, 1), b2 + hstep); STAGE(SA(0, 0), a2);
                if (lax) WAIT_V(24); else WAIT_V(8);
                lax = false;
                WAIT_L(0); BAR; SCHED; MMA2(1); BAR; SCHED;
                LDB(B0, 1, 0); LDB(B1, 1, 1); LDA(At, 1, 0); STAGE(SA(0, 1), a2 + hstep);
                WAIT_V(8); WAIT_L(0); BAR; SCHED; MMA2(0); BAR; SCHED;
                LDA(At, 1, 1); STAGE(SB(1, 0), b3); STAGE(SB(1, 1), b3 + hstep); STAGE(SA(1, 0), a3);
                WAIT_V(8); WAIT_L(0); BAR; SCHED; MMA2(1); BAR; SCHED;
            }
            const bool last = q0 == 5u, draw = NLAST < WB && last && round == NB - 2u;
            if (wid == 0) {
                u32 tk = 0;
                if (draw && lane == 0) tk = draw_ticket();
                const u32 q1 = arrived(last ? round + 1u : round);
                if (lane == 0) *(volatile LAS u32*)(ctl + (nwait & 1u)) = q1 | (tk << 8);
                asm volatile("s_waitcnt lgkmcnt(0)" ::: "memory");
            }
            WAIT_V(0);
            if (wr == 0) BAR;
            BAR;
            asm volatile("" ::: "memory");
            staged = lds_ld(ctl + (nwait & 1u));
            ++nwait;
            if (draw) last_tile(staged >> 8, ni, nj);
            staged &= 7u;
            if (staged <= (last ? 0u : q0) || (last && nj >= (u32)NP)) staged = 0u;
            if (staged) STAGE_FIRST(last ? wf + (size_t)nj * (2u << 20) : cA + (size_t)k1 * kstep, last ? xf + (size_t)ni * (2u << 20) : cB + (size_t)k1 * kstep);
        }
        if (round < 4u) STAMP_AT(4 + round * 2);
#pragma unroll
        for (int bj = 0; bj < 2; ++bj) {
            const Dest dst = dest_of(y_base(p.out, ti, gpanel(rank, tj), bj), 2u, (u32)wid);
            const u32 loff = (u32)(lane >> 4) * dst.LH + (u32)(lane & 15) * 16u;
            u32x4 v[8];
#pragma unroll
            for (int g = 0; g < 8; ++g)
#pragma unroll
                for (int q = 0; q < 4; ++q) {
                    typedef float f32x2 __attribute__((ext_vector_type(2)));
                    typedef __bf16 bf16x2 __attribute__((ext_vector_type(2)));
                    const f32x2 f = {acc[q >> 1][bj][(q & 1) * 2][g >> 2][g & 3], acc[q >> 1][bj][(q & 1) * 2 + 1][g >> 2][g & 3]};
                    v[g][q] = __builtin_bit_cast(u32, __builtin_convertvector(f, bf16x2));
                }
            if (bad) {
#pragma unroll
                for (int g = 0; g < 8; ++g) v[g] = (u32x4){0x7fc07fc0u, 0x7fc07fc0u, 0x7fc07fc0u, 0x7fc07fc0u};
            }
            if (tj < (u32)POWN) {
#pragma unroll
                for (int g = 0; g < 8; ++g) store_group<AUX_DEV>(dst, loff, g, v[g]);
            } else if (staged) {
#pragma unroll
                for (int g = 0; g < 8; ++g) store_group<AUX_NT>(dst, loff, g, v[g]);
            } else {
#pragma unroll
                for (int g = 0; g < 8; ++g) store_group<AUX_OUT>(dst, loff, g, v[g]);
            }
        }
        if (tj < (u32)POWN) {
            asm volatile("s_waitcnt vmcnt(0)" ::: "memory");
            if (lane == 0) __hip_atomic_store((GAS u32*)(ws + OFF_TILE + (tj * 16u + ti) * 128u + (u32)wid * 4u), call, __ATOMIC_RELAXED, __HIP_MEMORY_SCOPE_AGENT);
        } else if (staged) staged |= 8u;
        ti = ni; tj = nj;
        asm volatile(".Lcode_far:");
    }
    STAMP_AT(15);
}

static void build_tables(Params& p) {
    unsigned char* role = (unsigned char*)p.role;
    for (int i = 0; i < 64; ++i) role[i] = (unsigned char)(i < NPULL ? (1 + i % 7) | (i / 7) << 3 : (i - NPULL) << 3);
    {
        constexpr int SL = (256 - NGEMM) / 2;
        static_assert(4 * PULL_WG_PER_PEER <= SL, "roles");
        int cnt[2] = {0, 0}, np = 0;
        for (int j = 0; j < PULL_WG_PER_PEER; ++j)
            for (int rel = 1; rel < 8; ++rel) {
                const int h = (((0xb2 >> p.rank) ^ (0xb2 >> ((p.rank + rel) & 7))) & 1), e = cnt[h]++;
                role[(e >> 2) * 8 + h * 4 + ((e + (h ? e >> 2 : 0)) & 3)] = (unsigned char)(rel | j << 3);
            }
        for (int h = 0; h < 2; ++h) for (int e = cnt[h]; e < SL; ++e) role[(e >> 2) * 8 + h * 4 + (e & 3)] = (unsigned char)(np++ << 3);
    }
    unsigned short* t = (unsigned short*)p.task;
    int n = 0;
    for (u32 c = PUSHC; c < (u32)NCIN; ++c) {
        if (c == 16u) for (u32 e = 0; e < (u32)PUSHC; ++e) t[n++] = (unsigned short)(e << 4 | 8u);
        const InPos ip = in_of(c);
        if (ip.isw && ip.s < (u32)POWN) for (u32 k = 1; k <= 8; ++k) t[n++] = (unsigned short)(c << 4 | k);
        else t[n++] = (unsigned short)(c << 4);
    }
}

extern "C" void kernel_launch(int rank, int nranks, ncclComm_t comm, hipStream_t stream,
                              void* const* d_in, const int* in_sizes, int n_in, void* d_out, int out_size,
                              const int* in_dtypes, int out_dtype, void* const* d_out_all,
                              void* const* d_ws_all, size_t ws_bytes) {
    if (nranks != NR || (unsigned)rank >= (unsigned)NR || ws_bytes < WS_NEED || n_in != 2) { fprintf(stderr, "unsupported configuration\n"); return; }
    Params p;
    p.x = (const char*)d_in[0]; p.w = (const char*)d_in[1]; p.out = (char*)d_out; p.rank = rank; p.pad = 0;
    for (int i = 0; i < NR; ++i) p.ws[i] = (char*)d_ws_all[(rank + i) % NR];
    build_tables(p);
    hipLaunchKernelGGL(gemm_ar_fused, dim3(256), dim3(512), 0, stream, p);
    HIP_CHECK(hipGetLastError());
}
```

```cpp
#include <hip/hip_runtime.h>
#include <rccl/rccl.h>
#include <cstdio>

#define HIP_CHECK(expr) do { hipError_t _e = (expr); if (_e != hipSuccess) { \
    fprintf(stderr, "HIP error %s at %s:%d\n", hipGetErrorString(_e), __FILE__, __LINE__); \
    return; } } while (0)

typedef unsigned u32;
typedef unsigned long long u64;
typedef u32 u32x4 __attribute__((ext_vector_type(4)));
typedef u32 u32x2 __attribute__((ext_vector_type(2)));
typedef unsigned short u16x2 __attribute__((ext_vector_type(2)));
typedef short bf16x8 __attribute__((ext_vector_type(8)));
typedef float f32x4 __attribute__((ext_vector_type(4)));
#define LAS __attribute__((address_space(3)))
#define GAS __attribute__((address_space(1)))
typedef __amdgpu_buffer_rsrc_t rsrc_t;

constexpr int NR = 8;
#define PULL_WG_PER_PEER 8
#define TRIGV 1
#define XROT 0
#define PAT 56565
#define LVAR 5
#define PUSHC 5
#define PUSHW 1
#define PACK_IN 112
#define TOUCH 7
#define TOUCH_AT 100
#define CODEPF 1
#define PJOIN 1
#define DEC_SLEEP 2
#define RHO 6
#define ENC_DELAY 150
#define ENC_PACE 192
#define POLL_SLEEP 8
#define WB 12
constexpr int NPULL = 7 * PULL_WG_PER_PEER, NGEMM = 16 * WB, NPACK = 256 - NGEMM - NPULL;
constexpr int POWN = 16 - RHO, NP = POWN + 8 * RHO;
constexpr int NB = (NP + WB - 1) / WB, N0 = 16 + WB;
constexpr int NCIN = 64 + 4 * NP, NCY = 32 * POWN, NC = NCIN + NCY;
constexpr int NTASK = 16 * 36;
constexpr int NDEC = 6;
constexpr u64 DEADLINE = 100000000ull;
constexpr int PUSHB = (8 * PUSHC + PULL_WG_PER_PEER - 1) / PULL_WG_PER_PEER;
constexpr int NLAST = NP - (NB - 1) * WB;
static_assert(NGEMM == 192 && NPACK == 8 && (PACK_IN + NCY) / NPACK < 126 && NB <= 8 && POWN <= WB && PUSHB <= NDEC, "roles");

constexpr int FIXB = 5120, CVAR = 8 * FIXB, CHDR = 32, CH = 66560;
constexpr size_t OFF_CALL = 0;
constexpr size_t OFF_TMO = 131072;
constexpr size_t OFF_TICK = OFF_TMO + 256;
constexpr size_t OFF_RDY = OFF_TICK + 16 * 128;
constexpr size_t OFF_FLAG = OFF_RDY + 64 * 128;
constexpr size_t OFF_TILE = OFF_FLAG + 8 * 256;
constexpr size_t OFF_GRAN = 196608;
constexpr size_t OFF_DBG = 524288;
constexpr size_t OFF_XF = 1ull << 20;
constexpr size_t OFF_WF = OFF_XF + (64ull << 20);
constexpr size_t OFF_SY = OFF_WF + NP * (2ull << 20);
constexpr size_t OFF_SI = OFF_SY + (size_t)NCY * CH;
constexpr size_t WS_NEED = OFF_SI + 2ull * NTASK * CH;

constexpr int A2B = 2064, A3B = 528, VARB = 912, WLDS = A2B + A3B + VARB, N2MAX = 2032, N3MAX = 496, UBITS = 2016, NPATCH = 31;
constexpr int CTL_OFF = 2048, CTL_WORDS = 128, WL_OFF = 2560, BIG_OFF = 30720, LDS_BYTES = BIG_OFF + 131072;
static_assert(WL_OFF + 8 * WLDS <= BIG_OFF && LDS_BYTES <= 163840, "LDS map");
constexpr int NF = 16, NV = 4, VSLOT = 8192, VRING = NF * FIXB;
constexpr int AUX_SYS = 17, AUX_DEV = 16, AUX_OUT = 18, AUX_NT = 2;

struct Params {
    int rank;
    u32 links;
    const char* x; const char* w; char* out;
    char* ws[NR];
    u32 task[NTASK / 2];
};

__device__ __forceinline__ u32 rfl(u32 x) { return (u32)__builtin_amdgcn_readfirstlane((int)x); }
__device__ __forceinline__ rsrc_t mk_rsrc(const void* p, u32 bytes) {
    const u64 a = (u64)p; const u64 u = ((u64)rfl((u32)(a >> 32)) << 32) | rfl((u32)a);
    return __builtin_amdgcn_make_buffer_rsrc((void*)u, (short)0, (int)rfl(bytes), 0x00020000);
}
__device__ __forceinline__ u32 vperm(u32 a, u32 b, u32 s) { return __builtin_amdgcn_perm(a, b, s); }
__device__ __forceinline__ u32 wave_incl_scan(u32 x) {
    x += (u32)__builtin_amdgcn_update_dpp(0, (int)x, 0x111, 0xf, 0xf, false);
    x += (u32)__builtin_amdgcn_update_dpp(0, (int)x, 0x112, 0xf, 0xf, false);
    x += (u32)__builtin_amdgcn_update_dpp(0, (int)x, 0x114, 0xf, 0xf, false);
    x += (u32)__builtin_amdgcn_update_dpp(0, (int)x, 0x118, 0xf, 0xf, false);
    x += (u32)__builtin_amdgcn_update_dpp(0, (int)x, 0x142, 0xa, 0xf, false);
    x += (u32)__builtin_amdgcn_update_dpp(0, (int)x, 0x143, 0xc, 0xf, false);
    return x;
}
__device__ __forceinline__ u32 pk_sub(u32 a, u32 b) { return __builtin_bit_cast(u32, (u16x2)(__builtin_bit_cast(u16x2, a) - __builtin_bit_cast(u16x2, b))); }
template <int AUX>
__device__ __forceinline__ void st128(u32x4 x, rsrc_t r, int voff, int soff) {
    __builtin_amdgcn_raw_buffer_store_b128(x, r, voff, soff, AUX);
    asm volatile("s_nop 1" :: "v"(x) : "memory");
}
#define WAVE_FENCE() __builtin_amdgcn_fence(__ATOMIC_ACQ_REL, "wavefront")

struct TabEntry { u32 dep, cmp, n, r0, r1; };
__device__ __forceinline__ TabEntry table_entry(u32 i) {
    u32 n = 0, dep = 0, cmp = 0x0c0c0c0cu;
#pragma unroll
    for (u32 t = 0; t < 4; ++t) {
        const u32 c = (i >> (2 * t)) & 3u;
        dep |= (c ? 4u + c : n) << (8 * t);
        cmp ^= c ? 0u : (0x0cu ^ t) << (8 * n);
        n += c ? 0u : 1u;
    }
    u64 r = 0;
#pragma unroll
    for (u32 t = 0; t < 7; ++t) r += (u64)(((i >> t) & 1u) ^ ((i >> t) ? 1u : 0u)) << (8u * (u32)__builtin_popcount(i & ((1u << t) - 1u)));
    return TabEntry{dep, cmp, n, (u32)r, (u32)(r >> 32)};
}
__device__ __forceinline__ void or3(LAS u32* q, u64 x, u32 sh) {
    const u64 y = x << sh;
    __hip_atomic_fetch_or(q, (u32)y, __ATOMIC_RELAXED, __HIP_MEMORY_SCOPE_WORKGROUP);
    __hip_atomic_fetch_or(q + 1, (u32)(y >> 32), __ATOMIC_RELAXED, __HIP_MEMORY_SCOPE_WORKGROUP);
    __hip_atomic_fetch_or(q + 2, (u32)(((x >> 32) << sh) >> 32), __ATOMIC_RELAXED, __HIP_MEMORY_SCOPE_WORKGROUP);
}
__device__ __forceinline__ u32 code_byte(u32 hi, u32 lo, u32 x) {
    const u32 f = vperm(hi, lo, x);
    return __builtin_amdgcn_udot4((f ^ (f >> 2)) & 0x03030303u, 0x40100401u, 0u, false);
}
template <bool PRED>
__device__ __forceinline__ void or_bytes(u32 pos, u32 x) {
    const u64 y = (u64)x << ((pos & 3u) * 8u);
    LAS u32* q = (LAS u32*)(size_t)(pos & ~3u);
    if (!PRED || x) {
        __hip_atomic_fetch_or(q, (u32)y, __ATOMIC_RELAXED, __HIP_MEMORY_SCOPE_WORKGROUP);
        __hip_atomic_fetch_or(q + 1, (u32)(y >> 32), __ATOMIC_RELAXED, __HIP_MEMORY_SCOPE_WORKGROUP);
    }
}
__device__ __forceinline__ u32 get_bytes(u32 pos) {
    const LAS u32* q = (const LAS u32*)(size_t)(pos & ~3u);
    return __builtin_amdgcn_alignbyte(q[1], q[0], pos);
}
__device__ __forceinline__ u32 la(const LAS void* p) { return (u32)(size_t)p; }
__device__ __forceinline__ u32 pack_level2(const LAS u32x2* tab, const LAS char* src, u32 n, LAS u32* end, LAS char* dst, int lane) {
    u32 m = 0;
    for (u32 i0 = 0; i0 < n; i0 += 1024u) {
        const u32 i = i0 + (u32)lane * 16u, hi = 0x02010000u, lo = 0x00030000u;
        const u32x4 x = *(const LAS u32x4*)(src + (i < n ? i : 0u));
        u32 cw = code_byte(hi, lo, x[0]) | (code_byte(hi, lo, x[1]) << 8) | (code_byte(hi, lo, x[2]) << 16) | (code_byte(hi, lo, x[3]) << 24);
        const u32 valid = i < n ? n - i : 0u;
        if (valid < 16u) cw |= 0x55555555u << (2u * valid);
        u32x2 e[4]; u32 c[5]; c[0] = 0;
#pragma unroll
        for (int j = 0; j < 4; ++j) { e[j] = tab[(cw >> (8 * j)) & 0xffu]; c[j + 1] = c[j] + e[j][1]; }
        const u32 incl = wave_incl_scan(c[4]), p0 = la(dst) + m + incl - c[4];
        if (i < n) *(end - 1 - (i >> 4)) = cw;
        m += (u32)__builtin_amdgcn_readlane((int)incl, 63);
        if (m <= (u32)N3MAX) {
#pragma unroll
            for (int j = 0; j < 4; ++j) or_bytes<true>(p0 + c[j], vperm(0u, x[j], e[j][0]));
        }
    }
    WAVE_FENCE();
    return m;
}
__device__ __forceinline__ void pack_level3(const LAS char* src, u32 n, LAS u32* ub, LAS u32* pt, u32 T, int lane, u32& bits, u32& np) {
    for (u32 i0 = 0; i0 < n; i0 += 256u) {
        const u32 i = i0 + (u32)lane * 4u;
        const u32 x = *(const LAS u32*)(src + (i < n ? i : 0u));
        u32 cum = 0, pf = 0; u64 m = 0;
#pragma unroll
        for (u32 t = 0; t < 4; ++t) {
            const u32 r = ((x >> (8 * t)) - 8u) & 0xffu;
            const bool valid = i + t < n, big = r > 15u;
            cum += valid && !big ? r : 0u;
            m |= (u64)(valid ? 1u : 0u) << cum;
            cum += valid ? 1u : 0u;
            pf |= valid && big ? 1u << t : 0u;
        }
        const u32 incl = wave_incl_scan(cum), p = bits + incl - cum;
        bits += (u32)__builtin_amdgcn_readlane((int)incl, 63);
        if (bits <= (u32)UBITS && cum) or3(ub + (p >> 5), m, p & 31u);
        if (__builtin_amdgcn_ballot_w64(pf != 0u)) {
#pragma unroll
            for (u32 t = 0; t < 4; ++t) {
                const u64 bm = __builtin_amdgcn_ballot_w64((pf >> t) & 1u);
                const u32 at = np + __builtin_amdgcn_mbcnt_hi((u32)(bm >> 32), __builtin_amdgcn_mbcnt_lo((u32)bm, 0u));
                if ((pf >> t) & 1u) pt[at] = ((i + t) << 8) | ((T - (x >> (8 * t))) & 0xffu);
                np += (u32)__builtin_popcountll(bm);
            }
        }
    }
    WAVE_FENCE();
}
__device__ __forceinline__ void encode_block(const u32 (&v)[8][4], rsrc_t ch, u32 b, LAS u32* alloc, const LAS u32x2* tab, LAS char* wl, int lane) {
#pragma unroll
    for (int j = 0; j < 4; ++j) if (j < 3 || lane < (WLDS - 3072) / 16) *(LAS u32x4*)(wl + j * 1024 + lane * 16) = (u32x4){0, 0, 0, 0};
    u32 s = 0;
#pragma unroll
    for (int g = 0; g < 8; ++g) s = __builtin_amdgcn_udot4(__builtin_amdgcn_alignbit(v[g][0], v[g][0], 31), 0x01000100u, s, false);
    u32 T = ((u32)__builtin_amdgcn_readlane((int)wave_incl_scan(s), 63) >> 10) + 5u;
    T = T > 255u ? 255u : (T < 23u ? 23u : T);
    const u32 tt = (T * 256u + 255u) * 0x10001u;
    const int fo = (int)(b * FIXB);
    u32 U[16], l1[4] = {0, 0, 0, 0}, c[17];
    u32x2 e[16];
#pragma unroll
    for (int q = 0; q < 4; ++q) {
        u32x4 sm;
#pragma unroll
        for (int j = 0; j < 4; ++j) {
            const int k = 4 * q + j;
            const u32 a0 = v[k >> 1][(k & 1) * 2], a1 = v[k >> 1][(k & 1) * 2 + 1];
            const u32 r0 = __builtin_amdgcn_alignbit(a0, a0, 31), r1 = __builtin_amdgcn_alignbit(a1, a1, 31);
            sm[j] = vperm(r1, r0, 0x06040200u);
            U[k] = vperm(pk_sub(tt, r1), pk_sub(tt, r0), 0x07050301u);
            const u32 cb = code_byte(0x00000302u, 0x01000000u, U[k]);
            l1[q] |= cb << (8 * j);
            e[k] = tab[cb];
        }
        st128<AUX_SYS>(sm, ch, lane * 16 + q * 1024, fo);
    }
    st128<AUX_SYS>((u32x4){l1[0], l1[1], l1[2], l1[3]}, ch, lane * 16 + 4096, fo);
    __builtin_amdgcn_sched_barrier(0);
    c[0] = 0;
#pragma unroll
    for (int k = 0; k < 16; ++k) c[k + 1] = c[k] + e[k][1];
    const u32 incl = wave_incl_scan(c[16]), p0 = la(wl) + incl - c[16];
    const u32 n2 = (u32)__builtin_amdgcn_readlane((int)incl, 63);
    bool raw = n2 > (u32)N2MAX;
    u32 d2 = 0, d3 = 0, np = 0;
    LAS u32* var = (LAS u32*)(wl + A2B + A3B);
    if (!raw) {
        WAVE_FENCE();
#pragma unroll
        for (int k = 0; k < 16; ++k) or_bytes<false>(p0 + c[k], vperm(0u, U[k], e[k][0]));
        WAVE_FENCE();
        d2 = (n2 + 15u) >> 4;
        const u32 n3 = pack_level2(tab, wl, n2, var + d2, wl + A2B, lane);
        raw = n3 > (u32)N3MAX;
        if (!raw) {
            u32 bits = 0;
            pack_level3(wl + A2B, n3, var + d2, (LAS u32*)wl, T, lane, bits, np);
            raw = bits > (u32)UBITS || np > (u32)NPATCH;
            d3 = (bits + 31u) >> 5;
            if (!raw && (u32)lane < np) var[d2 + d3 + (u32)lane] = ((const LAS u32*)wl)[lane];
            WAVE_FENCE();
        }
    }
    const u32 vd = d2 + d3 + np;
    u32 off = 0;
    if (lane == 0) off = __hip_atomic_fetch_add(alloc, raw ? 3072u : vd * 4u, __ATOMIC_RELAXED, __HIP_MEMORY_SCOPE_WORKGROUP);
    off = rfl(off) + CHDR;
    const int vo = CVAR + (int)off;
    if (lane == 0) __builtin_amdgcn_raw_buffer_store_b32(raw ? off << 17 : (T | (d3 << 8) | (np << 14) | ((off + d2 * 4u) << 17)), ch, 0, CVAR + (int)b * 4, AUX_SYS);
    if (raw) {
        asm volatile("s_waitcnt vmcnt(0)" ::: "memory");
#pragma unroll
        for (int g = 0; g < 8; ++g)
            st128<AUX_SYS>((u32x4){v[g][0], v[g][1], v[g][2], v[g][3]}, ch, lane * 16 + (g < 5 ? g : g - 5) * 1024, g < 5 ? fo : vo);
    } else {
        for (u32 o = (u32)lane; o < vd; o += 64u) __builtin_amdgcn_raw_buffer_store_b32(var[o], ch, (int)o * 4, vo, AUX_SYS);
    }
    WAVE_FENCE();
}
__device__ __forceinline__ bool chunk_done(LAS u32* cnt, int lane, u32& total) {
    u32 n = 0;
    if (lane == 0) n = __hip_atomic_fetch_add(cnt + 1, 1u, __ATOMIC_RELAXED, __HIP_MEMORY_SCOPE_WORKGROUP);
    if (rfl(n) != 7u) return false;
    total = rfl(*(volatile LAS u32*)cnt) + CHDR;
    return true;
}

struct Dest { rsrc_t r; u32 A, B, C, LH; };
__device__ __forceinline__ Dest dest_of(char* base, u32 kind, u32 b) {
    if (kind == 0u) return Dest{mk_rsrc(base + b * 4096u, 32768u + 4096u), 1024u, 32768u, 2048u, 256u};
    if (kind == 1u) return Dest{mk_rsrc(base + (b >> 2) * 32768u + (b & 1u) * 8192u + (b & 2u) * 512u, 16384u + 6144u + 1024u), 2048u, 4096u, 16384u, 256u};
    return Dest{mk_rsrc(base + (b & 3u) * (32u * 65536u) + (b >> 2) * 256u, 31u * 65536u + 256u), 65536u, 131072u, 16u * 65536u, 4u * 65536u};
}
template <int AUX>
__device__ __forceinline__ void store_group(const Dest& d, u32 loff, int g, u32x4 x) {
    st128<AUX>(x, d.r, (int)loff, (int)((g & 1 ? d.A : 0u) + (g & 2 ? d.B : 0u) + (g & 4 ? d.C : 0u)));
}

#define SCHED __builtin_amdgcn_sched_barrier(0)
__device__ __forceinline__ u32x2 get2(u32 pos) { const LAS u32* q = (const LAS u32*)(size_t)(pos & ~3u); return (u32x2){q[0], q[1]}; }
__device__ __forceinline__ void spread3(u32 w, u32 start, u32 n, u32x2 runs, u32 t8, u32 p) {
    if (n) {
        const u32 carry = (u32)__builtin_clz((w << 8) | start);
        const u64 x = ((u64)(t8 - runs[1]) << 32) | (t8 - runs[0] - carry);
        const u32 sh = 64u - 8u * n;
        or3((LAS u32*)(size_t)(p & ~3u), (x << sh) >> sh, (p & 3u) * 8u);
    }
}
template <class Out>
__device__ __forceinline__ void decode_block(const u32x4 (&d)[5], u32 hdr, const LAS u32* ub, const LAS u32x2* tab, const LAS u32x2* tab3, LAS char* wl, int lane, Out out) {
    const u32 T = hdr & 0xffu, d3 = (hdr >> 8) & 0x3fu, np = (hdr >> 14) & 0x1fu;
    auto ex = [&](u32 a, u32 b, u32 c) { return ((T - a) << 8) | ((T - b) << 16) | ((T - c) << 24); };
    const u32 t1 = ex(3, 4, 5), t2 = ex(6, 7, 2), t8 = (T - 8u) * 0x01010101u;
    const u32 w2 = *(ub - 1 - lane), p3i = la(ub) + (u32)lane - 3u;
    u32x2 e[16], e2[4], g[16];
    g[0] = get2(p3i);
    if (lane < A3B / 16) *(LAS u32x4*)(wl + A2B + lane * 16) = (u32x4){0, 0, 0, 0};
#pragma unroll
    for (int k = 0; k < 8; ++k) e[k] = tab[(d[4][k >> 2] >> (8 * (k & 3))) & 0xffu];
    SCHED;
    const u32 w3 = __builtin_amdgcn_alignbyte(g[0][1], g[0][0], p3i), x3 = (u32)lane < d3 * 4u ? w3 >> 24 : 0u;
    const u32x2 e3 = tab3[x3];
#pragma unroll
    for (int j = 0; j < 4; ++j) e2[j] = tab[(w2 >> (8 * j)) & 0xffu];
#pragma unroll
    for (int k = 8; k < 16; ++k) e[k] = tab[(d[4][k >> 2] >> (8 * (k & 3))) & 0xffu];
    SCHED;
    u32 c[17], f[5];
    c[0] = 0; f[0] = 0;
#pragma unroll
    for (int k = 0; k < 16; ++k) c[k + 1] = c[k] + e[k][1];
#pragma unroll
    for (int j = 0; j < 4; ++j) f[j + 1] = f[j] + e2[j][1];
    const u32 n3 = (u32)__builtin_popcount(x3);
    const u32 pk = c[16] | (f[4] << 11) | (n3 << 22);
    const u32 incl = wave_incl_scan(pk), ex0 = incl - pk, tot = (u32)__builtin_amdgcn_readlane((int)incl, 63);
    const u32 a3 = la(wl) + A2B, p1 = la(wl) + (ex0 & 0x7ffu), p2 = a3 + ((ex0 >> 11) & 0x1ffu);
    WAVE_FENCE();
    spread3(w3, lane < 3 ? 0x80000000u >> (8 * lane) : 0x80u, n3, e3, t8, a3 + ((ex0 >> 22) & 0x1ffu));
    for (u32 i0 = 64u, m = tot >> 22; i0 < d3 * 4u; i0 += 64u) {
        const u32 w = get_bytes(p3i + i0), x = i0 + (u32)lane < d3 * 4u ? w >> 24 : 0u, n = (u32)__builtin_popcount(x), in = wave_incl_scan(n);
        spread3(w, 0x80u, n, tab3[x], t8, a3 + ((m + in - n) & 0x1ffu));
        m += (u32)__builtin_amdgcn_readlane((int)in, 63);
    }
    WAVE_FENCE();
    if ((u32)lane < np) { const u32 pw = ub[d3 + (u32)lane]; *(LAS unsigned char*)(wl + A2B + ((pw >> 8) & 0x1ffu)) = (unsigned char)pw; }
    WAVE_FENCE();
#pragma unroll
    for (int j = 0; j < 4; ++j) { f[j] += p2; g[j] = get2(f[j]); }
    SCHED;
#pragma unroll
    for (int k = 0; k < 16; ++k) c[k] += p1;
    SCHED;
    u32x4 a2;
#pragma unroll
    for (int j = 0; j < 4; ++j) a2[j] = vperm(t2, __builtin_amdgcn_alignbyte(g[j][1], g[j][0], f[j]), e2[j][0]);
    *(LAS u32x4*)(wl + lane * 16) = a2;
    if ((tot & 0x7ffu) > 1024u) {
        const u32 cw = *(ub - 65 - lane);
        f[0] = 0;
#pragma unroll
        for (int j = 0; j < 4; ++j) { e2[j] = tab[(cw >> (8 * j)) & 0xffu]; f[j + 1] = f[j] + e2[j][1]; }
        const u32 p = a3 + (((tot >> 11) + wave_incl_scan(f[4]) - f[4]) & 0x1ffu);
#pragma unroll
        for (int j = 0; j < 4; ++j) a2[j] = vperm(t2, get_bytes(p + f[j]), e2[j][0]);
        *(LAS u32x4*)(wl + 1024 + lane * 16) = a2;
    }
    WAVE_FENCE();
#pragma unroll
    for (int k = 0; k < 16; ++k) g[k] = get2(c[k]);
    SCHED;
    u32x4 o;
#pragma unroll
    for (int k = 0; k < 16; ++k) {
        const u32 x = vperm(t1, __builtin_amdgcn_alignbyte(g[k][1], g[k][0], c[k]), e[k][0]);
        const u32 smk = d[k >> 2][k & 3];
        const u32 r0 = vperm(x, smk, 0x05010400u), r1 = vperm(x, smk, 0x07030602u);
        o[2 * (k & 1)] = __builtin_amdgcn_alignbit(r0, r0, 1);
        o[2 * (k & 1) + 1] = __builtin_amdgcn_alignbit(r1, r1, 1);
        if (k & 1) out(k >> 1, o);
    }
    WAVE_FENCE();
}

__device__ __forceinline__ char* frag_base(char* ws, bool isw, u32 s, u32 q, u32 rel, u32 call) {
    return ws + (isw ? OFF_WF : OFF_XF + (size_t)(call & 1u) * (32u << 20)) + (size_t)(s * 64u + (rel ? 6u + 14u * q + 2u * rel : 2u * q)) * 32768u;
}
__device__ __forceinline__ char* y_base(char* out, u32 i, u32 gp, u32 bj) { return out + (size_t)(i * 256u + bj * 128u) * 65536u + gp * 512u; }
__device__ __forceinline__ u32 gpanel(u32 r, u32 j) { return j < (u32)POWN ? r * 16u + j : ((j - POWN) & 7u) * 16u + POWN + ((j - POWN) >> 3); }
__device__ __forceinline__ u32 slot_in(bool isw, u32 s, u32 q, u32 dst) {
    return !isw ? s * 4u + q : (s < (u32)POWN ? 64u + (dst * POWN + s) * 4u + q : 64u + 28u * POWN + s * 4u + q);
}
__host__ __device__ __forceinline__ u32 batch_size(u32 bt) { return bt == 0u ? (u32)N0 : (bt == NB - 1u ? (u32)(NP - (NB - 1) * WB) : (u32)WB); }
struct InPos { bool isw; u32 s, q, bt; };
__host__ __device__ __forceinline__ InPos in_of(u32 c) {
    if (c < 4u * N0) { const u32 q = c / N0, pp = c - q * N0; return InPos{pp >= 16u, pp >= 16u ? pp - 16u : pp, q, 0u}; }
    const u32 bt = (c - 64u) / (4u * WB), r = c - 64u - bt * 4u * WB, q = r / batch_size(bt);
    return InPos{true, bt * WB + r - q * batch_size(bt), q, bt};
}

__device__ __forceinline__ void arrive(char* ws, u32 ci, u32 n, u32 call, int lane) {
    u32 old = 0;
    if (lane == 0) old = __hip_atomic_fetch_add((GAS u32*)(ws + OFF_RDY + ci * 128u), n, __ATOMIC_RELAXED, __HIP_MEMORY_SCOPE_SYSTEM);
    if (rfl(old) + n == call * ((ci & 7u) ? 56u : 32u) * batch_size(ci >> 3) && lane < 8)
        __hip_atomic_store((GAS u32*)(ws + OFF_FLAG + (u32)lane * 256u + ci * 4u), call, __ATOMIC_RELAXED, __HIP_MEMORY_SCOPE_SYSTEM);
}
__device__ __forceinline__ void load_x(const char* x, u32 s, u32 q, u32 b, int lane, u32x4 (&a)[8], bool on = true) {
    const rsrc_t r = mk_rsrc(x + (size_t)((s & 15u) * 256u + b * 32u) * 1024u + q * 256u, on ? 31u * 1024u + 256u : 0u);
#pragma unroll
    for (int i = 0; i < 8; ++i) a[i] = __builtin_amdgcn_raw_buffer_load_b128(r, (lane >> 2) * 1024 + ((lane & 3) ^ ((lane >> 5) << 1)) * 16 + (i & 3) * 64, (i >> 2) * 16384, 0);
}

__device__ __forceinline__ bool give_up(GAS u32* tmo, u32 call, u64 t0) {
    if (rfl(__hip_atomic_load(tmo, __ATOMIC_RELAXED, __HIP_MEMORY_SCOPE_AGENT)) == call || __builtin_amdgcn_s_memrealtime() - t0 > DEADLINE) {
        __hip_atomic_store(tmo, call, __ATOMIC_RELAXED, __HIP_MEMORY_SCOPE_AGENT);
        return true;
    }
    return false;
}

#define STAMP_IT 0
#define STAMP_AT(i) do { } while (0)

__device__ __forceinline__ u32 lds_ld(LAS u32* p) { return rfl(*(volatile LAS u32*)p); }
__device__ __forceinline__ void dma_piece(rsrc_t r, u32 voff, u32 soff, u32 ldsaddr) {
    u32 keep;
    asm volatile("s_nop 4\n\ts_mov_b32 %0, m0\n\ts_mov_b32 m0, %1\n\ts_nop 0\n\tbuffer_load_dwordx4 %2, %3, %4 offen sc0 sc1 lds\n\ts_mov_b32 m0, %0"
                 : "=&s"(keep) : "s"(ldsaddr), "v"(voff), "s"(r), "s"(soff) : "memory");
}

#define DMA_P(j, w, off) "s_waitcnt vmcnt(" w ")\n\tbuffer_load_dwordx4 %2, %3, " off " sc0 sc1 lds\n\t.if %8 == " #j "\n\tds_max_u32 %6, %7\n\t.endif\n\t"
__device__ __forceinline__ void dma_block(rsrc_t r, u32 voff, u32 soff, u32 ldsaddr, u32 at, u32 val) {
    constexpr int W0 = PAT / 10000, W1 = PAT / 1000 % 10, W2 = PAT / 100 % 10, W3 = PAT / 10 % 10, W4 = PAT % 10;
    constexpr int JP = W0 <= 5 ? 0 : (W1 <= 6 ? 1 : (W2 <= 7 ? 2 : (W3 <= 8 ? 3 : 4)));
    u32 keep;
    asm volatile("s_nop 2\n\ts_mov_b32 %0, m0\n\ts_mov_b32 m0, %1\n\t"
                 DMA_P(0, "%9", "%4 offen") DMA_P(1, "%10", "%4 offen offset:1024") DMA_P(2, "%11", "%4 offen offset:2048") DMA_P(3, "%12", "%4 offen offset:3072")
                 "s_add_u32 m0, m0, 0x1000\n\t" DMA_P(4, "%13", "%5 offen") "s_mov_b32 m0, %0"
                 : "=&s"(keep) : "s"(ldsaddr), "v"(voff), "s"(r), "s"(soff), "s"(soff + 4096u), "v"(at), "v"(val), "n"(JP),
                   "n"(W0), "n"(W1), "n"(W2), "n"(W3), "n"(W4) : "memory", "scc");
}

__global__ void __launch_bounds__(512, 2) gemm_ar_fused(const Params p) {
    __shared__ __attribute__((aligned(16))) char smem[LDS_BYTES];
    LAS char* lds = (LAS char*)smem + BIG_OFF;
    LAS u32x2* tab = (LAS u32x2*)smem;
    LAS u32x2* tab3 = (LAS u32x2*)((LAS char*)smem + WL_OFF);
    const int tid = threadIdx.x, wid = (int)rfl((u32)tid >> 6), lane = tid & 63;
    const u32 bid = blockIdx.x;
    if (CODEPF && bid - 16u < 64u) {
        u64 pc; u32 left;
        asm volatile("s_getpc_b64 %0\n.Lcode_here:\n\ts_mov_b32 %1, .Lcode_far-.Lcode_here" : "=s"(pc), "=s"(left));
        const u32 o = ((bid - 16u) >> 3) * 8192u + (u32)tid * 16u;
        if (o + 16u <= left) { const u32x4 t = *(const GAS u32x4*)(pc + o); asm volatile("" :: "v"(t)); }
    }
    u32 rank = (u32)p.rank;
    char* ws = p.ws[0]; const char* px = p.x;
    SCHED;
    TabEntry te = table_entry((u32)tid & 255u);
    asm volatile("" : "+s"(rank), "+s"(ws), "+s"(px), "+v"(te.dep), "+v"(te.cmp), "+v"(te.n), "+v"(te.r0), "+v"(te.r1));
    LAS char* wl = (LAS char*)smem + WL_OFF + wid * WLDS;
    LAS u32* ctl = (LAS u32*)((LAS char*)smem + CTL_OFF);
    GAS u32* tmo = (GAS u32*)(ws + OFF_TMO);
    STAMP_AT(0);
    const u32 role = bid < (u32)NGEMM ? 0u : (bid < (u32)(NGEMM + NPULL) ? (1u + (bid - NGEMM) % 7u) | ((bid - NGEMM) / 7u) << 3 : (bid & 7u) << 3);
    const u32 pn = (role & 7u) ? (role >> 3) * PUSHB + (u32)wid - 2u : (bid + PUSHC) * 8u + (u32)wid;
    const bool early = (role & 7u) ? wid >= 2 && wid < 2 + PUSHB && pn < 8u * PUSHC : bid < 16u - PUSHC;
    GAS u32* callp = (GAS u32*)(ws + OFF_CALL + (bid * 8u + wid) * 64u);
    const u32 callv = __hip_atomic_load(callp, __ATOMIC_RELAXED, __HIP_MEMORY_SCOPE_AGENT);
    u32x4 a[8];
    load_x(px, pn >> 3, 0u, pn & 7u, lane, a, early);
    SCHED;
    char* wsl = ws;
    if (tid < 256) tab[tid] = (u32x2){(role & 7u) ? te.dep : te.cmp, te.n};
    else { if (tid < 256 + CTL_WORDS) ctl[tid - 256] = 0u; if (role & 7u) tab3[tid - 256] = (u32x2){te.r0, te.r1}; }
    if (!(role & 7u)) ((LAS u32*)lds)[tid] = 0u;
#pragma unroll
    for (int k = 1; k < 8; ++k) wsl = lane == k ? p.ws[k] : wsl;
    asm volatile("s_waitcnt lgkmcnt(0)" : "+v"(wsl) :: "memory"); __builtin_amdgcn_s_barrier(); asm volatile("" ::: "memory");

    if (role & 7u) {
        if ((TOUCH & 4) && wid == 1) {
            const u32 warm = __hip_atomic_load((const GAS u32*)(p.ws[role & 7u] + OFF_XF + (size_t)(lane & 1) * (32u << 20) + (size_t)(((role >> 3) * PUSHB) >> 3) * (2u << 20)), __ATOMIC_RELAXED, __HIP_MEMORY_SCOPE_SYSTEM);
            asm volatile("" :: "v"(warm));
        }
        if (early) {
            char* pws = p.ws[role & 7u];
            const u32 call = rfl(callv) + 1u;
            const Dest dst = dest_of(frag_base(pws, false, pn >> 3, 0u, 8u - (role & 7u), call), 0u, pn & 7u);
#pragma unroll
            for (int g = 0; g < 8; ++g) { store_group<AUX_SYS>(dst, (u32)lane * 16u, g, a[g]); asm volatile("s_waitcnt vmcnt(%0)" :: "n"(PUSHW - 1) : "memory"); }
            STAMP_AT(12);
            if (lane == 0) __hip_atomic_store(callp, call, __ATOMIC_RELAXED, __HIP_MEMORY_SCOPE_AGENT);
            asm volatile("s_waitcnt vmcnt(0)" ::: "memory");
            STAMP_AT(13);
            arrive(pws, 1u, 1u, call, lane);
        }
        LAS u32* vready = ctl; LAS u32* endk = ctl + 1; LAS u32* landed = ctl + 2; LAS u32* prog = ctl + 5;
        LAS u32* vdone = ctl + 8; LAS u32* sdone = ctl + 16; LAS u32* ffree = ctl + 32;
        LAS u32* q = ctl + 64;
        const u32 l0 = la(lds), voff = (u32)lane * 16u;
        if (wid == 0) {
            const u32 call = rfl(callv) + 1u;
            if (lane == 0) __hip_atomic_store(callp, call, __ATOMIC_RELAXED, __HIP_MEMORY_SCOPE_AGENT);
            u32 xcc;
            asm volatile("s_getreg_b32 %0, hwreg(HW_REG_XCC_ID)" : "=s"(xcc));
            xcc = (xcc + XROT) & 7u;
            const u32 row = (bid - NGEMM) >> 3;
            u32 rel = (p.links >> (4u * (xcc >= 4u ? 3u + ((row + xcc) & 3u) : (row < 6u ? (row + xcc) % 3u : 3u + xcc)))) & 7u;
            GAS u64* tick = (GAS u64*)(ws + OFF_TICK);
            bool fresh = true;
            for (u32 k = 0;; ++k) {
                while (lds_ld(prog) < 2u * k) __builtin_amdgcn_s_sleep(2);
                u32 c;
                for (;;) {
                    u32 tk = 0;
                    if (lane == 0) {
                        if (fresh) __hip_atomic_fetch_max(tick + rel * 16u, (u64)call << 32, __ATOMIC_RELAXED, __HIP_MEMORY_SCOPE_AGENT);
                        tk = (u32)__hip_atomic_fetch_add(tick + rel * 16u, 1ull, __ATOMIC_RELAXED, __HIP_MEMORY_SCOPE_AGENT);
                    }
                    c = rfl(tk) + PUSHC;
                    fresh = c >= (u32)NC;
                    if (!fresh) break;
                    if (k) while (lds_ld(prog) < 2u * k + 1u) __builtin_amdgcn_s_sleep(2);
                    const u64 o = __hip_atomic_load(tick + ((u32)lane & 7u) * 16u, __ATOMIC_RELAXED, __HIP_MEMORY_SCOPE_AGENT);
                    const u32 left = (u32)__builtin_amdgcn_ballot_w64(o < (((u64)call << 32) | (u32)(NC - PUSHC))) & 0xfeu;
                    if (!left) break;
                    rel = (u32)__builtin_ctz(left);
                }
                const u32 src = (rank + rel) & 7u;
                const char* pws = p.ws[rel];
                GAS u64* gran = (GAS u64*)(ws + OFF_GRAN) + (size_t)((call & 1u) * 8u + src) * NC;
                u32 fl = 2u, vb = 0, kc = 2u;
                const char* from = pws; char* to = ws;
                if (c < (u32)NC) {
                    if (c >= (u32)NCIN) {
                        const u32 yc = c - NCIN;
                        from = pws + OFF_SY + (size_t)yc * CH;
                        to = y_base(p.out, (yc >> 1) & 15u, src * 16u + (yc >> 5), yc & 1u);
                    } else {
                        const InPos ip = in_of(c);
                        from = pws + OFF_SI + (size_t)((call & 1u) * NTASK + slot_in(ip.isw, ip.s, ip.q, rank)) * CH;
                        to = frag_base(ws, ip.isw, ip.s, ip.q, rel, call);
                        kc = (ip.isw ? 1u : 0u) | ((ip.bt * 8u + ip.q + 1u) << 2);
                    }
                    if ((TOUCH & 1) && k == 0u) {
                        const u32 warm = __hip_atomic_load((const GAS u32*)((lane & 1) ? to : from), __ATOMIC_RELAXED, __HIP_MEMORY_SCOPE_SYSTEM);
                        asm volatile("" :: "v"(warm));
                    }
                    const u64 t0 = __builtin_amdgcn_s_memrealtime();
                    for (u32 spin = 0;; ++spin) {
                        const u64 g = __hip_atomic_load(gran + c, __ATOMIC_RELAXED, __HIP_MEMORY_SCOPE_SYSTEM);
                        if (rfl((u32)g) == call) { fl = 0u; vb = rfl((u32)(g >> 32)); break; }
                        if ((spin & 15u) == 15u && give_up(tmo, call, t0)) { fl = 1u; break; }
                        __builtin_amdgcn_s_sleep(1);
                    }
                    if (k == 0) STAMP_AT(1);
                }
                while (lds_ld(vdone + (k & (NV - 1))) < 8u * (k / NV)) __builtin_amdgcn_s_sleep(2);
                if (lane == 0) {
                    *(volatile LAS u32x4*)(q + (k & (NV - 1)) * 8u + 4u) = (u32x4){(u32)(u64)to, (u32)((u64)to >> 32), kc, call};
                    *(volatile LAS u32x4*)(q + (k & (NV - 1)) * 8u) = (u32x4){(u32)(u64)from, (u32)((u64)from >> 32), fl | (c % LVAR) << 2 | ((k + 1u) << 5), vb};
                }
                if (fl & 2u) { if (lane == 0) *(volatile LAS u32*)endk = k + 1u; break; }
                if (!(fl & 1u)) {
                    const u32 vl = vb > (u32)VSLOT ? (u32)CHDR : (vb + 15u) & ~15u;
                    const rsrc_t r = mk_rsrc(from, CVAR + vl);
                    for (u32 o = 0; o < vl; o += 1024u) { dma_piece(r, voff + CVAR + o, 0u, rfl(l0 + VRING + (k & (NV - 1)) * VSLOT + o)); if (k) asm volatile("s_waitcnt vmcnt(0)" ::: "memory"); }
                    asm volatile("s_waitcnt vmcnt(0)" ::: "memory");
                }
                if (lane == 0) *(volatile LAS u32*)vready = k + 1u;
            }
        } else if (wid == 1) {
            __builtin_amdgcn_s_setprio(3);
            const u32 la_landed = lane ? la(wl) + (u32)lane * 4u : la(landed);
            u32 n = 0;
            auto drain = [&]() { asm volatile("s_waitcnt vmcnt(0)" ::: "memory"); if (lane == 0) __hip_atomic_fetch_max(landed, n, __ATOMIC_RELAXED, __HIP_MEMORY_SCOPE_WORKGROUP); };
            u32x4 e = (u32x4){0, 0, 0, 0};
            u32 fr = 0;
            u32 trig = 0;
            for (u32 k = 0;; ++k) {
                if (rfl(e[2]) >> 5 != k + 1u) {
                    e = *(volatile LAS u32x4*)(q + (k & (NV - 1)) * 8u);
                    if (rfl(e[2]) >> 5 != k + 1u) {
                        drain();
                        do { __builtin_amdgcn_s_sleep(1); e = *(volatile LAS u32x4*)(q + (k & (NV - 1)) * 8u); } while (rfl(e[2]) >> 5 != k + 1u);
                    }
                }
                const u32 fl = rfl(e[2]);
                if (fl & 2u) break;
                if (TRIGV || k == 0u) trig = (fl >> 2) & 7u;
                const rsrc_t r = mk_rsrc((const void*)(((u64)e[1] << 32) | e[0]), (fl & 1u) ? 0u : (u32)CVAR);
#pragma unroll
                for (u32 b = 0; b < 8u; ++b, ++n) {
                    const u32 sl = n & (NF - 1), m = n + 1u;
                    if (rfl(fr) + NF <= n && lds_ld(ffree + sl) + NF <= n) { drain(); while (lds_ld(ffree + sl) + NF <= n) __builtin_amdgcn_s_sleep(1); }
                    fr = *(volatile LAS u32*)(ffree + (m & (NF - 1)));
                    if (b == 7u) e = *(volatile LAS u32x4*)(q + ((k + 1u) & (NV - 1)) * 8u);
                    if ((b == trig || b == 7u) && lane == 0) *(volatile LAS u32*)prog = 2u * k + (b == 7u ? 3u : 2u);
                    dma_block(r, voff, b * FIXB, rfl(l0 + sl * FIXB), la_landed, n ? n - 1u : 0u);
                }
            }
            drain();
            STAMP_AT(2);
        } else {
#define SEG(i)
            u32 owed = 0xffffffffu, call = 0;
            auto settle = [&]() {
                if (owed != 0xffffffffu) {
                    asm volatile("s_waitcnt vmcnt(0)" ::: "memory");
                    u32 old = 0;
                    if (lane == 0) old = __hip_atomic_fetch_add(sdone + (owed & 7u), 1u, __ATOMIC_RELAXED, __HIP_MEMORY_SCOPE_WORKGROUP);
                    if ((rfl(old) & 7u) == 7u) arrive(ws, owed >> 3, 8u, call, lane);
                    owed = 0xffffffffu;
                }
            };
            for (u32 n = (u32)wid - 2u;; n += NDEC) {
                const u32 k = n >> 3, b = n & 7u, sl = n & (NF - 1);
                bool end = false;
                for (;;) {
                    const u32x4 st = *(volatile LAS u32x4*)ctl;
                    if (rfl(st[0]) > k && rfl(st[2]) > n) break;
                    if (rfl(st[1]) - 1u <= k) { end = true; break; }
                    settle();
                    __builtin_amdgcn_s_sleep(DEC_SLEEP);
                }
                if (end) break;
                asm volatile("" ::: "memory");
                u32x4 d[5];
                const LAS char* vs = lds + VRING + (k & (NV - 1)) * VSLOT;
                const u32 h = *(const LAS u32*)(vs + b * 4u);
                const u32x4 e0 = *(const LAS u32x4*)(q + (k & (NV - 1)) * 8u), e1 = *(const LAS u32x4*)(q + (k & (NV - 1)) * 8u + 4u);
                d[4] = *(const LAS u32x4*)(lds + sl * FIXB + 4096 + lane * 16);
#pragma unroll
                for (int j = 0; j < 4; ++j) d[j] = *(const LAS u32x4*)(lds + sl * FIXB + j * 1024 + lane * 16);
                asm volatile("s_waitcnt lgkmcnt(0)" ::: "memory");
                if (lane == 0) *(volatile LAS u32*)(ffree + sl) = n + 1u;
                asm volatile("" : "+v"(d[0]), "+v"(d[1]), "+v"(d[2]), "+v"(d[3]), "+v"(d[4]) :: "memory");
                const u32 hdr = rfl(h), off = (hdr >> 19) * 4u, kc = rfl(e1[2]);
                SEG(0);
                call = rfl(e1[3]);
                const bool far = rfl(e0[3]) > (u32)VSLOT;
                const LAS u32* var = (const LAS u32*)(vs + (off <= (u32)VSLOT ? off : 0u));
                u32x4 o[8];
                if (rfl(e0[2]) & 1u) {
#pragma unroll
                    for (int g = 0; g < 8; ++g) o[g] = (u32x4){0x7fc07fc0u, 0x7fc07fc0u, 0x7fc07fc0u, 0x7fc07fc0u};
                } else {
                    if (far) {
                        const rsrc_t r = mk_rsrc((const void*)(((u64)e0[1] << 32) | e0[0]), CH);
#pragma unroll
                        for (int j = 0; j < 3; ++j) o[5 + j] = __builtin_amdgcn_raw_buffer_load_b128(r, (int)(CVAR + off) - ((hdr & 0xffu) ? 512 : 0) + j * 1024 + lane * 16, 0, AUX_SYS);
                        if (lane < VARB / 16) *(LAS u32x4*)(wl + A2B + A3B + lane * 16) = o[5];
                        WAVE_FENCE();
                        var = (const LAS u32*)(wl + A2B + A3B + 512);
                    }
                    if (hdr & 0xffu) decode_block(d, hdr, var, tab, tab3, wl, lane, [&](int g, u32x4 x) { o[g] = x; });
                    else {
#pragma unroll
                        for (int g = 0; g < 5; ++g) o[g] = d[g];
                        if (!far) {
#pragma unroll
                            for (int j = 0; j < 3; ++j)
#pragma unroll
                                for (int i = 0; i < 4; ++i) o[5 + j][i] = *(const volatile LAS u32*)(var + j * 256 + lane * 4 + i);
                        }
                    }
                }
                const Dest dst = dest_of((char*)(((u64)e1[1] << 32) | e1[0]), kc & 3u, b);
                asm volatile("s_waitcnt lgkmcnt(0)" ::: "memory");
                if (lane == 0) __hip_atomic_fetch_add(vdone + (k & (NV - 1)), 1u, __ATOMIC_RELAXED, __HIP_MEMORY_SCOPE_WORKGROUP);
                SEG(1);
                settle();
                SEG(3);
                const u32 loff = (u32)(lane >> 4) * dst.LH + (u32)(lane & 15) * 16u;
                if ((kc & 3u) == 2u) {
#pragma unroll
                    for (int g = 0; g < 8; ++g) store_group<AUX_OUT>(dst, loff, g, o[g]);
                } else {
#pragma unroll
                    for (int g = 0; g < 8; ++g) store_group<AUX_DEV>(dst, loff, g, o[g]);
                    owed = (k & 7u) | ((kc >> 2) << 3);
                }
                SEG(2);
            }
            settle();
            STAMP_AT(3);
        }
        return;
    }

    {
        const bool packer = bid >= (u32)NGEMM;
        const u64 tb = __builtin_amdgcn_s_memrealtime();
        if ((TOUCH & 2) && bid - 120u < 8u && wid == 0) {
            while (__builtin_amdgcn_s_memrealtime() - tb < TOUCH_AT) __builtin_amdgcn_s_sleep(2);
            const u32 warm = __hip_atomic_load((const GAS u32*)(wsl + OFF_GRAN), __ATOMIC_RELAXED, __HIP_MEMORY_SCOPE_SYSTEM);
            asm volatile("" :: "v"(warm));
        }
        const u32 tend = packer ? (u32)NTASK : (u32)(NTASK - PACK_IN), b = (u32)wid;
        u32 it = 0;
        for (u32 task = packer ? (role >> 3) + NTASK - PACK_IN : bid; task < tend; task += packer ? (u32)NPACK : (u32)NGEMM, ++it) {
            if (task >= 16u - PUSHC) while (__builtin_amdgcn_s_memrealtime() - tb < (u64)(ENC_DELAY + (packer ? 0u : task * ENC_PACE / 64u))) __builtin_amdgcn_s_sleep(4);
            if (it == STAMP_IT) STAMP_AT(12);
            const u32 tw = task < 16u - PUSHC ? (task + PUSHC) << 4 : (p.task[task >> 1] >> ((task & 1u) * 16u)) & 0xffffu, c = tw >> 4, k = tw & 15u;
            const InPos ip = in_of(c);
            const u32 dd = (rank + k) & 7u;
            if (ip.isw) {
                const rsrc_t r = mk_rsrc(p.w + (size_t)(ip.q * 128u + (b >> 1) * 32u) * 65536u + (gpanel(dd, ip.s) * 256u + (b & 1u) * 128u) * 2u, 31u * 65536u + 256u);
#pragma unroll
                for (int i = 0; i < 8; ++i) a[i] = __builtin_amdgcn_raw_buffer_load_b128(r, (lane >> 4) * 8 * 65536 + (lane & 15) * 16, i * 65536, AUX_NT);
            } else if (task >= 16u - PUSHC) load_x(px, ip.s, ip.q, b, lane, a);
            u32 v[8][4];
            if (ip.isw) {
#pragma unroll
                for (int j = 0; j < 8; ++j)
#pragma unroll
                    for (int e = 0; e < 4; ++e) v[j][e] = vperm(a[2 * e + 1][j >> 1], a[2 * e][j >> 1], (j & 1) ? 0x07060302u : 0x05040100u);
            } else {
#pragma unroll
                for (int j = 0; j < 8; ++j)
#pragma unroll
                    for (int e = 0; e < 4; ++e) v[j][e] = a[j][e];
            }
            const bool own = (k & 7u) == 0u, send = k < 8u;
            const u32 call = rfl(callv) + 1u;
            LAS u32* cnt = (LAS u32*)lds + 4u * it;
            u32 total;
            if (send) {
                encode_block(v, mk_rsrc(ws + OFF_SI + (size_t)((call & 1u) * NTASK + slot_in(ip.isw, ip.s, ip.q, dd)) * CH, CH), b, cnt, tab, wl, lane);
                if (it == STAMP_IT) STAMP_AT(14);
                asm volatile("s_waitcnt vmcnt(0)" ::: "memory");
                if (chunk_done(cnt, lane, total) && (k ? (u32)lane == k : (u32)lane - 1u < 7u))
                    __hip_atomic_store((GAS u64*)(wsl + OFF_GRAN) + (size_t)((call & 1u) * 8u + rank) * NC + c,
                                       ((u64)total << 32) | call, __ATOMIC_RELAXED, __HIP_MEMORY_SCOPE_SYSTEM);
            }
            if (own) {
                const Dest dst = dest_of(frag_base(ws, ip.isw, ip.s, ip.q, 0u, call), ip.isw ? 1u : 0u, b);
#pragma unroll
                for (int g = 0; g < 8; ++g) store_group<AUX_DEV>(dst, (u32)lane * 16u, g, (u32x4){v[g][0], v[g][1], v[g][2], v[g][3]});
                asm volatile("s_waitcnt vmcnt(0)" ::: "memory");
                if (chunk_done(cnt + 2, lane, total)) arrive(ws, ip.bt * 8u, 8u, call, lane);
            }
            if (it == STAMP_IT) STAMP_AT(1);
        }
        if (lane == 0) __hip_atomic_store(callp, callv + 1u, __ATOMIC_RELAXED, __HIP_MEMORY_SCOPE_AGENT);
        if (packer) {
            STAMP_AT(4);
            if (wid >= 4) __builtin_amdgcn_s_setprio(1);
            const u32 call = rfl(callv) + 1u;
            GAS u32* tf = (GAS u32*)(ws + OFF_TILE + b * 4u);
            u32 seen = 0, nanv = 0;
            auto fetch = [&](u32 yc) {
                if (rfl(seen) != call) {
                    const u64 t0 = __builtin_amdgcn_s_memrealtime();
                    while (rfl(__hip_atomic_load(tf + (yc >> 1) * 32u, __ATOMIC_RELAXED, __HIP_MEMORY_SCOPE_AGENT)) != call) {
                        if (give_up(tmo, call, t0)) { nanv = 0x7fc07fc0u; break; }
                        __builtin_amdgcn_s_sleep(POLL_SLEEP);
                    }
                }
                asm volatile("" ::: "memory");
                const Dest sy = dest_of(y_base(p.out, (yc >> 1) & 15u, rank * 16u + (yc >> 5), yc & 1u), 2u, b);
#pragma unroll
                for (int i = 0; i < 8; ++i) a[i] = __builtin_amdgcn_raw_buffer_load_b128(sy.r, (lane >> 4) * 4 * 65536 + (lane & 15) * 16, ((i & 3) + (i >> 2) * 16) * 65536, AUX_SYS);
                seen = __hip_atomic_load(tf + ((yc + NPACK) >> 1) * 32u, __ATOMIC_RELAXED, __HIP_MEMORY_SCOPE_AGENT);
            };
            auto publish = [&](u32 yc) {
                u32 total;
                if (chunk_done((LAS u32*)lds + 4u * it, lane, total) && (u32)lane - 1u < 7u)
                    __hip_atomic_store((GAS u64*)(wsl + OFF_GRAN) + (size_t)((call & 1u) * 8u + rank) * NC + NCIN + yc,
                                       ((u64)total << 32) | call, __ATOMIC_RELAXED, __HIP_MEMORY_SCOPE_SYSTEM);
                ++it;
            };
            u32 yc = role >> 3;
            fetch(yc);
            STAMP_AT(3);
            for (;; yc += NPACK) {
                u32 v[8][4];
#pragma unroll
                for (int j = 0; j < 8; ++j)
#pragma unroll
                    for (int e = 0; e < 4; ++e) v[j][e] = a[j][e] | nanv;
                if (yc + NPACK < (u32)NCY) fetch(yc + NPACK);
                encode_block(v, mk_rsrc(ws + OFF_SY + (size_t)yc * CH, CH), b, (LAS u32*)lds + 4u * it, tab, wl, lane);
                asm volatile("s_waitcnt vmcnt(0)" ::: "memory");
                publish(yc);
                if (yc + NPACK >= (u32)NCY) break;
            }
            __builtin_amdgcn_s_setprio(0);
            if (!PJOIN || NLAST == WB) { STAMP_AT(2); return; }
        }
        STAMP_AT(2);
    }

    const u32 call = rfl(callv) + 1u;
    const int wr = wid >> 2, wc = wid & 3;
    constexpr int HTB = 16384;
#define SA(b, h) (((b) * 2 + (h)) * HTB)
#define SB(b, h) ((4 + (b) * 2 + (h)) * HTB)
#define STAGE(bufoff, gbase) do { _Pragma("unroll") for (int _i = 0; _i < 2; ++_i) \
        __builtin_amdgcn_global_load_lds((const GAS u32*)((gbase) + voff + _i * 8192), (LAS u32*)(lds + (bufoff) + ldsw + _i * 8192), 16, 0, AUX_SYS); } while (0)
#define LDA(dst, b, h) do { _Pragma("unroll") for (int m = 0; m < 4; ++m) _Pragma("unroll") for (int k = 0; k < 2; ++k) dst[m][k] = *(const LAS bf16x8*)(lds + SA(b, h) + aoff + m * 2048 + k * 1024); } while (0)
#define LDB(dst, b, h) do { _Pragma("unroll") for (int n = 0; n < 2; ++n) _Pragma("unroll") for (int k = 0; k < 2; ++k) dst[n][k] = *(const LAS bf16x8*)(lds + SB(b, h) + boff + n * 2048 + k * 1024); } while (0)
#define MMA(ai, bj, At, Bt) do { _Pragma("unroll") for (int _x = 0; _x < 16; ++_x) { const int m = (_x >> 1) & 3, n = (_x ^ (_x >> 1)) & 1, k = _x >> 3; \
        acc[ai][bj][m][n] = __builtin_amdgcn_mfma_f32_16x16x32_bf16(Bt[n][k], At[m][k], acc[ai][bj][m][n], 0, 0, 0); SCHED; } } while (0)
#define MMA2(ai) do { __builtin_amdgcn_s_setprio(1); MMA(ai, 0, At, B0); MMA(ai, 1, At, B1); __builtin_amdgcn_s_setprio(0); } while (0)
#define WAIT_V(n) asm volatile("s_waitcnt vmcnt(" #n ")" ::: "memory")
#define WAIT_L(n) asm volatile("s_waitcnt lgkmcnt(" #n ")" ::: "memory")
#define BAR __builtin_amdgcn_s_barrier()
    const u32 voff = (u32)tid * 16u, ldsw = (u32)wid * 1024u;
    const int aoff = wr * 8192 + lane * 16, boff = wc * 4096 + ((lane & 15) * 4 + ((lane >> 4) ^ (((lane >> 3) & 1) << 1))) * 16;
    const u32 idx = bid >> 3, hh = (bid >> 2) & 1u;
    u32 ti = (bid & 3u) * 4u + (idx < 4u * (WB / 2) ? idx & 3u : 2u * hh + (idx & 1u)), tj = idx < 4u * (WB / 2) ? 2u * (idx >> 2) + hh : WB - 1u;
    constexpr size_t kstep = 32768, hstep = 16384;
    const GAS char* xf = (const GAS char*)frag_base(ws, false, 0u, 0u, 0u, call);
    const GAS char* wf = (const GAS char*)(ws + OFF_WF);
    GAS u32* flags = (GAS u32*)(ws + OFF_FLAG + (bid & 7u) * 256u) + ((u32)lane & 7u);
    auto arrived = [&](u32 bt) { return (u32)__builtin_ctz(((u32)__builtin_amdgcn_ballot_w64(__hip_atomic_load(flags + bt * 8u, __ATOMIC_RELAXED, __HIP_MEMORY_SCOPE_SYSTEM) != call) & 31u) | 32u); };
#define STAGE_FIRST(pA, pB) do { STAGE(SB(0, 0), pB); STAGE(SA(0, 0), pA); STAGE(SB(0, 1), (pB) + hstep); STAGE(SA(0, 1), (pA) + hstep); \
        STAGE(SB(1, 0), (pB) + kstep); STAGE(SA(1, 0), (pA) + kstep); STAGE(SB(1, 1), (pB) + hstep + kstep); } while (0)
    u32 nwait = 0, bad = 0, staged = 0;
    auto draw_ticket = [&]() { return __hip_atomic_fetch_add((GAS u32*)(ws + OFF_TICK + 8u * 128u), 1u, __ATOMIC_RELAXED, __HIP_MEMORY_SCOPE_AGENT) - (call - 1u) * (NGEMM + (PJOIN ? NPACK : 0)); };
    auto last_tile = [&](u32 tk, u32& i, u32& j) { const u32 gr = tk / (4u * NLAST), r = tk - gr * 4u * NLAST; i = (gr * 4u + (r & 3u)) & 15u; j = gr < 4u ? (NB - 1u) * WB + (r >> 2) : (u32)NP; };
    u32 round = 0;
    if (bid >= (u32)NGEMM) {
        if (tid == 0) *(volatile LAS u32*)(ctl + 2) = draw_ticket();
        asm volatile("s_waitcnt lgkmcnt(0)" ::: "memory"); BAR; asm volatile("" ::: "memory");
        last_tile(lds_ld(ctl + 2), ti, tj);
        round = NB - 1u;
    }
    for (; tj < (u32)NP; ++round) {
        const GAS char* cA = wf + (size_t)tj * (2u << 20);
        const GAS char* cB = xf + (size_t)ti * (2u << 20);
        u32 ni = ti, nj = tj + WB;
        f32x4 acc[2][2][4][2];
#pragma unroll
        for (int a = 0; a < 2; ++a)
#pragma unroll
            for (int b = 0; b < 2; ++b)
#pragma unroll
                for (int m = 0; m < 4; ++m)
#pragma unroll
                    for (int n = 0; n < 2; ++n) acc[a][b][m][n] = (f32x4){0.f, 0.f, 0.f, 0.f};
        bf16x8 At[4][2], B0[2][2], B1[2][2];
        for (u32 q0 = 0; q0 < 5u;) {
            const u32 k0 = q0 ? q0 * 14u - 6u : 0u;
            if (!staged) {
                if (wid == 0) {
                    const u64 t0 = __builtin_amdgcn_s_memrealtime();
                    u32 q1;
                    for (u32 spin = 0;; ++spin) {
                        q1 = arrived(round);
                        if (q1 > q0) break;
                        if ((spin & 7u) == 7u && give_up(tmo, call, t0)) { q1 = 5u | 8u; break; }
                        __builtin_amdgcn_s_sleep(POLL_SLEEP);
                    }
                    if (lane == 0) *(volatile LAS u32*)(ctl + (nwait & 1u)) = q1;
                    asm volatile("s_waitcnt lgkmcnt(0)" ::: "memory");
                }
                asm volatile("" ::: "memory"); BAR; asm volatile("" ::: "memory");
                const u32 te = lds_ld(ctl + (nwait & 1u));
                ++nwait; bad |= te >> 3;
                staged = te & 7u;
                STAGE_FIRST(cA + (size_t)k0 * kstep, cB + (size_t)k0 * kstep);
            }
            if (k0 == 0u && round < 4u) STAMP_AT(3 + round * 2);
            q0 = staged & 7u;
            const u32 k1 = q0 * 14u - 6u;
            if (wr == 1) BAR;
            bool lax = (staged >> 3) != 0u;
            if (lax) WAIT_V(22); else WAIT_V(6);
            BAR; BAR;
            for (u32 t = k0; t < k1; t += 2u) {
                const u32 t2 = t + 2u < k1 ? t + 2u : t;
                const GAS char* a1 = cA + (size_t)(t + 1u) * kstep;
                const GAS char* a2 = cA + (size_t)t2 * kstep; const GAS char* b2 = cB + (size_t)t2 * kstep;
                const GAS char* a3 = a2 + kstep; const GAS char* b3 = b2 + kstep;
                LDB(B0, 0, 0); LDB(B1, 0, 1); LDA(At, 0, 0); STAGE(SA(1, 1), a1 + hstep);
                if (lax) WAIT_V(24); else WAIT_V(8);
                WAIT_L(0); BAR; SCHED; MMA2(0); BAR; SCHED;
                LDA(At, 0, 1); STAGE(SB(0, 0), b2); STAGE(SB(0, 1), b2 + hstep); STAGE(SA(0, 0), a2);
                if (lax) WAIT_V(24); else WAIT_V(8);
                lax = false;
                WAIT_L(0); BAR; SCHED; MMA2(1); BAR; SCHED;
                LDB(B0, 1, 0); LDB(B1, 1, 1); LDA(At, 1, 0); STAGE(SA(0, 1), a2 + hstep);
                WAIT_V(8); WAIT_L(0); BAR; SCHED; MMA2(0); BAR; SCHED;
                LDA(At, 1, 1); STAGE(SB(1, 0), b3); STAGE(SB(1, 1), b3 + hstep); STAGE(SA(1, 0), a3);
                WAIT_V(8); WAIT_L(0); BAR; SCHED; MMA2(1); BAR; SCHED;
            }
            const bool last = q0 == 5u, draw = NLAST < WB && last && round == NB - 2u;
            if (wid == 0) {
                u32 tk = 0;
                if (draw && lane == 0) tk = draw_ticket();
                const u32 q1 = arrived(last ? round + 1u : round);
                if (lane == 0) *(volatile LAS u32*)(ctl + (nwait & 1u)) = q1 | (tk << 8);
                asm volatile("s_waitcnt lgkmcnt(0)" ::: "memory");
            }
            WAIT_V(0);
            if (wr == 0) BAR;
            BAR;
            asm volatile("" ::: "memory");
            staged = lds_ld(ctl + (nwait & 1u));
            ++nwait;
            if (draw) last_tile(staged >> 8, ni, nj);
            staged &= 7u;
            if (staged <= (last ? 0u : q0) || (last && nj >= (u32)NP)) staged = 0u;
            if (staged) STAGE_FIRST(last ? wf + (size_t)nj * (2u << 20) : cA + (size_t)k1 * kstep, last ? xf + (size_t)ni * (2u << 20) : cB + (size_t)k1 * kstep);
        }
        if (round < 4u) STAMP_AT(4 + round * 2);
#pragma unroll
        for (int bj = 0; bj < 2; ++bj) {
            const Dest dst = dest_of(y_base(p.out, ti, gpanel(rank, tj), bj), 2u, (u32)wid);
            const u32 loff = (u32)(lane >> 4) * dst.LH + (u32)(lane & 15) * 16u;
            u32x4 v[8];
#pragma unroll
            for (int g = 0; g < 8; ++g)
#pragma unroll
                for (int q = 0; q < 4; ++q) {
                    typedef float f32x2 __attribute__((ext_vector_type(2)));
                    typedef __bf16 bf16x2 __attribute__((ext_vector_type(2)));
                    const f32x2 f = {acc[q >> 1][bj][(q & 1) * 2][g >> 2][g & 3], acc[q >> 1][bj][(q & 1) * 2 + 1][g >> 2][g & 3]};
                    v[g][q] = __builtin_bit_cast(u32, __builtin_convertvector(f, bf16x2));
                }
            if (bad) {
#pragma unroll
                for (int g = 0; g < 8; ++g) v[g] = (u32x4){0x7fc07fc0u, 0x7fc07fc0u, 0x7fc07fc0u, 0x7fc07fc0u};
            }
            if (tj < (u32)POWN) {
#pragma unroll
                for (int g = 0; g < 8; ++g) store_group<AUX_DEV>(dst, loff, g, v[g]);
            } else if (staged) {
#pragma unroll
                for (int g = 0; g < 8; ++g) store_group<AUX_NT>(dst, loff, g, v[g]);
            } else {
#pragma unroll
                for (int g = 0; g < 8; ++g) store_group<AUX_OUT>(dst, loff, g, v[g]);
            }
        }
        if (tj < (u32)POWN) {
            asm volatile("s_waitcnt vmcnt(0)" ::: "memory");
            if (lane == 0) __hip_atomic_store((GAS u32*)(ws + OFF_TILE + (tj * 16u + ti) * 128u + (u32)wid * 4u), call, __ATOMIC_RELAXED, __HIP_MEMORY_SCOPE_AGENT);
        } else if (staged) staged |= 8u;
        ti = ni; tj = nj;
        asm volatile(".Lcode_far:");
    }
    STAMP_AT(15);
}

static void build_tables(Params& p) {
    int at[2] = {0, 3};
    p.links = 0;
    for (int rel = 1; rel < 8; ++rel) p.links |= (u32)rel << (4 * at[((0xb2 >> p.rank) ^ (0xb2 >> ((p.rank + rel) & 7))) & 1]++);
    unsigned short* t = (unsigned short*)p.task;
    int n = 0;
    for (u32 c = PUSHC; c < (u32)NCIN; ++c) {
        if (c == 16u) for (u32 e = 0; e < (u32)PUSHC; ++e) t[n++] = (unsigned short)(e << 4 | 8u);
        const InPos ip = in_of(c);
        if (ip.isw && ip.s < (u32)POWN) for (u32 k = 1; k <= 8; ++k) t[n++] = (unsigned short)(c << 4 | k);
        else t[n++] = (unsigned short)(c << 4);
    }
}

extern "C" void kernel_launch(int rank, int nranks, ncclComm_t comm, hipStream_t stream,
                              void* const* d_in, const int* in_sizes, int n_in, void* d_out, int out_size,
                              const int* in_dtypes, int out_dtype, void* const* d_out_all,
                              void* const* d_ws_all, size_t ws_bytes) {
    if (nranks != NR || (unsigned)rank >= (unsigned)NR || ws_bytes < WS_NEED || n_in != 2) { fprintf(stderr, "unsupported configuration\n"); return; }
    Params p;
    p.x = (const char*)d_in[0]; p.w = (const char*)d_in[1]; p.out = (char*)d_out; p.rank = rank;
    for (int i = 0; i < NR; ++i) p.ws[i] = (char*)d_ws_all[(rank + i) % NR];
    build_tables(p);
    hipLaunchKernelGGL(gemm_ar_fused, dim3(256), dim3(512), 0, stream, p);
    HIP_CHECK(hipGetLastError());
}
```

```cpp
#include <hip/hip_runtime.h>
#include <rccl/rccl.h>
#include <cstdio>

#define HIP_CHECK(expr) do { hipError_t _e = (expr); if (_e != hipSuccess) { \
    fprintf(stderr, "HIP error %s at %s:%d\n", hipGetErrorString(_e), __FILE__, __LINE__); \
    return; } } while (0)

typedef unsigned u32;
typedef unsigned long long u64;
typedef u32 u32x4 __attribute__((ext_vector_type(4)));
typedef u32 u32x2 __attribute__((ext_vector_type(2)));
typedef unsigned short u16x2 __attribute__((ext_vector_type(2)));
typedef short bf16x8 __attribute__((ext_vector_type(8)));
typedef float f32x4 __attribute__((ext_vector_type(4)));
#define LAS __attribute__((address_space(3)))
#define GAS __attribute__((address_space(1)))
typedef __amdgpu_buffer_rsrc_t rsrc_t;

constexpr int NR = 8;
constexpr int RHO = 6;
constexpr int WB = 12;
constexpr int PULL_WG_PER_PEER = 8;
constexpr int PAT = 56565;
constexpr int LVAR = 5;
constexpr int PUSHC = 8;
constexpr int PACK_IN = 112;
constexpr int ENC_DELAY = 150;
constexpr int ENC_PACE = 192;
constexpr int POLL_SLEEP = 8;
constexpr int NPULL = 7 * PULL_WG_PER_PEER, NGEMM = 16 * WB, NPACK = 256 - NGEMM - NPULL;
constexpr int POWN = 16 - RHO, NP = POWN + 8 * RHO;
constexpr int NB = (NP + WB - 1) / WB, N0 = 16 + WB;
constexpr int NCIN = 64 + 4 * NP, NCY = 32 * POWN, NC = NCIN + NCY;
constexpr int NTASK = 16 * 36;
constexpr int NDEC = 6;
constexpr u64 DEADLINE = 100000000ull;
constexpr int PUSHB = 7;
constexpr int NLAST = NP - (NB - 1) * WB;
static_assert(NGEMM == 192 && NPACK == 8 && (PACK_IN + NCY) / NPACK < 126 && NB <= 8 && POWN <= WB, "roles");

constexpr int FIXB = 5120, CVAR = 8 * FIXB, CHDR = 32, CH = 66560;
constexpr size_t OFF_CALL = 0;
constexpr size_t OFF_TMO = 131072;
constexpr size_t OFF_TICK = OFF_TMO + 256;
constexpr size_t OFF_RDY = OFF_TICK + 16 * 128;
constexpr size_t OFF_FLAG = OFF_RDY + 64 * 128;
constexpr size_t OFF_TILE = OFF_FLAG + 8 * 256;
constexpr size_t OFF_PF = 188416;
constexpr size_t OFF_GRAN = 196608;
constexpr size_t OFF_PS = 1ull << 20;
constexpr size_t OFF_XF = 8ull << 20;
constexpr size_t OFF_WF = OFF_XF + (32ull << 20);
constexpr size_t OFF_SY = OFF_WF + NP * (2ull << 20);
constexpr size_t OFF_SI = OFF_SY + (size_t)NCY * CH;
constexpr size_t WS_NEED = OFF_SI + 2ull * NTASK * CH;

constexpr int A2B = 2064, A3B = 528, VARB = 912, WLDS = A2B + A3B + VARB, N2MAX = 2032, N3MAX = 496, UBITS = 2016, NPATCH = 31;
constexpr int CTL_OFF = 2048, CTL_WORDS = 128, WL_OFF = 2560, BIG_OFF = 30720, LDS_BYTES = BIG_OFF + 131072;
static_assert(WL_OFF + 8 * WLDS <= BIG_OFF && LDS_BYTES <= 163840, "LDS map");
constexpr int NF = 16, NV = 4, VSLOT = 8192, VRING = NF * FIXB;
constexpr int AUX_SYS = 17, AUX_DEV = 16, AUX_OUT = 18, AUX_NT = 2;

struct Params {
    int rank;
    u32 links;
    const char* x; const char* w; char* out;
    char* ws[NR];
    u32 task[NTASK / 2];
};

__device__ __forceinline__ u32 rfl(u32 x) { return (u32)__builtin_amdgcn_readfirstlane((int)x); }
__device__ __forceinline__ rsrc_t mk_rsrc(const void* p, u32 bytes) {
    const u64 a = (u64)p; const u64 u = ((u64)rfl((u32)(a >> 32)) << 32) | rfl((u32)a);
    return __builtin_amdgcn_make_buffer_rsrc((void*)u, (short)0, (int)rfl(bytes), 0x00020000);
}
__device__ __forceinline__ u32 vperm(u32 a, u32 b, u32 s) { return __builtin_amdgcn_perm(a, b, s); }
__device__ __forceinline__ u32 wave_incl_scan(u32 x) {
    x += (u32)__builtin_amdgcn_update_dpp(0, (int)x, 0x111, 0xf, 0xf, false);
    x += (u32)__builtin_amdgcn_update_dpp(0, (int)x, 0x112, 0xf, 0xf, false);
    x += (u32)__builtin_amdgcn_update_dpp(0, (int)x, 0x114, 0xf, 0xf, false);
    x += (u32)__builtin_amdgcn_update_dpp(0, (int)x, 0x118, 0xf, 0xf, false);
    x += (u32)__builtin_amdgcn_update_dpp(0, (int)x, 0x142, 0xa, 0xf, false);
    x += (u32)__builtin_amdgcn_update_dpp(0, (int)x, 0x143, 0xc, 0xf, false);
    return x;
}
__device__ __forceinline__ u32 pk_sub(u32 a, u32 b) { return __builtin_bit_cast(u32, (u16x2)(__builtin_bit_cast(u16x2, a) - __builtin_bit_cast(u16x2, b))); }
template <int AUX>
__device__ __forceinline__ void st128(u32x4 x, rsrc_t r, int voff, int soff) {
    __builtin_amdgcn_raw_buffer_store_b128(x, r, voff, soff, AUX);
    asm volatile("s_nop 1" :: "v"(x) : "memory");
}
#define WAVE_FENCE() __builtin_amdgcn_fence(__ATOMIC_ACQ_REL, "wavefront")

struct TabEntry { u32 dep, cmp, n, r0, r1; };
__device__ __forceinline__ TabEntry table_entry(u32 i) {
    u32 n = 0, dep = 0, cmp = 0x0c0c0c0cu;
#pragma unroll
    for (u32 t = 0; t < 4; ++t) {
        const u32 c = (i >> (2 * t)) & 3u;
        dep |= (c ? 4u + c : n) << (8 * t);
        cmp ^= c ? 0u : (0x0cu ^ t) << (8 * n);
        n += c ? 0u : 1u;
    }
    u64 r = 0;
#pragma unroll
    for (u32 t = 0; t < 7; ++t) r += (u64)(((i >> t) & 1u) ^ ((i >> t) ? 1u : 0u)) << (8u * (u32)__builtin_popcount(i & ((1u << t) - 1u)));
    return TabEntry{dep, cmp, n, (u32)r, (u32)(r >> 32)};
}
__device__ __forceinline__ void or3(LAS u32* q, u64 x, u32 sh) {
    const u64 y = x << sh;
    __hip_atomic_fetch_or(q, (u32)y, __ATOMIC_RELAXED, __HIP_MEMORY_SCOPE_WORKGROUP);
    __hip_atomic_fetch_or(q + 1, (u32)(y >> 32), __ATOMIC_RELAXED, __HIP_MEMORY_SCOPE_WORKGROUP);
    __hip_atomic_fetch_or(q + 2, (u32)(((x >> 32) << sh) >> 32), __ATOMIC_RELAXED, __HIP_MEMORY_SCOPE_WORKGROUP);
}
__device__ __forceinline__ u32 code_byte(u32 hi, u32 lo, u32 x) {
    const u32 f = vperm(hi, lo, x);
    return __builtin_amdgcn_udot4((f ^ (f >> 2)) & 0x03030303u, 0x40100401u, 0u, false);
}
template <bool PRED>
__device__ __forceinline__ void or_bytes(u32 pos, u32 x) {
    const u64 y = (u64)x << ((pos & 3u) * 8u);
    LAS u32* q = (LAS u32*)(size_t)(pos & ~3u);
    if (!PRED || x) {
        __hip_atomic_fetch_or(q, (u32)y, __ATOMIC_RELAXED, __HIP_MEMORY_SCOPE_WORKGROUP);
        __hip_atomic_fetch_or(q + 1, (u32)(y >> 32), __ATOMIC_RELAXED, __HIP_MEMORY_SCOPE_WORKGROUP);
    }
}
__device__ __forceinline__ u32 get_bytes(u32 pos) {
    const LAS u32* q = (const LAS u32*)(size_t)(pos & ~3u);
    return __builtin_amdgcn_alignbyte(q[1], q[0], pos);
}
__device__ __forceinline__ u32 la(const LAS void* p) { return (u32)(size_t)p; }
__device__ __forceinline__ u32 pack_level2(const LAS u32x2* tab, const LAS char* src, u32 n, LAS u32* end, LAS char* dst, int lane) {
    u32 m = 0;
    for (u32 i0 = 0; i0 < n; i0 += 1024u) {
        const u32 i = i0 + (u32)lane * 16u, hi = 0x02010000u, lo = 0x00030000u;
        const u32x4 x = *(const LAS u32x4*)(src + (i < n ? i : 0u));
        u32 cw = code_byte(hi, lo, x[0]) | (code_byte(hi, lo, x[1]) << 8) | (code_byte(hi, lo, x[2]) << 16) | (code_byte(hi, lo, x[3]) << 24);
        const u32 valid = i < n ? n - i : 0u;
        if (valid < 16u) cw |= 0x55555555u << (2u * valid);
        u32x2 e[4]; u32 c[5]; c[0] = 0;
#pragma unroll
        for (int j = 0; j < 4; ++j) { e[j] = tab[(cw >> (8 * j)) & 0xffu]; c[j + 1] = c[j] + e[j][1]; }
        const u32 incl = wave_incl_scan(c[4]), p0 = la(dst) + m + incl - c[4];
        if (i < n) *(end - 1 - (i >> 4)) = cw;
        m += (u32)__builtin_amdgcn_readlane((int)incl, 63);
        if (m <= (u32)N3MAX) {
#pragma unroll
            for (int j = 0; j < 4; ++j) or_bytes<true>(p0 + c[j], vperm(0u, x[j], e[j][0]));
        }
    }
    WAVE_FENCE();
    return m;
}
__device__ __forceinline__ void pack_level3(const LAS char* src, u32 n, LAS u32* ub, LAS u32* pt, u32 T, int lane, u32& bits, u32& np) {
    for (u32 i0 = 0; i0 < n; i0 += 256u) {
        const u32 i = i0 + (u32)lane * 4u;
        const u32 x = *(const LAS u32*)(src + (i < n ? i : 0u));
        u32 cum = 0, pf = 0; u64 m = 0;
#pragma unroll
        for (u32 t = 0; t < 4; ++t) {
            const u32 r = ((x >> (8 * t)) - 8u) & 0xffu;
            const bool valid = i + t < n, big = r > 15u;
            cum += valid && !big ? r : 0u;
            m |= (u64)(valid ? 1u : 0u) << cum;
            cum += valid ? 1u : 0u;
            pf |= valid && big ? 1u << t : 0u;
        }
        const u32 incl = wave_incl_scan(cum), p = bits + incl - cum;
        bits += (u32)__builtin_amdgcn_readlane((int)incl, 63);
        if (bits <= (u32)UBITS && cum) or3(ub + (p >> 5), m, p & 31u);
        if (__builtin_amdgcn_ballot_w64(pf != 0u)) {
#pragma unroll
            for (u32 t = 0; t < 4; ++t) {
                const u64 bm = __builtin_amdgcn_ballot_w64((pf >> t) & 1u);
                const u32 at = np + __builtin_amdgcn_mbcnt_hi((u32)(bm >> 32), __builtin_amdgcn_mbcnt_lo((u32)bm, 0u));
                if ((pf >> t) & 1u) pt[at] = ((i + t) << 8) | ((T - (x >> (8 * t))) & 0xffu);
                np += (u32)__builtin_popcountll(bm);
            }
        }
    }
    WAVE_FENCE();
}
__device__ __forceinline__ u32 block_base(const u32 (&v)[8][4]) {
    u32 s = 0;
#pragma unroll
    for (int g = 0; g < 8; ++g) s = __builtin_amdgcn_udot4(__builtin_amdgcn_alignbit(v[g][0], v[g][0], 31), 0x01000100u, s, false);
    const u32 T = ((u32)__builtin_amdgcn_readlane((int)wave_incl_scan(s), 63) >> 10) + 5u;
    return T > 255u ? 255u : (T < 23u ? 23u : T);
}
__device__ __forceinline__ void push_fixed(const u32 (&v)[8][4], rsrc_t to, int lane) {
    u32x4 l1 = (u32x4){0, 0, 0, 0};
    u32 tt = 0;
#pragma unroll
    for (int q = 0; q < 4; ++q) {
        u32x4 sm;
#pragma unroll
        for (int j = 0; j < 4; ++j) {
            const int k = 4 * q + j;
            const u32 a0 = v[k >> 1][(k & 1) * 2], a1 = v[k >> 1][(k & 1) * 2 + 1];
            sm[j] = vperm(__builtin_amdgcn_alignbit(a1, a1, 31), __builtin_amdgcn_alignbit(a0, a0, 31), 0x06040200u);
        }
        if (q) asm volatile("s_waitcnt vmcnt(0)" ::: "memory");
        st128<AUX_SYS>(sm, to, lane * 16 + q * 1024, 0);
        if (q == 0) tt = (block_base(v) * 256u + 255u) * 0x10001u;
#pragma unroll
        for (int j = 0; j < 4; ++j) {
            const int k = 4 * q + j;
            const u32 a0 = v[k >> 1][(k & 1) * 2], a1 = v[k >> 1][(k & 1) * 2 + 1];
            l1[q] |= code_byte(0x00000302u, 0x01000000u, vperm(pk_sub(tt, __builtin_amdgcn_alignbit(a1, a1, 31)), pk_sub(tt, __builtin_amdgcn_alignbit(a0, a0, 31)), 0x07050301u)) << (8 * j);
        }
    }
    asm volatile("s_waitcnt vmcnt(0)" ::: "memory");
    st128<AUX_SYS>(l1, to, lane * 16 + 4096, 0);
}
__device__ __forceinline__ void encode_block(const u32 (&v)[8][4], rsrc_t ch, u32 b, LAS u32* alloc, const LAS u32x2* tab, LAS char* wl, int lane) {
#pragma unroll
    for (int j = 0; j < 4; ++j) if (j < 3 || lane < (WLDS - 3072) / 16) *(LAS u32x4*)(wl + j * 1024 + lane * 16) = (u32x4){0, 0, 0, 0};
    const u32 T = block_base(v);
    const u32 tt = (T * 256u + 255u) * 0x10001u;
    const int fo = (int)(b * FIXB);
    u32 U[16], l1[4] = {0, 0, 0, 0}, c[17];
    u32x2 e[16];
#pragma unroll
    for (int q = 0; q < 4; ++q) {
        u32x4 sm;
#pragma unroll
        for (int j = 0; j < 4; ++j) {
            const int k = 4 * q + j;
            const u32 a0 = v[k >> 1][(k & 1) * 2], a1 = v[k >> 1][(k & 1) * 2 + 1];
            const u32 r0 = __builtin_amdgcn_alignbit(a0, a0, 31), r1 = __builtin_amdgcn_alignbit(a1, a1, 31);
            sm[j] = vperm(r1, r0, 0x06040200u);
            U[k] = vperm(pk_sub(tt, r1), pk_sub(tt, r0), 0x07050301u);
            const u32 cb = code_byte(0x00000302u, 0x01000000u, U[k]);
            l1[q] |= cb << (8 * j);
            e[k] = tab[cb];
        }
        st128<AUX_SYS>(sm, ch, lane * 16 + q * 1024, fo);
    }
    st128<AUX_SYS>((u32x4){l1[0], l1[1], l1[2], l1[3]}, ch, lane * 16 + 4096, fo);
    __builtin_amdgcn_sched_barrier(0);
    c[0] = 0;
#pragma unroll
    for (int k = 0; k < 16; ++k) c[k + 1] = c[k] + e[k][1];
    const u32 incl = wave_incl_scan(c[16]), p0 = la(wl) + incl - c[16];
    const u32 n2 = (u32)__builtin_amdgcn_readlane((int)incl, 63);
    bool raw = n2 > (u32)N2MAX;
    u32 d2 = 0, d3 = 0, np = 0;
    LAS u32* var = (LAS u32*)(wl + A2B + A3B);
    if (!raw) {
        WAVE_FENCE();
#pragma unroll
        for (int k = 0; k < 16; ++k) or_bytes<false>(p0 + c[k], vperm(0u, U[k], e[k][0]));
        WAVE_FENCE();
        d2 = (n2 + 15u) >> 4;
        const u32 n3 = pack_level2(tab, wl, n2, var + d2, wl + A2B, lane);
        raw = n3 > (u32)N3MAX;
        if (!raw) {
            u32 bits = 0;
            pack_level3(wl + A2B, n3, var + d2, (LAS u32*)wl, T, lane, bits, np);
            raw = bits > (u32)UBITS || np > (u32)NPATCH;
            d3 = (bits + 31u) >> 5;
            if (!raw && (u32)lane < np) var[d2 + d3 + (u32)lane] = ((const LAS u32*)wl)[lane];
            WAVE_FENCE();
        }
    }
    const u32 vd = d2 + d3 + np;
    u32 off = 0;
    if (lane == 0) off = __hip_atomic_fetch_add(alloc, raw ? 3072u : vd * 4u, __ATOMIC_RELAXED, __HIP_MEMORY_SCOPE_WORKGROUP);
    off = rfl(off) + CHDR;
    const int vo = CVAR + (int)off;
    if (lane == 0) __builtin_amdgcn_raw_buffer_store_b32(raw ? off << 17 : (T | (d3 << 8) | (np << 14) | ((off + d2 * 4u) << 17)), ch, 0, CVAR + (int)b * 4, AUX_SYS);
    if (raw) {
        asm volatile("s_waitcnt vmcnt(0)" ::: "memory");
#pragma unroll
        for (int g = 0; g < 8; ++g)
            st128<AUX_SYS>((u32x4){v[g][0], v[g][1], v[g][2], v[g][3]}, ch, lane * 16 + (g < 5 ? g : g - 5) * 1024, g < 5 ? fo : vo);
    } else {
        for (u32 o = (u32)lane; o < vd; o += 64u) __builtin_amdgcn_raw_buffer_store_b32(var[o], ch, (int)o * 4, vo, AUX_SYS);
    }
    WAVE_FENCE();
}
__device__ __forceinline__ bool chunk_done(LAS u32* cnt, int lane, u32& total) {
    u32 n = 0;
    if (lane == 0) n = __hip_atomic_fetch_add(cnt + 1, 1u, __ATOMIC_RELAXED, __HIP_MEMORY_SCOPE_WORKGROUP);
    if (rfl(n) != 7u) return false;
    total = rfl(*(volatile LAS u32*)cnt) + CHDR;
    return true;
}

struct Dest { rsrc_t r; u32 A, B, C, LH; };
__device__ __forceinline__ Dest dest_of(char* base, u32 kind, u32 b) {
    if (kind == 0u) return Dest{mk_rsrc(base + b * 4096u, 32768u + 4096u), 1024u, 32768u, 2048u, 256u};
    if (kind == 1u) return Dest{mk_rsrc(base + (b >> 2) * 32768u + (b & 1u) * 8192u + (b & 2u) * 512u, 16384u + 6144u + 1024u), 2048u, 4096u, 16384u, 256u};
    return Dest{mk_rsrc(base + (b & 3u) * (32u * 65536u) + (b >> 2) * 256u, 31u * 65536u + 256u), 65536u, 131072u, 16u * 65536u, 4u * 65536u};
}
template <int AUX>
__device__ __forceinline__ void store_group(const Dest& d, u32 loff, int g, u32x4 x) {
    st128<AUX>(x, d.r, (int)loff, (int)((g & 1 ? d.A : 0u) + (g & 2 ? d.B : 0u) + (g & 4 ? d.C : 0u)));
}

#define SCHED __builtin_amdgcn_sched_barrier(0)
__device__ __forceinline__ u32x2 get2(u32 pos) { const LAS u32* q = (const LAS u32*)(size_t)(pos & ~3u); return (u32x2){q[0], q[1]}; }
__device__ __forceinline__ void spread3(u32 w, u32 start, u32 n, u32x2 runs, u32 t8, u32 p) {
    if (n) {
        const u32 carry = (u32)__builtin_clz((w << 8) | start);
        const u64 x = ((u64)(t8 - runs[1]) << 32) | (t8 - runs[0] - carry);
        const u32 sh = 64u - 8u * n;
        or3((LAS u32*)(size_t)(p & ~3u), (x << sh) >> sh, (p & 3u) * 8u);
    }
}
template <class Out>
__device__ __forceinline__ void decode_block(const u32x4 (&d)[5], u32 hdr, const LAS u32* ub, const LAS u32x2* tab, const LAS u32x2* tab3, LAS char* wl, int lane, Out out) {
    const u32 T = hdr & 0xffu, d3 = (hdr >> 8) & 0x3fu, np = (hdr >> 14) & 0x1fu;
    auto ex = [&](u32 a, u32 b, u32 c) { return ((T - a) << 8) | ((T - b) << 16) | ((T - c) << 24); };
    const u32 t1 = ex(3, 4, 5), t2 = ex(6, 7, 2), t8 = (T - 8u) * 0x01010101u;
    const u32 w2 = *(ub - 1 - lane), p3i = la(ub) + (u32)lane - 3u;
    u32x2 e[16], e2[4], g[16];
    g[0] = get2(p3i);
    if (lane < A3B / 16) *(LAS u32x4*)(wl + A2B + lane * 16) = (u32x4){0, 0, 0, 0};
#pragma unroll
    for (int k = 0; k < 8; ++k) e[k] = tab[(d[4][k >> 2] >> (8 * (k & 3))) & 0xffu];
    SCHED;
    const u32 w3 = __builtin_amdgcn_alignbyte(g[0][1], g[0][0], p3i), x3 = (u32)lane < d3 * 4u ? w3 >> 24 : 0u;
    const u32x2 e3 = tab3[x3];
#pragma unroll
    for (int j = 0; j < 4; ++j) e2[j] = tab[(w2 >> (8 * j)) & 0xffu];
#pragma unroll
    for (int k = 8; k < 16; ++k) e[k] = tab[(d[4][k >> 2] >> (8 * (k & 3))) & 0xffu];
    SCHED;
    u32 c[17], f[5];
    c[0] = 0; f[0] = 0;
#pragma unroll
    for (int k = 0; k < 16; ++k) c[k + 1] = c[k] + e[k][1];
#pragma unroll
    for (int j = 0; j < 4; ++j) f[j + 1] = f[j] + e2[j][1];
    const u32 n3 = (u32)__builtin_popcount(x3);
    const u32 pk = c[16] | (f[4] << 11) | (n3 << 22);
    const u32 incl = wave_incl_scan(pk), ex0 = incl - pk, tot = (u32)__builtin_amdgcn_readlane((int)incl, 63);
    const u32 a3 = la(wl) + A2B, p1 = la(wl) + (ex0 & 0x7ffu), p2 = a3 + ((ex0 >> 11) & 0x1ffu);
    WAVE_FENCE();
    spread3(w3, lane < 3 ? 0x80000000u >> (8 * lane) : 0x80u, n3, e3, t8, a3 + ((ex0 >> 22) & 0x1ffu));
    for (u32 i0 = 64u, m = tot >> 22; i0 < d3 * 4u; i0 += 64u) {
        const u32 w = get_bytes(p3i + i0), x = i0 + (u32)lane < d3 * 4u ? w >> 24 : 0u, n = (u32)__builtin_popcount(x), in = wave_incl_scan(n);
        spread3(w, 0x80u, n, tab3[x], t8, a3 + ((m + in - n) & 0x1ffu));
        m += (u32)__builtin_amdgcn_readlane((int)in, 63);
    }
    WAVE_FENCE();
    if ((u32)lane < np) { const u32 pw = ub[d3 + (u32)lane]; *(LAS unsigned char*)(wl + A2B + ((pw >> 8) & 0x1ffu)) = (unsigned char)pw; }
    WAVE_FENCE();
#pragma unroll
    for (int j = 0; j < 4; ++j) { f[j] += p2; g[j] = get2(f[j]); }
    SCHED;
#pragma unroll
    for (int k = 0; k < 16; ++k) c[k] += p1;
    SCHED;
    u32x4 a2;
#pragma unroll
    for (int j = 0; j < 4; ++j) a2[j] = vperm(t2, __builtin_amdgcn_alignbyte(g[j][1], g[j][0], f[j]), e2[j][0]);
    *(LAS u32x4*)(wl + lane * 16) = a2;
    if ((tot & 0x7ffu) > 1024u) {
        const u32 cw = *(ub - 65 - lane);
        f[0] = 0;
#pragma unroll
        for (int j = 0; j < 4; ++j) { e2[j] = tab[(cw >> (8 * j)) & 0xffu]; f[j + 1] = f[j] + e2[j][1]; }
        const u32 p = a3 + (((tot >> 11) + wave_incl_scan(f[4]) - f[4]) & 0x1ffu);
#pragma unroll
        for (int j = 0; j < 4; ++j) a2[j] = vperm(t2, get_bytes(p + f[j]), e2[j][0]);
        *(LAS u32x4*)(wl + 1024 + lane * 16) = a2;
    }
    WAVE_FENCE();
#pragma unroll
    for (int k = 0; k < 16; ++k) g[k] = get2(c[k]);
    SCHED;
    u32x4 o;
#pragma unroll
    for (int k = 0; k < 16; ++k) {
        const u32 x = vperm(t1, __builtin_amdgcn_alignbyte(g[k][1], g[k][0], c[k]), e[k][0]);
        const u32 smk = d[k >> 2][k & 3];
        const u32 r0 = vperm(x, smk, 0x05010400u), r1 = vperm(x, smk, 0x07030602u);
        o[2 * (k & 1)] = __builtin_amdgcn_alignbit(r0, r0, 1);
        o[2 * (k & 1) + 1] = __builtin_amdgcn_alignbit(r1, r1, 1);
        if (k & 1) out(k >> 1, o);
    }
    WAVE_FENCE();
}

__device__ __forceinline__ char* frag_base(char* ws, bool isw, u32 s, u32 q, u32 rel) {
    return ws + (isw ? OFF_WF : OFF_XF) + (size_t)(s * 64u + (rel ? 6u + 14u * q + 2u * rel : 2u * q)) * 32768u;
}
__device__ __forceinline__ char* y_base(char* out, u32 i, u32 gp, u32 bj) { return out + (size_t)(i * 256u + bj * 128u) * 65536u + gp * 512u; }
__device__ __forceinline__ u32 gpanel(u32 r, u32 j) { return j < (u32)POWN ? r * 16u + j : ((j - POWN) & 7u) * 16u + POWN + ((j - POWN) >> 3); }
__device__ __forceinline__ u32 slot_in(bool isw, u32 s, u32 q, u32 dst) {
    return !isw ? s * 4u + q : (s < (u32)POWN ? 64u + (dst * POWN + s) * 4u + q : 64u + 28u * POWN + s * 4u + q);
}
__host__ __device__ __forceinline__ u32 batch_size(u32 bt) { return bt == 0u ? (u32)N0 : (bt == NB - 1u ? (u32)(NP - (NB - 1) * WB) : (u32)WB); }
struct InPos { bool isw; u32 s, q, bt; };
__host__ __device__ __forceinline__ InPos in_of(u32 c) {
    if (c < 4u * N0) { const u32 q = c / N0, pp = c - q * N0; return InPos{pp >= 16u, pp >= 16u ? pp - 16u : pp, q, 0u}; }
    const u32 bt = (c - 64u) / (4u * WB), r = c - 64u - bt * 4u * WB, q = r / batch_size(bt);
    return InPos{true, bt * WB + r - q * batch_size(bt), q, bt};
}

__device__ __forceinline__ void arrive(char* ws, u32 ci, u32 n, u32 call, int lane) {
    u32 old = 0;
    if (lane == 0) old = __hip_atomic_fetch_add((GAS u32*)(ws + OFF_RDY + ci * 128u), n, __ATOMIC_RELAXED, __HIP_MEMORY_SCOPE_SYSTEM);
    if (rfl(old) + n == call * ((ci & 7u) ? 56u : 32u) * batch_size(ci >> 3) && lane < 8)
        __hip_atomic_store((GAS u32*)(ws + OFF_FLAG + (u32)lane * 256u + ci * 4u), call, __ATOMIC_RELAXED, __HIP_MEMORY_SCOPE_SYSTEM);
}
__device__ __forceinline__ void load_x(const char* x, u32 s, u32 q, u32 b, int lane, u32x4 (&a)[8], bool on = true) {
    const rsrc_t r = mk_rsrc(x + (size_t)((s & 15u) * 256u + b * 32u) * 1024u + q * 256u, on ? 31u * 1024u + 256u : 0u);
#pragma unroll
    for (int i = 0; i < 8; ++i) a[i] = __builtin_amdgcn_raw_buffer_load_b128(r, (lane >> 2) * 1024 + ((lane & 3) ^ ((lane >> 5) << 1)) * 16 + (i & 3) * 64, (i >> 2) * 16384, 0);
}

__device__ __forceinline__ bool give_up(GAS u32* tmo, u32 call, u64 t0) {
    if (rfl(__hip_atomic_load(tmo, __ATOMIC_RELAXED, __HIP_MEMORY_SCOPE_AGENT)) == call || __builtin_amdgcn_s_memrealtime() - t0 > DEADLINE) {
        __hip_atomic_store(tmo, call, __ATOMIC_RELAXED, __HIP_MEMORY_SCOPE_AGENT);
        return true;
    }
    return false;
}


#define LDS_WAIT(cond, zz) do { u32 _s = 0; while (cond) { __builtin_amdgcn_s_sleep(zz); if (++_s >> 25) return; } } while (0)
__device__ __forceinline__ u32 lds_ld(LAS u32* p) { return rfl(*(volatile LAS u32*)p); }
__device__ __forceinline__ void dma_piece(rsrc_t r, u32 voff, u32 soff, u32 ldsaddr) {
    u32 keep;
    asm volatile("s_nop 4\n\ts_mov_b32 %0, m0\n\ts_mov_b32 m0, %1\n\ts_nop 0\n\tbuffer_load_dwordx4 %2, %3, %4 offen sc0 sc1 lds\n\ts_mov_b32 m0, %0"
                 : "=&s"(keep) : "s"(ldsaddr), "v"(voff), "s"(r), "s"(soff) : "memory");
}

#define DMA_P(j, w, off) "s_waitcnt vmcnt(" w ")\n\tbuffer_load_dwordx4 %2, %3, " off " sc0 sc1 lds\n\t.if %8 == " #j "\n\tds_max_u32 %6, %7\n\t.endif\n\t"
__device__ __forceinline__ void dma_block(rsrc_t r, u32 voff, u32 soff, u32 ldsaddr, u32 at, u32 val) {
    constexpr int W0 = PAT / 10000, W1 = PAT / 1000 % 10, W2 = PAT / 100 % 10, W3 = PAT / 10 % 10, W4 = PAT % 10;
    constexpr int JP = W0 <= 5 ? 0 : (W1 <= 6 ? 1 : (W2 <= 7 ? 2 : (W3 <= 8 ? 3 : 4)));
    u32 keep;
    asm volatile("s_nop 2\n\ts_mov_b32 %0, m0\n\ts_mov_b32 m0, %1\n\t"
                 DMA_P(0, "%9", "%4 offen") DMA_P(1, "%10", "%4 offen offset:1024") DMA_P(2, "%11", "%4 offen offset:2048") DMA_P(3, "%12", "%4 offen offset:3072")
                 "s_add_u32 m0, m0, 0x1000\n\t" DMA_P(4, "%13", "%5 offen") "s_mov_b32 m0, %0"
                 : "=&s"(keep) : "s"(ldsaddr), "v"(voff), "s"(r), "s"(soff), "s"(soff + 4096u), "v"(at), "v"(val), "n"(JP),
                   "n"(W0), "n"(W1), "n"(W2), "n"(W3), "n"(W4) : "memory", "scc");
}

__global__ void __launch_bounds__(512, 2) gemm_ar_fused(const Params p) {
    __shared__ __attribute__((aligned(16))) char smem[LDS_BYTES];
    LAS char* lds = (LAS char*)smem + BIG_OFF;
    LAS u32x2* tab = (LAS u32x2*)smem;
    LAS u32x2* tab3 = (LAS u32x2*)((LAS char*)smem + WL_OFF);
    const int tid = threadIdx.x, wid = (int)rfl((u32)tid >> 6), lane = tid & 63;
    const u32 bid = blockIdx.x;
    if (bid - 16u < 64u) {
        u64 pc; u32 left;
        asm volatile("s_getpc_b64 %0\n.Lcode_here:\n\ts_mov_b32 %1, .Lcode_far-.Lcode_here" : "=s"(pc), "=s"(left));
        const u32 o = ((bid - 16u) >> 3) * 8192u + (u32)tid * 16u;
        if (o + 16u <= left) { const u32x4 t = *(const GAS u32x4*)(pc + o); asm volatile("" :: "v"(t)); }
    }
    u32 rank = (u32)p.rank;
    char* ws = p.ws[0]; const char* px = p.x;
    SCHED;
    TabEntry te = table_entry((u32)tid & 255u);
    asm volatile("" : "+s"(rank), "+s"(ws), "+s"(px), "+v"(te.dep), "+v"(te.cmp), "+v"(te.n), "+v"(te.r0), "+v"(te.r1));
    LAS char* wl = (LAS char*)smem + WL_OFF + wid * WLDS;
    LAS u32* ctl = (LAS u32*)((LAS char*)smem + CTL_OFF);
    GAS u32* tmo = (GAS u32*)(ws + OFF_TMO);
    const u32 role = bid < (u32)NGEMM ? 0u : (bid < (u32)(NGEMM + NPULL) ? (1u + (bid - NGEMM) % 7u) | ((bid - NGEMM) / 7u) << 3 : (bid & 7u) << 3);
    const u32 pn = (role & 7u) ? 64u + (role >> 3) * PUSHB + 7u - (u32)wid : (bid < (u32)NGEMM ? bid : 15u) * 8u + (u32)wid;
    const u32 prel = (role & 7u) ? role & 7u : role >> 3;
    const bool early = (role & 7u) ? wid >= 8 - PUSHB : bid < 16u || prel != 0u;
    GAS u32* callp = (GAS u32*)(ws + OFF_CALL + (bid * 8u + wid) * 64u);
    const u32 callv = __hip_atomic_load(callp, __ATOMIC_RELAXED, __HIP_MEMORY_SCOPE_AGENT);
    u32x4 a[8];
    load_x(px, pn >> 3, 0u, pn & 7u, lane, a, early);
    SCHED;
    char* wsl = ws;
    if (tid < 256) tab[tid] = (u32x2){(role & 7u) ? te.dep : te.cmp, te.n};
    else { if (tid < 256 + CTL_WORDS) ctl[tid - 256] = 0u; if (role & 7u) tab3[tid - 256] = (u32x2){te.r0, te.r1}; }
    if (!(role & 7u)) ((LAS u32*)lds)[tid] = 0u;
#pragma unroll
    for (int k = 1; k < 8; ++k) wsl = lane == k ? p.ws[k] : wsl;
    asm volatile("s_waitcnt lgkmcnt(0)" : "+v"(wsl) :: "memory"); __builtin_amdgcn_s_barrier(); asm volatile("" ::: "memory");

    if (wid == 0 && (role & 7u)) {
        const u32 warm = __hip_atomic_load((const GAS u32*)(p.ws[role & 7u] + OFF_PS + (size_t)((lane & 1) * 8u + 8u - (role & 7u)) * (64u * FIXB)), __ATOMIC_RELAXED, __HIP_MEMORY_SCOPE_SYSTEM);
        asm volatile("" :: "v"(warm));
    }
    if (early && prel) {
        char* pws = p.ws[prel];
        const u32 call = rfl(callv) + 1u, at = (8u - prel) * 64u + pn - 64u;
        u32 v[8][4];
#pragma unroll
        for (int j = 0; j < 8; ++j)
#pragma unroll
            for (int e = 0; e < 4; ++e) v[j][e] = a[j][e];
        push_fixed(v, mk_rsrc(pws + OFF_PS + (size_t)((call & 1u) * 512u + at) * FIXB, FIXB), lane);
        if (lane == 0) __hip_atomic_store(callp, call, __ATOMIC_RELAXED, __HIP_MEMORY_SCOPE_AGENT);
        asm volatile("s_waitcnt vmcnt(0)" ::: "memory");
        if (lane == 0) __hip_atomic_store((GAS u32*)(pws + OFF_PF) + at, call, __ATOMIC_RELAXED, __HIP_MEMORY_SCOPE_SYSTEM);
    }
    if (role & 7u) {
        LAS u32* vready = ctl; LAS u32* endk = ctl + 1; LAS u32* landed = ctl + 2; LAS u32* prog = ctl + 5;
        LAS u32* vdone = ctl + 8; LAS u32* sdone = ctl + 16; LAS u32* ffree = ctl + 32;
        LAS u32* q = ctl + 64;
        const u32 l0 = la(lds), voff = (u32)lane * 16u;
        if (wid == 0) {
            const u32 call = rfl(callv) + 1u;
            if (lane == 0) __hip_atomic_store(callp, call, __ATOMIC_RELAXED, __HIP_MEMORY_SCOPE_AGENT);
            u32 xcc;
            asm volatile("s_getreg_b32 %0, hwreg(HW_REG_XCC_ID)" : "=s"(xcc));
            xcc &= 7u;
            const u32 row = (bid - NGEMM) >> 3, e4 = row * 4u + xcc;
            const u32 li = row == 6u ? (0x62651430u >> (4u * (bid & 7u))) & 7u : (xcc >= 4u ? 3u + ((row + xcc) & 3u) : (e4 < 21u ? e4 % 3u : e4 - 18u));
            const u32 rel = (p.links >> (4u * li)) & 7u, src = (rank + rel) & 7u;
            const char* pws = p.ws[rel];
            GAS u64* gran = (GAS u64*)(ws + OFF_GRAN) + (size_t)((call & 1u) * 8u + src) * NC;
            GAS u64* tick = (GAS u64*)(ws + OFF_TICK + rel * 128u);
            if (lane == 0) __hip_atomic_fetch_max(tick, (u64)call << 32, __ATOMIC_RELAXED, __HIP_MEMORY_SCOPE_AGENT);
            u32 c = 24u;
            bool here = false;
            for (u32 k = 0;; ++k) {
                if (!here) LDS_WAIT(lds_ld(prog) < k, 2);
                if (c < 8u || c - 16u < 8u) c = c < 8u ? c + 16u : c - 8u;
                else {
                    u32 tk = 0;
                    if (lane == 0) tk = (u32)__hip_atomic_fetch_add(tick, 1ull, __ATOMIC_RELAXED, __HIP_MEMORY_SCOPE_AGENT);
                    c = rfl(tk);
                    if (c >= 8u) c += 16u;
                }
                here = c - 8u < (u32)PUSHC;
                u32 fl = 2u, vb = 0, kc = 2u;
                const char* from = pws; char* to = ws;
                if (c < (u32)NC) {
                    if (c >= (u32)NCIN) {
                        const u32 yc = c - NCIN;
                        from = pws + OFF_SY + (size_t)yc * CH;
                        to = y_base(p.out, (yc >> 1) & 15u, src * 16u + (yc >> 5), yc & 1u);
                    } else {
                        const InPos ip = in_of(c);
                        from = pws + OFF_SI + (size_t)((call & 1u) * NTASK + slot_in(ip.isw, ip.s, ip.q, rank)) * CH;
                        to = frag_base(ws, ip.isw, ip.s, ip.q, rel);
                        kc = (ip.isw ? 1u : 0u) | ((ip.bt * 8u + ip.q + 1u) << 2) | (here ? (rel * 8u + c - 8u) << 8 : 0u);
                    }
                    if (k == 0u) {
                        const u32 warm = __hip_atomic_load((const GAS u32*)((lane & 1) ? to : from), __ATOMIC_RELAXED, __HIP_MEMORY_SCOPE_SYSTEM);
                        asm volatile("" :: "v"(warm));
                    }
                    const u64 t0 = __builtin_amdgcn_s_memrealtime();
                    for (u32 spin = 0;; ++spin) {
                        const u64 g = __hip_atomic_load(gran + c, __ATOMIC_RELAXED, __HIP_MEMORY_SCOPE_SYSTEM);
                        if (rfl((u32)g) == call) { fl = 0u; vb = rfl((u32)(g >> 32)); break; }
                        if ((spin & 15u) == 15u && give_up(tmo, call, t0)) { fl = 1u; break; }
                        __builtin_amdgcn_s_sleep(1);
                    }
                }
                LDS_WAIT(lds_ld(vdone + (k & (NV - 1))) < 8u * (k / NV), 2);
                if (lane == 0) {
                    *(volatile LAS u32x4*)(q + (k & (NV - 1)) * 8u + 4u) = (u32x4){(u32)(u64)to, (u32)((u64)to >> 32), kc, call};
                    *(volatile LAS u32x4*)(q + (k & (NV - 1)) * 8u) = (u32x4){(u32)(u64)from, (u32)((u64)from >> 32), fl | (here ? 7u : (c - 16u < 8u ? 0u : c % LVAR)) << 2 | ((k + 1u) << 5), vb};
                }
                if (fl & 2u) { if (lane == 0) *(volatile LAS u32*)endk = k + 1u; break; }
                if (!(fl & 1u)) {
                    const u32 vl = vb > (u32)VSLOT ? (u32)CHDR : (vb + 15u) & ~15u;
                    const rsrc_t r = mk_rsrc(from, CVAR + vl);
                    for (u32 o = 0; o < vl; o += 1024u) { dma_piece(r, voff + CVAR + o, 0u, rfl(l0 + VRING + (k & (NV - 1)) * VSLOT + o)); if (k && !here) asm volatile("s_waitcnt vmcnt(0)" ::: "memory"); }
                    asm volatile("s_waitcnt vmcnt(0)" ::: "memory");
                }
                if (lane == 0) *(volatile LAS u32*)vready = k + 1u;
            }
        } else if (wid == 1) {
            __builtin_amdgcn_s_setprio(3);
            const u32 la_landed = lane ? la(wl) + (u32)lane * 4u : la(landed);
            u32 n = 0;
            auto drain = [&]() { asm volatile("s_waitcnt vmcnt(0)" ::: "memory"); if (lane == 0) __hip_atomic_fetch_max(landed, n, __ATOMIC_RELAXED, __HIP_MEMORY_SCOPE_WORKGROUP); };
            u32x4 e = (u32x4){0, 0, 0, 0};
            u32 fr = 0, m1 = 0, m2 = 0;
            for (u32 k = 0;; ++k) {
                if (rfl(e[2]) >> 5 != k + 1u) {
                    e = *(volatile LAS u32x4*)(q + (k & (NV - 1)) * 8u);
                    if (rfl(e[2]) >> 5 != k + 1u) {
                        drain();
                        LDS_WAIT((e = *(volatile LAS u32x4*)(q + (k & (NV - 1)) * 8u), rfl(e[2]) >> 5 != k + 1u), 1);
                    }
                }
                const u32 fl = rfl(e[2]);
                if (fl & 2u) break;
                if (((fl >> 2) & 7u) == 7u) { n += 8u; fr = 0u; continue; }
                const rsrc_t r = mk_rsrc((const void*)(((u64)e[1] << 32) | e[0]), (fl & 1u) ? 0u : (u32)CVAR);
#pragma unroll
                for (u32 b = 0; b < 8u; ++b, ++n) {
                    const u32 sl = n & (NF - 1), m = n + 1u;
                    if (rfl(fr) + NF <= n && lds_ld(ffree + sl) + NF <= n) { drain(); LDS_WAIT(lds_ld(ffree + sl) + NF <= n, 1); }
                    fr = *(volatile LAS u32*)(ffree + (m & (NF - 1)));
                    if (b == 7u) e = *(volatile LAS u32x4*)(q + ((k + 1u) & (NV - 1)) * 8u);
                    if (b == ((fl >> 2) & 7u) && lane == 0) *(volatile LAS u32*)prog = k + 1u;
                    dma_block(r, voff, b * FIXB, rfl(l0 + sl * FIXB), la_landed, m2);
                    m2 = m1; m1 = m;
                }
            }
            drain();
        } else {
            u32 owed = 0xffffffffu, call = 0;
            auto settle = [&]() {
                if (owed != 0xffffffffu) {
                    asm volatile("s_waitcnt vmcnt(0)" ::: "memory");
                    u32 old = 0;
                    if (lane == 0) old = __hip_atomic_fetch_add(sdone + (owed & 7u), 1u, __ATOMIC_RELAXED, __HIP_MEMORY_SCOPE_WORKGROUP);
                    if ((rfl(old) & 7u) == 7u) arrive(ws, owed >> 3, 8u, call, lane);
                    owed = 0xffffffffu;
                }
            };
            for (u32 n = (u32)wid - 2u;; n += NDEC) {
                const u32 k = n >> 3, b = n & 7u, sl = n & (NF - 1);
                bool end = false;
                for (u32 spin = 0;; ++spin) {
                    const u32x4 st = *(volatile LAS u32x4*)ctl;
                    if (rfl(st[0]) > k && rfl(st[2]) > n) break;
                    if (rfl(st[1]) - 1u <= k || spin >> 25) { end = true; break; }
                    settle();
                    __builtin_amdgcn_s_sleep(2);
                }
                if (end) break;
                asm volatile("" ::: "memory");
                u32x4 d[5];
                const LAS char* vs = lds + VRING + (k & (NV - 1)) * VSLOT;
                const u32 h = *(const LAS u32*)(vs + b * 4u);
                const u32x4 e0 = *(const LAS u32x4*)(q + (k & (NV - 1)) * 8u), e1 = *(const LAS u32x4*)(q + (k & (NV - 1)) * 8u + 4u);
                d[4] = *(const LAS u32x4*)(lds + sl * FIXB + 4096 + lane * 16);
#pragma unroll
                for (int j = 0; j < 4; ++j) d[j] = *(const LAS u32x4*)(lds + sl * FIXB + j * 1024 + lane * 16);
                asm volatile("s_waitcnt lgkmcnt(0)" ::: "memory");
                if (lane == 0) __hip_atomic_fetch_max(ffree + sl, n + 1u, __ATOMIC_RELAXED, __HIP_MEMORY_SCOPE_WORKGROUP);
                asm volatile("" : "+v"(d[0]), "+v"(d[1]), "+v"(d[2]), "+v"(d[3]), "+v"(d[4]) :: "memory");
                const u32 hdr = rfl(h), off = (hdr >> 19) * 4u, kc = rfl(e1[2]);
                call = rfl(e1[3]);
                const bool far = rfl(e0[3]) > (u32)VSLOT;
                const LAS u32* var = (const LAS u32*)(vs + (off <= (u32)VSLOT ? off : 0u));
                u32x4 o[8];
                bool lost = (rfl(e0[2]) & 1u) != 0u;
                if ((kc >> 8) && !lost) {
                    const u32 at = (kc >> 8) * 8u + b;
                    const u64 t0 = __builtin_amdgcn_s_memrealtime();
                    while (rfl(__hip_atomic_load((GAS u32*)(ws + OFF_PF) + at, __ATOMIC_RELAXED, __HIP_MEMORY_SCOPE_SYSTEM)) != call) {
                        if (give_up(tmo, call, t0)) { lost = true; break; }
                        __builtin_amdgcn_s_sleep(2);
                    }
                    asm volatile("" ::: "memory");
                    const rsrc_t r = mk_rsrc((hdr & 0xffu) ? ws + OFF_PS + (size_t)((call & 1u) * 512u + at) * FIXB : (const char*)(((u64)e0[1] << 32) | e0[0]) + b * FIXB, FIXB);
#pragma unroll
                    for (int j = 0; j < 5; ++j) d[j] = __builtin_amdgcn_raw_buffer_load_b128(r, lane * 16 + j * 1024, 0, AUX_SYS);
                }
                if (lost) {
#pragma unroll
                    for (int g = 0; g < 8; ++g) o[g] = (u32x4){0x7fc07fc0u, 0x7fc07fc0u, 0x7fc07fc0u, 0x7fc07fc0u};
                } else {
                    if (far) {
                        const rsrc_t r = mk_rsrc((const void*)(((u64)e0[1] << 32) | e0[0]), CH);
#pragma unroll
                        for (int j = 0; j < 3; ++j) o[5 + j] = __builtin_amdgcn_raw_buffer_load_b128(r, (int)(CVAR + off) - ((hdr & 0xffu) ? 512 : 0) + j * 1024 + lane * 16, 0, AUX_SYS);
                        if (lane < VARB / 16) *(LAS u32x4*)(wl + A2B + A3B + lane * 16) = o[5];
                        WAVE_FENCE();
                        var = (const LAS u32*)(wl + A2B + A3B + 512);
                    }
                    if (hdr & 0xffu) decode_block(d, hdr, var, tab, tab3, wl, lane, [&](int g, u32x4 x) { o[g] = x; });
                    else {
#pragma unroll
                        for (int g = 0; g < 5; ++g) o[g] = d[g];
                        if (!far) {
#pragma unroll
                            for (int j = 0; j < 3; ++j)
#pragma unroll
                                for (int i = 0; i < 4; ++i) o[5 + j][i] = *(const volatile LAS u32*)(var + j * 256 + lane * 4 + i);
                        }
                    }
                }
                const Dest dst = dest_of((char*)(((u64)e1[1] << 32) | e1[0]), kc & 3u, b);
                asm volatile("s_waitcnt lgkmcnt(0)" ::: "memory");
                if (lane == 0) __hip_atomic_fetch_add(vdone + (k & (NV - 1)), 1u, __ATOMIC_RELAXED, __HIP_MEMORY_SCOPE_WORKGROUP);
                settle();
                const u32 loff = (u32)(lane >> 4) * dst.LH + (u32)(lane & 15) * 16u;
                if ((kc & 3u) == 2u) {
#pragma unroll
                    for (int g = 0; g < 8; ++g) store_group<AUX_OUT>(dst, loff, g, o[g]);
                } else {
#pragma unroll
                    for (int g = 0; g < 8; ++g) store_group<AUX_DEV>(dst, loff, g, o[g]);
                    owed = (k & 7u) | (((kc >> 2) & 63u) << 3);
                }
            }
            settle();
        }
        return;
    }

    {
        const bool packer = bid >= (u32)NGEMM;
        const u64 tb = __builtin_amdgcn_s_memrealtime();
        if (bid - 120u < 8u && wid == 0) {
            while (__builtin_amdgcn_s_memrealtime() - tb < 100u) __builtin_amdgcn_s_sleep(2);
            const u32 warm = __hip_atomic_load((const GAS u32*)(wsl + OFF_GRAN), __ATOMIC_RELAXED, __HIP_MEMORY_SCOPE_SYSTEM);
            asm volatile("" :: "v"(warm));
        }
        const u32 tend = packer ? (u32)NTASK : (u32)(NTASK - PACK_IN), b = (u32)wid;
        u32 it = 0;
        for (u32 task = packer ? (role >> 3) + NTASK - PACK_IN : bid; task < tend; task += packer ? (u32)NPACK : (u32)NGEMM, ++it) {
            if (task >= 16u) while (__builtin_amdgcn_s_memrealtime() - tb < (u64)(ENC_DELAY + (packer ? 0u : task * ENC_PACE / 64u))) __builtin_amdgcn_s_sleep(4);
            const u32 tw = task < 16u ? task << 4 : (p.task[task >> 1] >> ((task & 1u) * 16u)) & 0xffffu, c = tw >> 4, k = tw & 15u;
            const InPos ip = in_of(c);
            const u32 dd = (rank + k) & 7u;
            if (ip.isw) {
                const rsrc_t r = mk_rsrc(p.w + (size_t)(ip.q * 128u + (b >> 1) * 32u) * 65536u + (gpanel(dd, ip.s) * 256u + (b & 1u) * 128u) * 2u, 31u * 65536u + 256u);
#pragma unroll
                for (int i = 0; i < 8; ++i) a[i] = __builtin_amdgcn_raw_buffer_load_b128(r, (lane >> 4) * 8 * 65536 + (lane & 15) * 16, i * 65536, AUX_NT);
            } else if (task >= 16u) load_x(px, ip.s, ip.q, b, lane, a);
            u32 v[8][4];
            if (ip.isw) {
#pragma unroll
                for (int j = 0; j < 8; ++j)
#pragma unroll
                    for (int e = 0; e < 4; ++e) v[j][e] = vperm(a[2 * e + 1][j >> 1], a[2 * e][j >> 1], (j & 1) ? 0x07060302u : 0x05040100u);
            } else {
#pragma unroll
                for (int j = 0; j < 8; ++j)
#pragma unroll
                    for (int e = 0; e < 4; ++e) v[j][e] = a[j][e];
            }
            const bool own = (k & 7u) == 0u, send = k < 8u;
            const u32 call = rfl(callv) + 1u;
            LAS u32* cnt = (LAS u32*)lds + 4u * it;
            u32 total;
            if (send) {
                encode_block(v, mk_rsrc(ws + OFF_SI + (size_t)((call & 1u) * NTASK + slot_in(ip.isw, ip.s, ip.q, dd)) * CH, CH), b, cnt, tab, wl, lane);
                asm volatile("s_waitcnt vmcnt(0)" ::: "memory");
                if (chunk_done(cnt, lane, total) && (k ? (u32)lane == k : (u32)lane - 1u < 7u))
                    __hip_atomic_store((GAS u64*)(wsl + OFF_GRAN) + (size_t)((call & 1u) * 8u + rank) * NC + c,
                                       ((u64)total << 32) | call, __ATOMIC_RELAXED, __HIP_MEMORY_SCOPE_SYSTEM);
            }
            if (own) {
                const Dest dst = dest_of(frag_base(ws, ip.isw, ip.s, ip.q, 0u), ip.isw ? 1u : 0u, b);
#pragma unroll
                for (int g = 0; g < 8; ++g) store_group<AUX_DEV>(dst, (u32)lane * 16u, g, (u32x4){v[g][0], v[g][1], v[g][2], v[g][3]});
                asm volatile("s_waitcnt vmcnt(0)" ::: "memory");
                if (chunk_done(cnt + 2, lane, total)) arrive(ws, ip.bt * 8u, 8u, call, lane);
            }
        }
        if (lane == 0) __hip_atomic_store(callp, callv + 1u, __ATOMIC_RELAXED, __HIP_MEMORY_SCOPE_AGENT);
        if (packer) {
            if (wid >= 4) __builtin_amdgcn_s_setprio(1);
            const u32 call = rfl(callv) + 1u;
            GAS u32* tf = (GAS u32*)(ws + OFF_TILE + b * 4u);
            u32 seen = 0, nanv = 0;
            auto fetch = [&](u32 yc) {
                if (rfl(seen) != call) {
                    const u64 t0 = __builtin_amdgcn_s_memrealtime();
                    while (rfl(__hip_atomic_load(tf + (yc >> 1) * 32u, __ATOMIC_RELAXED, __HIP_MEMORY_SCOPE_AGENT)) != call) {
                        if (give_up(tmo, call, t0)) { nanv = 0x7fc07fc0u; break; }
                        __builtin_amdgcn_s_sleep(POLL_SLEEP);
                    }
                }
                asm volatile("" ::: "memory");
                const Dest sy = dest_of(y_base(p.out, (yc >> 1) & 15u, rank * 16u + (yc >> 5), yc & 1u), 2u, b);
#pragma unroll
                for (int i = 0; i < 8; ++i) a[i] = __builtin_amdgcn_raw_buffer_load_b128(sy.r, (lane >> 4) * 4 * 65536 + (lane & 15) * 16, ((i & 3) + (i >> 2) * 16) * 65536, AUX_SYS);
                seen = __hip_atomic_load(tf + ((yc + NPACK) >> 1) * 32u, __ATOMIC_RELAXED, __HIP_MEMORY_SCOPE_AGENT);
            };
            auto publish = [&](u32 yc) {
                u32 total;
                if (chunk_done((LAS u32*)lds + 4u * it, lane, total) && (u32)lane - 1u < 7u)
                    __hip_atomic_store((GAS u64*)(wsl + OFF_GRAN) + (size_t)((call & 1u) * 8u + rank) * NC + NCIN + yc,
                                       ((u64)total << 32) | call, __ATOMIC_RELAXED, __HIP_MEMORY_SCOPE_SYSTEM);
                ++it;
            };
            u32 yc = role >> 3;
            fetch(yc);
            for (;; yc += NPACK) {
                u32 v[8][4];
#pragma unroll
                for (int j = 0; j < 8; ++j)
#pragma unroll
                    for (int e = 0; e < 4; ++e) v[j][e] = a[j][e] | nanv;
                if (yc + NPACK < (u32)NCY) fetch(yc + NPACK);
                encode_block(v, mk_rsrc(ws + OFF_SY + (size_t)yc * CH, CH), b, (LAS u32*)lds + 4u * it, tab, wl, lane);
                asm volatile("s_waitcnt vmcnt(0)" ::: "memory");
                publish(yc);
                if (yc + NPACK >= (u32)NCY) break;
            }
            __builtin_amdgcn_s_setprio(0);
            if (NLAST == WB) return;
        }
    }

    const u32 call = rfl(callv) + 1u;
    const int wr = wid >> 2, wc = wid & 3;
    constexpr int HTB = 16384;
#define SA(b, h) (((b) * 2 + (h)) * HTB)
#define SB(b, h) ((4 + (b) * 2 + (h)) * HTB)
#define STAGE(bufoff, gbase) do { _Pragma("unroll") for (int _i = 0; _i < 2; ++_i) \
        __builtin_amdgcn_global_load_lds((const GAS u32*)((gbase) + voff + _i * 8192), (LAS u32*)(lds + (bufoff) + ldsw + _i * 8192), 16, 0, AUX_SYS); } while (0)
#define LDA(dst, b, h) do { _Pragma("unroll") for (int m = 0; m < 4; ++m) _Pragma("unroll") for (int k = 0; k < 2; ++k) dst[m][k] = *(const LAS bf16x8*)(lds + SA(b, h) + aoff + m * 2048 + k * 1024); } while (0)
#define LDB(dst, b, h) do { _Pragma("unroll") for (int n = 0; n < 2; ++n) _Pragma("unroll") for (int k = 0; k < 2; ++k) dst[n][k] = *(const LAS bf16x8*)(lds + SB(b, h) + boff + n * 2048 + k * 1024); } while (0)
#define MMA(ai, bj, At, Bt) do { _Pragma("unroll") for (int _x = 0; _x < 16; ++_x) { const int n = _x >> 3, m = (_x >> 1) & 3, k = (_x ^ (_x >> 1)) & 1; \
        acc[ai][bj][m][n] = __builtin_amdgcn_mfma_f32_16x16x32_bf16(Bt[n][k], At[m][k], acc[ai][bj][m][n], 0, 0, 0); SCHED; } } while (0)
#define MMA2(ai) do { __builtin_amdgcn_s_setprio(1); MMA(ai, 0, At, B0); MMA(ai, 1, At, B1); __builtin_amdgcn_s_setprio(0); } while (0)
#define WAIT_V(n) asm volatile("s_waitcnt vmcnt(" #n ")" ::: "memory")
#define WAIT_L(n) asm volatile("s_waitcnt lgkmcnt(" #n ")" ::: "memory")
#define BAR __builtin_amdgcn_s_barrier()
    const u32 voff = (u32)tid * 16u, ldsw = (u32)wid * 1024u;
    const int aoff = wr * 8192 + lane * 16, boff = wc * 4096 + ((lane & 15) * 4 + ((lane >> 4) ^ (((lane >> 3) & 1) << 1))) * 16;
    const u32 idx = bid >> 3, hh = (bid >> 2) & 1u;
    u32 ti = (bid & 3u) * 4u + (idx < 4u * (WB / 2) ? idx & 3u : 2u * hh + (idx & 1u)), tj = idx < 4u * (WB / 2) ? 2u * (idx >> 2) + hh : WB - 1u;
    constexpr size_t kstep = 32768, hstep = 16384;
    const GAS char* xf = (const GAS char*)frag_base(ws, false, 0u, 0u, 0u);
    const GAS char* wf = (const GAS char*)(ws + OFF_WF);
    GAS u32* flags = (GAS u32*)(ws + OFF_FLAG + (bid & 7u) * 256u) + ((u32)lane & 7u);
    auto arrived = [&](u32 bt) { return (u32)__builtin_ctz(((u32)__builtin_amdgcn_ballot_w64(__hip_atomic_load(flags + bt * 8u, __ATOMIC_RELAXED, __HIP_MEMORY_SCOPE_SYSTEM) != call) & 31u) | 32u); };
#define STAGE_FIRST(pA, pB) do { STAGE(SB(0, 0), pB); STAGE(SA(0, 0), pA); STAGE(SB(0, 1), (pB) + hstep); STAGE(SA(0, 1), (pA) + hstep); \
        STAGE(SB(1, 0), (pB) + kstep); STAGE(SA(1, 0), (pA) + kstep); STAGE(SB(1, 1), (pB) + hstep + kstep); } while (0)
    u32 nwait = 0, bad = 0, staged = 0;
    auto draw_ticket = [&]() { return __hip_atomic_fetch_add((GAS u32*)(ws + OFF_TICK + 8u * 128u), 1u, __ATOMIC_RELAXED, __HIP_MEMORY_SCOPE_AGENT) - (call - 1u) * (NGEMM + NPACK); };
    auto last_tile = [&](u32 tk, u32& i, u32& j) { const u32 gr = tk / (4u * NLAST), r = tk - gr * 4u * NLAST; i = (gr * 4u + (r & 3u)) & 15u; j = gr < 4u ? (NB - 1u) * WB + (r >> 2) : (u32)NP; };
    u32 round = 0;
    if (bid >= (u32)NGEMM) {
        if (tid == 0) *(volatile LAS u32*)(ctl + 2) = draw_ticket();
        asm volatile("s_waitcnt lgkmcnt(0)" ::: "memory"); BAR; asm volatile("" ::: "memory");
        last_tile(lds_ld(ctl + 2), ti, tj);
        round = NB - 1u;
    }
    for (; tj < (u32)NP; ++round) {
        const GAS char* cA = wf + (size_t)tj * (2u << 20);
        const GAS char* cB = xf + (size_t)ti * (2u << 20);
        u32 ni = ti, nj = tj + WB;
        f32x4 acc[2][2][4][2];
#pragma unroll
        for (int a = 0; a < 2; ++a)
#pragma unroll
            for (int b = 0; b < 2; ++b)
#pragma unroll
                for (int m = 0; m < 4; ++m)
#pragma unroll
                    for (int n = 0; n < 2; ++n) acc[a][b][m][n] = (f32x4){0.f, 0.f, 0.f, 0.f};
        bf16x8 At[4][2], B0[2][2], B1[2][2];
        for (u32 q0 = 0; q0 < 5u;) {
            const u32 k0 = q0 ? q0 * 14u - 6u : 0u;
            if (!staged) {
                if (wid == 0) {
                    const u64 t0 = __builtin_amdgcn_s_memrealtime();
                    u32 q1;
                    for (u32 spin = 0;; ++spin) {
                        q1 = arrived(round);
                        if (q1 > q0) break;
                        if ((spin & 7u) == 7u && give_up(tmo, call, t0)) { q1 = 5u | 8u; break; }
                        __builtin_amdgcn_s_sleep(POLL_SLEEP);
                    }
                    if (lane == 0) *(volatile LAS u32*)(ctl + (nwait & 1u)) = q1;
                    asm volatile("s_waitcnt lgkmcnt(0)" ::: "memory");
                }
                asm volatile("" ::: "memory"); BAR; asm volatile("" ::: "memory");
                const u32 te = lds_ld(ctl + (nwait & 1u));
                ++nwait; bad |= te >> 3;
                staged = te & 7u;
                STAGE_FIRST(cA + (size_t)k0 * kstep, cB + (size_t)k0 * kstep);
            }
            q0 = staged & 7u;
            const u32 k1 = q0 * 14u - 6u;
            if (wr == 1) BAR;
            bool lax = (staged >> 3) != 0u;
            if (lax) WAIT_V(22); else WAIT_V(6);
            BAR; BAR;
            for (u32 t = k0; t < k1; t += 2u) {
                const u32 t2 = t + 2u < k1 ? t + 2u : t;
                const GAS char* a1 = cA + (size_t)(t + 1u) * kstep;
                const GAS char* a2 = cA + (size_t)t2 * kstep; const GAS char* b2 = cB + (size_t)t2 * kstep;
                const GAS char* a3 = a2 + kstep; const GAS char* b3 = b2 + kstep;
                LDB(B0, 0, 0); LDB(B1, 0, 1); LDA(At, 0, 0); STAGE(SA(1, 1), a1 + hstep);
                if (lax) WAIT_V(24); else WAIT_V(8);
                WAIT_L(0); BAR; SCHED; MMA2(0); BAR; SCHED;
                LDA(At, 0, 1); STAGE(SB(0, 0), b2); STAGE(SB(0, 1), b2 + hstep); STAGE(SA(0, 0), a2);
                if (lax) WAIT_V(24); else WAIT_V(8);
                lax = false;
                WAIT_L(0); BAR; SCHED; MMA2(1); BAR; SCHED;
                LDB(B0, 1, 0); LDB(B1, 1, 1); LDA(At, 1, 0); STAGE(SA(0, 1), a2 + hstep);
                WAIT_V(8); WAIT_L(0); BAR; SCHED; MMA2(0); BAR; SCHED;
                LDA(At, 1, 1); STAGE(SB(1, 0), b3); STAGE(SB(1, 1), b3 + hstep); STAGE(SA(1, 0), a3);
                WAIT_V(8); WAIT_L(0); BAR; SCHED; MMA2(1); BAR; SCHED;
            }
            const bool last = q0 == 5u, draw = NLAST < WB && last && round == NB - 2u;
            if (wid == 0) {
                u32 tk = 0;
                if (draw && lane == 0) tk = draw_ticket();
                const u32 q1 = arrived(last ? round + 1u : round);
                if (lane == 0) *(volatile LAS u32*)(ctl + (nwait & 1u)) = q1 | (tk << 8);
                asm volatile("s_waitcnt lgkmcnt(0)" ::: "memory");
            }
            WAIT_V(0);
            if (wr == 0) BAR;
            BAR;
            asm volatile("" ::: "memory");
            staged = lds_ld(ctl + (nwait & 1u));
            ++nwait;
            if (draw) last_tile(staged >> 8, ni, nj);
            staged &= 7u;
            if (staged <= (last ? 0u : q0) || (last && nj >= (u32)NP)) staged = 0u;
            if (staged) STAGE_FIRST(last ? wf + (size_t)nj * (2u << 20) : cA + (size_t)k1 * kstep, last ? xf + (size_t)ni * (2u << 20) : cB + (size_t)k1 * kstep);
        }
#pragma unroll
        for (int bj = 0; bj < 2; ++bj) {
            const Dest dst = dest_of(y_base(p.out, ti, gpanel(rank, tj), bj), 2u, (u32)wid);
            const u32 loff = (u32)(lane >> 4) * dst.LH + (u32)(lane & 15) * 16u;
            u32x4 v[8];
#pragma unroll
            for (int g = 0; g < 8; ++g)
#pragma unroll
                for (int q = 0; q < 4; ++q) {
                    typedef float f32x2 __attribute__((ext_vector_type(2)));
                    typedef __bf16 bf16x2 __attribute__((ext_vector_type(2)));
                    const f32x2 f = {acc[q >> 1][bj][(q & 1) * 2][g >> 2][g & 3], acc[q >> 1][bj][(q & 1) * 2 + 1][g >> 2][g & 3]};
                    v[g][q] = __builtin_bit_cast(u32, __builtin_convertvector(f, bf16x2));
                }
            if (bad) {
#pragma unroll
                for (int g = 0; g < 8; ++g) v[g] = (u32x4){0x7fc07fc0u, 0x7fc07fc0u, 0x7fc07fc0u, 0x7fc07fc0u};
            }
            if (tj < (u32)POWN) {
#pragma unroll
                for (int g = 0; g < 8; ++g) store_group<AUX_DEV>(dst, loff, g, v[g]);
            } else if (staged) {
#pragma unroll
                for (int g = 0; g < 8; ++g) store_group<AUX_NT>(dst, loff, g, v[g]);
            } else {
#pragma unroll
                for (int g = 0; g < 8; ++g) store_group<AUX_OUT>(dst, loff, g, v[g]);
            }
        }
        if (tj < (u32)POWN) {
            asm volatile("s_waitcnt vmcnt(0)" ::: "memory");
            if (lane == 0) __hip_atomic_store((GAS u32*)(ws + OFF_TILE + (tj * 16u + ti) * 128u + (u32)wid * 4u), call, __ATOMIC_RELAXED, __HIP_MEMORY_SCOPE_AGENT);
        } else if (staged) staged |= 8u;
        ti = ni; tj = nj;
        asm volatile(".Lcode_far:");
    }
}

static void build_tables(Params& p) {
    int at[2] = {0, 3};
    p.links = 0;
    for (int rel = 1; rel < 8; ++rel) p.links |= (u32)rel << (4 * at[((0xb2 >> p.rank) ^ (0xb2 >> ((p.rank + rel) & 7))) & 1]++);
    unsigned short* t = (unsigned short*)p.task;
    int n = 0;
    for (u32 c = 0; c < (u32)NCIN; ++c) {
        const InPos ip = in_of(c);
        if (ip.isw && ip.s < (u32)POWN) for (u32 k = 1; k <= 8; ++k) t[n++] = (unsigned short)(c << 4 | k);
        else t[n++] = (unsigned short)(c << 4);
    }
}

extern "C" void kernel_launch(int rank, int nranks, ncclComm_t comm, hipStream_t stream,
                              void* const* d_in, const int* in_sizes, int n_in, void* d_out, int out_size,
                              const int* in_dtypes, int out_dtype, void* const* d_out_all,
                              void* const* d_ws_all, size_t ws_bytes) {
    if (nranks != NR || (unsigned)rank >= (unsigned)NR || ws_bytes < WS_NEED || n_in != 2) { fprintf(stderr, "unsupported configuration\n"); return; }
    Params p;
    p.x = (const char*)d_in[0]; p.w = (const char*)d_in[1]; p.out = (char*)d_out; p.rank = rank;
    for (int i = 0; i < NR; ++i) p.ws[i] = (char*)d_ws_all[(rank + i) % NR];
    build_tables(p);
    hipLaunchKernelGGL(gemm_ar_fused, dim3(256), dim3(512), 0, stream, p);
    HIP_CHECK(hipGetLastError());
}
```
